```python
import numpy as np

kernel_name = "gpu_a2a_gemm_m4096_k4096_n32768_f32_relu"


ABSMAX_RTOL = 0.001

COMPUTE_DTYPE = "float32"
EPILOGUE = "relu"
IN_QUANT = None
OUT_QUANT = None
IN_DTYPES = {"x": "float32", "w_mat": "float32"}
WORLD_SIZE = 8

def setup_inputs():

    x = np.random.randn(4096, 4096).astype(np.float32)
    w_mat = (np.random.randn(4096, 32768) / np.sqrt(4096)).astype(np.float32)
    return {"x": x, "w_mat": w_mat}

def _epilogue(y):
    return np.maximum(y, 0.0).astype(np.float32)

def reference(x, w_mat):
    y = (x @ w_mat).astype(np.float32)
    return _epilogue(y)
```

```cpp
#include <hip/hip_runtime.h>
#include <rccl/rccl.h>
#include <cstdio>
#include <cstdint>

#define HIP_CHECK(expr) do { hipError_t _e = (expr); if (_e != hipSuccess) { \
    fprintf(stderr, "HIP error %s at %s:%d\n", hipGetErrorString(_e), __FILE__, __LINE__); \
    return; } } while (0)

namespace {

typedef _Float16 f16x8 __attribute__((ext_vector_type(8)));
typedef _Float16 f16x4 __attribute__((ext_vector_type(4)));
typedef _Float16 f16x2 __attribute__((ext_vector_type(2)));
typedef float f32x4 __attribute__((ext_vector_type(4)));
typedef unsigned u32x4 __attribute__((ext_vector_type(4)));
typedef unsigned u32x2 __attribute__((ext_vector_type(2)));

constexpr int NR = 8;
constexpr int M_LOC = 512;
constexpr int KTOT = 4096, NTOT = 32768, KSL = KTOT / NR;
constexpr int BN = 128, BK = 64, NT = KTOT / BK;
constexpr int NWG = NTOT / BN;
constexpr int NTHR = 512;
constexpr int NBUF = 4;
constexpr int BBUF = BK * BN * 2;
constexpr int LDS_BYTES = NBUF * BBUF;

constexpr size_t WS_CTR = 0;
constexpr size_t WS_TMO = 4096;
constexpr size_t WS_FLAGS = 8192;
constexpr size_t WS_A = 1u << 20;
constexpr unsigned A_BYTES = M_LOC * KTOT * 2;
constexpr size_t WS_NEED = WS_A + 2 * (size_t)A_BYTES;
constexpr unsigned W_BYTES = (unsigned)((size_t)KTOT * NTOT * 4);
constexpr unsigned OOB = 0x40000000u;
constexpr unsigned long long DEADLINE_TICKS = 40000000ull;

struct Params {
    const float* x;
    const float* w;
    float* out;
    char* ws[NR];
    int rank;
};

#define SYS_RLX __ATOMIC_RELAXED, __HIP_MEMORY_SCOPE_SYSTEM
#define AGT_RLX __ATOMIC_RELAXED, __HIP_MEMORY_SCOPE_AGENT

__device__ __forceinline__ unsigned pk16(float a, float b) {
    f16x2 v; v[0] = (_Float16)a; v[1] = (_Float16)b;
    return __builtin_bit_cast(unsigned, v);
}

__device__ __forceinline__ bool wait_flags(const unsigned* fp, unsigned* tmo, unsigned call) {
    unsigned long long t0 = __builtin_amdgcn_s_memrealtime();
    for (;;) {
        unsigned v = __hip_atomic_load(fp, SYS_RLX);
        if (__all(v >= call)) return true;
        if (__hip_atomic_load(tmo, AGT_RLX) == call ||
            __builtin_amdgcn_s_memrealtime() - t0 > DEADLINE_TICKS) {
            __hip_atomic_store(tmo, call, AGT_RLX);
            return false;
        }
        __builtin_amdgcn_s_sleep(2);
    }
}

template <int V> struct IC { static constexpr int value = V; };

__global__ void __launch_bounds__(NTHR, 2) a2a_gemm_kernel(Params p) {
    extern __shared__ __attribute__((aligned(16))) char lds[];
    const int tid = threadIdx.x;
    const int lane = tid & 63;
    const int wave = __builtin_amdgcn_readfirstlane(tid >> 6);
    const int bid = blockIdx.x;
    const int rank = p.rank;
    char* myws = p.ws[rank];

    unsigned* ctr = (unsigned*)(myws + WS_CTR) + bid;
    unsigned call = __builtin_amdgcn_readfirstlane(__hip_atomic_load(ctr, AGT_RLX)) + 1u;

    const auto rsW = __builtin_amdgcn_make_buffer_rsrc((void*)p.w, 0, W_BYTES, 0x00020000);
    const auto rsA = __builtin_amdgcn_make_buffer_rsrc((void*)(myws + WS_A), 0, 2 * A_BYTES, 0x00020000);

    const int cgrp = lane & 31;
    const int hsel = ((lane >> 5) ^ (lane >> 3)) & 1;
    const unsigned wvoff = (unsigned)(((wave * 8 + hsel * 4) * NTOT + bid * BN + cgrp * 4) * 4);
    const unsigned bwr = (unsigned)((((wave >> 2) * 8 + (cgrp >> 4) * 4) * 1024) +
                                    ((wave & 3) * 16 + (cgrp & 15)) * 16 + hsel * 8);
    const unsigned brd = (unsigned)(lane * 16);
    const unsigned avoff = (unsigned)(wave * 8192 + lane * 16);

    auto kstep_of = [&](int t) { return (((rank + (t >> 3)) & 7) << 3) + (t & 7); };

    f32x4 wr[2][4];
    auto issue_w = [&](auto P, int t) {
        const unsigned so = (t < NT) ? (unsigned)kstep_of(t) * (unsigned)(BK * NTOT * 4) : OOB;
#pragma unroll
        for (int q = 0; q < 4; ++q)
            wr[P.value][q] = __builtin_bit_cast(f32x4,
                __builtin_amdgcn_raw_buffer_load_b128(rsW, wvoff, so + q * (NTOT * 4), 0));
    };
    auto write_b = [&](auto P, unsigned bufoff) {
#pragma unroll
        for (int e = 0; e < 4; ++e) {
            u32x2 v;
            v[0] = pk16(wr[P.value][0][e], wr[P.value][1][e]);
            v[1] = pk16(wr[P.value][2][e], wr[P.value][3][e]);
            *(u32x2*)(lds + bufoff + bwr + e * 1024) = v;
        }
    };

    {
        const int dest = bid >> 5, mt = bid & 31;
        const float* xs = p.x + (size_t)(bid * 16 + (lane & 15)) * KSL + (lane >> 4) * 8;
        f32x4 xv[2][2];
#pragma unroll
        for (int b = 0; b < 2; ++b) {
            const int uu = wave * 2 + b;
            xv[b][0] = *(const f32x4*)(xs + uu * 32);
            xv[b][1] = *(const f32x4*)(xs + uu * 32 + 4);
        }
        issue_w(IC<0>{}, 0);
        issue_w(IC<1>{}, 1);
        char* dws = p.ws[dest];
        const auto rsD = __builtin_amdgcn_make_buffer_rsrc((void*)(dws + WS_A), 0, 2 * A_BYTES, 0x00020000);
        const unsigned par = (call & 1u) * A_BYTES;
#pragma unroll
        for (int b = 0; b < 2; ++b) {
            u32x4 v;
            v[0] = pk16(xv[b][0][0], xv[b][0][1]);
            v[1] = pk16(xv[b][0][2], xv[b][0][3]);
            v[2] = pk16(xv[b][1][0], xv[b][1][1]);
            v[3] = pk16(xv[b][1][2], xv[b][1][3]);
            const unsigned so = par + (unsigned)(((((rank * 8 + wave) * 8 + (mt >> 2)) * 2 + b) * 4 + (mt & 3)) * 1024);
            __builtin_amdgcn_raw_buffer_store_b128(v, rsD, (unsigned)(lane * 16), so, 17  );
        }
        asm volatile("s_waitcnt vmcnt(0)" ::: "memory");
        __syncthreads();
        if (tid == 0) {
            __builtin_amdgcn_fence(__ATOMIC_RELEASE, "");
            asm volatile("s_waitcnt vmcnt(0)" ::: "memory");
            __hip_atomic_store((unsigned*)(dws + WS_FLAGS) + rank * 32 + mt, call, SYS_RLX);
            __hip_atomic_store(ctr, call, AGT_RLX);
        }
    }

    unsigned* tmo = (unsigned*)(myws + WS_TMO);
    const unsigned* flags = (const unsigned*)(myws + WS_FLAGS) + wave * 4 + (lane & 3);
    const unsigned apar = (call & 1u) * A_BYTES;
    bool ok = true;

    f16x8 ar[2][8];
    auto issue_a = [&](auto P, auto H, unsigned so) {
#pragma unroll
        for (int i = 0; i < 4; ++i)
            ar[P.value][H.value * 4 + i] = __builtin_bit_cast(f16x8,
                __builtin_amdgcn_raw_buffer_load_b128(rsA, avoff + i * 1024, so + H.value * 4096, 0));
    };
    auto issue_w2 = [&](auto P, unsigned so) {
#pragma unroll
        for (int q = 0; q < 4; ++q)
            wr[P.value][q] = __builtin_bit_cast(f32x4,
                __builtin_amdgcn_raw_buffer_load_b128(rsW, wvoff, so + q * (NTOT * 4), 0));
    };
    auto write_b2 = [&](auto P, auto E, unsigned bufoff) {
        constexpr int e = E.value;
        u32x2 v;
        v[0] = pk16(wr[P.value][0][e], wr[P.value][1][e]);
        v[1] = pk16(wr[P.value][2][e], wr[P.value][3][e]);
        *(u32x2*)(lds + bufoff + bwr + e * 1024) = v;
    };
#define SCHED() __builtin_amdgcn_sched_barrier(0)

    write_b(IC<0>{}, 0);
    issue_w(IC<0>{}, 2);
    write_b(IC<1>{}, BBUF);
    issue_w(IC<1>{}, 3);
    ok = wait_flags(flags + rank * 32, tmo, call);
    asm volatile("" ::: "memory");
    unsigned a_cur = apar + (unsigned)rank * (8u * 65536u);
    unsigned w_cur = (unsigned)rank * (8u * BK * NTOT * 4);
    issue_a(IC<0>{}, IC<0>{}, a_cur);
    issue_a(IC<0>{}, IC<1>{}, a_cur);

    f32x4 acc[4][8];
#pragma unroll
    for (int m = 0; m < 4; ++m)
#pragma unroll
        for (int n = 0; n < 8; ++n) acc[m][n] = f32x4{0.f, 0.f, 0.f, 0.f};

    asm volatile("s_waitcnt lgkmcnt(0)" ::: "memory");
    __builtin_amdgcn_s_barrier();
    asm volatile("" ::: "memory");

    f16x8 bq[4];
#pragma unroll
    for (int g = 0; g < 3; ++g) bq[g] = *(const f16x8*)(lds + brd + g * 1024);
    unsigned fl = 0;
    SCHED();

    for (int j = 0; j < NR; ++j) {
        const int nsrc = (rank + j + 1) & 7;
        const unsigned a_nxt = (j + 1 < NR) ? apar + (unsigned)nsrc * (8u * 65536u) : OOB;
        const unsigned w_nxt = (j + 1 < NR) ? (unsigned)nsrc * (8u * BK * NTOT * 4) : OOB;
        auto step = [&](auto I) {
            constexpr int i = I.value;
            constexpr int c = i & 1;
            constexpr unsigned rbuf = (i & 3) * BBUF, nbuf = ((i + 1) & 3) * BBUF, wbuf = ((i + 2) & 3) * BBUF;
            const unsigned aso = (i < 7) ? a_cur + (i + 1) * 65536u : a_nxt;
            const unsigned wso = (i < 4) ? w_cur + (i + 4) * (unsigned)(BK * NTOT * 4)
                                         : w_nxt + (i - 4) * (unsigned)(BK * NTOT * 4);
            if constexpr (i == 7) {
                if (!__all(fl >= call)) ok &= wait_flags(flags + nsrc * 32, tmo, call);
                asm volatile("" ::: "memory");
            }
#pragma unroll
            for (int g = 0; g < 16; ++g) {
                const int s = g >> 3, n = g & 7;
                bq[(g + 3) & 3] = *(const f16x8*)(lds + ((g + 3 < 16) ? rbuf : nbuf) + brd + ((g + 3) & 15) * 1024);
                if (g == 0) {
                    if constexpr (i == 5) fl = __hip_atomic_load(flags + nsrc * 32, SYS_RLX);
                    issue_a(IC<c ^ 1>{}, IC<0>{}, aso);
                }
                if (g == 1) issue_a(IC<c ^ 1>{}, IC<1>{}, aso);
                if (g == 2) { write_b2(IC<c>{}, IC<0>{}, wbuf); write_b2(IC<c>{}, IC<1>{}, wbuf); }
                if (g == 3) { write_b2(IC<c>{}, IC<2>{}, wbuf); write_b2(IC<c>{}, IC<3>{}, wbuf); }
                if (g == 4) issue_w2(IC<c>{}, wso);
#pragma unroll
                for (int m = 0; m < 4; ++m)
                    acc[m][n] = __builtin_amdgcn_mfma_f32_16x16x32_f16(ar[c][s * 4 + m], bq[g & 3], acc[m][n], 0, 0, 0);
                SCHED();
            }
            asm volatile("s_waitcnt lgkmcnt(3)" ::: "memory");
            __builtin_amdgcn_s_barrier();
            asm volatile("" ::: "memory");
            SCHED();
        };
        step(IC<0>{}); step(IC<1>{}); step(IC<2>{}); step(IC<3>{});
        step(IC<4>{}); step(IC<5>{}); step(IC<6>{}); step(IC<7>{});
        a_cur = a_nxt; w_cur = w_nxt;
    }

    if (__hip_atomic_load(tmo, AGT_RLX) == call) ok = false;
    const float bad = ok ? 0.f : __builtin_nanf("");
    float* op = p.out + (size_t)(wave * 64 + (lane >> 4) * 4) * NTOT + bid * BN + (lane & 15) * 4;
#pragma unroll
    for (int m = 0; m < 4; ++m)
#pragma unroll
        for (int r = 0; r < 4; ++r)
#pragma unroll
            for (int h = 0; h < 2; ++h) {
                f32x4 v;
#pragma unroll
                for (int e = 0; e < 4; ++e) v[e] = fmaxf(acc[m][h * 4 + e][r], 0.f) + bad;
                *(f32x4*)(op + (size_t)(m * 16 + r) * NTOT + h * 64) = v;
            }
}

}

extern "C" void kernel_launch(int rank, int nranks, ncclComm_t comm,
                              hipStream_t stream,
                              void* const* d_in, const int* in_sizes, int n_in,
                              void* d_out, int out_size,
                              const int* in_dtypes, int out_dtype,
                              void* const* d_out_all,
                              void* const* d_ws_all, size_t ws_bytes) {
    (void)comm; (void)in_sizes; (void)n_in; (void)out_size; (void)in_dtypes; (void)out_dtype; (void)d_out_all;
    if (nranks != NR || (unsigned)rank >= (unsigned)NR || ws_bytes < WS_NEED) {
        fprintf(stderr, "unsupported configuration\n");
        return;
    }
    Params p;
    p.x = (const float*)d_in[0];
    p.w = (const float*)d_in[1];
    p.out = (float*)d_out;
    for (int i = 0; i < NR; ++i) p.ws[i] = (char*)d_ws_all[i];
    p.rank = rank;
    a2a_gemm_kernel<<<dim3(NWG), dim3(NTHR), LDS_BYTES, stream>>>(p);
    HIP_CHECK(hipGetLastError());
}
```

```cpp
#include <hip/hip_runtime.h>
#include <rccl/rccl.h>
#include <cstdio>
#include <cstdint>

#define HIP_CHECK(expr) do { hipError_t _e = (expr); if (_e != hipSuccess) { \
    fprintf(stderr, "HIP error %s at %s:%d\n", hipGetErrorString(_e), __FILE__, __LINE__); \
    return; } } while (0)

namespace {

typedef unsigned u32x4 __attribute__((ext_vector_type(4)));

constexpr int NR = 8;
constexpr int KTOT = 4096, NTOT = 32768, KSL = KTOT / NR, M_LOC = 512;
constexpr int BN = 128;
constexpr int NWG = NTOT / BN;
constexpr int NTHR = 512;
constexpr int LDS_BYTES = 148480;

constexpr unsigned WS_CTRL_BYTES = 65536;
constexpr size_t WS_A = 1u << 20;
constexpr unsigned A_BYTES = M_LOC * KTOT * 2;
constexpr size_t WS_NEED = WS_A + 2 * (size_t)A_BYTES;

struct Params {
    const float* x;
    const float* w;
    float* out;
    char* ws[NR];
    int rank;
};

__device__ __forceinline__ u32x4 make_rsrc(const void* p, unsigned bytes) {
    const unsigned long long a = (unsigned long long)p;
    u32x4 r;
    r[0] = (unsigned)a;
    r[1] = (unsigned)(a >> 32) & 0xffffu;
    r[2] = bytes;
    r[3] = 0x00020000u;
    return r;
}

__global__ void __launch_bounds__(NTHR, 2) a2a_gemm_kernel(Params p) {
    __shared__ __attribute__((aligned(1024))) char lds[LDS_BYTES];
    const int tid = threadIdx.x;
    const int lane = tid & 63;
    const int wave = __builtin_amdgcn_readfirstlane(tid >> 6);
    const int bid = blockIdx.x;
    const int rank = p.rank;
    const int ct = (bid & 7) * 32 + (bid >> 3);
    char* myws = p.ws[0];
    const int dest = (rank + wave) & 7;
    char* dws = p.ws[wave & 7];

    const u32x4 rsW = make_rsrc(p.w, (unsigned)((size_t)KTOT * NTOT * 4));
    const u32x4 rsA = make_rsrc(myws + WS_A, 2 * A_BYTES);
    const u32x4 rsC = make_rsrc(myws, WS_CTRL_BYTES);
    const u32x4 rsD = make_rsrc(dws + WS_A, 2 * A_BYTES);
    const u32x4 rsDC = make_rsrc(dws, WS_CTRL_BYTES);
    const u32x4 rsX = make_rsrc(p.x, (unsigned)(KTOT * KSL * 4));
    const u32x4 rsO = make_rsrc(p.out, (unsigned)((size_t)M_LOC * NTOT * 4));

    const unsigned ldsb = (unsigned)(uintptr_t)(__attribute__((address_space(3))) char*)lds;
    const int cgrp = lane & 31;
    const int hsel = ((lane >> 5) ^ (lane >> 3)) & 1;
    constexpr int ILV = 0;
    const unsigned cbase = ILV ? (unsigned)((ct >> 1) * 1024 + (ct & 1) * 256) : (unsigned)(ct * 512);
    const unsigned wvoff = (unsigned)((wave * 8 + hsel * 4) * NTOT * 4) + cbase + (unsigned)((cgrp >> 4) * (ILV ? 512 : 256) + (cgrp & 15) * 16);
    const unsigned bwr = ldsb + (unsigned)((((wave >> 2) * 8 + (cgrp >> 4) * 4) * 1024) +
                                           ((wave & 3) * 16 + (cgrp & 15)) * 16 + hsel * 8);
    const unsigned brd = ldsb + (unsigned)(lane * 16);
    const unsigned avoff = (unsigned)(wave * 8192 + lane * 16);
    unsigned xvoff = (unsigned)(((dest * M_LOC + ((bid & 7) + 8 * (bid >> 6)) * 16 + (lane & 15)) * KSL + ((bid >> 3) & 7) * 64 + (lane >> 4) * 8) * 4);
    const unsigned ooff = (unsigned)((wave * 64 + (lane >> 4) * 4) * NTOT * 4) + cbase + (unsigned)((lane & 15) * 16);

    const int tt = (lane - 8) & 63, tq = (tt * 37) >> 8, tj = tt - 7 * tq + 1;
    const int ksrc = lane < 8 ? 0 : tj, kchunk = lane < 8 ? lane : tq;
    const unsigned cvoff = 128u * (unsigned)(((rank + ksrc) & 7) * 8 + kchunk);

    asm volatile(
        "s_mov_b32 s61, 0\n"
        "s_mov_b32 s58, 0\n"
        "s_lshl_b32 s63, %[bid], 3\n"
        "buffer_load_dwordx2 v[246:247], off, %[rsC], s63 sc1\n"
        "buffer_load_dwordx4 v[192:195], %[xvoff], %[rsX], 0 offen offset:0 nt\n"
        "buffer_load_dwordx4 v[196:199], %[xvoff], %[rsX], 0 offen offset:16 nt\n"
        "buffer_load_dwordx4 v[200:203], %[xvoff], %[rsX], 0 offen offset:128 nt\n"
        "buffer_load_dwordx4 v[204:207], %[xvoff], %[rsX], 0 offen offset:144 nt\n"
        "v_add_u32 v244, 0x10000, %[brd]\n"
        "v_add_u32 v245, 0x10000, %[bwr]\n"
        "s_lshl_b32 s72, %[rank], 3\n"
        "s_lshl_b32 s73, %[rank], 3\n"
        "s_add_u32 s73, s73, 1\n"
        "s_lshl_b32 s74, %[rank], 3\n"
        "s_add_u32 s74, s74, 2\n"
        "s_lshl_b32 s75, %[rank], 3\n"
        "s_add_u32 s75, s75, 3\n"
        "s_lshl_b32 s76, %[rank], 3\n"
        "s_add_u32 s76, s76, 4\n"
        "s_lshl_b32 s77, %[rank], 3\n"
        "s_add_u32 s77, s77, 5\n"
        "s_lshl_b32 s78, %[rank], 3\n"
        "s_add_u32 s78, s78, 6\n"
        "s_lshl_b32 s79, %[rank], 3\n"
        "s_add_u32 s79, s79, 7\n"
        "s_bfe_u32 s85, %[bid], 0x30003\n"
        "s_lshl_b32 s68, %[rank], 3\n"
        "s_add_u32 s68, s68, s85\n"
        "s_lshl_b32 s67, s68, 16\n"
        "s_lshl_b32 s68, s68, 7\n"
        "s_add_u32 s68, s68, 8192\n"
        "s_lshr_b32 s82, %[bid], 6\n"
        "s_lshl_b32 s82, s82, 14\n"
        "s_add_u32 s67, s67, s82\n"
        "s_bfe_u32 s82, %[bid], 0x10002\n"
        "s_lshl_b32 s82, s82, 13\n"
        "s_add_u32 s67, s67, s82\n"
        "s_and_b32 s82, %[bid], 3\n"
        "s_lshl_b32 s82, s82, 10\n"
        "s_add_u32 s67, s67, s82\n"
        "s_lshl_b32 s69, %[wave], 11\n"
        "s_add_u32 s69, s69, 131072\n"
        "s_lshl_b32 s83, %[rank], 26\n"
        "s_mov_b32 s65, 0x7fffffff\n"
        "s_mov_b32 s66, 0x7fffffff\n"
        "s_cmp_eq_u32 %[wave], 0\n"
        "s_cbranch_scc1 L_imm%=\n"
        "s_cmp_lt_u32 s85, 1\n"
        "s_cbranch_scc1 L_early%=\n"
        "s_add_u32 s80, s83, 0\n"
        "buffer_load_dwordx4 v[0:3], %[wvoff], %[rsW], s80 offen nt\n"
        "s_add_u32 s80, s83, 131072\n"
        "buffer_load_dwordx4 v[4:7], %[wvoff], %[rsW], s80 offen nt\n"
        "s_add_u32 s80, s83, 262144\n"
        "buffer_load_dwordx4 v[8:11], %[wvoff], %[rsW], s80 offen nt\n"
        "s_add_u32 s80, s83, 393216\n"
        "buffer_load_dwordx4 v[12:15], %[wvoff], %[rsW], s80 offen nt\n"
        "s_add_u32 s80, s83, 8388608\n"
        "buffer_load_dwordx4 v[16:19], %[wvoff], %[rsW], s80 offen nt\n"
        "s_add_u32 s80, s83, 8519680\n"
        "buffer_load_dwordx4 v[20:23], %[wvoff], %[rsW], s80 offen nt\n"
        "s_add_u32 s80, s83, 8650752\n"
        "buffer_load_dwordx4 v[24:27], %[wvoff], %[rsW], s80 offen nt\n"
        "s_add_u32 s80, s83, 8781824\n"
        "buffer_load_dwordx4 v[28:31], %[wvoff], %[rsW], s80 offen nt\n"
        "s_add_u32 s80, s83, 16777216\n"
        "buffer_load_dwordx4 v[32:35], %[wvoff], %[rsW], s80 offen nt\n"
        "s_add_u32 s80, s83, 16908288\n"
        "buffer_load_dwordx4 v[36:39], %[wvoff], %[rsW], s80 offen nt\n"
        "s_add_u32 s80, s83, 17039360\n"
        "buffer_load_dwordx4 v[40:43], %[wvoff], %[rsW], s80 offen nt\n"
        "s_add_u32 s80, s83, 17170432\n"
        "buffer_load_dwordx4 v[44:47], %[wvoff], %[rsW], s80 offen nt\n"
        "s_waitcnt vmcnt(12)\n"
        "v_readfirstlane_b32 s60, v246\n"
        "v_readfirstlane_b32 s59, v247\n"
        "s_nop 3\n"
        "s_add_u32 s60, s60, 1\n"
        "s_and_b32 s62, s60, 1\n"
        "s_lshl_b32 s62, s62, 22\n"
        "s_add_u32 s67, s67, s62\n"
        "s_add_u32 s84, s67, 4096\n"
        "s_lshl_b32 s71, s60, 5\n"
        "v_cvt_pk_f16_f32 v208, v192, v193\n"
        "v_cvt_pk_f16_f32 v209, v194, v195\n"
        "v_cvt_pk_f16_f32 v210, v196, v197\n"
        "v_cvt_pk_f16_f32 v211, v198, v199\n"
        "v_cvt_pk_f16_f32 v212, v200, v201\n"
        "v_cvt_pk_f16_f32 v213, v202, v203\n"
        "v_cvt_pk_f16_f32 v214, v204, v205\n"
        "v_cvt_pk_f16_f32 v215, v206, v207\n"
        "v_add_u32 v247, s69, %[brd]\n"
        "ds_write_b128 v247, v[208:211]\n"
        "ds_write_b128 v247, v[212:215] offset:1024\n"
        "s_sub_u32 s65, s85, 1\n"
        "s_lshl_b32 s65, s65, 1\n"
        "s_add_u32 s66, s65, 2\n"
        "s_branch L_wcvt%=\n"
        "L_early%=:\n"
        "s_waitcnt vmcnt(0)\n"
        "v_readfirstlane_b32 s60, v246\n"
        "v_readfirstlane_b32 s59, v247\n"
        "s_nop 3\n"
        "s_add_u32 s60, s60, 1\n"
        "s_and_b32 s62, s60, 1\n"
        "s_lshl_b32 s62, s62, 22\n"
        "s_add_u32 s67, s67, s62\n"
        "s_add_u32 s84, s67, 4096\n"
        "s_lshl_b32 s71, s60, 5\n"
        "v_cvt_pk_f16_f32 v208, v192, v193\n"
        "v_cvt_pk_f16_f32 v209, v194, v195\n"
        "v_cvt_pk_f16_f32 v210, v196, v197\n"
        "v_cvt_pk_f16_f32 v211, v198, v199\n"
        "v_cvt_pk_f16_f32 v212, v200, v201\n"
        "v_cvt_pk_f16_f32 v213, v202, v203\n"
        "v_cvt_pk_f16_f32 v214, v204, v205\n"
        "v_cvt_pk_f16_f32 v215, v206, v207\n"
        "buffer_store_dwordx4 v[208:211], %[brd], %[rsD], s67 offen sc0 sc1\n"
        "buffer_store_dwordx4 v[212:215], %[brd], %[rsD], s84 offen sc0 sc1\n"
        "s_add_u32 s80, s83, 0\n"
        "buffer_load_dwordx4 v[0:3], %[wvoff], %[rsW], s80 offen nt\n"
        "s_add_u32 s80, s83, 131072\n"
        "buffer_load_dwordx4 v[4:7], %[wvoff], %[rsW], s80 offen nt\n"
        "s_add_u32 s80, s83, 262144\n"
        "buffer_load_dwordx4 v[8:11], %[wvoff], %[rsW], s80 offen nt\n"
        "s_add_u32 s80, s83, 393216\n"
        "buffer_load_dwordx4 v[12:15], %[wvoff], %[rsW], s80 offen nt\n"
        "s_add_u32 s80, s83, 8388608\n"
        "buffer_load_dwordx4 v[16:19], %[wvoff], %[rsW], s80 offen nt\n"
        "s_add_u32 s80, s83, 8519680\n"
        "buffer_load_dwordx4 v[20:23], %[wvoff], %[rsW], s80 offen nt\n"
        "s_add_u32 s80, s83, 8650752\n"
        "buffer_load_dwordx4 v[24:27], %[wvoff], %[rsW], s80 offen nt\n"
        "s_add_u32 s80, s83, 8781824\n"
        "buffer_load_dwordx4 v[28:31], %[wvoff], %[rsW], s80 offen nt\n"
        "s_add_u32 s80, s83, 16777216\n"
        "buffer_load_dwordx4 v[32:35], %[wvoff], %[rsW], s80 offen nt\n"
        "s_add_u32 s80, s83, 16908288\n"
        "buffer_load_dwordx4 v[36:39], %[wvoff], %[rsW], s80 offen nt\n"
        "s_add_u32 s80, s83, 17039360\n"
        "buffer_load_dwordx4 v[40:43], %[wvoff], %[rsW], s80 offen nt\n"
        "s_add_u32 s80, s83, 17170432\n"
        "buffer_load_dwordx4 v[44:47], %[wvoff], %[rsW], s80 offen nt\n"
        "s_waitcnt vmcnt(12)\n"
        "v_mov_b32 v247, 1\n"
        "s_mov_b64 exec, 1\n"
        "buffer_atomic_add v247, off, %[rsDC], s68 sc1\n"
        "s_mov_b64 exec, -1\n"
        "s_branch L_wcvt%=\n"
        "L_imm%=:\n"
        "s_setprio 3\n"
        "s_waitcnt vmcnt(0)\n"
        "v_readfirstlane_b32 s60, v246\n"
        "v_readfirstlane_b32 s59, v247\n"
        "s_nop 3\n"
        "s_add_u32 s60, s60, 1\n"
        "s_and_b32 s62, s60, 1\n"
        "s_lshl_b32 s62, s62, 22\n"
        "s_add_u32 s67, s67, s62\n"
        "s_add_u32 s84, s67, 4096\n"
        "s_lshl_b32 s71, s60, 5\n"
        "v_cvt_pk_f16_f32 v208, v192, v193\n"
        "v_cvt_pk_f16_f32 v209, v194, v195\n"
        "v_cvt_pk_f16_f32 v210, v196, v197\n"
        "v_cvt_pk_f16_f32 v211, v198, v199\n"
        "v_cvt_pk_f16_f32 v212, v200, v201\n"
        "v_cvt_pk_f16_f32 v213, v202, v203\n"
        "v_cvt_pk_f16_f32 v214, v204, v205\n"
        "v_cvt_pk_f16_f32 v215, v206, v207\n"
        "buffer_store_dwordx4 v[208:211], %[brd], %[rsD], s67 offen sc0 sc1\n"
        "buffer_store_dwordx4 v[212:215], %[brd], %[rsD], s84 offen sc0 sc1\n"
        "s_waitcnt vmcnt(0)\n"
        "v_mov_b32 v247, 1\n"
        "s_mov_b64 exec, 1\n"
        "buffer_atomic_add v247, off, %[rsDC], s68 sc1\n"
        "s_mov_b64 exec, -1\n"
        "s_setprio 0\n"
        "s_add_u32 s80, s83, 0\n"
        "buffer_load_dwordx4 v[0:3], %[wvoff], %[rsW], s80 offen nt\n"
        "s_add_u32 s80, s83, 131072\n"
        "buffer_load_dwordx4 v[4:7], %[wvoff], %[rsW], s80 offen nt\n"
        "s_add_u32 s80, s83, 262144\n"
        "buffer_load_dwordx4 v[8:11], %[wvoff], %[rsW], s80 offen nt\n"
        "s_add_u32 s80, s83, 393216\n"
        "buffer_load_dwordx4 v[12:15], %[wvoff], %[rsW], s80 offen nt\n"
        "s_add_u32 s80, s83, 8388608\n"
        "buffer_load_dwordx4 v[16:19], %[wvoff], %[rsW], s80 offen nt\n"
        "s_add_u32 s80, s83, 8519680\n"
        "buffer_load_dwordx4 v[20:23], %[wvoff], %[rsW], s80 offen nt\n"
        "s_add_u32 s80, s83, 8650752\n"
        "buffer_load_dwordx4 v[24:27], %[wvoff], %[rsW], s80 offen nt\n"
        "s_add_u32 s80, s83, 8781824\n"
        "buffer_load_dwordx4 v[28:31], %[wvoff], %[rsW], s80 offen nt\n"
        "s_add_u32 s80, s83, 16777216\n"
        "buffer_load_dwordx4 v[32:35], %[wvoff], %[rsW], s80 offen nt\n"
        "s_add_u32 s80, s83, 16908288\n"
        "buffer_load_dwordx4 v[36:39], %[wvoff], %[rsW], s80 offen nt\n"
        "s_add_u32 s80, s83, 17039360\n"
        "buffer_load_dwordx4 v[40:43], %[wvoff], %[rsW], s80 offen nt\n"
        "s_add_u32 s80, s83, 17170432\n"
        "buffer_load_dwordx4 v[44:47], %[wvoff], %[rsW], s80 offen nt\n"
        "L_wcvt%=:\n"
        "s_waitcnt vmcnt(8)\n"
        "v_cvt_pk_f16_f32 v240, v0, v4\n"
        "v_cvt_pk_f16_f32 v241, v8, v12\n"
        "ds_write_b64 %[bwr], v[240:241] offset:0\n"
        "v_cvt_pk_f16_f32 v242, v1, v5\n"
        "v_cvt_pk_f16_f32 v243, v9, v13\n"
        "ds_write_b64 %[bwr], v[242:243] offset:1024\n"
        "v_cvt_pk_f16_f32 v240, v2, v6\n"
        "v_cvt_pk_f16_f32 v241, v10, v14\n"
        "ds_write_b64 %[bwr], v[240:241] offset:2048\n"
        "v_cvt_pk_f16_f32 v242, v3, v7\n"
        "v_cvt_pk_f16_f32 v243, v11, v15\n"
        "ds_write_b64 %[bwr], v[242:243] offset:3072\n"
        "s_waitcnt vmcnt(4)\n"
        "v_cvt_pk_f16_f32 v240, v16, v20\n"
        "v_cvt_pk_f16_f32 v241, v24, v28\n"
        "ds_write_b64 %[bwr], v[240:241] offset:16384\n"
        "v_cvt_pk_f16_f32 v242, v17, v21\n"
        "v_cvt_pk_f16_f32 v243, v25, v29\n"
        "ds_write_b64 %[bwr], v[242:243] offset:17408\n"
        "v_cvt_pk_f16_f32 v240, v18, v22\n"
        "v_cvt_pk_f16_f32 v241, v26, v30\n"
        "ds_write_b64 %[bwr], v[240:241] offset:18432\n"
        "v_cvt_pk_f16_f32 v242, v19, v23\n"
        "v_cvt_pk_f16_f32 v243, v27, v31\n"
        "ds_write_b64 %[bwr], v[242:243] offset:19456\n"
        "s_waitcnt vmcnt(0)\n"
        "v_cvt_pk_f16_f32 v240, v32, v36\n"
        "v_cvt_pk_f16_f32 v241, v40, v44\n"
        "ds_write_b64 %[bwr], v[240:241] offset:32768\n"
        "v_cvt_pk_f16_f32 v242, v33, v37\n"
        "v_cvt_pk_f16_f32 v243, v41, v45\n"
        "ds_write_b64 %[bwr], v[242:243] offset:33792\n"
        "v_cvt_pk_f16_f32 v240, v34, v38\n"
        "v_cvt_pk_f16_f32 v241, v42, v46\n"
        "ds_write_b64 %[bwr], v[240:241] offset:34816\n"
        "v_cvt_pk_f16_f32 v242, v35, v39\n"
        "v_cvt_pk_f16_f32 v243, v43, v47\n"
        "ds_write_b64 %[bwr], v[242:243] offset:35840\n"
        "L_wdone%=:\n"
        "s_cmp_lg_u32 %[wave], 4\n"
        "s_cbranch_scc1 L_ready%=\n"
        "v_mov_b32 v246, 0\n"
        "s_mov_b64 s[44:45], 0\n"
        "s_mov_b64 s[46:47], 0xff\n"
        "s_mov_b64 s[48:49], 255\n"
        "s_mov_b32 s50, 0\n"
        "s_mov_b32 s51, 1\n"
        "s_mov_b32 s52, 2\n"
        "s_mov_b32 s53, 3\n"
        "s_mov_b32 s54, 4\n"
        "s_mov_b32 s55, 5\n"
        "s_mov_b32 s56, 6\n"
        "s_mov_b32 s57, 7\n"
        "s_branch L_poll0%=\n"
        "L_poll1%=:\n"
        "s_mov_b32 s58, 1\n"
        "s_max_u32 s90, s58, 15\n"
        "s_bfm_b64 s[98:99], s90, 0\n"
        "s_cmp_ge_u32 s58, 64\n"
        "s_cselect_b64 s[98:99], -1, s[98:99]\n"
        "s_add_u32 s58, s58, 7\n"
        "s_andn2_b64 s[98:99], s[98:99], s[44:45]\n"
        "s_cbranch_scc0 L_ready%=\n"
        "s_mov_b64 exec, s[98:99]\n"
        "s_movk_i32 s90, 8192\n"
        "buffer_load_dword v246, %[cvoff], %[rsC], s90 offen sc0 sc1\n"
        "s_mov_b64 exec, -1\n"
        "L_ready%=:\n"
        "s_waitcnt lgkmcnt(0)\n"
        "s_barrier\n"
        "s_cmp_lg_u32 %[wave], 0\n"
        "s_cbranch_scc1 L_noctr%=\n"
        "v_mov_b32 v247, s60\n"
        "s_mov_b64 s[96:97], exec\n"
        "s_mov_b64 exec, 1\n"
        "buffer_store_dword v247, off, %[rsC], s63 sc1\n"
        "s_mov_b64 exec, s[96:97]\n"
        "L_noctr%=:\n"
        "s_lshr_b32 s70, %[bid], 3\n"
        "s_lshl_b32 s70, s70, 11\n"
        "s_add_u32 s70, s70, s62\n"
        "s_mov_b32 s64, 0\n"
        "s_cmp_lg_u32 %[wave], 5\n"
        "s_cbranch_scc1 L_nohot%=\n"
        "s_memrealtime s[92:93]\n"
        "s_waitcnt lgkmcnt(0)\n"
        "s_sub_u32 s93, s92, s59\n"
        "s_cmp_lt_u32 s93, 6000\n"
        "s_cselect_b32 s58, 1000, 0\n"
        "s_movk_i32 s50, 110\n"
        "s_cselect_b32 s50, 100, s50\n"
        "L_nohot%=:\n"
        "s_sub_u32 s70, s70, 32768\n"
        "s_add_u32 s70, s70, 0\n"
        "s_cmp_ge_u32 %[wave], 4\n"
        "s_cbranch_scc1 L_ypre%=\n"
        "s_lshl_b32 s82, s75, 23\n"
        "buffer_load_dwordx4 v[192:195], %[wvoff], %[rsW], s82 offen nt\n"
        "s_lshl_b32 s82, s75, 23\n"
        "s_add_u32 s82, s82, 131072\n"
        "buffer_load_dwordx4 v[196:199], %[wvoff], %[rsW], s82 offen nt\n"
        "s_lshl_b32 s80, s72, 16\n"
        "s_add_u32 s80, s80, s62\n"
        "s_add_u32 s86, s80, 2048\n"
        "buffer_load_dwordx4 v[128:131], %[avoff], %[rsA], s80 offen offset:0\n"
        "buffer_load_dwordx4 v[132:135], %[avoff], %[rsA], s80 offen offset:1024\n"
        "buffer_load_dwordx4 v[136:139], %[avoff], %[rsA], s86 offen offset:0\n"
        "buffer_load_dwordx4 v[140:143], %[avoff], %[rsA], s86 offen offset:1024\n"
        "s_lshl_b32 s82, s75, 23\n"
        "s_add_u32 s82, s82, 262144\n"
        "buffer_load_dwordx4 v[200:203], %[wvoff], %[rsW], s82 offen nt\n"
        "s_lshl_b32 s82, s75, 23\n"
        "s_add_u32 s82, s82, 393216\n"
        "buffer_load_dwordx4 v[204:207], %[wvoff], %[rsW], s82 offen nt\n"
        "s_lshl_b32 s82, s76, 23\n"
        "buffer_load_dwordx4 v[208:211], %[wvoff], %[rsW], s82 offen nt\n"
        "s_lshl_b32 s81, s72, 16\n"
        "s_add_u32 s81, s81, s62\n"
        "s_add_u32 s81, s81, 4096\n"
        "s_add_u32 s87, s81, 2048\n"
        "buffer_load_dwordx4 v[144:147], %[avoff], %[rsA], s81 offen offset:0\n"
        "buffer_load_dwordx4 v[148:151], %[avoff], %[rsA], s81 offen offset:1024\n"
        "buffer_load_dwordx4 v[152:155], %[avoff], %[rsA], s87 offen offset:0\n"
        "buffer_load_dwordx4 v[156:159], %[avoff], %[rsA], s87 offen offset:1024\n"
        "s_lshl_b32 s82, s76, 23\n"
        "s_add_u32 s82, s82, 131072\n"
        "buffer_load_dwordx4 v[212:215], %[wvoff], %[rsW], s82 offen nt\n"
        "s_lshl_b32 s82, s76, 23\n"
        "s_add_u32 s82, s82, 262144\n"
        "buffer_load_dwordx4 v[216:219], %[wvoff], %[rsW], s82 offen nt\n"
        "s_lshl_b32 s82, s76, 23\n"
        "s_add_u32 s82, s82, 393216\n"
        "buffer_load_dwordx4 v[220:223], %[wvoff], %[rsW], s82 offen nt\n"
        "v_mov_b32 v0, 0\n"
        "v_mov_b32 v1, 0\n"
        "v_mov_b32 v2, 0\n"
        "v_mov_b32 v3, 0\n"
        "v_mov_b32 v4, 0\n"
        "v_mov_b32 v5, 0\n"
        "v_mov_b32 v6, 0\n"
        "v_mov_b32 v7, 0\n"
        "v_mov_b32 v8, 0\n"
        "v_mov_b32 v9, 0\n"
        "v_mov_b32 v10, 0\n"
        "v_mov_b32 v11, 0\n"
        "v_mov_b32 v12, 0\n"
        "v_mov_b32 v13, 0\n"
        "v_mov_b32 v14, 0\n"
        "v_mov_b32 v15, 0\n"
        "v_mov_b32 v16, 0\n"
        "v_mov_b32 v17, 0\n"
        "v_mov_b32 v18, 0\n"
        "v_mov_b32 v19, 0\n"
        "v_mov_b32 v20, 0\n"
        "v_mov_b32 v21, 0\n"
        "v_mov_b32 v22, 0\n"
        "v_mov_b32 v23, 0\n"
        "v_mov_b32 v24, 0\n"
        "v_mov_b32 v25, 0\n"
        "v_mov_b32 v26, 0\n"
        "v_mov_b32 v27, 0\n"
        "v_mov_b32 v28, 0\n"
        "v_mov_b32 v29, 0\n"
        "v_mov_b32 v30, 0\n"
        "v_mov_b32 v31, 0\n"
        "v_mov_b32 v32, 0\n"
        "v_mov_b32 v33, 0\n"
        "v_mov_b32 v34, 0\n"
        "v_mov_b32 v35, 0\n"
        "v_mov_b32 v36, 0\n"
        "v_mov_b32 v37, 0\n"
        "v_mov_b32 v38, 0\n"
        "v_mov_b32 v39, 0\n"
        "v_mov_b32 v40, 0\n"
        "v_mov_b32 v41, 0\n"
        "v_mov_b32 v42, 0\n"
        "v_mov_b32 v43, 0\n"
        "v_mov_b32 v44, 0\n"
        "v_mov_b32 v45, 0\n"
        "v_mov_b32 v46, 0\n"
        "v_mov_b32 v47, 0\n"
        "v_mov_b32 v48, 0\n"
        "v_mov_b32 v49, 0\n"
        "v_mov_b32 v50, 0\n"
        "v_mov_b32 v51, 0\n"
        "v_mov_b32 v52, 0\n"
        "v_mov_b32 v53, 0\n"
        "v_mov_b32 v54, 0\n"
        "v_mov_b32 v55, 0\n"
        "v_mov_b32 v56, 0\n"
        "v_mov_b32 v57, 0\n"
        "v_mov_b32 v58, 0\n"
        "v_mov_b32 v59, 0\n"
        "v_mov_b32 v60, 0\n"
        "v_mov_b32 v61, 0\n"
        "v_mov_b32 v62, 0\n"
        "v_mov_b32 v63, 0\n"
        "v_mov_b32 v64, 0\n"
        "v_mov_b32 v65, 0\n"
        "v_mov_b32 v66, 0\n"
        "v_mov_b32 v67, 0\n"
        "v_mov_b32 v68, 0\n"
        "v_mov_b32 v69, 0\n"
        "v_mov_b32 v70, 0\n"
        "v_mov_b32 v71, 0\n"
        "v_mov_b32 v72, 0\n"
        "v_mov_b32 v73, 0\n"
        "v_mov_b32 v74, 0\n"
        "v_mov_b32 v75, 0\n"
        "v_mov_b32 v76, 0\n"
        "v_mov_b32 v77, 0\n"
        "v_mov_b32 v78, 0\n"
        "v_mov_b32 v79, 0\n"
        "v_mov_b32 v80, 0\n"
        "v_mov_b32 v81, 0\n"
        "v_mov_b32 v82, 0\n"
        "v_mov_b32 v83, 0\n"
        "v_mov_b32 v84, 0\n"
        "v_mov_b32 v85, 0\n"
        "v_mov_b32 v86, 0\n"
        "v_mov_b32 v87, 0\n"
        "v_mov_b32 v88, 0\n"
        "v_mov_b32 v89, 0\n"
        "v_mov_b32 v90, 0\n"
        "v_mov_b32 v91, 0\n"
        "v_mov_b32 v92, 0\n"
        "v_mov_b32 v93, 0\n"
        "v_mov_b32 v94, 0\n"
        "v_mov_b32 v95, 0\n"
        "v_mov_b32 v96, 0\n"
        "v_mov_b32 v97, 0\n"
        "v_mov_b32 v98, 0\n"
        "v_mov_b32 v99, 0\n"
        "v_mov_b32 v100, 0\n"
        "v_mov_b32 v101, 0\n"
        "v_mov_b32 v102, 0\n"
        "v_mov_b32 v103, 0\n"
        "v_mov_b32 v104, 0\n"
        "v_mov_b32 v105, 0\n"
        "v_mov_b32 v106, 0\n"
        "v_mov_b32 v107, 0\n"
        "v_mov_b32 v108, 0\n"
        "v_mov_b32 v109, 0\n"
        "v_mov_b32 v110, 0\n"
        "v_mov_b32 v111, 0\n"
        "v_mov_b32 v112, 0\n"
        "v_mov_b32 v113, 0\n"
        "v_mov_b32 v114, 0\n"
        "v_mov_b32 v115, 0\n"
        "v_mov_b32 v116, 0\n"
        "v_mov_b32 v117, 0\n"
        "v_mov_b32 v118, 0\n"
        "v_mov_b32 v119, 0\n"
        "v_mov_b32 v120, 0\n"
        "v_mov_b32 v121, 0\n"
        "v_mov_b32 v122, 0\n"
        "v_mov_b32 v123, 0\n"
        "v_mov_b32 v124, 0\n"
        "v_mov_b32 v125, 0\n"
        "v_mov_b32 v126, 0\n"
        "v_mov_b32 v127, 0\n"
        "ds_read_b128 v[224:227], %[brd] offset:0\n"
        "ds_read_b128 v[228:231], %[brd] offset:1024\n"
        "ds_read_b128 v[232:235], %[brd] offset:2048\n"
        "L_loopX%=:\n"
        "s_barrier\n"
        "s_setprio 2\n"
        "s_lshl_b32 s80, s73, 16\n"
        "s_add_u32 s80, s80, s62\n"
        "s_add_u32 s86, s80, 2048\n"
        "buffer_load_dwordx4 v[160:163], %[avoff], %[rsA], s80 offen offset:0\n"
        "s_waitcnt vmcnt(9)\n"
        "v_cvt_pk_f16_f32 v240, v192, v196\n"
        "v_cvt_pk_f16_f32 v241, v200, v204\n"
        "ds_write_b64 %[bwr], v[240:241] offset:49152\n"
        "buffer_load_dwordx4 v[164:167], %[avoff], %[rsA], s80 offen offset:1024\n"
        "v_cvt_pk_f16_f32 v242, v193, v197\n"
        "v_cvt_pk_f16_f32 v243, v201, v205\n"
        "ds_write_b64 %[bwr], v[242:243] offset:50176\n"
        "buffer_load_dwordx4 v[168:171], %[avoff], %[rsA], s86 offen offset:0\n"
        "v_cvt_pk_f16_f32 v240, v194, v198\n"
        "v_cvt_pk_f16_f32 v241, v202, v206\n"
        "ds_write_b64 %[bwr], v[240:241] offset:51200\n"
        "buffer_load_dwordx4 v[172:175], %[avoff], %[rsA], s86 offen offset:1024\n"
        "v_cvt_pk_f16_f32 v242, v195, v199\n"
        "v_cvt_pk_f16_f32 v243, v203, v207\n"
        "ds_write_b64 %[bwr], v[242:243] offset:52224\n"
        "s_lshl_b32 s81, s73, 16\n"
        "s_add_u32 s81, s81, s62\n"
        "s_add_u32 s81, s81, 4096\n"
        "s_add_u32 s87, s81, 2048\n"
        "buffer_load_dwordx4 v[176:179], %[avoff], %[rsA], s81 offen offset:0\n"
        "buffer_load_dwordx4 v[180:183], %[avoff], %[rsA], s81 offen offset:1024\n"
        "s_barrier\n"
        "s_cmp_eq_u32 s66, 0\n"
        "s_cbranch_scc1 L_flagcX0%=\n"
        "L_flagretcX0%=:\n"
        "s_cmp_eq_u32 s65, 0\n"
        "s_cbranch_scc1 L_pushcX0%=\n"
        "L_pushretcX0%=:\n"
        "s_lshl_b32 s82, s77, 23\n"
        "buffer_load_dwordx4 v[192:195], %[wvoff], %[rsW], s82 offen nt\n"
        "s_lshl_b32 s82, s77, 23\n"
        "s_add_u32 s82, s82, 131072\n"
        "buffer_load_dwordx4 v[196:199], %[wvoff], %[rsW], s82 offen nt\n"
        "s_setprio 0\n"
        "s_waitcnt lgkmcnt(6)\n"
        "v_mfma_f32_16x16x32_f16 v[0:3], v[128:131], v[224:227], v[0:3]\n"
        "ds_read_b128 v[236:239], %[brd] offset:3072\n"
        "v_mfma_f32_16x16x32_f16 v[32:35], v[132:135], v[224:227], v[32:35]\n"
        "v_mfma_f32_16x16x32_f16 v[64:67], v[136:139], v[224:227], v[64:67]\n"
        "v_mfma_f32_16x16x32_f16 v[96:99], v[140:143], v[224:227], v[96:99]\n"
        "s_waitcnt lgkmcnt(6)\n"
        "v_mfma_f32_16x16x32_f16 v[4:7], v[128:131], v[228:231], v[4:7]\n"
        "ds_read_b128 v[224:227], %[brd] offset:4096\n"
        "v_mfma_f32_16x16x32_f16 v[36:39], v[132:135], v[228:231], v[36:39]\n"
        "v_mfma_f32_16x16x32_f16 v[68:71], v[136:139], v[228:231], v[68:71]\n"
        "v_mfma_f32_16x16x32_f16 v[100:103], v[140:143], v[228:231], v[100:103]\n"
        "s_waitcnt lgkmcnt(6)\n"
        "v_mfma_f32_16x16x32_f16 v[8:11], v[128:131], v[232:235], v[8:11]\n"
        "ds_read_b128 v[228:231], %[brd] offset:5120\n"
        "v_mfma_f32_16x16x32_f16 v[40:43], v[132:135], v[232:235], v[40:43]\n"
        "v_mfma_f32_16x16x32_f16 v[72:75], v[136:139], v[232:235], v[72:75]\n"
        "v_mfma_f32_16x16x32_f16 v[104:107], v[140:143], v[232:235], v[104:107]\n"
        "s_waitcnt lgkmcnt(2)\n"
        "v_mfma_f32_16x16x32_f16 v[12:15], v[128:131], v[236:239], v[12:15]\n"
        "ds_read_b128 v[232:235], %[brd] offset:6144\n"
        "v_mfma_f32_16x16x32_f16 v[44:47], v[132:135], v[236:239], v[44:47]\n"
        "v_mfma_f32_16x16x32_f16 v[76:79], v[136:139], v[236:239], v[76:79]\n"
        "v_mfma_f32_16x16x32_f16 v[108:111], v[140:143], v[236:239], v[108:111]\n"
        "s_waitcnt lgkmcnt(2)\n"
        "v_mfma_f32_16x16x32_f16 v[16:19], v[128:131], v[224:227], v[16:19]\n"
        "ds_read_b128 v[236:239], %[brd] offset:7168\n"
        "v_mfma_f32_16x16x32_f16 v[48:51], v[132:135], v[224:227], v[48:51]\n"
        "v_mfma_f32_16x16x32_f16 v[80:83], v[136:139], v[224:227], v[80:83]\n"
        "v_mfma_f32_16x16x32_f16 v[112:115], v[140:143], v[224:227], v[112:115]\n"
        "s_waitcnt lgkmcnt(2)\n"
        "v_mfma_f32_16x16x32_f16 v[20:23], v[128:131], v[228:231], v[20:23]\n"
        "ds_read_b128 v[224:227], %[brd] offset:8192\n"
        "v_mfma_f32_16x16x32_f16 v[52:55], v[132:135], v[228:231], v[52:55]\n"
        "v_mfma_f32_16x16x32_f16 v[84:87], v[136:139], v[228:231], v[84:87]\n"
        "v_mfma_f32_16x16x32_f16 v[116:119], v[140:143], v[228:231], v[116:119]\n"
        "s_waitcnt lgkmcnt(2)\n"
        "v_mfma_f32_16x16x32_f16 v[24:27], v[128:131], v[232:235], v[24:27]\n"
        "ds_read_b128 v[228:231], %[brd] offset:9216\n"
        "v_mfma_f32_16x16x32_f16 v[56:59], v[132:135], v[232:235], v[56:59]\n"
        "v_mfma_f32_16x16x32_f16 v[88:91], v[136:139], v[232:235], v[88:91]\n"
        "v_mfma_f32_16x16x32_f16 v[120:123], v[140:143], v[232:235], v[120:123]\n"
        "s_waitcnt lgkmcnt(2)\n"
        "v_mfma_f32_16x16x32_f16 v[28:31], v[128:131], v[236:239], v[28:31]\n"
        "ds_read_b128 v[232:235], %[brd] offset:10240\n"
        "v_mfma_f32_16x16x32_f16 v[60:63], v[132:135], v[236:239], v[60:63]\n"
        "v_mfma_f32_16x16x32_f16 v[92:95], v[136:139], v[236:239], v[92:95]\n"
        "v_mfma_f32_16x16x32_f16 v[124:127], v[140:143], v[236:239], v[124:127]\n"
        "s_waitcnt vmcnt(11) lgkmcnt(2)\n"
        "v_mfma_f32_16x16x32_f16 v[0:3], v[144:147], v[224:227], v[0:3]\n"
        "ds_read_b128 v[236:239], %[brd] offset:11264\n"
        "v_mfma_f32_16x16x32_f16 v[32:35], v[148:151], v[224:227], v[32:35]\n"
        "v_mfma_f32_16x16x32_f16 v[64:67], v[152:155], v[224:227], v[64:67]\n"
        "v_mfma_f32_16x16x32_f16 v[96:99], v[156:159], v[224:227], v[96:99]\n"
        "s_waitcnt lgkmcnt(2)\n"
        "v_mfma_f32_16x16x32_f16 v[4:7], v[144:147], v[228:231], v[4:7]\n"
        "ds_read_b128 v[224:227], %[brd] offset:12288\n"
        "v_mfma_f32_16x16x32_f16 v[36:39], v[148:151], v[228:231], v[36:39]\n"
        "v_mfma_f32_16x16x32_f16 v[68:71], v[152:155], v[228:231], v[68:71]\n"
        "v_mfma_f32_16x16x32_f16 v[100:103], v[156:159], v[228:231], v[100:103]\n"
        "s_waitcnt lgkmcnt(2)\n"
        "v_mfma_f32_16x16x32_f16 v[8:11], v[144:147], v[232:235], v[8:11]\n"
        "ds_read_b128 v[228:231], %[brd] offset:13312\n"
        "v_mfma_f32_16x16x32_f16 v[40:43], v[148:151], v[232:235], v[40:43]\n"
        "v_mfma_f32_16x16x32_f16 v[72:75], v[152:155], v[232:235], v[72:75]\n"
        "v_mfma_f32_16x16x32_f16 v[104:107], v[156:159], v[232:235], v[104:107]\n"
        "s_barrier\n"
        "s_setprio 2\n"
        "buffer_load_dwordx4 v[184:187], %[avoff], %[rsA], s87 offen offset:0\n"
        "s_waitcnt vmcnt(9)\n"
        "v_cvt_pk_f16_f32 v240, v208, v212\n"
        "v_cvt_pk_f16_f32 v241, v216, v220\n"
        "ds_write_b64 v245, v[240:241] offset:0\n"
        "buffer_load_dwordx4 v[188:191], %[avoff], %[rsA], s87 offen offset:1024\n"
        "v_cvt_pk_f16_f32 v242, v209, v213\n"
        "v_cvt_pk_f16_f32 v243, v217, v221\n"
        "ds_write_b64 v245, v[242:243] offset:1024\n"
        "s_lshl_b32 s80, s74, 16\n"
        "s_add_u32 s80, s80, s62\n"
        "s_add_u32 s86, s80, 2048\n"
        "buffer_load_dwordx4 v[128:131], %[avoff], %[rsA], s80 offen offset:0\n"
        "v_cvt_pk_f16_f32 v240, v210, v214\n"
        "v_cvt_pk_f16_f32 v241, v218, v222\n"
        "ds_write_b64 v245, v[240:241] offset:2048\n"
        "buffer_load_dwordx4 v[132:135], %[avoff], %[rsA], s80 offen offset:1024\n"
        "v_cvt_pk_f16_f32 v242, v211, v215\n"
        "v_cvt_pk_f16_f32 v243, v219, v223\n"
        "ds_write_b64 v245, v[242:243] offset:3072\n"
        "buffer_load_dwordx4 v[136:139], %[avoff], %[rsA], s86 offen offset:0\n"
        "buffer_load_dwordx4 v[140:143], %[avoff], %[rsA], s86 offen offset:1024\n"
        "s_barrier\n"
        "s_lshl_b32 s82, s77, 23\n"
        "s_add_u32 s82, s82, 262144\n"
        "buffer_load_dwordx4 v[200:203], %[wvoff], %[rsW], s82 offen nt\n"
        "s_lshl_b32 s82, s77, 23\n"
        "s_add_u32 s82, s82, 393216\n"
        "buffer_load_dwordx4 v[204:207], %[wvoff], %[rsW], s82 offen nt\n"
        "s_lshl_b32 s82, s78, 23\n"
        "buffer_load_dwordx4 v[208:211], %[wvoff], %[rsW], s82 offen nt\n"
        "s_setprio 0\n"
        "s_waitcnt lgkmcnt(6)\n"
        "v_mfma_f32_16x16x32_f16 v[12:15], v[144:147], v[236:239], v[12:15]\n"
        "ds_read_b128 v[232:235], %[brd] offset:14336\n"
        "v_mfma_f32_16x16x32_f16 v[44:47], v[148:151], v[236:239], v[44:47]\n"
        "v_mov_b32 v247, 147456\n"
        "ds_read_b32 v240, v247\n"
        "v_mfma_f32_16x16x32_f16 v[76:79], v[152:155], v[236:239], v[76:79]\n"
        "v_mfma_f32_16x16x32_f16 v[108:111], v[156:159], v[236:239], v[108:111]\n"
        "s_waitcnt lgkmcnt(7)\n"
        "v_mfma_f32_16x16x32_f16 v[16:19], v[144:147], v[224:227], v[16:19]\n"
        "ds_read_b128 v[236:239], %[brd] offset:15360\n"
        "v_mfma_f32_16x16x32_f16 v[48:51], v[148:151], v[224:227], v[48:51]\n"
        "v_mfma_f32_16x16x32_f16 v[80:83], v[152:155], v[224:227], v[80:83]\n"
        "v_mfma_f32_16x16x32_f16 v[112:115], v[156:159], v[224:227], v[112:115]\n"
        "s_waitcnt lgkmcnt(7)\n"
        "v_mfma_f32_16x16x32_f16 v[20:23], v[144:147], v[228:231], v[20:23]\n"
        "ds_read_b128 v[224:227], %[brd] offset:16384\n"
        "v_mfma_f32_16x16x32_f16 v[52:55], v[148:151], v[228:231], v[52:55]\n"
        "v_mfma_f32_16x16x32_f16 v[84:87], v[152:155], v[228:231], v[84:87]\n"
        "v_mfma_f32_16x16x32_f16 v[116:119], v[156:159], v[228:231], v[116:119]\n"
        "s_waitcnt lgkmcnt(3)\n"
        "v_mfma_f32_16x16x32_f16 v[24:27], v[144:147], v[232:235], v[24:27]\n"
        "ds_read_b128 v[228:231], %[brd] offset:17408\n"
        "v_mfma_f32_16x16x32_f16 v[56:59], v[148:151], v[232:235], v[56:59]\n"
        "s_waitcnt lgkmcnt(3)\n"
        "v_readfirstlane_b32 s72, v240\n"
        "v_mfma_f32_16x16x32_f16 v[88:91], v[152:155], v[232:235], v[88:91]\n"
        "v_mfma_f32_16x16x32_f16 v[120:123], v[156:159], v[232:235], v[120:123]\n"
        "s_waitcnt lgkmcnt(2)\n"
        "v_mfma_f32_16x16x32_f16 v[28:31], v[144:147], v[236:239], v[28:31]\n"
        "ds_read_b128 v[232:235], %[brd] offset:18432\n"
        "v_mfma_f32_16x16x32_f16 v[60:63], v[148:151], v[236:239], v[60:63]\n"
        "v_mfma_f32_16x16x32_f16 v[92:95], v[152:155], v[236:239], v[92:95]\n"
        "v_mfma_f32_16x16x32_f16 v[124:127], v[156:159], v[236:239], v[124:127]\n"
        "s_waitcnt vmcnt(13) lgkmcnt(2)\n"
        "v_mfma_f32_16x16x32_f16 v[0:3], v[160:163], v[224:227], v[0:3]\n"
        "ds_read_b128 v[236:239], %[brd] offset:19456\n"
        "v_mfma_f32_16x16x32_f16 v[32:35], v[164:167], v[224:227], v[32:35]\n"
        "v_mfma_f32_16x16x32_f16 v[64:67], v[168:171], v[224:227], v[64:67]\n"
        "v_mfma_f32_16x16x32_f16 v[96:99], v[172:175], v[224:227], v[96:99]\n"
        "s_waitcnt lgkmcnt(2)\n"
        "v_mfma_f32_16x16x32_f16 v[4:7], v[160:163], v[228:231], v[4:7]\n"
        "ds_read_b128 v[224:227], %[brd] offset:20480\n"
        "v_mfma_f32_16x16x32_f16 v[36:39], v[164:167], v[228:231], v[36:39]\n"
        "v_mfma_f32_16x16x32_f16 v[68:71], v[168:171], v[228:231], v[68:71]\n"
        "v_mfma_f32_16x16x32_f16 v[100:103], v[172:175], v[228:231], v[100:103]\n"
        "s_waitcnt lgkmcnt(2)\n"
        "v_mfma_f32_16x16x32_f16 v[8:11], v[160:163], v[232:235], v[8:11]\n"
        "ds_read_b128 v[228:231], %[brd] offset:21504\n"
        "v_mfma_f32_16x16x32_f16 v[40:43], v[164:167], v[232:235], v[40:43]\n"
        "v_mfma_f32_16x16x32_f16 v[72:75], v[168:171], v[232:235], v[72:75]\n"
        "v_mfma_f32_16x16x32_f16 v[104:107], v[172:175], v[232:235], v[104:107]\n"
        "s_waitcnt lgkmcnt(2)\n"
        "v_mfma_f32_16x16x32_f16 v[12:15], v[160:163], v[236:239], v[12:15]\n"
        "ds_read_b128 v[232:235], %[brd] offset:22528\n"
        "v_mfma_f32_16x16x32_f16 v[44:47], v[164:167], v[236:239], v[44:47]\n"
        "v_mfma_f32_16x16x32_f16 v[76:79], v[168:171], v[236:239], v[76:79]\n"
        "v_mfma_f32_16x16x32_f16 v[108:111], v[172:175], v[236:239], v[108:111]\n"
        "s_waitcnt lgkmcnt(2)\n"
        "v_mfma_f32_16x16x32_f16 v[16:19], v[160:163], v[224:227], v[16:19]\n"
        "ds_read_b128 v[236:239], %[brd] offset:23552\n"
        "v_mfma_f32_16x16x32_f16 v[48:51], v[164:167], v[224:227], v[48:51]\n"
        "v_mfma_f32_16x16x32_f16 v[80:83], v[168:171], v[224:227], v[80:83]\n"
        "v_mfma_f32_16x16x32_f16 v[112:115], v[172:175], v[224:227], v[112:115]\n"
        "s_barrier\n"
        "s_setprio 2\n"
        "s_lshl_b32 s81, s74, 16\n"
        "s_add_u32 s81, s81, s62\n"
        "s_add_u32 s81, s81, 4096\n"
        "s_add_u32 s87, s81, 2048\n"
        "buffer_load_dwordx4 v[144:147], %[avoff], %[rsA], s81 offen offset:0\n"
        "buffer_load_dwordx4 v[148:151], %[avoff], %[rsA], s81 offen offset:1024\n"
        "buffer_load_dwordx4 v[152:155], %[avoff], %[rsA], s87 offen offset:0\n"
        "buffer_load_dwordx4 v[156:159], %[avoff], %[rsA], s87 offen offset:1024\n"
        "s_barrier\n"
        "s_lshl_b32 s82, s78, 23\n"
        "s_add_u32 s82, s82, 131072\n"
        "buffer_load_dwordx4 v[212:215], %[wvoff], %[rsW], s82 offen nt\n"
        "s_lshl_b32 s82, s78, 23\n"
        "s_add_u32 s82, s82, 262144\n"
        "buffer_load_dwordx4 v[216:219], %[wvoff], %[rsW], s82 offen nt\n"
        "s_lshl_b32 s82, s78, 23\n"
        "s_add_u32 s82, s82, 393216\n"
        "buffer_load_dwordx4 v[220:223], %[wvoff], %[rsW], s82 offen nt\n"
        "s_setprio 0\n"
        "s_waitcnt lgkmcnt(2)\n"
        "v_mfma_f32_16x16x32_f16 v[20:23], v[160:163], v[228:231], v[20:23]\n"
        "ds_read_b128 v[224:227], %[brd] offset:24576\n"
        "v_mfma_f32_16x16x32_f16 v[52:55], v[164:167], v[228:231], v[52:55]\n"
        "v_mfma_f32_16x16x32_f16 v[84:87], v[168:171], v[228:231], v[84:87]\n"
        "v_mfma_f32_16x16x32_f16 v[116:119], v[172:175], v[228:231], v[116:119]\n"
        "s_waitcnt lgkmcnt(2)\n"
        "v_mfma_f32_16x16x32_f16 v[24:27], v[160:163], v[232:235], v[24:27]\n"
        "ds_read_b128 v[228:231], %[brd] offset:25600\n"
        "v_mfma_f32_16x16x32_f16 v[56:59], v[164:167], v[232:235], v[56:59]\n"
        "v_mfma_f32_16x16x32_f16 v[88:91], v[168:171], v[232:235], v[88:91]\n"
        "v_mfma_f32_16x16x32_f16 v[120:123], v[172:175], v[232:235], v[120:123]\n"
        "s_waitcnt lgkmcnt(2)\n"
        "v_mfma_f32_16x16x32_f16 v[28:31], v[160:163], v[236:239], v[28:31]\n"
        "ds_read_b128 v[232:235], %[brd] offset:26624\n"
        "v_mfma_f32_16x16x32_f16 v[60:63], v[164:167], v[236:239], v[60:63]\n"
        "v_mfma_f32_16x16x32_f16 v[92:95], v[168:171], v[236:239], v[92:95]\n"
        "v_mfma_f32_16x16x32_f16 v[124:127], v[172:175], v[236:239], v[124:127]\n"
        "s_waitcnt vmcnt(14) lgkmcnt(2)\n"
        "v_mfma_f32_16x16x32_f16 v[0:3], v[176:179], v[224:227], v[0:3]\n"
        "ds_read_b128 v[236:239], %[brd] offset:27648\n"
        "v_mfma_f32_16x16x32_f16 v[32:35], v[180:183], v[224:227], v[32:35]\n"
        "v_mfma_f32_16x16x32_f16 v[64:67], v[184:187], v[224:227], v[64:67]\n"
        "v_mfma_f32_16x16x32_f16 v[96:99], v[188:191], v[224:227], v[96:99]\n"
        "s_waitcnt lgkmcnt(2)\n"
        "v_mfma_f32_16x16x32_f16 v[4:7], v[176:179], v[228:231], v[4:7]\n"
        "ds_read_b128 v[224:227], %[brd] offset:28672\n"
        "v_mfma_f32_16x16x32_f16 v[36:39], v[180:183], v[228:231], v[36:39]\n"
        "v_mov_b32 v247, 147460\n"
        "ds_read_b32 v240, v247\n"
        "v_mfma_f32_16x16x32_f16 v[68:71], v[184:187], v[228:231], v[68:71]\n"
        "v_mfma_f32_16x16x32_f16 v[100:103], v[188:191], v[228:231], v[100:103]\n"
        "s_waitcnt lgkmcnt(3)\n"
        "v_mfma_f32_16x16x32_f16 v[8:11], v[176:179], v[232:235], v[8:11]\n"
        "ds_read_b128 v[228:231], %[brd] offset:29696\n"
        "v_mfma_f32_16x16x32_f16 v[40:43], v[180:183], v[232:235], v[40:43]\n"
        "v_mfma_f32_16x16x32_f16 v[72:75], v[184:187], v[232:235], v[72:75]\n"
        "v_mfma_f32_16x16x32_f16 v[104:107], v[188:191], v[232:235], v[104:107]\n"
        "s_waitcnt lgkmcnt(3)\n"
        "v_mfma_f32_16x16x32_f16 v[12:15], v[176:179], v[236:239], v[12:15]\n"
        "ds_read_b128 v[232:235], %[brd] offset:30720\n"
        "v_mfma_f32_16x16x32_f16 v[44:47], v[180:183], v[236:239], v[44:47]\n"
        "v_mfma_f32_16x16x32_f16 v[76:79], v[184:187], v[236:239], v[76:79]\n"
        "v_mfma_f32_16x16x32_f16 v[108:111], v[188:191], v[236:239], v[108:111]\n"
        "s_waitcnt lgkmcnt(3)\n"
        "v_mfma_f32_16x16x32_f16 v[16:19], v[176:179], v[224:227], v[16:19]\n"
        "ds_read_b128 v[236:239], %[brd] offset:31744\n"
        "v_mfma_f32_16x16x32_f16 v[48:51], v[180:183], v[224:227], v[48:51]\n"
        "s_waitcnt lgkmcnt(3)\n"
        "v_readfirstlane_b32 s73, v240\n"
        "v_mfma_f32_16x16x32_f16 v[80:83], v[184:187], v[224:227], v[80:83]\n"
        "v_mfma_f32_16x16x32_f16 v[112:115], v[188:191], v[224:227], v[112:115]\n"
        "s_waitcnt lgkmcnt(2)\n"
        "v_mfma_f32_16x16x32_f16 v[20:23], v[176:179], v[228:231], v[20:23]\n"
        "ds_read_b128 v[224:227], %[brd] offset:32768\n"
        "v_mfma_f32_16x16x32_f16 v[52:55], v[180:183], v[228:231], v[52:55]\n"
        "v_mfma_f32_16x16x32_f16 v[84:87], v[184:187], v[228:231], v[84:87]\n"
        "v_mfma_f32_16x16x32_f16 v[116:119], v[188:191], v[228:231], v[116:119]\n"
        "s_waitcnt lgkmcnt(2)\n"
        "v_mfma_f32_16x16x32_f16 v[24:27], v[176:179], v[232:235], v[24:27]\n"
        "ds_read_b128 v[228:231], %[brd] offset:33792\n"
        "v_mfma_f32_16x16x32_f16 v[56:59], v[180:183], v[232:235], v[56:59]\n"
        "v_mfma_f32_16x16x32_f16 v[88:91], v[184:187], v[232:235], v[88:91]\n"
        "v_mfma_f32_16x16x32_f16 v[120:123], v[188:191], v[232:235], v[120:123]\n"
        "s_waitcnt lgkmcnt(2)\n"
        "v_mfma_f32_16x16x32_f16 v[28:31], v[176:179], v[236:239], v[28:31]\n"
        "ds_read_b128 v[232:235], %[brd] offset:34816\n"
        "v_mfma_f32_16x16x32_f16 v[60:63], v[180:183], v[236:239], v[60:63]\n"
        "v_mfma_f32_16x16x32_f16 v[92:95], v[184:187], v[236:239], v[92:95]\n"
        "v_mfma_f32_16x16x32_f16 v[124:127], v[188:191], v[236:239], v[124:127]\n"
        "s_barrier\n"
        "s_setprio 2\n"
        "s_lshl_b32 s80, s75, 16\n"
        "s_add_u32 s80, s80, s62\n"
        "s_add_u32 s86, s80, 2048\n"
        "buffer_load_dwordx4 v[160:163], %[avoff], %[rsA], s80 offen offset:0\n"
        "s_waitcnt vmcnt(9)\n"
        "v_cvt_pk_f16_f32 v240, v192, v196\n"
        "v_cvt_pk_f16_f32 v241, v200, v204\n"
        "ds_write_b64 v245, v[240:241] offset:16384\n"
        "buffer_load_dwordx4 v[164:167], %[avoff], %[rsA], s80 offen offset:1024\n"
        "v_cvt_pk_f16_f32 v242, v193, v197\n"
        "v_cvt_pk_f16_f32 v243, v201, v205\n"
        "ds_write_b64 v245, v[242:243] offset:17408\n"
        "buffer_load_dwordx4 v[168:171], %[avoff], %[rsA], s86 offen offset:0\n"
        "v_cvt_pk_f16_f32 v240, v194, v198\n"
        "v_cvt_pk_f16_f32 v241, v202, v206\n"
        "ds_write_b64 v245, v[240:241] offset:18432\n"
        "buffer_load_dwordx4 v[172:175], %[avoff], %[rsA], s86 offen offset:1024\n"
        "v_cvt_pk_f16_f32 v242, v195, v199\n"
        "v_cvt_pk_f16_f32 v243, v203, v207\n"
        "ds_write_b64 v245, v[242:243] offset:19456\n"
        "s_lshl_b32 s81, s75, 16\n"
        "s_add_u32 s81, s81, s62\n"
        "s_add_u32 s81, s81, 4096\n"
        "s_add_u32 s87, s81, 2048\n"
        "buffer_load_dwordx4 v[176:179], %[avoff], %[rsA], s81 offen offset:0\n"
        "buffer_load_dwordx4 v[180:183], %[avoff], %[rsA], s81 offen offset:1024\n"
        "s_barrier\n"
        "s_cmp_eq_u32 s66, 2\n"
        "s_cbranch_scc1 L_flagcX1%=\n"
        "L_flagretcX1%=:\n"
        "s_cmp_eq_u32 s65, 2\n"
        "s_cbranch_scc1 L_pushcX1%=\n"
        "L_pushretcX1%=:\n"
        "s_lshl_b32 s82, s79, 23\n"
        "buffer_load_dwordx4 v[192:195], %[wvoff], %[rsW], s82 offen nt\n"
        "s_lshl_b32 s82, s79, 23\n"
        "s_add_u32 s82, s82, 131072\n"
        "buffer_load_dwordx4 v[196:199], %[wvoff], %[rsW], s82 offen nt\n"
        "s_setprio 0\n"
        "s_waitcnt lgkmcnt(6)\n"
        "v_mfma_f32_16x16x32_f16 v[0:3], v[128:131], v[224:227], v[0:3]\n"
        "ds_read_b128 v[236:239], %[brd] offset:35840\n"
        "v_mfma_f32_16x16x32_f16 v[32:35], v[132:135], v[224:227], v[32:35]\n"
        "v_mfma_f32_16x16x32_f16 v[64:67], v[136:139], v[224:227], v[64:67]\n"
        "v_mfma_f32_16x16x32_f16 v[96:99], v[140:143], v[224:227], v[96:99]\n"
        "s_waitcnt lgkmcnt(6)\n"
        "v_mfma_f32_16x16x32_f16 v[4:7], v[128:131], v[228:231], v[4:7]\n"
        "ds_read_b128 v[224:227], %[brd] offset:36864\n"
        "v_mfma_f32_16x16x32_f16 v[36:39], v[132:135], v[228:231], v[36:39]\n"
        "v_mfma_f32_16x16x32_f16 v[68:71], v[136:139], v[228:231], v[68:71]\n"
        "v_mfma_f32_16x16x32_f16 v[100:103], v[140:143], v[228:231], v[100:103]\n"
        "s_waitcnt lgkmcnt(6)\n"
        "v_mfma_f32_16x16x32_f16 v[8:11], v[128:131], v[232:235], v[8:11]\n"
        "ds_read_b128 v[228:231], %[brd] offset:37888\n"
        "v_mfma_f32_16x16x32_f16 v[40:43], v[132:135], v[232:235], v[40:43]\n"
        "v_mfma_f32_16x16x32_f16 v[72:75], v[136:139], v[232:235], v[72:75]\n"
        "v_mfma_f32_16x16x32_f16 v[104:107], v[140:143], v[232:235], v[104:107]\n"
        "s_waitcnt lgkmcnt(2)\n"
        "v_mfma_f32_16x16x32_f16 v[12:15], v[128:131], v[236:239], v[12:15]\n"
        "ds_read_b128 v[232:235], %[brd] offset:38912\n"
        "v_mfma_f32_16x16x32_f16 v[44:47], v[132:135], v[236:239], v[44:47]\n"
        "v_mfma_f32_16x16x32_f16 v[76:79], v[136:139], v[236:239], v[76:79]\n"
        "v_mfma_f32_16x16x32_f16 v[108:111], v[140:143], v[236:239], v[108:111]\n"
        "s_waitcnt lgkmcnt(2)\n"
        "v_mfma_f32_16x16x32_f16 v[16:19], v[128:131], v[224:227], v[16:19]\n"
        "ds_read_b128 v[236:239], %[brd] offset:39936\n"
        "v_mfma_f32_16x16x32_f16 v[48:51], v[132:135], v[224:227], v[48:51]\n"
        "v_mfma_f32_16x16x32_f16 v[80:83], v[136:139], v[224:227], v[80:83]\n"
        "v_mfma_f32_16x16x32_f16 v[112:115], v[140:143], v[224:227], v[112:115]\n"
        "s_waitcnt lgkmcnt(2)\n"
        "v_mfma_f32_16x16x32_f16 v[20:23], v[128:131], v[228:231], v[20:23]\n"
        "ds_read_b128 v[224:227], %[brd] offset:40960\n"
        "v_mfma_f32_16x16x32_f16 v[52:55], v[132:135], v[228:231], v[52:55]\n"
        "v_mfma_f32_16x16x32_f16 v[84:87], v[136:139], v[228:231], v[84:87]\n"
        "v_mfma_f32_16x16x32_f16 v[116:119], v[140:143], v[228:231], v[116:119]\n"
        "s_waitcnt lgkmcnt(2)\n"
        "v_mfma_f32_16x16x32_f16 v[24:27], v[128:131], v[232:235], v[24:27]\n"
        "ds_read_b128 v[228:231], %[brd] offset:41984\n"
        "v_mfma_f32_16x16x32_f16 v[56:59], v[132:135], v[232:235], v[56:59]\n"
        "v_mfma_f32_16x16x32_f16 v[88:91], v[136:139], v[232:235], v[88:91]\n"
        "v_mfma_f32_16x16x32_f16 v[120:123], v[140:143], v[232:235], v[120:123]\n"
        "s_waitcnt lgkmcnt(2)\n"
        "v_mfma_f32_16x16x32_f16 v[28:31], v[128:131], v[236:239], v[28:31]\n"
        "ds_read_b128 v[232:235], %[brd] offset:43008\n"
        "v_mfma_f32_16x16x32_f16 v[60:63], v[132:135], v[236:239], v[60:63]\n"
        "v_mfma_f32_16x16x32_f16 v[92:95], v[136:139], v[236:239], v[92:95]\n"
        "v_mfma_f32_16x16x32_f16 v[124:127], v[140:143], v[236:239], v[124:127]\n"
        "s_waitcnt vmcnt(11) lgkmcnt(2)\n"
        "v_mfma_f32_16x16x32_f16 v[0:3], v[144:147], v[224:227], v[0:3]\n"
        "ds_read_b128 v[236:239], %[brd] offset:44032\n"
        "v_mfma_f32_16x16x32_f16 v[32:35], v[148:151], v[224:227], v[32:35]\n"
        "v_mfma_f32_16x16x32_f16 v[64:67], v[152:155], v[224:227], v[64:67]\n"
        "v_mfma_f32_16x16x32_f16 v[96:99], v[156:159], v[224:227], v[96:99]\n"
        "s_waitcnt lgkmcnt(2)\n"
        "v_mfma_f32_16x16x32_f16 v[4:7], v[144:147], v[228:231], v[4:7]\n"
        "ds_read_b128 v[224:227], %[brd] offset:45056\n"
        "v_mfma_f32_16x16x32_f16 v[36:39], v[148:151], v[228:231], v[36:39]\n"
        "v_mfma_f32_16x16x32_f16 v[68:71], v[152:155], v[228:231], v[68:71]\n"
        "v_mfma_f32_16x16x32_f16 v[100:103], v[156:159], v[228:231], v[100:103]\n"
        "s_waitcnt lgkmcnt(2)\n"
        "v_mfma_f32_16x16x32_f16 v[8:11], v[144:147], v[232:235], v[8:11]\n"
        "ds_read_b128 v[228:231], %[brd] offset:46080\n"
        "v_mfma_f32_16x16x32_f16 v[40:43], v[148:151], v[232:235], v[40:43]\n"
        "v_mfma_f32_16x16x32_f16 v[72:75], v[152:155], v[232:235], v[72:75]\n"
        "v_mfma_f32_16x16x32_f16 v[104:107], v[156:159], v[232:235], v[104:107]\n"
        "s_barrier\n"
        "s_setprio 2\n"
        "buffer_load_dwordx4 v[184:187], %[avoff], %[rsA], s87 offen offset:0\n"
        "s_waitcnt vmcnt(9)\n"
        "v_cvt_pk_f16_f32 v240, v208, v212\n"
        "v_cvt_pk_f16_f32 v241, v216, v220\n"
        "ds_write_b64 v245, v[240:241] offset:32768\n"
        "buffer_load_dwordx4 v[188:191], %[avoff], %[rsA], s87 offen offset:1024\n"
        "v_cvt_pk_f16_f32 v242, v209, v213\n"
        "v_cvt_pk_f16_f32 v243, v217, v221\n"
        "ds_write_b64 v245, v[242:243] offset:33792\n"
        "s_lshl_b32 s80, s76, 16\n"
        "s_add_u32 s80, s80, s62\n"
        "s_add_u32 s86, s80, 2048\n"
        "buffer_load_dwordx4 v[128:131], %[avoff], %[rsA], s80 offen offset:0\n"
        "v_cvt_pk_f16_f32 v240, v210, v214\n"
        "v_cvt_pk_f16_f32 v241, v218, v222\n"
        "ds_write_b64 v245, v[240:241] offset:34816\n"
        "buffer_load_dwordx4 v[132:135], %[avoff], %[rsA], s80 offen offset:1024\n"
        "v_cvt_pk_f16_f32 v242, v211, v215\n"
        "v_cvt_pk_f16_f32 v243, v219, v223\n"
        "ds_write_b64 v245, v[242:243] offset:35840\n"
        "buffer_load_dwordx4 v[136:139], %[avoff], %[rsA], s86 offen offset:0\n"
        "buffer_load_dwordx4 v[140:143], %[avoff], %[rsA], s86 offen offset:1024\n"
        "s_barrier\n"
        "s_lshl_b32 s82, s79, 23\n"
        "s_add_u32 s82, s82, 262144\n"
        "buffer_load_dwordx4 v[200:203], %[wvoff], %[rsW], s82 offen nt\n"
        "s_lshl_b32 s82, s79, 23\n"
        "s_add_u32 s82, s82, 393216\n"
        "buffer_load_dwordx4 v[204:207], %[wvoff], %[rsW], s82 offen nt\n"
        "s_lshl_b32 s82, s72, 23\n"
        "buffer_load_dwordx4 v[208:211], %[wvoff], %[rsW], s82 offen nt\n"
        "s_setprio 0\n"
        "s_waitcnt lgkmcnt(6)\n"
        "v_mfma_f32_16x16x32_f16 v[12:15], v[144:147], v[236:239], v[12:15]\n"
        "ds_read_b128 v[232:235], %[brd] offset:47104\n"
        "v_mfma_f32_16x16x32_f16 v[44:47], v[148:151], v[236:239], v[44:47]\n"
        "v_mov_b32 v247, 147464\n"
        "ds_read_b32 v240, v247\n"
        "v_mfma_f32_16x16x32_f16 v[76:79], v[152:155], v[236:239], v[76:79]\n"
        "v_mfma_f32_16x16x32_f16 v[108:111], v[156:159], v[236:239], v[108:111]\n"
        "s_waitcnt lgkmcnt(7)\n"
        "v_mfma_f32_16x16x32_f16 v[16:19], v[144:147], v[224:227], v[16:19]\n"
        "ds_read_b128 v[236:239], %[brd] offset:48128\n"
        "v_mfma_f32_16x16x32_f16 v[48:51], v[148:151], v[224:227], v[48:51]\n"
        "v_mfma_f32_16x16x32_f16 v[80:83], v[152:155], v[224:227], v[80:83]\n"
        "v_mfma_f32_16x16x32_f16 v[112:115], v[156:159], v[224:227], v[112:115]\n"
        "s_waitcnt lgkmcnt(7)\n"
        "v_mfma_f32_16x16x32_f16 v[20:23], v[144:147], v[228:231], v[20:23]\n"
        "ds_read_b128 v[224:227], %[brd] offset:49152\n"
        "v_mfma_f32_16x16x32_f16 v[52:55], v[148:151], v[228:231], v[52:55]\n"
        "v_mfma_f32_16x16x32_f16 v[84:87], v[152:155], v[228:231], v[84:87]\n"
        "v_mfma_f32_16x16x32_f16 v[116:119], v[156:159], v[228:231], v[116:119]\n"
        "s_waitcnt lgkmcnt(3)\n"
        "v_mfma_f32_16x16x32_f16 v[24:27], v[144:147], v[232:235], v[24:27]\n"
        "ds_read_b128 v[228:231], %[brd] offset:50176\n"
        "v_mfma_f32_16x16x32_f16 v[56:59], v[148:151], v[232:235], v[56:59]\n"
        "s_waitcnt lgkmcnt(3)\n"
        "v_readfirstlane_b32 s74, v240\n"
        "v_mfma_f32_16x16x32_f16 v[88:91], v[152:155], v[232:235], v[88:91]\n"
        "v_mfma_f32_16x16x32_f16 v[120:123], v[156:159], v[232:235], v[120:123]\n"
        "s_waitcnt lgkmcnt(2)\n"
        "v_mfma_f32_16x16x32_f16 v[28:31], v[144:147], v[236:239], v[28:31]\n"
        "ds_read_b128 v[232:235], %[brd] offset:51200\n"
        "v_mfma_f32_16x16x32_f16 v[60:63], v[148:151], v[236:239], v[60:63]\n"
        "v_mfma_f32_16x16x32_f16 v[92:95], v[152:155], v[236:239], v[92:95]\n"
        "v_mfma_f32_16x16x32_f16 v[124:127], v[156:159], v[236:239], v[124:127]\n"
        "s_waitcnt vmcnt(13) lgkmcnt(2)\n"
        "v_mfma_f32_16x16x32_f16 v[0:3], v[160:163], v[224:227], v[0:3]\n"
        "ds_read_b128 v[236:239], %[brd] offset:52224\n"
        "v_mfma_f32_16x16x32_f16 v[32:35], v[164:167], v[224:227], v[32:35]\n"
        "v_mfma_f32_16x16x32_f16 v[64:67], v[168:171], v[224:227], v[64:67]\n"
        "v_mfma_f32_16x16x32_f16 v[96:99], v[172:175], v[224:227], v[96:99]\n"
        "s_waitcnt lgkmcnt(2)\n"
        "v_mfma_f32_16x16x32_f16 v[4:7], v[160:163], v[228:231], v[4:7]\n"
        "ds_read_b128 v[224:227], %[brd] offset:53248\n"
        "v_mfma_f32_16x16x32_f16 v[36:39], v[164:167], v[228:231], v[36:39]\n"
        "v_mfma_f32_16x16x32_f16 v[68:71], v[168:171], v[228:231], v[68:71]\n"
        "v_mfma_f32_16x16x32_f16 v[100:103], v[172:175], v[228:231], v[100:103]\n"
        "s_waitcnt lgkmcnt(2)\n"
        "v_mfma_f32_16x16x32_f16 v[8:11], v[160:163], v[232:235], v[8:11]\n"
        "ds_read_b128 v[228:231], %[brd] offset:54272\n"
        "v_mfma_f32_16x16x32_f16 v[40:43], v[164:167], v[232:235], v[40:43]\n"
        "v_mfma_f32_16x16x32_f16 v[72:75], v[168:171], v[232:235], v[72:75]\n"
        "v_mfma_f32_16x16x32_f16 v[104:107], v[172:175], v[232:235], v[104:107]\n"
        "s_waitcnt lgkmcnt(2)\n"
        "v_mfma_f32_16x16x32_f16 v[12:15], v[160:163], v[236:239], v[12:15]\n"
        "ds_read_b128 v[232:235], %[brd] offset:55296\n"
        "v_mfma_f32_16x16x32_f16 v[44:47], v[164:167], v[236:239], v[44:47]\n"
        "v_mfma_f32_16x16x32_f16 v[76:79], v[168:171], v[236:239], v[76:79]\n"
        "v_mfma_f32_16x16x32_f16 v[108:111], v[172:175], v[236:239], v[108:111]\n"
        "s_waitcnt lgkmcnt(2)\n"
        "v_mfma_f32_16x16x32_f16 v[16:19], v[160:163], v[224:227], v[16:19]\n"
        "ds_read_b128 v[236:239], %[brd] offset:56320\n"
        "v_mfma_f32_16x16x32_f16 v[48:51], v[164:167], v[224:227], v[48:51]\n"
        "v_mfma_f32_16x16x32_f16 v[80:83], v[168:171], v[224:227], v[80:83]\n"
        "v_mfma_f32_16x16x32_f16 v[112:115], v[172:175], v[224:227], v[112:115]\n"
        "s_barrier\n"
        "s_setprio 2\n"
        "s_lshl_b32 s81, s76, 16\n"
        "s_add_u32 s81, s81, s62\n"
        "s_add_u32 s81, s81, 4096\n"
        "s_add_u32 s87, s81, 2048\n"
        "buffer_load_dwordx4 v[144:147], %[avoff], %[rsA], s81 offen offset:0\n"
        "buffer_load_dwordx4 v[148:151], %[avoff], %[rsA], s81 offen offset:1024\n"
        "buffer_load_dwordx4 v[152:155], %[avoff], %[rsA], s87 offen offset:0\n"
        "buffer_load_dwordx4 v[156:159], %[avoff], %[rsA], s87 offen offset:1024\n"
        "s_barrier\n"
        "s_lshl_b32 s82, s72, 23\n"
        "s_add_u32 s82, s82, 131072\n"
        "buffer_load_dwordx4 v[212:215], %[wvoff], %[rsW], s82 offen nt\n"
        "s_lshl_b32 s82, s72, 23\n"
        "s_add_u32 s82, s82, 262144\n"
        "buffer_load_dwordx4 v[216:219], %[wvoff], %[rsW], s82 offen nt\n"
        "s_lshl_b32 s82, s72, 23\n"
        "s_add_u32 s82, s82, 393216\n"
        "buffer_load_dwordx4 v[220:223], %[wvoff], %[rsW], s82 offen nt\n"
        "s_setprio 0\n"
        "s_waitcnt lgkmcnt(2)\n"
        "v_mfma_f32_16x16x32_f16 v[20:23], v[160:163], v[228:231], v[20:23]\n"
        "ds_read_b128 v[224:227], %[brd] offset:57344\n"
        "v_mfma_f32_16x16x32_f16 v[52:55], v[164:167], v[228:231], v[52:55]\n"
        "v_mfma_f32_16x16x32_f16 v[84:87], v[168:171], v[228:231], v[84:87]\n"
        "v_mfma_f32_16x16x32_f16 v[116:119], v[172:175], v[228:231], v[116:119]\n"
        "s_waitcnt lgkmcnt(2)\n"
        "v_mfma_f32_16x16x32_f16 v[24:27], v[160:163], v[232:235], v[24:27]\n"
        "ds_read_b128 v[228:231], %[brd] offset:58368\n"
        "v_mfma_f32_16x16x32_f16 v[56:59], v[164:167], v[232:235], v[56:59]\n"
        "v_mfma_f32_16x16x32_f16 v[88:91], v[168:171], v[232:235], v[88:91]\n"
        "v_mfma_f32_16x16x32_f16 v[120:123], v[172:175], v[232:235], v[120:123]\n"
        "s_waitcnt lgkmcnt(2)\n"
        "v_mfma_f32_16x16x32_f16 v[28:31], v[160:163], v[236:239], v[28:31]\n"
        "ds_read_b128 v[232:235], %[brd] offset:59392\n"
        "v_mfma_f32_16x16x32_f16 v[60:63], v[164:167], v[236:239], v[60:63]\n"
        "v_mfma_f32_16x16x32_f16 v[92:95], v[168:171], v[236:239], v[92:95]\n"
        "v_mfma_f32_16x16x32_f16 v[124:127], v[172:175], v[236:239], v[124:127]\n"
        "s_waitcnt vmcnt(14) lgkmcnt(2)\n"
        "v_mfma_f32_16x16x32_f16 v[0:3], v[176:179], v[224:227], v[0:3]\n"
        "ds_read_b128 v[236:239], %[brd] offset:60416\n"
        "v_mfma_f32_16x16x32_f16 v[32:35], v[180:183], v[224:227], v[32:35]\n"
        "v_mfma_f32_16x16x32_f16 v[64:67], v[184:187], v[224:227], v[64:67]\n"
        "v_mfma_f32_16x16x32_f16 v[96:99], v[188:191], v[224:227], v[96:99]\n"
        "s_waitcnt lgkmcnt(2)\n"
        "v_mfma_f32_16x16x32_f16 v[4:7], v[176:179], v[228:231], v[4:7]\n"
        "ds_read_b128 v[224:227], %[brd] offset:61440\n"
        "v_mfma_f32_16x16x32_f16 v[36:39], v[180:183], v[228:231], v[36:39]\n"
        "v_mov_b32 v247, 147468\n"
        "ds_read_b32 v240, v247\n"
        "v_mfma_f32_16x16x32_f16 v[68:71], v[184:187], v[228:231], v[68:71]\n"
        "v_mfma_f32_16x16x32_f16 v[100:103], v[188:191], v[228:231], v[100:103]\n"
        "s_waitcnt lgkmcnt(3)\n"
        "v_mfma_f32_16x16x32_f16 v[8:11], v[176:179], v[232:235], v[8:11]\n"
        "ds_read_b128 v[228:231], %[brd] offset:62464\n"
        "v_mfma_f32_16x16x32_f16 v[40:43], v[180:183], v[232:235], v[40:43]\n"
        "v_mfma_f32_16x16x32_f16 v[72:75], v[184:187], v[232:235], v[72:75]\n"
        "v_mfma_f32_16x16x32_f16 v[104:107], v[188:191], v[232:235], v[104:107]\n"
        "s_waitcnt lgkmcnt(3)\n"
        "v_mfma_f32_16x16x32_f16 v[12:15], v[176:179], v[236:239], v[12:15]\n"
        "ds_read_b128 v[232:235], %[brd] offset:63488\n"
        "v_mfma_f32_16x16x32_f16 v[44:47], v[180:183], v[236:239], v[44:47]\n"
        "v_mfma_f32_16x16x32_f16 v[76:79], v[184:187], v[236:239], v[76:79]\n"
        "v_mfma_f32_16x16x32_f16 v[108:111], v[188:191], v[236:239], v[108:111]\n"
        "s_waitcnt lgkmcnt(3)\n"
        "v_mfma_f32_16x16x32_f16 v[16:19], v[176:179], v[224:227], v[16:19]\n"
        "ds_read_b128 v[236:239], %[brd] offset:64512\n"
        "v_mfma_f32_16x16x32_f16 v[48:51], v[180:183], v[224:227], v[48:51]\n"
        "s_waitcnt lgkmcnt(3)\n"
        "v_readfirstlane_b32 s75, v240\n"
        "v_mfma_f32_16x16x32_f16 v[80:83], v[184:187], v[224:227], v[80:83]\n"
        "v_mfma_f32_16x16x32_f16 v[112:115], v[188:191], v[224:227], v[112:115]\n"
        "s_waitcnt lgkmcnt(2)\n"
        "v_mfma_f32_16x16x32_f16 v[20:23], v[176:179], v[228:231], v[20:23]\n"
        "ds_read_b128 v[224:227], v244 offset:0\n"
        "v_mfma_f32_16x16x32_f16 v[52:55], v[180:183], v[228:231], v[52:55]\n"
        "v_mfma_f32_16x16x32_f16 v[84:87], v[184:187], v[228:231], v[84:87]\n"
        "v_mfma_f32_16x16x32_f16 v[116:119], v[188:191], v[228:231], v[116:119]\n"
        "s_waitcnt lgkmcnt(2)\n"
        "v_mfma_f32_16x16x32_f16 v[24:27], v[176:179], v[232:235], v[24:27]\n"
        "ds_read_b128 v[228:231], v244 offset:1024\n"
        "v_mfma_f32_16x16x32_f16 v[56:59], v[180:183], v[232:235], v[56:59]\n"
        "v_mfma_f32_16x16x32_f16 v[88:91], v[184:187], v[232:235], v[88:91]\n"
        "v_mfma_f32_16x16x32_f16 v[120:123], v[188:191], v[232:235], v[120:123]\n"
        "s_waitcnt lgkmcnt(2)\n"
        "v_mfma_f32_16x16x32_f16 v[28:31], v[176:179], v[236:239], v[28:31]\n"
        "ds_read_b128 v[232:235], v244 offset:2048\n"
        "v_mfma_f32_16x16x32_f16 v[60:63], v[180:183], v[236:239], v[60:63]\n"
        "v_mfma_f32_16x16x32_f16 v[92:95], v[184:187], v[236:239], v[92:95]\n"
        "v_mfma_f32_16x16x32_f16 v[124:127], v[188:191], v[236:239], v[124:127]\n"
        "s_barrier\n"
        "s_setprio 2\n"
        "s_lshl_b32 s80, s77, 16\n"
        "s_add_u32 s80, s80, s62\n"
        "s_add_u32 s86, s80, 2048\n"
        "buffer_load_dwordx4 v[160:163], %[avoff], %[rsA], s80 offen offset:0\n"
        "s_waitcnt vmcnt(9)\n"
        "v_cvt_pk_f16_f32 v240, v192, v196\n"
        "v_cvt_pk_f16_f32 v241, v200, v204\n"
        "ds_write_b64 v245, v[240:241] offset:49152\n"
        "buffer_load_dwordx4 v[164:167], %[avoff], %[rsA], s80 offen offset:1024\n"
        "v_cvt_pk_f16_f32 v242, v193, v197\n"
        "v_cvt_pk_f16_f32 v243, v201, v205\n"
        "ds_write_b64 v245, v[242:243] offset:50176\n"
        "buffer_load_dwordx4 v[168:171], %[avoff], %[rsA], s86 offen offset:0\n"
        "v_cvt_pk_f16_f32 v240, v194, v198\n"
        "v_cvt_pk_f16_f32 v241, v202, v206\n"
        "ds_write_b64 v245, v[240:241] offset:51200\n"
        "buffer_load_dwordx4 v[172:175], %[avoff], %[rsA], s86 offen offset:1024\n"
        "v_cvt_pk_f16_f32 v242, v195, v199\n"
        "v_cvt_pk_f16_f32 v243, v203, v207\n"
        "ds_write_b64 v245, v[242:243] offset:52224\n"
        "s_lshl_b32 s81, s77, 16\n"
        "s_add_u32 s81, s81, s62\n"
        "s_add_u32 s81, s81, 4096\n"
        "s_add_u32 s87, s81, 2048\n"
        "buffer_load_dwordx4 v[176:179], %[avoff], %[rsA], s81 offen offset:0\n"
        "buffer_load_dwordx4 v[180:183], %[avoff], %[rsA], s81 offen offset:1024\n"
        "s_barrier\n"
        "s_cmp_eq_u32 s66, 4\n"
        "s_cbranch_scc1 L_flagcX2%=\n"
        "L_flagretcX2%=:\n"
        "s_cmp_eq_u32 s65, 4\n"
        "s_cbranch_scc1 L_pushcX2%=\n"
        "L_pushretcX2%=:\n"
        "s_lshl_b32 s82, s73, 23\n"
        "buffer_load_dwordx4 v[192:195], %[wvoff], %[rsW], s82 offen nt\n"
        "s_lshl_b32 s82, s73, 23\n"
        "s_add_u32 s82, s82, 131072\n"
        "buffer_load_dwordx4 v[196:199], %[wvoff], %[rsW], s82 offen nt\n"
        "s_setprio 0\n"
        "s_waitcnt lgkmcnt(6)\n"
        "v_mfma_f32_16x16x32_f16 v[0:3], v[128:131], v[224:227], v[0:3]\n"
        "ds_read_b128 v[236:239], v244 offset:3072\n"
        "v_mfma_f32_16x16x32_f16 v[32:35], v[132:135], v[224:227], v[32:35]\n"
        "v_mfma_f32_16x16x32_f16 v[64:67], v[136:139], v[224:227], v[64:67]\n"
        "v_mfma_f32_16x16x32_f16 v[96:99], v[140:143], v[224:227], v[96:99]\n"
        "s_waitcnt lgkmcnt(6)\n"
        "v_mfma_f32_16x16x32_f16 v[4:7], v[128:131], v[228:231], v[4:7]\n"
        "ds_read_b128 v[224:227], v244 offset:4096\n"
        "v_mfma_f32_16x16x32_f16 v[36:39], v[132:135], v[228:231], v[36:39]\n"
        "v_mfma_f32_16x16x32_f16 v[68:71], v[136:139], v[228:231], v[68:71]\n"
        "v_mfma_f32_16x16x32_f16 v[100:103], v[140:143], v[228:231], v[100:103]\n"
        "s_waitcnt lgkmcnt(6)\n"
        "v_mfma_f32_16x16x32_f16 v[8:11], v[128:131], v[232:235], v[8:11]\n"
        "ds_read_b128 v[228:231], v244 offset:5120\n"
        "v_mfma_f32_16x16x32_f16 v[40:43], v[132:135], v[232:235], v[40:43]\n"
        "v_mfma_f32_16x16x32_f16 v[72:75], v[136:139], v[232:235], v[72:75]\n"
        "v_mfma_f32_16x16x32_f16 v[104:107], v[140:143], v[232:235], v[104:107]\n"
        "s_waitcnt lgkmcnt(2)\n"
        "v_mfma_f32_16x16x32_f16 v[12:15], v[128:131], v[236:239], v[12:15]\n"
        "ds_read_b128 v[232:235], v244 offset:6144\n"
        "v_mfma_f32_16x16x32_f16 v[44:47], v[132:135], v[236:239], v[44:47]\n"
        "v_mfma_f32_16x16x32_f16 v[76:79], v[136:139], v[236:239], v[76:79]\n"
        "v_mfma_f32_16x16x32_f16 v[108:111], v[140:143], v[236:239], v[108:111]\n"
        "s_waitcnt lgkmcnt(2)\n"
        "v_mfma_f32_16x16x32_f16 v[16:19], v[128:131], v[224:227], v[16:19]\n"
        "ds_read_b128 v[236:239], v244 offset:7168\n"
        "v_mfma_f32_16x16x32_f16 v[48:51], v[132:135], v[224:227], v[48:51]\n"
        "v_mfma_f32_16x16x32_f16 v[80:83], v[136:139], v[224:227], v[80:83]\n"
        "v_mfma_f32_16x16x32_f16 v[112:115], v[140:143], v[224:227], v[112:115]\n"
        "s_waitcnt lgkmcnt(2)\n"
        "v_mfma_f32_16x16x32_f16 v[20:23], v[128:131], v[228:231], v[20:23]\n"
        "ds_read_b128 v[224:227], v244 offset:8192\n"
        "v_mfma_f32_16x16x32_f16 v[52:55], v[132:135], v[228:231], v[52:55]\n"
        "v_mfma_f32_16x16x32_f16 v[84:87], v[136:139], v[228:231], v[84:87]\n"
        "v_mfma_f32_16x16x32_f16 v[116:119], v[140:143], v[228:231], v[116:119]\n"
        "s_waitcnt lgkmcnt(2)\n"
        "v_mfma_f32_16x16x32_f16 v[24:27], v[128:131], v[232:235], v[24:27]\n"
        "ds_read_b128 v[228:231], v244 offset:9216\n"
        "v_mfma_f32_16x16x32_f16 v[56:59], v[132:135], v[232:235], v[56:59]\n"
        "v_mfma_f32_16x16x32_f16 v[88:91], v[136:139], v[232:235], v[88:91]\n"
        "v_mfma_f32_16x16x32_f16 v[120:123], v[140:143], v[232:235], v[120:123]\n"
        "s_waitcnt lgkmcnt(2)\n"
        "v_mfma_f32_16x16x32_f16 v[28:31], v[128:131], v[236:239], v[28:31]\n"
        "ds_read_b128 v[232:235], v244 offset:10240\n"
        "v_mfma_f32_16x16x32_f16 v[60:63], v[132:135], v[236:239], v[60:63]\n"
        "v_mfma_f32_16x16x32_f16 v[92:95], v[136:139], v[236:239], v[92:95]\n"
        "v_mfma_f32_16x16x32_f16 v[124:127], v[140:143], v[236:239], v[124:127]\n"
        "s_waitcnt vmcnt(11) lgkmcnt(2)\n"
        "v_mfma_f32_16x16x32_f16 v[0:3], v[144:147], v[224:227], v[0:3]\n"
        "ds_read_b128 v[236:239], v244 offset:11264\n"
        "v_mfma_f32_16x16x32_f16 v[32:35], v[148:151], v[224:227], v[32:35]\n"
        "v_mfma_f32_16x16x32_f16 v[64:67], v[152:155], v[224:227], v[64:67]\n"
        "v_mfma_f32_16x16x32_f16 v[96:99], v[156:159], v[224:227], v[96:99]\n"
        "s_waitcnt lgkmcnt(2)\n"
        "v_mfma_f32_16x16x32_f16 v[4:7], v[144:147], v[228:231], v[4:7]\n"
        "ds_read_b128 v[224:227], v244 offset:12288\n"
        "v_mfma_f32_16x16x32_f16 v[36:39], v[148:151], v[228:231], v[36:39]\n"
        "v_mfma_f32_16x16x32_f16 v[68:71], v[152:155], v[228:231], v[68:71]\n"
        "v_mfma_f32_16x16x32_f16 v[100:103], v[156:159], v[228:231], v[100:103]\n"
        "s_waitcnt lgkmcnt(2)\n"
        "v_mfma_f32_16x16x32_f16 v[8:11], v[144:147], v[232:235], v[8:11]\n"
        "ds_read_b128 v[228:231], v244 offset:13312\n"
        "v_mfma_f32_16x16x32_f16 v[40:43], v[148:151], v[232:235], v[40:43]\n"
        "v_mfma_f32_16x16x32_f16 v[72:75], v[152:155], v[232:235], v[72:75]\n"
        "v_mfma_f32_16x16x32_f16 v[104:107], v[156:159], v[232:235], v[104:107]\n"
        "s_barrier\n"
        "s_setprio 2\n"
        "buffer_load_dwordx4 v[184:187], %[avoff], %[rsA], s87 offen offset:0\n"
        "s_waitcnt vmcnt(9)\n"
        "v_cvt_pk_f16_f32 v240, v208, v212\n"
        "v_cvt_pk_f16_f32 v241, v216, v220\n"
        "ds_write_b64 %[bwr], v[240:241] offset:0\n"
        "buffer_load_dwordx4 v[188:191], %[avoff], %[rsA], s87 offen offset:1024\n"
        "v_cvt_pk_f16_f32 v242, v209, v213\n"
        "v_cvt_pk_f16_f32 v243, v217, v221\n"
        "ds_write_b64 %[bwr], v[242:243] offset:1024\n"
        "s_lshl_b32 s80, s78, 16\n"
        "s_add_u32 s80, s80, s62\n"
        "s_add_u32 s86, s80, 2048\n"
        "buffer_load_dwordx4 v[128:131], %[avoff], %[rsA], s80 offen offset:0\n"
        "v_cvt_pk_f16_f32 v240, v210, v214\n"
        "v_cvt_pk_f16_f32 v241, v218, v222\n"
        "ds_write_b64 %[bwr], v[240:241] offset:2048\n"
        "buffer_load_dwordx4 v[132:135], %[avoff], %[rsA], s80 offen offset:1024\n"
        "v_cvt_pk_f16_f32 v242, v211, v215\n"
        "v_cvt_pk_f16_f32 v243, v219, v223\n"
        "ds_write_b64 %[bwr], v[242:243] offset:3072\n"
        "buffer_load_dwordx4 v[136:139], %[avoff], %[rsA], s86 offen offset:0\n"
        "buffer_load_dwordx4 v[140:143], %[avoff], %[rsA], s86 offen offset:1024\n"
        "s_barrier\n"
        "s_lshl_b32 s82, s73, 23\n"
        "s_add_u32 s82, s82, 262144\n"
        "buffer_load_dwordx4 v[200:203], %[wvoff], %[rsW], s82 offen nt\n"
        "s_lshl_b32 s82, s73, 23\n"
        "s_add_u32 s82, s82, 393216\n"
        "buffer_load_dwordx4 v[204:207], %[wvoff], %[rsW], s82 offen nt\n"
        "s_lshl_b32 s82, s74, 23\n"
        "buffer_load_dwordx4 v[208:211], %[wvoff], %[rsW], s82 offen nt\n"
        "s_setprio 0\n"
        "s_waitcnt lgkmcnt(6)\n"
        "v_mfma_f32_16x16x32_f16 v[12:15], v[144:147], v[236:239], v[12:15]\n"
        "ds_read_b128 v[232:235], v244 offset:14336\n"
        "v_mfma_f32_16x16x32_f16 v[44:47], v[148:151], v[236:239], v[44:47]\n"
        "v_mov_b32 v247, 147472\n"
        "ds_read_b32 v240, v247\n"
        "v_mfma_f32_16x16x32_f16 v[76:79], v[152:155], v[236:239], v[76:79]\n"
        "v_mfma_f32_16x16x32_f16 v[108:111], v[156:159], v[236:239], v[108:111]\n"
        "s_waitcnt lgkmcnt(7)\n"
        "v_mfma_f32_16x16x32_f16 v[16:19], v[144:147], v[224:227], v[16:19]\n"
        "ds_read_b128 v[236:239], v244 offset:15360\n"
        "v_mfma_f32_16x16x32_f16 v[48:51], v[148:151], v[224:227], v[48:51]\n"
        "v_mfma_f32_16x16x32_f16 v[80:83], v[152:155], v[224:227], v[80:83]\n"
        "v_mfma_f32_16x16x32_f16 v[112:115], v[156:159], v[224:227], v[112:115]\n"
        "s_waitcnt lgkmcnt(7)\n"
        "v_mfma_f32_16x16x32_f16 v[20:23], v[144:147], v[228:231], v[20:23]\n"
        "ds_read_b128 v[224:227], v244 offset:16384\n"
        "v_mfma_f32_16x16x32_f16 v[52:55], v[148:151], v[228:231], v[52:55]\n"
        "v_mfma_f32_16x16x32_f16 v[84:87], v[152:155], v[228:231], v[84:87]\n"
        "v_mfma_f32_16x16x32_f16 v[116:119], v[156:159], v[228:231], v[116:119]\n"
        "s_waitcnt lgkmcnt(3)\n"
        "v_mfma_f32_16x16x32_f16 v[24:27], v[144:147], v[232:235], v[24:27]\n"
        "ds_read_b128 v[228:231], v244 offset:17408\n"
        "v_mfma_f32_16x16x32_f16 v[56:59], v[148:151], v[232:235], v[56:59]\n"
        "s_waitcnt lgkmcnt(3)\n"
        "v_readfirstlane_b32 s76, v240\n"
        "v_mfma_f32_16x16x32_f16 v[88:91], v[152:155], v[232:235], v[88:91]\n"
        "v_mfma_f32_16x16x32_f16 v[120:123], v[156:159], v[232:235], v[120:123]\n"
        "s_waitcnt lgkmcnt(2)\n"
        "v_mfma_f32_16x16x32_f16 v[28:31], v[144:147], v[236:239], v[28:31]\n"
        "ds_read_b128 v[232:235], v244 offset:18432\n"
        "v_mfma_f32_16x16x32_f16 v[60:63], v[148:151], v[236:239], v[60:63]\n"
        "v_mfma_f32_16x16x32_f16 v[92:95], v[152:155], v[236:239], v[92:95]\n"
        "v_mfma_f32_16x16x32_f16 v[124:127], v[156:159], v[236:239], v[124:127]\n"
        "s_waitcnt vmcnt(13) lgkmcnt(2)\n"
        "v_mfma_f32_16x16x32_f16 v[0:3], v[160:163], v[224:227], v[0:3]\n"
        "ds_read_b128 v[236:239], v244 offset:19456\n"
        "v_mfma_f32_16x16x32_f16 v[32:35], v[164:167], v[224:227], v[32:35]\n"
        "v_mfma_f32_16x16x32_f16 v[64:67], v[168:171], v[224:227], v[64:67]\n"
        "v_mfma_f32_16x16x32_f16 v[96:99], v[172:175], v[224:227], v[96:99]\n"
        "s_waitcnt lgkmcnt(2)\n"
        "v_mfma_f32_16x16x32_f16 v[4:7], v[160:163], v[228:231], v[4:7]\n"
        "ds_read_b128 v[224:227], v244 offset:20480\n"
        "v_mfma_f32_16x16x32_f16 v[36:39], v[164:167], v[228:231], v[36:39]\n"
        "v_mfma_f32_16x16x32_f16 v[68:71], v[168:171], v[228:231], v[68:71]\n"
        "v_mfma_f32_16x16x32_f16 v[100:103], v[172:175], v[228:231], v[100:103]\n"
        "s_waitcnt lgkmcnt(2)\n"
        "v_mfma_f32_16x16x32_f16 v[8:11], v[160:163], v[232:235], v[8:11]\n"
        "ds_read_b128 v[228:231], v244 offset:21504\n"
        "v_mfma_f32_16x16x32_f16 v[40:43], v[164:167], v[232:235], v[40:43]\n"
        "v_mfma_f32_16x16x32_f16 v[72:75], v[168:171], v[232:235], v[72:75]\n"
        "v_mfma_f32_16x16x32_f16 v[104:107], v[172:175], v[232:235], v[104:107]\n"
        "s_waitcnt lgkmcnt(2)\n"
        "v_mfma_f32_16x16x32_f16 v[12:15], v[160:163], v[236:239], v[12:15]\n"
        "ds_read_b128 v[232:235], v244 offset:22528\n"
        "v_mfma_f32_16x16x32_f16 v[44:47], v[164:167], v[236:239], v[44:47]\n"
        "v_mfma_f32_16x16x32_f16 v[76:79], v[168:171], v[236:239], v[76:79]\n"
        "v_mfma_f32_16x16x32_f16 v[108:111], v[172:175], v[236:239], v[108:111]\n"
        "s_waitcnt lgkmcnt(2)\n"
        "v_mfma_f32_16x16x32_f16 v[16:19], v[160:163], v[224:227], v[16:19]\n"
        "ds_read_b128 v[236:239], v244 offset:23552\n"
        "v_mfma_f32_16x16x32_f16 v[48:51], v[164:167], v[224:227], v[48:51]\n"
        "v_mfma_f32_16x16x32_f16 v[80:83], v[168:171], v[224:227], v[80:83]\n"
        "v_mfma_f32_16x16x32_f16 v[112:115], v[172:175], v[224:227], v[112:115]\n"
        "s_barrier\n"
        "s_setprio 2\n"
        "s_lshl_b32 s81, s78, 16\n"
        "s_add_u32 s81, s81, s62\n"
        "s_add_u32 s81, s81, 4096\n"
        "s_add_u32 s87, s81, 2048\n"
        "buffer_load_dwordx4 v[144:147], %[avoff], %[rsA], s81 offen offset:0\n"
        "buffer_load_dwordx4 v[148:151], %[avoff], %[rsA], s81 offen offset:1024\n"
        "buffer_load_dwordx4 v[152:155], %[avoff], %[rsA], s87 offen offset:0\n"
        "buffer_load_dwordx4 v[156:159], %[avoff], %[rsA], s87 offen offset:1024\n"
        "s_barrier\n"
        "s_lshl_b32 s82, s74, 23\n"
        "s_add_u32 s82, s82, 131072\n"
        "buffer_load_dwordx4 v[212:215], %[wvoff], %[rsW], s82 offen nt\n"
        "s_lshl_b32 s82, s74, 23\n"
        "s_add_u32 s82, s82, 262144\n"
        "buffer_load_dwordx4 v[216:219], %[wvoff], %[rsW], s82 offen nt\n"
        "s_lshl_b32 s82, s74, 23\n"
        "s_add_u32 s82, s82, 393216\n"
        "buffer_load_dwordx4 v[220:223], %[wvoff], %[rsW], s82 offen nt\n"
        "s_setprio 0\n"
        "s_waitcnt lgkmcnt(2)\n"
        "v_mfma_f32_16x16x32_f16 v[20:23], v[160:163], v[228:231], v[20:23]\n"
        "ds_read_b128 v[224:227], v244 offset:24576\n"
        "v_mfma_f32_16x16x32_f16 v[52:55], v[164:167], v[228:231], v[52:55]\n"
        "v_mfma_f32_16x16x32_f16 v[84:87], v[168:171], v[228:231], v[84:87]\n"
        "v_mfma_f32_16x16x32_f16 v[116:119], v[172:175], v[228:231], v[116:119]\n"
        "s_waitcnt lgkmcnt(2)\n"
        "v_mfma_f32_16x16x32_f16 v[24:27], v[160:163], v[232:235], v[24:27]\n"
        "ds_read_b128 v[228:231], v244 offset:25600\n"
        "v_mfma_f32_16x16x32_f16 v[56:59], v[164:167], v[232:235], v[56:59]\n"
        "v_mfma_f32_16x16x32_f16 v[88:91], v[168:171], v[232:235], v[88:91]\n"
        "v_mfma_f32_16x16x32_f16 v[120:123], v[172:175], v[232:235], v[120:123]\n"
        "s_waitcnt lgkmcnt(2)\n"
        "v_mfma_f32_16x16x32_f16 v[28:31], v[160:163], v[236:239], v[28:31]\n"
        "ds_read_b128 v[232:235], v244 offset:26624\n"
        "v_mfma_f32_16x16x32_f16 v[60:63], v[164:167], v[236:239], v[60:63]\n"
        "v_mfma_f32_16x16x32_f16 v[92:95], v[168:171], v[236:239], v[92:95]\n"
        "v_mfma_f32_16x16x32_f16 v[124:127], v[172:175], v[236:239], v[124:127]\n"
        "s_waitcnt vmcnt(14) lgkmcnt(2)\n"
        "v_mfma_f32_16x16x32_f16 v[0:3], v[176:179], v[224:227], v[0:3]\n"
        "ds_read_b128 v[236:239], v244 offset:27648\n"
        "v_mfma_f32_16x16x32_f16 v[32:35], v[180:183], v[224:227], v[32:35]\n"
        "v_mfma_f32_16x16x32_f16 v[64:67], v[184:187], v[224:227], v[64:67]\n"
        "v_mfma_f32_16x16x32_f16 v[96:99], v[188:191], v[224:227], v[96:99]\n"
        "s_waitcnt lgkmcnt(2)\n"
        "v_mfma_f32_16x16x32_f16 v[4:7], v[176:179], v[228:231], v[4:7]\n"
        "ds_read_b128 v[224:227], v244 offset:28672\n"
        "v_mfma_f32_16x16x32_f16 v[36:39], v[180:183], v[228:231], v[36:39]\n"
        "v_mov_b32 v247, 147476\n"
        "ds_read_b32 v240, v247\n"
        "v_mfma_f32_16x16x32_f16 v[68:71], v[184:187], v[228:231], v[68:71]\n"
        "v_mfma_f32_16x16x32_f16 v[100:103], v[188:191], v[228:231], v[100:103]\n"
        "s_waitcnt lgkmcnt(3)\n"
        "v_mfma_f32_16x16x32_f16 v[8:11], v[176:179], v[232:235], v[8:11]\n"
        "ds_read_b128 v[228:231], v244 offset:29696\n"
        "v_mfma_f32_16x16x32_f16 v[40:43], v[180:183], v[232:235], v[40:43]\n"
        "v_mfma_f32_16x16x32_f16 v[72:75], v[184:187], v[232:235], v[72:75]\n"
        "v_mfma_f32_16x16x32_f16 v[104:107], v[188:191], v[232:235], v[104:107]\n"
        "s_waitcnt lgkmcnt(3)\n"
        "v_mfma_f32_16x16x32_f16 v[12:15], v[176:179], v[236:239], v[12:15]\n"
        "ds_read_b128 v[232:235], v244 offset:30720\n"
        "v_mfma_f32_16x16x32_f16 v[44:47], v[180:183], v[236:239], v[44:47]\n"
        "v_mfma_f32_16x16x32_f16 v[76:79], v[184:187], v[236:239], v[76:79]\n"
        "v_mfma_f32_16x16x32_f16 v[108:111], v[188:191], v[236:239], v[108:111]\n"
        "s_waitcnt lgkmcnt(3)\n"
        "v_mfma_f32_16x16x32_f16 v[16:19], v[176:179], v[224:227], v[16:19]\n"
        "ds_read_b128 v[236:239], v244 offset:31744\n"
        "v_mfma_f32_16x16x32_f16 v[48:51], v[180:183], v[224:227], v[48:51]\n"
        "s_waitcnt lgkmcnt(3)\n"
        "v_readfirstlane_b32 s77, v240\n"
        "v_mfma_f32_16x16x32_f16 v[80:83], v[184:187], v[224:227], v[80:83]\n"
        "v_mfma_f32_16x16x32_f16 v[112:115], v[188:191], v[224:227], v[112:115]\n"
        "s_waitcnt lgkmcnt(2)\n"
        "v_mfma_f32_16x16x32_f16 v[20:23], v[176:179], v[228:231], v[20:23]\n"
        "ds_read_b128 v[224:227], v244 offset:32768\n"
        "v_mfma_f32_16x16x32_f16 v[52:55], v[180:183], v[228:231], v[52:55]\n"
        "v_mfma_f32_16x16x32_f16 v[84:87], v[184:187], v[228:231], v[84:87]\n"
        "v_mfma_f32_16x16x32_f16 v[116:119], v[188:191], v[228:231], v[116:119]\n"
        "s_waitcnt lgkmcnt(2)\n"
        "v_mfma_f32_16x16x32_f16 v[24:27], v[176:179], v[232:235], v[24:27]\n"
        "ds_read_b128 v[228:231], v244 offset:33792\n"
        "v_mfma_f32_16x16x32_f16 v[56:59], v[180:183], v[232:235], v[56:59]\n"
        "v_mfma_f32_16x16x32_f16 v[88:91], v[184:187], v[232:235], v[88:91]\n"
        "v_mfma_f32_16x16x32_f16 v[120:123], v[188:191], v[232:235], v[120:123]\n"
        "s_waitcnt lgkmcnt(2)\n"
        "v_mfma_f32_16x16x32_f16 v[28:31], v[176:179], v[236:239], v[28:31]\n"
        "ds_read_b128 v[232:235], v244 offset:34816\n"
        "v_mfma_f32_16x16x32_f16 v[60:63], v[180:183], v[236:239], v[60:63]\n"
        "v_mfma_f32_16x16x32_f16 v[92:95], v[184:187], v[236:239], v[92:95]\n"
        "v_mfma_f32_16x16x32_f16 v[124:127], v[188:191], v[236:239], v[124:127]\n"
        "s_barrier\n"
        "s_setprio 2\n"
        "s_lshl_b32 s80, s79, 16\n"
        "s_add_u32 s80, s80, s62\n"
        "s_add_u32 s86, s80, 2048\n"
        "buffer_load_dwordx4 v[160:163], %[avoff], %[rsA], s80 offen offset:0\n"
        "s_waitcnt vmcnt(9)\n"
        "v_cvt_pk_f16_f32 v240, v192, v196\n"
        "v_cvt_pk_f16_f32 v241, v200, v204\n"
        "ds_write_b64 %[bwr], v[240:241] offset:16384\n"
        "buffer_load_dwordx4 v[164:167], %[avoff], %[rsA], s80 offen offset:1024\n"
        "v_cvt_pk_f16_f32 v242, v193, v197\n"
        "v_cvt_pk_f16_f32 v243, v201, v205\n"
        "ds_write_b64 %[bwr], v[242:243] offset:17408\n"
        "buffer_load_dwordx4 v[168:171], %[avoff], %[rsA], s86 offen offset:0\n"
        "v_cvt_pk_f16_f32 v240, v194, v198\n"
        "v_cvt_pk_f16_f32 v241, v202, v206\n"
        "ds_write_b64 %[bwr], v[240:241] offset:18432\n"
        "buffer_load_dwordx4 v[172:175], %[avoff], %[rsA], s86 offen offset:1024\n"
        "v_cvt_pk_f16_f32 v242, v195, v199\n"
        "v_cvt_pk_f16_f32 v243, v203, v207\n"
        "ds_write_b64 %[bwr], v[242:243] offset:19456\n"
        "s_lshl_b32 s81, s79, 16\n"
        "s_add_u32 s81, s81, s62\n"
        "s_add_u32 s81, s81, 4096\n"
        "s_add_u32 s87, s81, 2048\n"
        "buffer_load_dwordx4 v[176:179], %[avoff], %[rsA], s81 offen offset:0\n"
        "buffer_load_dwordx4 v[180:183], %[avoff], %[rsA], s81 offen offset:1024\n"
        "s_barrier\n"
        "s_cmp_eq_u32 s66, 6\n"
        "s_cbranch_scc1 L_flagcX3%=\n"
        "L_flagretcX3%=:\n"
        "s_cmp_eq_u32 s65, 6\n"
        "s_cbranch_scc1 L_pushcX3%=\n"
        "L_pushretcX3%=:\n"
        "s_lshl_b32 s82, s75, 23\n"
        "buffer_load_dwordx4 v[192:195], %[wvoff], %[rsW], s82 offen nt\n"
        "s_lshl_b32 s82, s75, 23\n"
        "s_add_u32 s82, s82, 131072\n"
        "buffer_load_dwordx4 v[196:199], %[wvoff], %[rsW], s82 offen nt\n"
        "s_setprio 0\n"
        "s_waitcnt lgkmcnt(6)\n"
        "v_mfma_f32_16x16x32_f16 v[0:3], v[128:131], v[224:227], v[0:3]\n"
        "ds_read_b128 v[236:239], v244 offset:35840\n"
        "v_mfma_f32_16x16x32_f16 v[32:35], v[132:135], v[224:227], v[32:35]\n"
        "v_mfma_f32_16x16x32_f16 v[64:67], v[136:139], v[224:227], v[64:67]\n"
        "v_mfma_f32_16x16x32_f16 v[96:99], v[140:143], v[224:227], v[96:99]\n"
        "s_waitcnt lgkmcnt(6)\n"
        "v_mfma_f32_16x16x32_f16 v[4:7], v[128:131], v[228:231], v[4:7]\n"
        "ds_read_b128 v[224:227], v244 offset:36864\n"
        "v_mfma_f32_16x16x32_f16 v[36:39], v[132:135], v[228:231], v[36:39]\n"
        "v_mfma_f32_16x16x32_f16 v[68:71], v[136:139], v[228:231], v[68:71]\n"
        "v_mfma_f32_16x16x32_f16 v[100:103], v[140:143], v[228:231], v[100:103]\n"
        "s_waitcnt lgkmcnt(6)\n"
        "v_mfma_f32_16x16x32_f16 v[8:11], v[128:131], v[232:235], v[8:11]\n"
        "ds_read_b128 v[228:231], v244 offset:37888\n"
        "v_mfma_f32_16x16x32_f16 v[40:43], v[132:135], v[232:235], v[40:43]\n"
        "v_mfma_f32_16x16x32_f16 v[72:75], v[136:139], v[232:235], v[72:75]\n"
        "v_mfma_f32_16x16x32_f16 v[104:107], v[140:143], v[232:235], v[104:107]\n"
        "s_waitcnt lgkmcnt(2)\n"
        "v_mfma_f32_16x16x32_f16 v[12:15], v[128:131], v[236:239], v[12:15]\n"
        "ds_read_b128 v[232:235], v244 offset:38912\n"
        "v_mfma_f32_16x16x32_f16 v[44:47], v[132:135], v[236:239], v[44:47]\n"
        "v_mfma_f32_16x16x32_f16 v[76:79], v[136:139], v[236:239], v[76:79]\n"
        "v_mfma_f32_16x16x32_f16 v[108:111], v[140:143], v[236:239], v[108:111]\n"
        "s_waitcnt lgkmcnt(2)\n"
        "v_mfma_f32_16x16x32_f16 v[16:19], v[128:131], v[224:227], v[16:19]\n"
        "ds_read_b128 v[236:239], v244 offset:39936\n"
        "v_mfma_f32_16x16x32_f16 v[48:51], v[132:135], v[224:227], v[48:51]\n"
        "v_mfma_f32_16x16x32_f16 v[80:83], v[136:139], v[224:227], v[80:83]\n"
        "v_mfma_f32_16x16x32_f16 v[112:115], v[140:143], v[224:227], v[112:115]\n"
        "s_waitcnt lgkmcnt(2)\n"
        "v_mfma_f32_16x16x32_f16 v[20:23], v[128:131], v[228:231], v[20:23]\n"
        "ds_read_b128 v[224:227], v244 offset:40960\n"
        "v_mfma_f32_16x16x32_f16 v[52:55], v[132:135], v[228:231], v[52:55]\n"
        "v_mfma_f32_16x16x32_f16 v[84:87], v[136:139], v[228:231], v[84:87]\n"
        "v_mfma_f32_16x16x32_f16 v[116:119], v[140:143], v[228:231], v[116:119]\n"
        "s_waitcnt lgkmcnt(2)\n"
        "v_mfma_f32_16x16x32_f16 v[24:27], v[128:131], v[232:235], v[24:27]\n"
        "ds_read_b128 v[228:231], v244 offset:41984\n"
        "v_mfma_f32_16x16x32_f16 v[56:59], v[132:135], v[232:235], v[56:59]\n"
        "v_mfma_f32_16x16x32_f16 v[88:91], v[136:139], v[232:235], v[88:91]\n"
        "v_mfma_f32_16x16x32_f16 v[120:123], v[140:143], v[232:235], v[120:123]\n"
        "s_waitcnt lgkmcnt(2)\n"
        "v_mfma_f32_16x16x32_f16 v[28:31], v[128:131], v[236:239], v[28:31]\n"
        "ds_read_b128 v[232:235], v244 offset:43008\n"
        "v_mfma_f32_16x16x32_f16 v[60:63], v[132:135], v[236:239], v[60:63]\n"
        "v_mfma_f32_16x16x32_f16 v[92:95], v[136:139], v[236:239], v[92:95]\n"
        "v_mfma_f32_16x16x32_f16 v[124:127], v[140:143], v[236:239], v[124:127]\n"
        "s_waitcnt vmcnt(11) lgkmcnt(2)\n"
        "v_mfma_f32_16x16x32_f16 v[0:3], v[144:147], v[224:227], v[0:3]\n"
        "ds_read_b128 v[236:239], v244 offset:44032\n"
        "v_mfma_f32_16x16x32_f16 v[32:35], v[148:151], v[224:227], v[32:35]\n"
        "v_mfma_f32_16x16x32_f16 v[64:67], v[152:155], v[224:227], v[64:67]\n"
        "v_mfma_f32_16x16x32_f16 v[96:99], v[156:159], v[224:227], v[96:99]\n"
        "s_waitcnt lgkmcnt(2)\n"
        "v_mfma_f32_16x16x32_f16 v[4:7], v[144:147], v[228:231], v[4:7]\n"
        "ds_read_b128 v[224:227], v244 offset:45056\n"
        "v_mfma_f32_16x16x32_f16 v[36:39], v[148:151], v[228:231], v[36:39]\n"
        "v_mfma_f32_16x16x32_f16 v[68:71], v[152:155], v[228:231], v[68:71]\n"
        "v_mfma_f32_16x16x32_f16 v[100:103], v[156:159], v[228:231], v[100:103]\n"
        "s_waitcnt lgkmcnt(2)\n"
        "v_mfma_f32_16x16x32_f16 v[8:11], v[144:147], v[232:235], v[8:11]\n"
        "ds_read_b128 v[228:231], v244 offset:46080\n"
        "v_mfma_f32_16x16x32_f16 v[40:43], v[148:151], v[232:235], v[40:43]\n"
        "v_mfma_f32_16x16x32_f16 v[72:75], v[152:155], v[232:235], v[72:75]\n"
        "v_mfma_f32_16x16x32_f16 v[104:107], v[156:159], v[232:235], v[104:107]\n"
        "s_barrier\n"
        "s_setprio 2\n"
        "buffer_load_dwordx4 v[184:187], %[avoff], %[rsA], s87 offen offset:0\n"
        "s_waitcnt vmcnt(9)\n"
        "v_cvt_pk_f16_f32 v240, v208, v212\n"
        "v_cvt_pk_f16_f32 v241, v216, v220\n"
        "ds_write_b64 %[bwr], v[240:241] offset:32768\n"
        "buffer_load_dwordx4 v[188:191], %[avoff], %[rsA], s87 offen offset:1024\n"
        "v_cvt_pk_f16_f32 v242, v209, v213\n"
        "v_cvt_pk_f16_f32 v243, v217, v221\n"
        "ds_write_b64 %[bwr], v[242:243] offset:33792\n"
        "s_lshl_b32 s80, s72, 16\n"
        "s_lshl_b32 s86, s72, 16\n"
        "s_cmp_eq_u32 s64, 48\n"
        "s_cselect_b32 s80, s86, s80\n"
        "s_cselect_b32 s86, 2, 1\n"
        "s_lshl_b32 s86, s86, 11\n"
        "s_add_u32 s80, s80, s62\n"
        "s_add_u32 s86, s86, s80\n"
        "buffer_load_dwordx4 v[128:131], %[avoff], %[rsA], s80 offen offset:0\n"
        "v_cvt_pk_f16_f32 v240, v210, v214\n"
        "v_cvt_pk_f16_f32 v241, v218, v222\n"
        "ds_write_b64 %[bwr], v[240:241] offset:34816\n"
        "buffer_load_dwordx4 v[132:135], %[avoff], %[rsA], s80 offen offset:1024\n"
        "v_cvt_pk_f16_f32 v242, v211, v215\n"
        "v_cvt_pk_f16_f32 v243, v219, v223\n"
        "ds_write_b64 %[bwr], v[242:243] offset:35840\n"
        "buffer_load_dwordx4 v[136:139], %[avoff], %[rsA], s86 offen offset:0\n"
        "buffer_load_dwordx4 v[140:143], %[avoff], %[rsA], s86 offen offset:1024\n"
        "s_barrier\n"
        "s_lshl_b32 s82, s75, 23\n"
        "s_add_u32 s82, s82, 262144\n"
        "buffer_load_dwordx4 v[200:203], %[wvoff], %[rsW], s82 offen nt\n"
        "s_lshl_b32 s82, s75, 23\n"
        "s_add_u32 s82, s82, 393216\n"
        "buffer_load_dwordx4 v[204:207], %[wvoff], %[rsW], s82 offen nt\n"
        "s_lshl_b32 s82, s76, 23\n"
        "buffer_load_dwordx4 v[208:211], %[wvoff], %[rsW], s82 offen nt\n"
        "s_setprio 0\n"
        "s_waitcnt lgkmcnt(6)\n"
        "v_mfma_f32_16x16x32_f16 v[12:15], v[144:147], v[236:239], v[12:15]\n"
        "ds_read_b128 v[232:235], v244 offset:47104\n"
        "v_mfma_f32_16x16x32_f16 v[44:47], v[148:151], v[236:239], v[44:47]\n"
        "v_mov_b32 v247, 147480\n"
        "ds_read_b32 v240, v247\n"
        "v_mfma_f32_16x16x32_f16 v[76:79], v[152:155], v[236:239], v[76:79]\n"
        "v_mfma_f32_16x16x32_f16 v[108:111], v[156:159], v[236:239], v[108:111]\n"
        "s_waitcnt lgkmcnt(7)\n"
        "v_mfma_f32_16x16x32_f16 v[16:19], v[144:147], v[224:227], v[16:19]\n"
        "ds_read_b128 v[236:239], v244 offset:48128\n"
        "v_mfma_f32_16x16x32_f16 v[48:51], v[148:151], v[224:227], v[48:51]\n"
        "v_mfma_f32_16x16x32_f16 v[80:83], v[152:155], v[224:227], v[80:83]\n"
        "v_mfma_f32_16x16x32_f16 v[112:115], v[156:159], v[224:227], v[112:115]\n"
        "s_waitcnt lgkmcnt(7)\n"
        "v_mfma_f32_16x16x32_f16 v[20:23], v[144:147], v[228:231], v[20:23]\n"
        "ds_read_b128 v[224:227], v244 offset:49152\n"
        "v_mfma_f32_16x16x32_f16 v[52:55], v[148:151], v[228:231], v[52:55]\n"
        "v_mfma_f32_16x16x32_f16 v[84:87], v[152:155], v[228:231], v[84:87]\n"
        "v_mfma_f32_16x16x32_f16 v[116:119], v[156:159], v[228:231], v[116:119]\n"
        "s_waitcnt lgkmcnt(3)\n"
        "v_mfma_f32_16x16x32_f16 v[24:27], v[144:147], v[232:235], v[24:27]\n"
        "ds_read_b128 v[228:231], v244 offset:50176\n"
        "v_mfma_f32_16x16x32_f16 v[56:59], v[148:151], v[232:235], v[56:59]\n"
        "s_waitcnt lgkmcnt(3)\n"
        "v_readfirstlane_b32 s78, v240\n"
        "v_mfma_f32_16x16x32_f16 v[88:91], v[152:155], v[232:235], v[88:91]\n"
        "v_mfma_f32_16x16x32_f16 v[120:123], v[156:159], v[232:235], v[120:123]\n"
        "s_waitcnt lgkmcnt(2)\n"
        "v_mfma_f32_16x16x32_f16 v[28:31], v[144:147], v[236:239], v[28:31]\n"
        "ds_read_b128 v[232:235], v244 offset:51200\n"
        "v_mfma_f32_16x16x32_f16 v[60:63], v[148:151], v[236:239], v[60:63]\n"
        "v_mfma_f32_16x16x32_f16 v[92:95], v[152:155], v[236:239], v[92:95]\n"
        "v_mfma_f32_16x16x32_f16 v[124:127], v[156:159], v[236:239], v[124:127]\n"
        "s_waitcnt vmcnt(13) lgkmcnt(2)\n"
        "v_mfma_f32_16x16x32_f16 v[0:3], v[160:163], v[224:227], v[0:3]\n"
        "ds_read_b128 v[236:239], v244 offset:52224\n"
        "v_mfma_f32_16x16x32_f16 v[32:35], v[164:167], v[224:227], v[32:35]\n"
        "v_mfma_f32_16x16x32_f16 v[64:67], v[168:171], v[224:227], v[64:67]\n"
        "v_mfma_f32_16x16x32_f16 v[96:99], v[172:175], v[224:227], v[96:99]\n"
        "s_waitcnt lgkmcnt(2)\n"
        "v_mfma_f32_16x16x32_f16 v[4:7], v[160:163], v[228:231], v[4:7]\n"
        "ds_read_b128 v[224:227], v244 offset:53248\n"
        "v_mfma_f32_16x16x32_f16 v[36:39], v[164:167], v[228:231], v[36:39]\n"
        "v_mfma_f32_16x16x32_f16 v[68:71], v[168:171], v[228:231], v[68:71]\n"
        "v_mfma_f32_16x16x32_f16 v[100:103], v[172:175], v[228:231], v[100:103]\n"
        "s_waitcnt lgkmcnt(2)\n"
        "v_mfma_f32_16x16x32_f16 v[8:11], v[160:163], v[232:235], v[8:11]\n"
        "ds_read_b128 v[228:231], v244 offset:54272\n"
        "v_mfma_f32_16x16x32_f16 v[40:43], v[164:167], v[232:235], v[40:43]\n"
        "v_mfma_f32_16x16x32_f16 v[72:75], v[168:171], v[232:235], v[72:75]\n"
        "v_mfma_f32_16x16x32_f16 v[104:107], v[172:175], v[232:235], v[104:107]\n"
        "s_waitcnt lgkmcnt(2)\n"
        "v_mfma_f32_16x16x32_f16 v[12:15], v[160:163], v[236:239], v[12:15]\n"
        "ds_read_b128 v[232:235], v244 offset:55296\n"
        "v_mfma_f32_16x16x32_f16 v[44:47], v[164:167], v[236:239], v[44:47]\n"
        "v_mfma_f32_16x16x32_f16 v[76:79], v[168:171], v[236:239], v[76:79]\n"
        "v_mfma_f32_16x16x32_f16 v[108:111], v[172:175], v[236:239], v[108:111]\n"
        "s_waitcnt lgkmcnt(2)\n"
        "v_mfma_f32_16x16x32_f16 v[16:19], v[160:163], v[224:227], v[16:19]\n"
        "ds_read_b128 v[236:239], v244 offset:56320\n"
        "v_mfma_f32_16x16x32_f16 v[48:51], v[164:167], v[224:227], v[48:51]\n"
        "v_mfma_f32_16x16x32_f16 v[80:83], v[168:171], v[224:227], v[80:83]\n"
        "v_mfma_f32_16x16x32_f16 v[112:115], v[172:175], v[224:227], v[112:115]\n"
        "s_barrier\n"
        "s_setprio 2\n"
        "s_lshl_b32 s81, s72, 16\n"
        "s_add_u32 s81, s81, 4096\n"
        "s_lshl_b32 s87, s73, 16\n"
        "s_cmp_eq_u32 s64, 48\n"
        "s_cselect_b32 s81, s87, s81\n"
        "s_cselect_b32 s87, 2, 1\n"
        "s_lshl_b32 s87, s87, 11\n"
        "s_add_u32 s81, s81, s62\n"
        "s_add_u32 s87, s87, s81\n"
        "buffer_load_dwordx4 v[144:147], %[avoff], %[rsA], s81 offen offset:0\n"
        "buffer_load_dwordx4 v[148:151], %[avoff], %[rsA], s81 offen offset:1024\n"
        "buffer_load_dwordx4 v[152:155], %[avoff], %[rsA], s87 offen offset:0\n"
        "buffer_load_dwordx4 v[156:159], %[avoff], %[rsA], s87 offen offset:1024\n"
        "s_barrier\n"
        "s_lshl_b32 s82, s76, 23\n"
        "s_add_u32 s82, s82, 131072\n"
        "buffer_load_dwordx4 v[212:215], %[wvoff], %[rsW], s82 offen nt\n"
        "s_lshl_b32 s82, s76, 23\n"
        "s_add_u32 s82, s82, 262144\n"
        "buffer_load_dwordx4 v[216:219], %[wvoff], %[rsW], s82 offen nt\n"
        "s_lshl_b32 s82, s76, 23\n"
        "s_add_u32 s82, s82, 393216\n"
        "buffer_load_dwordx4 v[220:223], %[wvoff], %[rsW], s82 offen nt\n"
        "s_setprio 0\n"
        "s_waitcnt lgkmcnt(2)\n"
        "v_mfma_f32_16x16x32_f16 v[20:23], v[160:163], v[228:231], v[20:23]\n"
        "ds_read_b128 v[224:227], v244 offset:57344\n"
        "v_mfma_f32_16x16x32_f16 v[52:55], v[164:167], v[228:231], v[52:55]\n"
        "v_mfma_f32_16x16x32_f16 v[84:87], v[168:171], v[228:231], v[84:87]\n"
        "v_mfma_f32_16x16x32_f16 v[116:119], v[172:175], v[228:231], v[116:119]\n"
        "s_waitcnt lgkmcnt(2)\n"
        "v_mfma_f32_16x16x32_f16 v[24:27], v[160:163], v[232:235], v[24:27]\n"
        "ds_read_b128 v[228:231], v244 offset:58368\n"
        "v_mfma_f32_16x16x32_f16 v[56:59], v[164:167], v[232:235], v[56:59]\n"
        "v_mfma_f32_16x16x32_f16 v[88:91], v[168:171], v[232:235], v[88:91]\n"
        "v_mfma_f32_16x16x32_f16 v[120:123], v[172:175], v[232:235], v[120:123]\n"
        "s_waitcnt lgkmcnt(2)\n"
        "v_mfma_f32_16x16x32_f16 v[28:31], v[160:163], v[236:239], v[28:31]\n"
        "ds_read_b128 v[232:235], v244 offset:59392\n"
        "v_mfma_f32_16x16x32_f16 v[60:63], v[164:167], v[236:239], v[60:63]\n"
        "v_mfma_f32_16x16x32_f16 v[92:95], v[168:171], v[236:239], v[92:95]\n"
        "v_mfma_f32_16x16x32_f16 v[124:127], v[172:175], v[236:239], v[124:127]\n"
        "s_waitcnt vmcnt(14) lgkmcnt(2)\n"
        "v_mfma_f32_16x16x32_f16 v[0:3], v[176:179], v[224:227], v[0:3]\n"
        "ds_read_b128 v[236:239], v244 offset:60416\n"
        "v_mfma_f32_16x16x32_f16 v[32:35], v[180:183], v[224:227], v[32:35]\n"
        "v_mfma_f32_16x16x32_f16 v[64:67], v[184:187], v[224:227], v[64:67]\n"
        "v_mfma_f32_16x16x32_f16 v[96:99], v[188:191], v[224:227], v[96:99]\n"
        "s_waitcnt lgkmcnt(2)\n"
        "v_mfma_f32_16x16x32_f16 v[4:7], v[176:179], v[228:231], v[4:7]\n"
        "ds_read_b128 v[224:227], v244 offset:61440\n"
        "v_mfma_f32_16x16x32_f16 v[36:39], v[180:183], v[228:231], v[36:39]\n"
        "v_mov_b32 v247, 147484\n"
        "ds_read_b32 v240, v247\n"
        "v_mfma_f32_16x16x32_f16 v[68:71], v[184:187], v[228:231], v[68:71]\n"
        "v_mfma_f32_16x16x32_f16 v[100:103], v[188:191], v[228:231], v[100:103]\n"
        "s_waitcnt lgkmcnt(3)\n"
        "v_mfma_f32_16x16x32_f16 v[8:11], v[176:179], v[232:235], v[8:11]\n"
        "ds_read_b128 v[228:231], v244 offset:62464\n"
        "v_mfma_f32_16x16x32_f16 v[40:43], v[180:183], v[232:235], v[40:43]\n"
        "v_mfma_f32_16x16x32_f16 v[72:75], v[184:187], v[232:235], v[72:75]\n"
        "v_mfma_f32_16x16x32_f16 v[104:107], v[188:191], v[232:235], v[104:107]\n"
        "s_waitcnt lgkmcnt(3)\n"
        "v_mfma_f32_16x16x32_f16 v[12:15], v[176:179], v[236:239], v[12:15]\n"
        "ds_read_b128 v[232:235], v244 offset:63488\n"
        "v_mfma_f32_16x16x32_f16 v[44:47], v[180:183], v[236:239], v[44:47]\n"
        "v_mfma_f32_16x16x32_f16 v[76:79], v[184:187], v[236:239], v[76:79]\n"
        "v_mfma_f32_16x16x32_f16 v[108:111], v[188:191], v[236:239], v[108:111]\n"
        "s_waitcnt lgkmcnt(3)\n"
        "v_mfma_f32_16x16x32_f16 v[16:19], v[176:179], v[224:227], v[16:19]\n"
        "ds_read_b128 v[236:239], v244 offset:64512\n"
        "v_mfma_f32_16x16x32_f16 v[48:51], v[180:183], v[224:227], v[48:51]\n"
        "s_waitcnt lgkmcnt(3)\n"
        "v_readfirstlane_b32 s79, v240\n"
        "v_mfma_f32_16x16x32_f16 v[80:83], v[184:187], v[224:227], v[80:83]\n"
        "v_mfma_f32_16x16x32_f16 v[112:115], v[188:191], v[224:227], v[112:115]\n"
        "s_waitcnt lgkmcnt(2)\n"
        "v_mfma_f32_16x16x32_f16 v[20:23], v[176:179], v[228:231], v[20:23]\n"
        "ds_read_b128 v[224:227], %[brd] offset:0\n"
        "v_mfma_f32_16x16x32_f16 v[52:55], v[180:183], v[228:231], v[52:55]\n"
        "v_mfma_f32_16x16x32_f16 v[84:87], v[184:187], v[228:231], v[84:87]\n"
        "v_mfma_f32_16x16x32_f16 v[116:119], v[188:191], v[228:231], v[116:119]\n"
        "s_waitcnt lgkmcnt(2)\n"
        "v_mfma_f32_16x16x32_f16 v[24:27], v[176:179], v[232:235], v[24:27]\n"
        "ds_read_b128 v[228:231], %[brd] offset:1024\n"
        "v_mfma_f32_16x16x32_f16 v[56:59], v[180:183], v[232:235], v[56:59]\n"
        "v_mfma_f32_16x16x32_f16 v[88:91], v[184:187], v[232:235], v[88:91]\n"
        "v_mfma_f32_16x16x32_f16 v[120:123], v[188:191], v[232:235], v[120:123]\n"
        "s_waitcnt lgkmcnt(2)\n"
        "v_mfma_f32_16x16x32_f16 v[28:31], v[176:179], v[236:239], v[28:31]\n"
        "ds_read_b128 v[232:235], %[brd] offset:2048\n"
        "v_mfma_f32_16x16x32_f16 v[60:63], v[180:183], v[236:239], v[60:63]\n"
        "v_mfma_f32_16x16x32_f16 v[92:95], v[184:187], v[236:239], v[92:95]\n"
        "v_mfma_f32_16x16x32_f16 v[124:127], v[188:191], v[236:239], v[124:127]\n"
        "s_add_u32 s64, s64, 8\n"
        "s_sub_u32 s65, s65, 8\n"
        "s_sub_u32 s66, s66, 8\n"
        "s_cmp_lt_u32 s64, 56\n"
        "s_cbranch_scc1 L_loopX%=\n"
        "s_barrier\n"
        "s_setprio 2\n"
        "s_lshl_b32 s80, s74, 16\n"
        "s_add_u32 s80, s80, s62\n"
        "s_add_u32 s86, s80, 4096\n"
        "buffer_load_dwordx4 v[160:163], %[avoff], %[rsA], s80 offen offset:0\n"
        "s_waitcnt vmcnt(9)\n"
        "v_cvt_pk_f16_f32 v240, v192, v196\n"
        "v_cvt_pk_f16_f32 v241, v200, v204\n"
        "ds_write_b64 %[bwr], v[240:241] offset:49152\n"
        "buffer_load_dwordx4 v[164:167], %[avoff], %[rsA], s80 offen offset:1024\n"
        "v_cvt_pk_f16_f32 v242, v193, v197\n"
        "v_cvt_pk_f16_f32 v243, v201, v205\n"
        "ds_write_b64 %[bwr], v[242:243] offset:50176\n"
        "buffer_load_dwordx4 v[168:171], %[avoff], %[rsA], s86 offen offset:0\n"
        "v_cvt_pk_f16_f32 v240, v194, v198\n"
        "v_cvt_pk_f16_f32 v241, v202, v206\n"
        "ds_write_b64 %[bwr], v[240:241] offset:51200\n"
        "buffer_load_dwordx4 v[172:175], %[avoff], %[rsA], s86 offen offset:1024\n"
        "v_cvt_pk_f16_f32 v242, v195, v199\n"
        "v_cvt_pk_f16_f32 v243, v203, v207\n"
        "ds_write_b64 %[bwr], v[242:243] offset:52224\n"
        "s_lshl_b32 s81, s75, 16\n"
        "s_add_u32 s81, s81, s62\n"
        "s_add_u32 s87, s81, 4096\n"
        "buffer_load_dwordx4 v[176:179], %[avoff], %[rsA], s81 offen offset:0\n"
        "buffer_load_dwordx4 v[180:183], %[avoff], %[rsA], s81 offen offset:1024\n"
        "s_barrier\n"
        "s_lshl_b32 s82, s77, 23\n"
        "buffer_load_dwordx4 v[192:195], %[wvoff], %[rsW], s82 offen nt\n"
        "s_lshl_b32 s82, s77, 23\n"
        "s_add_u32 s82, s82, 131072\n"
        "buffer_load_dwordx4 v[196:199], %[wvoff], %[rsW], s82 offen nt\n"
        "s_lshl_b32 s82, s77, 23\n"
        "s_add_u32 s82, s82, 262144\n"
        "buffer_load_dwordx4 v[200:203], %[wvoff], %[rsW], s82 offen nt\n"
        "s_setprio 0\n"
        "s_waitcnt lgkmcnt(6)\n"
        "v_mfma_f32_16x16x32_f16 v[0:3], v[128:131], v[224:227], v[0:3]\n"
        "ds_read_b128 v[236:239], %[brd] offset:3072\n"
        "v_mfma_f32_16x16x32_f16 v[32:35], v[132:135], v[224:227], v[32:35]\n"
        "s_waitcnt lgkmcnt(6)\n"
        "v_mfma_f32_16x16x32_f16 v[4:7], v[128:131], v[228:231], v[4:7]\n"
        "ds_read_b128 v[224:227], %[brd] offset:4096\n"
        "v_mfma_f32_16x16x32_f16 v[36:39], v[132:135], v[228:231], v[36:39]\n"
        "s_waitcnt lgkmcnt(6)\n"
        "v_mfma_f32_16x16x32_f16 v[8:11], v[128:131], v[232:235], v[8:11]\n"
        "ds_read_b128 v[228:231], %[brd] offset:5120\n"
        "v_mfma_f32_16x16x32_f16 v[40:43], v[132:135], v[232:235], v[40:43]\n"
        "s_waitcnt lgkmcnt(2)\n"
        "v_mfma_f32_16x16x32_f16 v[12:15], v[128:131], v[236:239], v[12:15]\n"
        "ds_read_b128 v[232:235], %[brd] offset:6144\n"
        "v_mfma_f32_16x16x32_f16 v[44:47], v[132:135], v[236:239], v[44:47]\n"
        "s_waitcnt lgkmcnt(2)\n"
        "v_mfma_f32_16x16x32_f16 v[16:19], v[128:131], v[224:227], v[16:19]\n"
        "ds_read_b128 v[236:239], %[brd] offset:7168\n"
        "v_mfma_f32_16x16x32_f16 v[48:51], v[132:135], v[224:227], v[48:51]\n"
        "s_waitcnt lgkmcnt(2)\n"
        "v_mfma_f32_16x16x32_f16 v[20:23], v[128:131], v[228:231], v[20:23]\n"
        "ds_read_b128 v[224:227], %[brd] offset:8192\n"
        "v_mfma_f32_16x16x32_f16 v[52:55], v[132:135], v[228:231], v[52:55]\n"
        "s_waitcnt lgkmcnt(2)\n"
        "v_mfma_f32_16x16x32_f16 v[24:27], v[128:131], v[232:235], v[24:27]\n"
        "ds_read_b128 v[228:231], %[brd] offset:9216\n"
        "v_mfma_f32_16x16x32_f16 v[56:59], v[132:135], v[232:235], v[56:59]\n"
        "s_waitcnt lgkmcnt(2)\n"
        "v_mfma_f32_16x16x32_f16 v[28:31], v[128:131], v[236:239], v[28:31]\n"
        "ds_read_b128 v[232:235], %[brd] offset:10240\n"
        "v_mfma_f32_16x16x32_f16 v[60:63], v[132:135], v[236:239], v[60:63]\n"
        "s_waitcnt lgkmcnt(2)\n"
        "v_mfma_f32_16x16x32_f16 v[0:3], v[136:139], v[224:227], v[0:3]\n"
        "ds_read_b128 v[236:239], %[brd] offset:11264\n"
        "v_mfma_f32_16x16x32_f16 v[32:35], v[140:143], v[224:227], v[32:35]\n"
        "s_waitcnt lgkmcnt(2)\n"
        "v_mfma_f32_16x16x32_f16 v[4:7], v[136:139], v[228:231], v[4:7]\n"
        "ds_read_b128 v[224:227], %[brd] offset:12288\n"
        "v_mfma_f32_16x16x32_f16 v[36:39], v[140:143], v[228:231], v[36:39]\n"
        "s_waitcnt lgkmcnt(2)\n"
        "v_mfma_f32_16x16x32_f16 v[8:11], v[136:139], v[232:235], v[8:11]\n"
        "ds_read_b128 v[228:231], %[brd] offset:13312\n"
        "v_mfma_f32_16x16x32_f16 v[40:43], v[140:143], v[232:235], v[40:43]\n"
        "s_waitcnt lgkmcnt(2)\n"
        "v_mfma_f32_16x16x32_f16 v[12:15], v[136:139], v[236:239], v[12:15]\n"
        "ds_read_b128 v[232:235], %[brd] offset:14336\n"
        "v_mfma_f32_16x16x32_f16 v[44:47], v[140:143], v[236:239], v[44:47]\n"
        "s_waitcnt lgkmcnt(2)\n"
        "v_mfma_f32_16x16x32_f16 v[16:19], v[136:139], v[224:227], v[16:19]\n"
        "ds_read_b128 v[236:239], %[brd] offset:15360\n"
        "v_mfma_f32_16x16x32_f16 v[48:51], v[140:143], v[224:227], v[48:51]\n"
        "s_waitcnt lgkmcnt(2)\n"
        "v_mfma_f32_16x16x32_f16 v[20:23], v[136:139], v[228:231], v[20:23]\n"
        "ds_read_b128 v[224:227], %[brd] offset:16384\n"
        "v_mfma_f32_16x16x32_f16 v[52:55], v[140:143], v[228:231], v[52:55]\n"
        "s_waitcnt lgkmcnt(2)\n"
        "v_mfma_f32_16x16x32_f16 v[24:27], v[136:139], v[232:235], v[24:27]\n"
        "ds_read_b128 v[228:231], %[brd] offset:17408\n"
        "v_mfma_f32_16x16x32_f16 v[56:59], v[140:143], v[232:235], v[56:59]\n"
        "s_waitcnt lgkmcnt(2)\n"
        "v_mfma_f32_16x16x32_f16 v[28:31], v[136:139], v[236:239], v[28:31]\n"
        "ds_read_b128 v[232:235], %[brd] offset:18432\n"
        "v_mfma_f32_16x16x32_f16 v[60:63], v[140:143], v[236:239], v[60:63]\n"
        "s_waitcnt vmcnt(12) lgkmcnt(2)\n"
        "v_mfma_f32_16x16x32_f16 v[0:3], v[144:147], v[224:227], v[0:3]\n"
        "ds_read_b128 v[236:239], %[brd] offset:19456\n"
        "v_mfma_f32_16x16x32_f16 v[32:35], v[148:151], v[224:227], v[32:35]\n"
        "s_waitcnt lgkmcnt(2)\n"
        "v_mfma_f32_16x16x32_f16 v[4:7], v[144:147], v[228:231], v[4:7]\n"
        "ds_read_b128 v[224:227], %[brd] offset:20480\n"
        "v_mfma_f32_16x16x32_f16 v[36:39], v[148:151], v[228:231], v[36:39]\n"
        "s_waitcnt lgkmcnt(2)\n"
        "v_mfma_f32_16x16x32_f16 v[8:11], v[144:147], v[232:235], v[8:11]\n"
        "ds_read_b128 v[228:231], %[brd] offset:21504\n"
        "v_mfma_f32_16x16x32_f16 v[40:43], v[148:151], v[232:235], v[40:43]\n"
        "s_waitcnt lgkmcnt(2)\n"
        "v_mfma_f32_16x16x32_f16 v[12:15], v[144:147], v[236:239], v[12:15]\n"
        "ds_read_b128 v[232:235], %[brd] offset:22528\n"
        "v_mfma_f32_16x16x32_f16 v[44:47], v[148:151], v[236:239], v[44:47]\n"
        "s_waitcnt lgkmcnt(2)\n"
        "v_mfma_f32_16x16x32_f16 v[16:19], v[144:147], v[224:227], v[16:19]\n"
        "ds_read_b128 v[236:239], %[brd] offset:23552\n"
        "v_mfma_f32_16x16x32_f16 v[48:51], v[148:151], v[224:227], v[48:51]\n"
        "s_waitcnt lgkmcnt(2)\n"
        "v_mfma_f32_16x16x32_f16 v[20:23], v[144:147], v[228:231], v[20:23]\n"
        "ds_read_b128 v[224:227], %[brd] offset:24576\n"
        "v_mfma_f32_16x16x32_f16 v[52:55], v[148:151], v[228:231], v[52:55]\n"
        "s_barrier\n"
        "s_setprio 2\n"
        "buffer_load_dwordx4 v[184:187], %[avoff], %[rsA], s87 offen offset:0\n"
        "s_waitcnt vmcnt(10)\n"
        "v_cvt_pk_f16_f32 v240, v208, v212\n"
        "v_cvt_pk_f16_f32 v241, v216, v220\n"
        "ds_write_b64 v245, v[240:241] offset:0\n"
        "buffer_load_dwordx4 v[188:191], %[avoff], %[rsA], s87 offen offset:1024\n"
        "v_cvt_pk_f16_f32 v242, v209, v213\n"
        "v_cvt_pk_f16_f32 v243, v217, v221\n"
        "ds_write_b64 v245, v[242:243] offset:1024\n"
        "s_lshl_b32 s80, s76, 16\n"
        "s_add_u32 s80, s80, s62\n"
        "s_add_u32 s86, s80, 4096\n"
        "buffer_load_dwordx4 v[128:131], %[avoff], %[rsA], s80 offen offset:0\n"
        "v_cvt_pk_f16_f32 v240, v210, v214\n"
        "v_cvt_pk_f16_f32 v241, v218, v222\n"
        "ds_write_b64 v245, v[240:241] offset:2048\n"
        "buffer_load_dwordx4 v[132:135], %[avoff], %[rsA], s80 offen offset:1024\n"
        "v_cvt_pk_f16_f32 v242, v211, v215\n"
        "v_cvt_pk_f16_f32 v243, v219, v223\n"
        "ds_write_b64 v245, v[242:243] offset:3072\n"
        "buffer_load_dwordx4 v[136:139], %[avoff], %[rsA], s86 offen offset:0\n"
        "buffer_load_dwordx4 v[140:143], %[avoff], %[rsA], s86 offen offset:1024\n"
        "s_barrier\n"
        "s_lshl_b32 s82, s77, 23\n"
        "s_add_u32 s82, s82, 393216\n"
        "buffer_load_dwordx4 v[204:207], %[wvoff], %[rsW], s82 offen nt\n"
        "s_lshl_b32 s82, s78, 23\n"
        "buffer_load_dwordx4 v[208:211], %[wvoff], %[rsW], s82 offen nt\n"
        "s_lshl_b32 s82, s78, 23\n"
        "s_add_u32 s82, s82, 131072\n"
        "buffer_load_dwordx4 v[212:215], %[wvoff], %[rsW], s82 offen nt\n"
        "s_setprio 0\n"
        "s_waitcnt lgkmcnt(6)\n"
        "v_mfma_f32_16x16x32_f16 v[24:27], v[144:147], v[232:235], v[24:27]\n"
        "ds_read_b128 v[228:231], %[brd] offset:25600\n"
        "v_mfma_f32_16x16x32_f16 v[56:59], v[148:151], v[232:235], v[56:59]\n"
        "s_waitcnt lgkmcnt(6)\n"
        "v_mfma_f32_16x16x32_f16 v[28:31], v[144:147], v[236:239], v[28:31]\n"
        "ds_read_b128 v[232:235], %[brd] offset:26624\n"
        "v_mfma_f32_16x16x32_f16 v[60:63], v[148:151], v[236:239], v[60:63]\n"
        "s_waitcnt lgkmcnt(6)\n"
        "v_mfma_f32_16x16x32_f16 v[0:3], v[152:155], v[224:227], v[0:3]\n"
        "ds_read_b128 v[236:239], %[brd] offset:27648\n"
        "v_mfma_f32_16x16x32_f16 v[32:35], v[156:159], v[224:227], v[32:35]\n"
        "s_waitcnt lgkmcnt(2)\n"
        "v_mfma_f32_16x16x32_f16 v[4:7], v[152:155], v[228:231], v[4:7]\n"
        "ds_read_b128 v[224:227], %[brd] offset:28672\n"
        "v_mfma_f32_16x16x32_f16 v[36:39], v[156:159], v[228:231], v[36:39]\n"
        "s_waitcnt lgkmcnt(2)\n"
        "v_mfma_f32_16x16x32_f16 v[8:11], v[152:155], v[232:235], v[8:11]\n"
        "ds_read_b128 v[228:231], %[brd] offset:29696\n"
        "v_mfma_f32_16x16x32_f16 v[40:43], v[156:159], v[232:235], v[40:43]\n"
        "s_waitcnt lgkmcnt(2)\n"
        "v_mfma_f32_16x16x32_f16 v[12:15], v[152:155], v[236:239], v[12:15]\n"
        "ds_read_b128 v[232:235], %[brd] offset:30720\n"
        "v_mfma_f32_16x16x32_f16 v[44:47], v[156:159], v[236:239], v[44:47]\n"
        "s_waitcnt lgkmcnt(2)\n"
        "v_mfma_f32_16x16x32_f16 v[16:19], v[152:155], v[224:227], v[16:19]\n"
        "ds_read_b128 v[236:239], %[brd] offset:31744\n"
        "v_mfma_f32_16x16x32_f16 v[48:51], v[156:159], v[224:227], v[48:51]\n"
        "s_waitcnt lgkmcnt(2)\n"
        "v_mfma_f32_16x16x32_f16 v[20:23], v[152:155], v[228:231], v[20:23]\n"
        "ds_read_b128 v[224:227], %[brd] offset:32768\n"
        "v_mfma_f32_16x16x32_f16 v[52:55], v[156:159], v[228:231], v[52:55]\n"
        "s_waitcnt lgkmcnt(2)\n"
        "v_mfma_f32_16x16x32_f16 v[24:27], v[152:155], v[232:235], v[24:27]\n"
        "ds_read_b128 v[228:231], %[brd] offset:33792\n"
        "v_mfma_f32_16x16x32_f16 v[56:59], v[156:159], v[232:235], v[56:59]\n"
        "s_waitcnt lgkmcnt(2)\n"
        "v_mfma_f32_16x16x32_f16 v[28:31], v[152:155], v[236:239], v[28:31]\n"
        "ds_read_b128 v[232:235], %[brd] offset:34816\n"
        "v_mfma_f32_16x16x32_f16 v[60:63], v[156:159], v[236:239], v[60:63]\n"
        "s_waitcnt vmcnt(14) lgkmcnt(2)\n"
        "v_mfma_f32_16x16x32_f16 v[0:3], v[160:163], v[224:227], v[0:3]\n"
        "ds_read_b128 v[236:239], %[brd] offset:35840\n"
        "v_mfma_f32_16x16x32_f16 v[32:35], v[164:167], v[224:227], v[32:35]\n"
        "s_waitcnt lgkmcnt(2)\n"
        "v_mfma_f32_16x16x32_f16 v[4:7], v[160:163], v[228:231], v[4:7]\n"
        "ds_read_b128 v[224:227], %[brd] offset:36864\n"
        "v_mfma_f32_16x16x32_f16 v[36:39], v[164:167], v[228:231], v[36:39]\n"
        "s_waitcnt lgkmcnt(2)\n"
        "v_mfma_f32_16x16x32_f16 v[8:11], v[160:163], v[232:235], v[8:11]\n"
        "ds_read_b128 v[228:231], %[brd] offset:37888\n"
        "v_mfma_f32_16x16x32_f16 v[40:43], v[164:167], v[232:235], v[40:43]\n"
        "s_waitcnt lgkmcnt(2)\n"
        "v_mfma_f32_16x16x32_f16 v[12:15], v[160:163], v[236:239], v[12:15]\n"
        "ds_read_b128 v[232:235], %[brd] offset:38912\n"
        "v_mfma_f32_16x16x32_f16 v[44:47], v[164:167], v[236:239], v[44:47]\n"
        "s_waitcnt lgkmcnt(2)\n"
        "v_mfma_f32_16x16x32_f16 v[16:19], v[160:163], v[224:227], v[16:19]\n"
        "ds_read_b128 v[236:239], %[brd] offset:39936\n"
        "v_mfma_f32_16x16x32_f16 v[48:51], v[164:167], v[224:227], v[48:51]\n"
        "s_waitcnt lgkmcnt(2)\n"
        "v_mfma_f32_16x16x32_f16 v[20:23], v[160:163], v[228:231], v[20:23]\n"
        "ds_read_b128 v[224:227], %[brd] offset:40960\n"
        "v_mfma_f32_16x16x32_f16 v[52:55], v[164:167], v[228:231], v[52:55]\n"
        "s_waitcnt lgkmcnt(2)\n"
        "v_mfma_f32_16x16x32_f16 v[24:27], v[160:163], v[232:235], v[24:27]\n"
        "ds_read_b128 v[228:231], %[brd] offset:41984\n"
        "v_mfma_f32_16x16x32_f16 v[56:59], v[164:167], v[232:235], v[56:59]\n"
        "s_waitcnt lgkmcnt(2)\n"
        "v_mfma_f32_16x16x32_f16 v[28:31], v[160:163], v[236:239], v[28:31]\n"
        "ds_read_b128 v[232:235], %[brd] offset:43008\n"
        "v_mfma_f32_16x16x32_f16 v[60:63], v[164:167], v[236:239], v[60:63]\n"
        "s_waitcnt lgkmcnt(2)\n"
        "v_mfma_f32_16x16x32_f16 v[0:3], v[168:171], v[224:227], v[0:3]\n"
        "ds_read_b128 v[236:239], %[brd] offset:44032\n"
        "v_mfma_f32_16x16x32_f16 v[32:35], v[172:175], v[224:227], v[32:35]\n"
        "s_waitcnt lgkmcnt(2)\n"
        "v_mfma_f32_16x16x32_f16 v[4:7], v[168:171], v[228:231], v[4:7]\n"
        "ds_read_b128 v[224:227], %[brd] offset:45056\n"
        "v_mfma_f32_16x16x32_f16 v[36:39], v[172:175], v[228:231], v[36:39]\n"
        "s_barrier\n"
        "s_setprio 2\n"
        "s_lshl_b32 s81, s77, 16\n"
        "s_add_u32 s81, s81, s62\n"
        "s_add_u32 s87, s81, 4096\n"
        "buffer_load_dwordx4 v[144:147], %[avoff], %[rsA], s81 offen offset:0\n"
        "s_waitcnt vmcnt(3)\n"
        "v_cvt_pk_f16_f32 v240, v192, v196\n"
        "v_cvt_pk_f16_f32 v241, v200, v204\n"
        "ds_write_b64 v245, v[240:241] offset:16384\n"
        "buffer_load_dwordx4 v[148:151], %[avoff], %[rsA], s81 offen offset:1024\n"
        "v_cvt_pk_f16_f32 v242, v193, v197\n"
        "v_cvt_pk_f16_f32 v243, v201, v205\n"
        "ds_write_b64 v245, v[242:243] offset:17408\n"
        "buffer_load_dwordx4 v[152:155], %[avoff], %[rsA], s87 offen offset:0\n"
        "v_cvt_pk_f16_f32 v240, v194, v198\n"
        "v_cvt_pk_f16_f32 v241, v202, v206\n"
        "ds_write_b64 v245, v[240:241] offset:18432\n"
        "buffer_load_dwordx4 v[156:159], %[avoff], %[rsA], s87 offen offset:1024\n"
        "v_cvt_pk_f16_f32 v242, v195, v199\n"
        "v_cvt_pk_f16_f32 v243, v203, v207\n"
        "ds_write_b64 v245, v[242:243] offset:19456\n"
        "s_barrier\n"
        "s_lshl_b32 s82, s78, 23\n"
        "s_add_u32 s82, s82, 262144\n"
        "buffer_load_dwordx4 v[216:219], %[wvoff], %[rsW], s82 offen nt\n"
        "s_lshl_b32 s82, s78, 23\n"
        "s_add_u32 s82, s82, 393216\n"
        "buffer_load_dwordx4 v[220:223], %[wvoff], %[rsW], s82 offen nt\n"
        "s_lshl_b32 s82, s79, 23\n"
        "buffer_load_dwordx4 v[192:195], %[wvoff], %[rsW], s82 offen nt\n"
        "s_setprio 0\n"
        "s_waitcnt lgkmcnt(6)\n"
        "v_mfma_f32_16x16x32_f16 v[8:11], v[168:171], v[232:235], v[8:11]\n"
        "ds_read_b128 v[228:231], %[brd] offset:46080\n"
        "v_mfma_f32_16x16x32_f16 v[40:43], v[172:175], v[232:235], v[40:43]\n"
        "s_waitcnt lgkmcnt(6)\n"
        "v_mfma_f32_16x16x32_f16 v[12:15], v[168:171], v[236:239], v[12:15]\n"
        "ds_read_b128 v[232:235], %[brd] offset:47104\n"
        "v_mfma_f32_16x16x32_f16 v[44:47], v[172:175], v[236:239], v[44:47]\n"
        "s_waitcnt lgkmcnt(6)\n"
        "v_mfma_f32_16x16x32_f16 v[16:19], v[168:171], v[224:227], v[16:19]\n"
        "ds_read_b128 v[236:239], %[brd] offset:48128\n"
        "v_mfma_f32_16x16x32_f16 v[48:51], v[172:175], v[224:227], v[48:51]\n"
        "s_waitcnt lgkmcnt(2)\n"
        "v_mfma_f32_16x16x32_f16 v[20:23], v[168:171], v[228:231], v[20:23]\n"
        "ds_read_b128 v[224:227], %[brd] offset:49152\n"
        "v_mfma_f32_16x16x32_f16 v[52:55], v[172:175], v[228:231], v[52:55]\n"
        "s_waitcnt lgkmcnt(2)\n"
        "v_mfma_f32_16x16x32_f16 v[24:27], v[168:171], v[232:235], v[24:27]\n"
        "ds_read_b128 v[228:231], %[brd] offset:50176\n"
        "v_mfma_f32_16x16x32_f16 v[56:59], v[172:175], v[232:235], v[56:59]\n"
        "s_waitcnt lgkmcnt(2)\n"
        "v_mfma_f32_16x16x32_f16 v[28:31], v[168:171], v[236:239], v[28:31]\n"
        "ds_read_b128 v[232:235], %[brd] offset:51200\n"
        "v_mfma_f32_16x16x32_f16 v[60:63], v[172:175], v[236:239], v[60:63]\n"
        "s_waitcnt lgkmcnt(2)\n"
        "v_mfma_f32_16x16x32_f16 v[0:3], v[176:179], v[224:227], v[0:3]\n"
        "ds_read_b128 v[236:239], %[brd] offset:52224\n"
        "v_mfma_f32_16x16x32_f16 v[32:35], v[180:183], v[224:227], v[32:35]\n"
        "s_waitcnt lgkmcnt(2)\n"
        "v_mfma_f32_16x16x32_f16 v[4:7], v[176:179], v[228:231], v[4:7]\n"
        "ds_read_b128 v[224:227], %[brd] offset:53248\n"
        "v_mfma_f32_16x16x32_f16 v[36:39], v[180:183], v[228:231], v[36:39]\n"
        "s_waitcnt lgkmcnt(2)\n"
        "v_mfma_f32_16x16x32_f16 v[8:11], v[176:179], v[232:235], v[8:11]\n"
        "ds_read_b128 v[228:231], %[brd] offset:54272\n"
        "v_mfma_f32_16x16x32_f16 v[40:43], v[180:183], v[232:235], v[40:43]\n"
        "s_waitcnt lgkmcnt(2)\n"
        "v_mfma_f32_16x16x32_f16 v[12:15], v[176:179], v[236:239], v[12:15]\n"
        "ds_read_b128 v[232:235], %[brd] offset:55296\n"
        "v_mfma_f32_16x16x32_f16 v[44:47], v[180:183], v[236:239], v[44:47]\n"
        "s_waitcnt lgkmcnt(2)\n"
        "v_mfma_f32_16x16x32_f16 v[16:19], v[176:179], v[224:227], v[16:19]\n"
        "ds_read_b128 v[236:239], %[brd] offset:56320\n"
        "v_mfma_f32_16x16x32_f16 v[48:51], v[180:183], v[224:227], v[48:51]\n"
        "s_waitcnt lgkmcnt(2)\n"
        "v_mfma_f32_16x16x32_f16 v[20:23], v[176:179], v[228:231], v[20:23]\n"
        "ds_read_b128 v[224:227], %[brd] offset:57344\n"
        "v_mfma_f32_16x16x32_f16 v[52:55], v[180:183], v[228:231], v[52:55]\n"
        "s_waitcnt lgkmcnt(2)\n"
        "v_mfma_f32_16x16x32_f16 v[24:27], v[176:179], v[232:235], v[24:27]\n"
        "ds_read_b128 v[228:231], %[brd] offset:58368\n"
        "v_mfma_f32_16x16x32_f16 v[56:59], v[180:183], v[232:235], v[56:59]\n"
        "s_waitcnt lgkmcnt(2)\n"
        "v_mfma_f32_16x16x32_f16 v[28:31], v[176:179], v[236:239], v[28:31]\n"
        "ds_read_b128 v[232:235], %[brd] offset:59392\n"
        "v_mfma_f32_16x16x32_f16 v[60:63], v[180:183], v[236:239], v[60:63]\n"
        "s_waitcnt lgkmcnt(2)\n"
        "v_mfma_f32_16x16x32_f16 v[0:3], v[184:187], v[224:227], v[0:3]\n"
        "ds_read_b128 v[236:239], %[brd] offset:60416\n"
        "v_mfma_f32_16x16x32_f16 v[32:35], v[188:191], v[224:227], v[32:35]\n"
        "s_waitcnt lgkmcnt(2)\n"
        "v_mfma_f32_16x16x32_f16 v[4:7], v[184:187], v[228:231], v[4:7]\n"
        "ds_read_b128 v[224:227], %[brd] offset:61440\n"
        "v_mfma_f32_16x16x32_f16 v[36:39], v[188:191], v[228:231], v[36:39]\n"
        "s_waitcnt lgkmcnt(2)\n"
        "v_mfma_f32_16x16x32_f16 v[8:11], v[184:187], v[232:235], v[8:11]\n"
        "ds_read_b128 v[228:231], %[brd] offset:62464\n"
        "v_mfma_f32_16x16x32_f16 v[40:43], v[188:191], v[232:235], v[40:43]\n"
        "s_waitcnt lgkmcnt(2)\n"
        "v_mfma_f32_16x16x32_f16 v[12:15], v[184:187], v[236:239], v[12:15]\n"
        "ds_read_b128 v[232:235], %[brd] offset:63488\n"
        "v_mfma_f32_16x16x32_f16 v[44:47], v[188:191], v[236:239], v[44:47]\n"
        "s_waitcnt lgkmcnt(2)\n"
        "v_mfma_f32_16x16x32_f16 v[16:19], v[184:187], v[224:227], v[16:19]\n"
        "ds_read_b128 v[236:239], %[brd] offset:64512\n"
        "v_mfma_f32_16x16x32_f16 v[48:51], v[188:191], v[224:227], v[48:51]\n"
        "s_waitcnt lgkmcnt(2)\n"
        "v_mfma_f32_16x16x32_f16 v[20:23], v[184:187], v[228:231], v[20:23]\n"
        "ds_read_b128 v[224:227], v244 offset:0\n"
        "v_mfma_f32_16x16x32_f16 v[52:55], v[188:191], v[228:231], v[52:55]\n"
        "s_waitcnt lgkmcnt(2)\n"
        "v_mfma_f32_16x16x32_f16 v[24:27], v[184:187], v[232:235], v[24:27]\n"
        "ds_read_b128 v[228:231], v244 offset:1024\n"
        "v_mfma_f32_16x16x32_f16 v[56:59], v[188:191], v[232:235], v[56:59]\n"
        "s_waitcnt lgkmcnt(2)\n"
        "v_mfma_f32_16x16x32_f16 v[28:31], v[184:187], v[236:239], v[28:31]\n"
        "ds_read_b128 v[232:235], v244 offset:2048\n"
        "v_mfma_f32_16x16x32_f16 v[60:63], v[188:191], v[236:239], v[60:63]\n"
        "s_barrier\n"
        "s_setprio 2\n"
        "s_lshl_b32 s80, s78, 16\n"
        "s_add_u32 s80, s80, s62\n"
        "s_add_u32 s86, s80, 4096\n"
        "buffer_load_dwordx4 v[160:163], %[avoff], %[rsA], s80 offen offset:0\n"
        "s_waitcnt vmcnt(2)\n"
        "v_cvt_pk_f16_f32 v240, v208, v212\n"
        "v_cvt_pk_f16_f32 v241, v216, v220\n"
        "ds_write_b64 v245, v[240:241] offset:32768\n"
        "buffer_load_dwordx4 v[164:167], %[avoff], %[rsA], s80 offen offset:1024\n"
        "v_cvt_pk_f16_f32 v242, v209, v213\n"
        "v_cvt_pk_f16_f32 v243, v217, v221\n"
        "ds_write_b64 v245, v[242:243] offset:33792\n"
        "buffer_load_dwordx4 v[168:171], %[avoff], %[rsA], s86 offen offset:0\n"
        "v_cvt_pk_f16_f32 v240, v210, v214\n"
        "v_cvt_pk_f16_f32 v241, v218, v222\n"
        "ds_write_b64 v245, v[240:241] offset:34816\n"
        "buffer_load_dwordx4 v[172:175], %[avoff], %[rsA], s86 offen offset:1024\n"
        "v_cvt_pk_f16_f32 v242, v211, v215\n"
        "v_cvt_pk_f16_f32 v243, v219, v223\n"
        "ds_write_b64 v245, v[242:243] offset:35840\n"
        "s_lshl_b32 s81, s79, 16\n"
        "s_add_u32 s81, s81, s62\n"
        "s_add_u32 s87, s81, 4096\n"
        "buffer_load_dwordx4 v[176:179], %[avoff], %[rsA], s81 offen offset:0\n"
        "buffer_load_dwordx4 v[180:183], %[avoff], %[rsA], s81 offen offset:1024\n"
        "s_barrier\n"
        "s_lshl_b32 s82, s79, 23\n"
        "s_add_u32 s82, s82, 131072\n"
        "buffer_load_dwordx4 v[196:199], %[wvoff], %[rsW], s82 offen nt\n"
        "s_lshl_b32 s82, s79, 23\n"
        "s_add_u32 s82, s82, 262144\n"
        "buffer_load_dwordx4 v[200:203], %[wvoff], %[rsW], s82 offen nt\n"
        "s_lshl_b32 s82, s79, 23\n"
        "s_add_u32 s82, s82, 393216\n"
        "buffer_load_dwordx4 v[204:207], %[wvoff], %[rsW], s82 offen nt\n"
        "s_setprio 0\n"
        "s_waitcnt lgkmcnt(6)\n"
        "v_mfma_f32_16x16x32_f16 v[0:3], v[128:131], v[224:227], v[0:3]\n"
        "ds_read_b128 v[236:239], v244 offset:3072\n"
        "v_mfma_f32_16x16x32_f16 v[32:35], v[132:135], v[224:227], v[32:35]\n"
        "s_waitcnt lgkmcnt(6)\n"
        "v_mfma_f32_16x16x32_f16 v[4:7], v[128:131], v[228:231], v[4:7]\n"
        "ds_read_b128 v[224:227], v244 offset:4096\n"
        "v_mfma_f32_16x16x32_f16 v[36:39], v[132:135], v[228:231], v[36:39]\n"
        "s_waitcnt lgkmcnt(6)\n"
        "v_mfma_f32_16x16x32_f16 v[8:11], v[128:131], v[232:235], v[8:11]\n"
        "ds_read_b128 v[228:231], v244 offset:5120\n"
        "v_mfma_f32_16x16x32_f16 v[40:43], v[132:135], v[232:235], v[40:43]\n"
        "s_waitcnt lgkmcnt(2)\n"
        "v_mfma_f32_16x16x32_f16 v[12:15], v[128:131], v[236:239], v[12:15]\n"
        "ds_read_b128 v[232:235], v244 offset:6144\n"
        "v_mfma_f32_16x16x32_f16 v[44:47], v[132:135], v[236:239], v[44:47]\n"
        "s_waitcnt lgkmcnt(2)\n"
        "v_mfma_f32_16x16x32_f16 v[16:19], v[128:131], v[224:227], v[16:19]\n"
        "ds_read_b128 v[236:239], v244 offset:7168\n"
        "v_mfma_f32_16x16x32_f16 v[48:51], v[132:135], v[224:227], v[48:51]\n"
        "s_waitcnt lgkmcnt(2)\n"
        "v_mfma_f32_16x16x32_f16 v[20:23], v[128:131], v[228:231], v[20:23]\n"
        "ds_read_b128 v[224:227], v244 offset:8192\n"
        "v_mfma_f32_16x16x32_f16 v[52:55], v[132:135], v[228:231], v[52:55]\n"
        "s_waitcnt lgkmcnt(2)\n"
        "v_mfma_f32_16x16x32_f16 v[24:27], v[128:131], v[232:235], v[24:27]\n"
        "ds_read_b128 v[228:231], v244 offset:9216\n"
        "v_mfma_f32_16x16x32_f16 v[56:59], v[132:135], v[232:235], v[56:59]\n"
        "s_waitcnt lgkmcnt(2)\n"
        "v_mfma_f32_16x16x32_f16 v[28:31], v[128:131], v[236:239], v[28:31]\n"
        "ds_read_b128 v[232:235], v244 offset:10240\n"
        "v_mfma_f32_16x16x32_f16 v[60:63], v[132:135], v[236:239], v[60:63]\n"
        "s_waitcnt lgkmcnt(2)\n"
        "v_mfma_f32_16x16x32_f16 v[0:3], v[136:139], v[224:227], v[0:3]\n"
        "ds_read_b128 v[236:239], v244 offset:11264\n"
        "v_mfma_f32_16x16x32_f16 v[32:35], v[140:143], v[224:227], v[32:35]\n"
        "s_waitcnt lgkmcnt(2)\n"
        "v_mfma_f32_16x16x32_f16 v[4:7], v[136:139], v[228:231], v[4:7]\n"
        "ds_read_b128 v[224:227], v244 offset:12288\n"
        "v_mfma_f32_16x16x32_f16 v[36:39], v[140:143], v[228:231], v[36:39]\n"
        "s_waitcnt lgkmcnt(2)\n"
        "v_mfma_f32_16x16x32_f16 v[8:11], v[136:139], v[232:235], v[8:11]\n"
        "ds_read_b128 v[228:231], v244 offset:13312\n"
        "v_mfma_f32_16x16x32_f16 v[40:43], v[140:143], v[232:235], v[40:43]\n"
        "s_waitcnt lgkmcnt(2)\n"
        "v_mfma_f32_16x16x32_f16 v[12:15], v[136:139], v[236:239], v[12:15]\n"
        "ds_read_b128 v[232:235], v244 offset:14336\n"
        "v_mfma_f32_16x16x32_f16 v[44:47], v[140:143], v[236:239], v[44:47]\n"
        "s_waitcnt lgkmcnt(2)\n"
        "v_mfma_f32_16x16x32_f16 v[16:19], v[136:139], v[224:227], v[16:19]\n"
        "ds_read_b128 v[236:239], v244 offset:15360\n"
        "v_mfma_f32_16x16x32_f16 v[48:51], v[140:143], v[224:227], v[48:51]\n"
        "s_waitcnt lgkmcnt(2)\n"
        "v_mfma_f32_16x16x32_f16 v[20:23], v[136:139], v[228:231], v[20:23]\n"
        "ds_read_b128 v[224:227], v244 offset:16384\n"
        "v_mfma_f32_16x16x32_f16 v[52:55], v[140:143], v[228:231], v[52:55]\n"
        "s_waitcnt lgkmcnt(2)\n"
        "v_mfma_f32_16x16x32_f16 v[24:27], v[136:139], v[232:235], v[24:27]\n"
        "ds_read_b128 v[228:231], v244 offset:17408\n"
        "v_mfma_f32_16x16x32_f16 v[56:59], v[140:143], v[232:235], v[56:59]\n"
        "s_waitcnt lgkmcnt(2)\n"
        "v_mfma_f32_16x16x32_f16 v[28:31], v[136:139], v[236:239], v[28:31]\n"
        "ds_read_b128 v[232:235], v244 offset:18432\n"
        "v_mfma_f32_16x16x32_f16 v[60:63], v[140:143], v[236:239], v[60:63]\n"
        "s_waitcnt lgkmcnt(2)\n"
        "v_mfma_f32_16x16x32_f16 v[0:3], v[144:147], v[224:227], v[0:3]\n"
        "ds_read_b128 v[236:239], v244 offset:19456\n"
        "v_mfma_f32_16x16x32_f16 v[32:35], v[148:151], v[224:227], v[32:35]\n"
        "s_waitcnt lgkmcnt(2)\n"
        "v_mfma_f32_16x16x32_f16 v[4:7], v[144:147], v[228:231], v[4:7]\n"
        "ds_read_b128 v[224:227], v244 offset:20480\n"
        "v_mfma_f32_16x16x32_f16 v[36:39], v[148:151], v[228:231], v[36:39]\n"
        "s_waitcnt lgkmcnt(2)\n"
        "v_mfma_f32_16x16x32_f16 v[8:11], v[144:147], v[232:235], v[8:11]\n"
        "ds_read_b128 v[228:231], v244 offset:21504\n"
        "v_mfma_f32_16x16x32_f16 v[40:43], v[148:151], v[232:235], v[40:43]\n"
        "s_waitcnt lgkmcnt(2)\n"
        "v_mfma_f32_16x16x32_f16 v[12:15], v[144:147], v[236:239], v[12:15]\n"
        "ds_read_b128 v[232:235], v244 offset:22528\n"
        "v_mfma_f32_16x16x32_f16 v[44:47], v[148:151], v[236:239], v[44:47]\n"
        "s_waitcnt lgkmcnt(2)\n"
        "v_mfma_f32_16x16x32_f16 v[16:19], v[144:147], v[224:227], v[16:19]\n"
        "ds_read_b128 v[236:239], v244 offset:23552\n"
        "v_mfma_f32_16x16x32_f16 v[48:51], v[148:151], v[224:227], v[48:51]\n"
        "s_waitcnt lgkmcnt(2)\n"
        "v_mfma_f32_16x16x32_f16 v[20:23], v[144:147], v[228:231], v[20:23]\n"
        "ds_read_b128 v[224:227], v244 offset:24576\n"
        "v_mfma_f32_16x16x32_f16 v[52:55], v[148:151], v[228:231], v[52:55]\n"
        "s_barrier\n"
        "s_setprio 2\n"
        "buffer_load_dwordx4 v[184:187], %[avoff], %[rsA], s87 offen offset:0\n"
        "s_waitcnt vmcnt(1)\n"
        "v_cvt_pk_f16_f32 v240, v192, v196\n"
        "v_cvt_pk_f16_f32 v241, v200, v204\n"
        "ds_write_b64 v245, v[240:241] offset:49152\n"
        "buffer_load_dwordx4 v[188:191], %[avoff], %[rsA], s87 offen offset:1024\n"
        "v_cvt_pk_f16_f32 v242, v193, v197\n"
        "v_cvt_pk_f16_f32 v243, v201, v205\n"
        "ds_write_b64 v245, v[242:243] offset:50176\n"
        "s_lshl_b32 s80, s72, 16\n"
        "s_add_u32 s80, s80, s62\n"
        "s_add_u32 s80, s80, 2048\n"
        "s_add_u32 s86, s80, 4096\n"
        "buffer_load_dwordx4 v[128:131], %[avoff], %[rsA], s80 offen offset:0\n"
        "v_cvt_pk_f16_f32 v240, v194, v198\n"
        "v_cvt_pk_f16_f32 v241, v202, v206\n"
        "ds_write_b64 v245, v[240:241] offset:51200\n"
        "buffer_load_dwordx4 v[132:135], %[avoff], %[rsA], s80 offen offset:1024\n"
        "v_cvt_pk_f16_f32 v242, v195, v199\n"
        "v_cvt_pk_f16_f32 v243, v203, v207\n"
        "ds_write_b64 v245, v[242:243] offset:52224\n"
        "buffer_load_dwordx4 v[136:139], %[avoff], %[rsA], s86 offen offset:0\n"
        "buffer_load_dwordx4 v[140:143], %[avoff], %[rsA], s86 offen offset:1024\n"
        "s_setprio 0\n"
        "s_waitcnt lgkmcnt(6)\n"
        "v_mfma_f32_16x16x32_f16 v[24:27], v[144:147], v[232:235], v[24:27]\n"
        "ds_read_b128 v[228:231], v244 offset:25600\n"
        "v_mfma_f32_16x16x32_f16 v[56:59], v[148:151], v[232:235], v[56:59]\n"
        "s_waitcnt lgkmcnt(6)\n"
        "v_mfma_f32_16x16x32_f16 v[28:31], v[144:147], v[236:239], v[28:31]\n"
        "ds_read_b128 v[232:235], v244 offset:26624\n"
        "v_mfma_f32_16x16x32_f16 v[60:63], v[148:151], v[236:239], v[60:63]\n"
        "s_waitcnt lgkmcnt(6)\n"
        "v_mfma_f32_16x16x32_f16 v[0:3], v[152:155], v[224:227], v[0:3]\n"
        "ds_read_b128 v[236:239], v244 offset:27648\n"
        "v_mfma_f32_16x16x32_f16 v[32:35], v[156:159], v[224:227], v[32:35]\n"
        "s_waitcnt lgkmcnt(2)\n"
        "v_mfma_f32_16x16x32_f16 v[4:7], v[152:155], v[228:231], v[4:7]\n"
        "ds_read_b128 v[224:227], v244 offset:28672\n"
        "v_mfma_f32_16x16x32_f16 v[36:39], v[156:159], v[228:231], v[36:39]\n"
        "s_waitcnt lgkmcnt(2)\n"
        "v_mfma_f32_16x16x32_f16 v[8:11], v[152:155], v[232:235], v[8:11]\n"
        "ds_read_b128 v[228:231], v244 offset:29696\n"
        "v_mfma_f32_16x16x32_f16 v[40:43], v[156:159], v[232:235], v[40:43]\n"
        "s_waitcnt lgkmcnt(2)\n"
        "v_mfma_f32_16x16x32_f16 v[12:15], v[152:155], v[236:239], v[12:15]\n"
        "ds_read_b128 v[232:235], v244 offset:30720\n"
        "v_mfma_f32_16x16x32_f16 v[44:47], v[156:159], v[236:239], v[44:47]\n"
        "s_waitcnt lgkmcnt(2)\n"
        "v_mfma_f32_16x16x32_f16 v[16:19], v[152:155], v[224:227], v[16:19]\n"
        "ds_read_b128 v[236:239], v244 offset:31744\n"
        "v_mfma_f32_16x16x32_f16 v[48:51], v[156:159], v[224:227], v[48:51]\n"
        "s_waitcnt lgkmcnt(2)\n"
        "v_mfma_f32_16x16x32_f16 v[20:23], v[152:155], v[228:231], v[20:23]\n"
        "ds_read_b128 v[224:227], v244 offset:32768\n"
        "v_mfma_f32_16x16x32_f16 v[52:55], v[156:159], v[228:231], v[52:55]\n"
        "s_waitcnt lgkmcnt(2)\n"
        "v_mfma_f32_16x16x32_f16 v[24:27], v[152:155], v[232:235], v[24:27]\n"
        "ds_read_b128 v[228:231], v244 offset:33792\n"
        "v_mfma_f32_16x16x32_f16 v[56:59], v[156:159], v[232:235], v[56:59]\n"
        "s_waitcnt lgkmcnt(2)\n"
        "v_mfma_f32_16x16x32_f16 v[28:31], v[152:155], v[236:239], v[28:31]\n"
        "ds_read_b128 v[232:235], v244 offset:34816\n"
        "v_mfma_f32_16x16x32_f16 v[60:63], v[156:159], v[236:239], v[60:63]\n"
        "s_waitcnt lgkmcnt(2)\n"
        "v_mfma_f32_16x16x32_f16 v[0:3], v[160:163], v[224:227], v[0:3]\n"
        "ds_read_b128 v[236:239], v244 offset:35840\n"
        "v_mfma_f32_16x16x32_f16 v[32:35], v[164:167], v[224:227], v[32:35]\n"
        "s_waitcnt lgkmcnt(2)\n"
        "v_mfma_f32_16x16x32_f16 v[4:7], v[160:163], v[228:231], v[4:7]\n"
        "ds_read_b128 v[224:227], v244 offset:36864\n"
        "v_mfma_f32_16x16x32_f16 v[36:39], v[164:167], v[228:231], v[36:39]\n"
        "s_waitcnt lgkmcnt(2)\n"
        "v_mfma_f32_16x16x32_f16 v[8:11], v[160:163], v[232:235], v[8:11]\n"
        "ds_read_b128 v[228:231], v244 offset:37888\n"
        "v_mfma_f32_16x16x32_f16 v[40:43], v[164:167], v[232:235], v[40:43]\n"
        "s_waitcnt lgkmcnt(2)\n"
        "v_mfma_f32_16x16x32_f16 v[12:15], v[160:163], v[236:239], v[12:15]\n"
        "ds_read_b128 v[232:235], v244 offset:38912\n"
        "v_mfma_f32_16x16x32_f16 v[44:47], v[164:167], v[236:239], v[44:47]\n"
        "s_waitcnt lgkmcnt(2)\n"
        "v_mfma_f32_16x16x32_f16 v[16:19], v[160:163], v[224:227], v[16:19]\n"
        "ds_read_b128 v[236:239], v244 offset:39936\n"
        "v_mfma_f32_16x16x32_f16 v[48:51], v[164:167], v[224:227], v[48:51]\n"
        "s_waitcnt lgkmcnt(2)\n"
        "v_mfma_f32_16x16x32_f16 v[20:23], v[160:163], v[228:231], v[20:23]\n"
        "ds_read_b128 v[224:227], v244 offset:40960\n"
        "v_mfma_f32_16x16x32_f16 v[52:55], v[164:167], v[228:231], v[52:55]\n"
        "s_waitcnt lgkmcnt(2)\n"
        "v_mfma_f32_16x16x32_f16 v[24:27], v[160:163], v[232:235], v[24:27]\n"
        "ds_read_b128 v[228:231], v244 offset:41984\n"
        "v_mfma_f32_16x16x32_f16 v[56:59], v[164:167], v[232:235], v[56:59]\n"
        "s_waitcnt lgkmcnt(2)\n"
        "v_mfma_f32_16x16x32_f16 v[28:31], v[160:163], v[236:239], v[28:31]\n"
        "ds_read_b128 v[232:235], v244 offset:43008\n"
        "v_mfma_f32_16x16x32_f16 v[60:63], v[164:167], v[236:239], v[60:63]\n"
        "s_waitcnt lgkmcnt(2)\n"
        "v_mfma_f32_16x16x32_f16 v[0:3], v[168:171], v[224:227], v[0:3]\n"
        "ds_read_b128 v[236:239], v244 offset:44032\n"
        "v_mfma_f32_16x16x32_f16 v[32:35], v[172:175], v[224:227], v[32:35]\n"
        "s_waitcnt lgkmcnt(2)\n"
        "v_mfma_f32_16x16x32_f16 v[4:7], v[168:171], v[228:231], v[4:7]\n"
        "ds_read_b128 v[224:227], v244 offset:45056\n"
        "v_mfma_f32_16x16x32_f16 v[36:39], v[172:175], v[228:231], v[36:39]\n"
        "s_barrier\n"
        "s_setprio 2\n"
        "s_lshl_b32 s81, s73, 16\n"
        "s_add_u32 s81, s81, s62\n"
        "s_add_u32 s81, s81, 2048\n"
        "s_add_u32 s87, s81, 4096\n"
        "buffer_load_dwordx4 v[144:147], %[avoff], %[rsA], s81 offen offset:0\n"
        "buffer_load_dwordx4 v[148:151], %[avoff], %[rsA], s81 offen offset:1024\n"
        "buffer_load_dwordx4 v[152:155], %[avoff], %[rsA], s87 offen offset:0\n"
        "buffer_load_dwordx4 v[156:159], %[avoff], %[rsA], s87 offen offset:1024\n"
        "s_setprio 0\n"
        "s_waitcnt lgkmcnt(2)\n"
        "v_mfma_f32_16x16x32_f16 v[8:11], v[168:171], v[232:235], v[8:11]\n"
        "ds_read_b128 v[228:231], v244 offset:46080\n"
        "v_mfma_f32_16x16x32_f16 v[40:43], v[172:175], v[232:235], v[40:43]\n"
        "s_waitcnt lgkmcnt(2)\n"
        "v_mfma_f32_16x16x32_f16 v[12:15], v[168:171], v[236:239], v[12:15]\n"
        "ds_read_b128 v[232:235], v244 offset:47104\n"
        "v_mfma_f32_16x16x32_f16 v[44:47], v[172:175], v[236:239], v[44:47]\n"
        "s_waitcnt lgkmcnt(2)\n"
        "v_mfma_f32_16x16x32_f16 v[16:19], v[168:171], v[224:227], v[16:19]\n"
        "ds_read_b128 v[236:239], v244 offset:48128\n"
        "v_mfma_f32_16x16x32_f16 v[48:51], v[172:175], v[224:227], v[48:51]\n"
        "s_waitcnt lgkmcnt(2)\n"
        "v_mfma_f32_16x16x32_f16 v[20:23], v[168:171], v[228:231], v[20:23]\n"
        "ds_read_b128 v[224:227], v244 offset:49152\n"
        "v_mfma_f32_16x16x32_f16 v[52:55], v[172:175], v[228:231], v[52:55]\n"
        "s_waitcnt lgkmcnt(2)\n"
        "v_mfma_f32_16x16x32_f16 v[24:27], v[168:171], v[232:235], v[24:27]\n"
        "ds_read_b128 v[228:231], v244 offset:50176\n"
        "v_mfma_f32_16x16x32_f16 v[56:59], v[172:175], v[232:235], v[56:59]\n"
        "s_waitcnt lgkmcnt(2)\n"
        "v_mfma_f32_16x16x32_f16 v[28:31], v[168:171], v[236:239], v[28:31]\n"
        "ds_read_b128 v[232:235], v244 offset:51200\n"
        "v_mfma_f32_16x16x32_f16 v[60:63], v[172:175], v[236:239], v[60:63]\n"
        "s_waitcnt vmcnt(8) lgkmcnt(2)\n"
        "v_mfma_f32_16x16x32_f16 v[0:3], v[176:179], v[224:227], v[0:3]\n"
        "ds_read_b128 v[236:239], v244 offset:52224\n"
        "v_mfma_f32_16x16x32_f16 v[32:35], v[180:183], v[224:227], v[32:35]\n"
        "s_waitcnt lgkmcnt(2)\n"
        "v_mfma_f32_16x16x32_f16 v[4:7], v[176:179], v[228:231], v[4:7]\n"
        "ds_read_b128 v[224:227], v244 offset:53248\n"
        "v_mfma_f32_16x16x32_f16 v[36:39], v[180:183], v[228:231], v[36:39]\n"
        "s_waitcnt lgkmcnt(2)\n"
        "v_mfma_f32_16x16x32_f16 v[8:11], v[176:179], v[232:235], v[8:11]\n"
        "ds_read_b128 v[228:231], v244 offset:54272\n"
        "v_mfma_f32_16x16x32_f16 v[40:43], v[180:183], v[232:235], v[40:43]\n"
        "s_waitcnt lgkmcnt(2)\n"
        "v_mfma_f32_16x16x32_f16 v[12:15], v[176:179], v[236:239], v[12:15]\n"
        "ds_read_b128 v[232:235], v244 offset:55296\n"
        "v_mfma_f32_16x16x32_f16 v[44:47], v[180:183], v[236:239], v[44:47]\n"
        "s_waitcnt lgkmcnt(2)\n"
        "v_mfma_f32_16x16x32_f16 v[16:19], v[176:179], v[224:227], v[16:19]\n"
        "ds_read_b128 v[236:239], v244 offset:56320\n"
        "v_mfma_f32_16x16x32_f16 v[48:51], v[180:183], v[224:227], v[48:51]\n"
        "s_waitcnt lgkmcnt(2)\n"
        "v_mfma_f32_16x16x32_f16 v[20:23], v[176:179], v[228:231], v[20:23]\n"
        "ds_read_b128 v[224:227], v244 offset:57344\n"
        "v_mfma_f32_16x16x32_f16 v[52:55], v[180:183], v[228:231], v[52:55]\n"
        "s_waitcnt lgkmcnt(2)\n"
        "v_mfma_f32_16x16x32_f16 v[24:27], v[176:179], v[232:235], v[24:27]\n"
        "ds_read_b128 v[228:231], v244 offset:58368\n"
        "v_mfma_f32_16x16x32_f16 v[56:59], v[180:183], v[232:235], v[56:59]\n"
        "s_waitcnt lgkmcnt(2)\n"
        "v_mfma_f32_16x16x32_f16 v[28:31], v[176:179], v[236:239], v[28:31]\n"
        "ds_read_b128 v[232:235], v244 offset:59392\n"
        "v_mfma_f32_16x16x32_f16 v[60:63], v[180:183], v[236:239], v[60:63]\n"
        "s_waitcnt lgkmcnt(2)\n"
        "v_mfma_f32_16x16x32_f16 v[0:3], v[184:187], v[224:227], v[0:3]\n"
        "ds_read_b128 v[236:239], v244 offset:60416\n"
        "v_mfma_f32_16x16x32_f16 v[32:35], v[188:191], v[224:227], v[32:35]\n"
        "s_waitcnt lgkmcnt(2)\n"
        "v_mfma_f32_16x16x32_f16 v[4:7], v[184:187], v[228:231], v[4:7]\n"
        "ds_read_b128 v[224:227], v244 offset:61440\n"
        "v_mfma_f32_16x16x32_f16 v[36:39], v[188:191], v[228:231], v[36:39]\n"
        "s_waitcnt lgkmcnt(2)\n"
        "v_mfma_f32_16x16x32_f16 v[8:11], v[184:187], v[232:235], v[8:11]\n"
        "ds_read_b128 v[228:231], v244 offset:62464\n"
        "v_mfma_f32_16x16x32_f16 v[40:43], v[188:191], v[232:235], v[40:43]\n"
        "s_waitcnt lgkmcnt(2)\n"
        "v_mfma_f32_16x16x32_f16 v[12:15], v[184:187], v[236:239], v[12:15]\n"
        "ds_read_b128 v[232:235], v244 offset:63488\n"
        "v_mfma_f32_16x16x32_f16 v[44:47], v[188:191], v[236:239], v[44:47]\n"
        "s_waitcnt lgkmcnt(2)\n"
        "v_mfma_f32_16x16x32_f16 v[16:19], v[184:187], v[224:227], v[16:19]\n"
        "ds_read_b128 v[236:239], v244 offset:64512\n"
        "v_mfma_f32_16x16x32_f16 v[48:51], v[188:191], v[224:227], v[48:51]\n"
        "s_waitcnt lgkmcnt(2)\n"
        "v_mfma_f32_16x16x32_f16 v[20:23], v[184:187], v[228:231], v[20:23]\n"
        "ds_read_b128 v[224:227], %[brd] offset:0\n"
        "v_mfma_f32_16x16x32_f16 v[52:55], v[188:191], v[228:231], v[52:55]\n"
        "s_waitcnt lgkmcnt(2)\n"
        "v_mfma_f32_16x16x32_f16 v[24:27], v[184:187], v[232:235], v[24:27]\n"
        "ds_read_b128 v[228:231], %[brd] offset:1024\n"
        "v_mfma_f32_16x16x32_f16 v[56:59], v[188:191], v[232:235], v[56:59]\n"
        "s_waitcnt lgkmcnt(2)\n"
        "v_mfma_f32_16x16x32_f16 v[28:31], v[184:187], v[236:239], v[28:31]\n"
        "ds_read_b128 v[232:235], %[brd] offset:2048\n"
        "v_mfma_f32_16x16x32_f16 v[60:63], v[188:191], v[236:239], v[60:63]\n"
        "s_setprio 2\n"
        "s_lshl_b32 s80, s74, 16\n"
        "s_add_u32 s80, s80, s62\n"
        "s_add_u32 s80, s80, 2048\n"
        "s_add_u32 s86, s80, 4096\n"
        "buffer_load_dwordx4 v[160:163], %[avoff], %[rsA], s80 offen offset:0\n"
        "buffer_load_dwordx4 v[164:167], %[avoff], %[rsA], s80 offen offset:1024\n"
        "buffer_load_dwordx4 v[168:171], %[avoff], %[rsA], s86 offen offset:0\n"
        "buffer_load_dwordx4 v[172:175], %[avoff], %[rsA], s86 offen offset:1024\n"
        "s_lshl_b32 s81, s75, 16\n"
        "s_add_u32 s81, s81, s62\n"
        "s_add_u32 s81, s81, 2048\n"
        "s_add_u32 s87, s81, 4096\n"
        "buffer_load_dwordx4 v[176:179], %[avoff], %[rsA], s81 offen offset:0\n"
        "buffer_load_dwordx4 v[180:183], %[avoff], %[rsA], s81 offen offset:1024\n"
        "s_setprio 0\n"
        "s_waitcnt vmcnt(10) lgkmcnt(2)\n"
        "v_mfma_f32_16x16x32_f16 v[64:67], v[128:131], v[224:227], v[64:67]\n"
        "ds_read_b128 v[236:239], %[brd] offset:3072\n"
        "v_mfma_f32_16x16x32_f16 v[96:99], v[132:135], v[224:227], v[96:99]\n"
        "s_waitcnt lgkmcnt(2)\n"
        "v_mfma_f32_16x16x32_f16 v[68:71], v[128:131], v[228:231], v[68:71]\n"
        "ds_read_b128 v[224:227], %[brd] offset:4096\n"
        "v_mfma_f32_16x16x32_f16 v[100:103], v[132:135], v[228:231], v[100:103]\n"
        "s_mov_b32 s88, 0\n"
        "v_max_f32 v192, 0, v0\n"
        "v_max_f32 v193, 0, v4\n"
        "v_max_f32 v194, 0, v8\n"
        "v_max_f32 v195, 0, v12\n"
        "buffer_store_dwordx4 v[192:195], %[ooff], %[rsO], s88 offen offset:0 nt\n"
        "s_waitcnt lgkmcnt(2)\n"
        "v_mfma_f32_16x16x32_f16 v[72:75], v[128:131], v[232:235], v[72:75]\n"
        "ds_read_b128 v[228:231], %[brd] offset:5120\n"
        "v_mfma_f32_16x16x32_f16 v[104:107], v[132:135], v[232:235], v[104:107]\n"
        "s_waitcnt lgkmcnt(2)\n"
        "v_mfma_f32_16x16x32_f16 v[76:79], v[128:131], v[236:239], v[76:79]\n"
        "ds_read_b128 v[232:235], %[brd] offset:6144\n"
        "v_mfma_f32_16x16x32_f16 v[108:111], v[132:135], v[236:239], v[108:111]\n"
        "s_waitcnt lgkmcnt(2)\n"
        "v_mfma_f32_16x16x32_f16 v[80:83], v[128:131], v[224:227], v[80:83]\n"
        "ds_read_b128 v[236:239], %[brd] offset:7168\n"
        "v_mfma_f32_16x16x32_f16 v[112:115], v[132:135], v[224:227], v[112:115]\n"
        "s_waitcnt lgkmcnt(2)\n"
        "v_mfma_f32_16x16x32_f16 v[84:87], v[128:131], v[228:231], v[84:87]\n"
        "ds_read_b128 v[224:227], %[brd] offset:8192\n"
        "v_mfma_f32_16x16x32_f16 v[116:119], v[132:135], v[228:231], v[116:119]\n"
        "s_mov_b32 s89, 0\n"
        "v_max_f32 v196, 0, v16\n"
        "v_max_f32 v197, 0, v20\n"
        "v_max_f32 v198, 0, v24\n"
        "v_max_f32 v199, 0, v28\n"
        "buffer_store_dwordx4 v[196:199], %[ooff], %[rsO], s89 offen offset:256 nt\n"
        "s_waitcnt lgkmcnt(2)\n"
        "v_mfma_f32_16x16x32_f16 v[88:91], v[128:131], v[232:235], v[88:91]\n"
        "ds_read_b128 v[228:231], %[brd] offset:9216\n"
        "v_mfma_f32_16x16x32_f16 v[120:123], v[132:135], v[232:235], v[120:123]\n"
        "s_waitcnt lgkmcnt(2)\n"
        "v_mfma_f32_16x16x32_f16 v[92:95], v[128:131], v[236:239], v[92:95]\n"
        "ds_read_b128 v[232:235], %[brd] offset:10240\n"
        "v_mfma_f32_16x16x32_f16 v[124:127], v[132:135], v[236:239], v[124:127]\n"
        "s_waitcnt lgkmcnt(2)\n"
        "v_mfma_f32_16x16x32_f16 v[64:67], v[136:139], v[224:227], v[64:67]\n"
        "ds_read_b128 v[236:239], %[brd] offset:11264\n"
        "v_mfma_f32_16x16x32_f16 v[96:99], v[140:143], v[224:227], v[96:99]\n"
        "s_waitcnt lgkmcnt(2)\n"
        "v_mfma_f32_16x16x32_f16 v[68:71], v[136:139], v[228:231], v[68:71]\n"
        "ds_read_b128 v[224:227], %[brd] offset:12288\n"
        "v_mfma_f32_16x16x32_f16 v[100:103], v[140:143], v[228:231], v[100:103]\n"
        "s_mov_b32 s88, 131072\n"
        "v_max_f32 v200, 0, v1\n"
        "v_max_f32 v201, 0, v5\n"
        "v_max_f32 v202, 0, v9\n"
        "v_max_f32 v203, 0, v13\n"
        "buffer_store_dwordx4 v[200:203], %[ooff], %[rsO], s88 offen offset:0 nt\n"
        "s_waitcnt lgkmcnt(2)\n"
        "v_mfma_f32_16x16x32_f16 v[72:75], v[136:139], v[232:235], v[72:75]\n"
        "ds_read_b128 v[228:231], %[brd] offset:13312\n"
        "v_mfma_f32_16x16x32_f16 v[104:107], v[140:143], v[232:235], v[104:107]\n"
        "s_waitcnt lgkmcnt(2)\n"
        "v_mfma_f32_16x16x32_f16 v[76:79], v[136:139], v[236:239], v[76:79]\n"
        "ds_read_b128 v[232:235], %[brd] offset:14336\n"
        "v_mfma_f32_16x16x32_f16 v[108:111], v[140:143], v[236:239], v[108:111]\n"
        "s_waitcnt lgkmcnt(2)\n"
        "v_mfma_f32_16x16x32_f16 v[80:83], v[136:139], v[224:227], v[80:83]\n"
        "ds_read_b128 v[236:239], %[brd] offset:15360\n"
        "v_mfma_f32_16x16x32_f16 v[112:115], v[140:143], v[224:227], v[112:115]\n"
        "s_waitcnt lgkmcnt(2)\n"
        "v_mfma_f32_16x16x32_f16 v[84:87], v[136:139], v[228:231], v[84:87]\n"
        "ds_read_b128 v[224:227], %[brd] offset:16384\n"
        "v_mfma_f32_16x16x32_f16 v[116:119], v[140:143], v[228:231], v[116:119]\n"
        "s_mov_b32 s89, 131072\n"
        "v_max_f32 v204, 0, v17\n"
        "v_max_f32 v205, 0, v21\n"
        "v_max_f32 v206, 0, v25\n"
        "v_max_f32 v207, 0, v29\n"
        "buffer_store_dwordx4 v[204:207], %[ooff], %[rsO], s89 offen offset:256 nt\n"
        "s_waitcnt lgkmcnt(2)\n"
        "v_mfma_f32_16x16x32_f16 v[88:91], v[136:139], v[232:235], v[88:91]\n"
        "ds_read_b128 v[228:231], %[brd] offset:17408\n"
        "v_mfma_f32_16x16x32_f16 v[120:123], v[140:143], v[232:235], v[120:123]\n"
        "s_waitcnt lgkmcnt(2)\n"
        "v_mfma_f32_16x16x32_f16 v[92:95], v[136:139], v[236:239], v[92:95]\n"
        "ds_read_b128 v[232:235], %[brd] offset:18432\n"
        "v_mfma_f32_16x16x32_f16 v[124:127], v[140:143], v[236:239], v[124:127]\n"
        "s_waitcnt vmcnt(10) lgkmcnt(2)\n"
        "v_mfma_f32_16x16x32_f16 v[64:67], v[144:147], v[224:227], v[64:67]\n"
        "ds_read_b128 v[236:239], %[brd] offset:19456\n"
        "v_mfma_f32_16x16x32_f16 v[96:99], v[148:151], v[224:227], v[96:99]\n"
        "s_waitcnt lgkmcnt(2)\n"
        "v_mfma_f32_16x16x32_f16 v[68:71], v[144:147], v[228:231], v[68:71]\n"
        "ds_read_b128 v[224:227], %[brd] offset:20480\n"
        "v_mfma_f32_16x16x32_f16 v[100:103], v[148:151], v[228:231], v[100:103]\n"
        "s_mov_b32 s88, 262144\n"
        "v_max_f32 v208, 0, v2\n"
        "v_max_f32 v209, 0, v6\n"
        "v_max_f32 v210, 0, v10\n"
        "v_max_f32 v211, 0, v14\n"
        "buffer_store_dwordx4 v[208:211], %[ooff], %[rsO], s88 offen offset:0 nt\n"
        "s_waitcnt lgkmcnt(2)\n"
        "v_mfma_f32_16x16x32_f16 v[72:75], v[144:147], v[232:235], v[72:75]\n"
        "ds_read_b128 v[228:231], %[brd] offset:21504\n"
        "v_mfma_f32_16x16x32_f16 v[104:107], v[148:151], v[232:235], v[104:107]\n"
        "s_waitcnt lgkmcnt(2)\n"
        "v_mfma_f32_16x16x32_f16 v[76:79], v[144:147], v[236:239], v[76:79]\n"
        "ds_read_b128 v[232:235], %[brd] offset:22528\n"
        "v_mfma_f32_16x16x32_f16 v[108:111], v[148:151], v[236:239], v[108:111]\n"
        "s_waitcnt lgkmcnt(2)\n"
        "v_mfma_f32_16x16x32_f16 v[80:83], v[144:147], v[224:227], v[80:83]\n"
        "ds_read_b128 v[236:239], %[brd] offset:23552\n"
        "v_mfma_f32_16x16x32_f16 v[112:115], v[148:151], v[224:227], v[112:115]\n"
        "s_waitcnt lgkmcnt(2)\n"
        "v_mfma_f32_16x16x32_f16 v[84:87], v[144:147], v[228:231], v[84:87]\n"
        "ds_read_b128 v[224:227], %[brd] offset:24576\n"
        "v_mfma_f32_16x16x32_f16 v[116:119], v[148:151], v[228:231], v[116:119]\n"
        "s_setprio 2\n"
        "buffer_load_dwordx4 v[184:187], %[avoff], %[rsA], s87 offen offset:0\n"
        "buffer_load_dwordx4 v[188:191], %[avoff], %[rsA], s87 offen offset:1024\n"
        "s_lshl_b32 s80, s76, 16\n"
        "s_add_u32 s80, s80, s62\n"
        "s_add_u32 s80, s80, 2048\n"
        "s_add_u32 s86, s80, 4096\n"
        "buffer_load_dwordx4 v[128:131], %[avoff], %[rsA], s80 offen offset:0\n"
        "buffer_load_dwordx4 v[132:135], %[avoff], %[rsA], s80 offen offset:1024\n"
        "buffer_load_dwordx4 v[136:139], %[avoff], %[rsA], s86 offen offset:0\n"
        "buffer_load_dwordx4 v[140:143], %[avoff], %[rsA], s86 offen offset:1024\n"
        "s_setprio 0\n"
        "s_mov_b32 s89, 262144\n"
        "v_max_f32 v212, 0, v18\n"
        "v_max_f32 v213, 0, v22\n"
        "v_max_f32 v214, 0, v26\n"
        "v_max_f32 v215, 0, v30\n"
        "buffer_store_dwordx4 v[212:215], %[ooff], %[rsO], s89 offen offset:256 nt\n"
        "s_waitcnt lgkmcnt(2)\n"
        "v_mfma_f32_16x16x32_f16 v[88:91], v[144:147], v[232:235], v[88:91]\n"
        "ds_read_b128 v[228:231], %[brd] offset:25600\n"
        "v_mfma_f32_16x16x32_f16 v[120:123], v[148:151], v[232:235], v[120:123]\n"
        "s_waitcnt lgkmcnt(2)\n"
        "v_mfma_f32_16x16x32_f16 v[92:95], v[144:147], v[236:239], v[92:95]\n"
        "ds_read_b128 v[232:235], %[brd] offset:26624\n"
        "v_mfma_f32_16x16x32_f16 v[124:127], v[148:151], v[236:239], v[124:127]\n"
        "s_waitcnt lgkmcnt(2)\n"
        "v_mfma_f32_16x16x32_f16 v[64:67], v[152:155], v[224:227], v[64:67]\n"
        "ds_read_b128 v[236:239], %[brd] offset:27648\n"
        "v_mfma_f32_16x16x32_f16 v[96:99], v[156:159], v[224:227], v[96:99]\n"
        "s_waitcnt lgkmcnt(2)\n"
        "v_mfma_f32_16x16x32_f16 v[68:71], v[152:155], v[228:231], v[68:71]\n"
        "ds_read_b128 v[224:227], %[brd] offset:28672\n"
        "v_mfma_f32_16x16x32_f16 v[100:103], v[156:159], v[228:231], v[100:103]\n"
        "s_mov_b32 s88, 393216\n"
        "v_max_f32 v216, 0, v3\n"
        "v_max_f32 v217, 0, v7\n"
        "v_max_f32 v218, 0, v11\n"
        "v_max_f32 v219, 0, v15\n"
        "buffer_store_dwordx4 v[216:219], %[ooff], %[rsO], s88 offen offset:0 nt\n"
        "s_waitcnt lgkmcnt(2)\n"
        "v_mfma_f32_16x16x32_f16 v[72:75], v[152:155], v[232:235], v[72:75]\n"
        "ds_read_b128 v[228:231], %[brd] offset:29696\n"
        "v_mfma_f32_16x16x32_f16 v[104:107], v[156:159], v[232:235], v[104:107]\n"
        "s_waitcnt lgkmcnt(2)\n"
        "v_mfma_f32_16x16x32_f16 v[76:79], v[152:155], v[236:239], v[76:79]\n"
        "ds_read_b128 v[232:235], %[brd] offset:30720\n"
        "v_mfma_f32_16x16x32_f16 v[108:111], v[156:159], v[236:239], v[108:111]\n"
        "s_waitcnt lgkmcnt(2)\n"
        "v_mfma_f32_16x16x32_f16 v[80:83], v[152:155], v[224:227], v[80:83]\n"
        "ds_read_b128 v[236:239], %[brd] offset:31744\n"
        "v_mfma_f32_16x16x32_f16 v[112:115], v[156:159], v[224:227], v[112:115]\n"
        "s_waitcnt lgkmcnt(2)\n"
        "v_mfma_f32_16x16x32_f16 v[84:87], v[152:155], v[228:231], v[84:87]\n"
        "ds_read_b128 v[224:227], %[brd] offset:32768\n"
        "v_mfma_f32_16x16x32_f16 v[116:119], v[156:159], v[228:231], v[116:119]\n"
        "s_mov_b32 s89, 393216\n"
        "v_max_f32 v220, 0, v19\n"
        "v_max_f32 v221, 0, v23\n"
        "v_max_f32 v222, 0, v27\n"
        "v_max_f32 v223, 0, v31\n"
        "buffer_store_dwordx4 v[220:223], %[ooff], %[rsO], s89 offen offset:256 nt\n"
        "s_waitcnt lgkmcnt(2)\n"
        "v_mfma_f32_16x16x32_f16 v[88:91], v[152:155], v[232:235], v[88:91]\n"
        "ds_read_b128 v[228:231], %[brd] offset:33792\n"
        "v_mfma_f32_16x16x32_f16 v[120:123], v[156:159], v[232:235], v[120:123]\n"
        "s_waitcnt lgkmcnt(2)\n"
        "v_mfma_f32_16x16x32_f16 v[92:95], v[152:155], v[236:239], v[92:95]\n"
        "ds_read_b128 v[232:235], %[brd] offset:34816\n"
        "v_mfma_f32_16x16x32_f16 v[124:127], v[156:159], v[236:239], v[124:127]\n"
        "s_waitcnt vmcnt(16) lgkmcnt(2)\n"
        "v_mfma_f32_16x16x32_f16 v[64:67], v[160:163], v[224:227], v[64:67]\n"
        "ds_read_b128 v[236:239], %[brd] offset:35840\n"
        "v_mfma_f32_16x16x32_f16 v[96:99], v[164:167], v[224:227], v[96:99]\n"
        "s_waitcnt lgkmcnt(2)\n"
        "v_mfma_f32_16x16x32_f16 v[68:71], v[160:163], v[228:231], v[68:71]\n"
        "ds_read_b128 v[224:227], %[brd] offset:36864\n"
        "v_mfma_f32_16x16x32_f16 v[100:103], v[164:167], v[228:231], v[100:103]\n"
        "s_mov_b32 s88, 2097152\n"
        "v_max_f32 v192, 0, v32\n"
        "v_max_f32 v193, 0, v36\n"
        "v_max_f32 v194, 0, v40\n"
        "v_max_f32 v195, 0, v44\n"
        "buffer_store_dwordx4 v[192:195], %[ooff], %[rsO], s88 offen offset:0 nt\n"
        "s_waitcnt lgkmcnt(2)\n"
        "v_mfma_f32_16x16x32_f16 v[72:75], v[160:163], v[232:235], v[72:75]\n"
        "ds_read_b128 v[228:231], %[brd] offset:37888\n"
        "v_mfma_f32_16x16x32_f16 v[104:107], v[164:167], v[232:235], v[104:107]\n"
        "s_waitcnt lgkmcnt(2)\n"
        "v_mfma_f32_16x16x32_f16 v[76:79], v[160:163], v[236:239], v[76:79]\n"
        "ds_read_b128 v[232:235], %[brd] offset:38912\n"
        "v_mfma_f32_16x16x32_f16 v[108:111], v[164:167], v[236:239], v[108:111]\n"
        "s_waitcnt lgkmcnt(2)\n"
        "v_mfma_f32_16x16x32_f16 v[80:83], v[160:163], v[224:227], v[80:83]\n"
        "ds_read_b128 v[236:239], %[brd] offset:39936\n"
        "v_mfma_f32_16x16x32_f16 v[112:115], v[164:167], v[224:227], v[112:115]\n"
        "s_waitcnt lgkmcnt(2)\n"
        "v_mfma_f32_16x16x32_f16 v[84:87], v[160:163], v[228:231], v[84:87]\n"
        "ds_read_b128 v[224:227], %[brd] offset:40960\n"
        "v_mfma_f32_16x16x32_f16 v[116:119], v[164:167], v[228:231], v[116:119]\n"
        "s_mov_b32 s89, 2097152\n"
        "v_max_f32 v196, 0, v48\n"
        "v_max_f32 v197, 0, v52\n"
        "v_max_f32 v198, 0, v56\n"
        "v_max_f32 v199, 0, v60\n"
        "buffer_store_dwordx4 v[196:199], %[ooff], %[rsO], s89 offen offset:256 nt\n"
        "s_waitcnt lgkmcnt(2)\n"
        "v_mfma_f32_16x16x32_f16 v[88:91], v[160:163], v[232:235], v[88:91]\n"
        "ds_read_b128 v[228:231], %[brd] offset:41984\n"
        "v_mfma_f32_16x16x32_f16 v[120:123], v[164:167], v[232:235], v[120:123]\n"
        "s_waitcnt lgkmcnt(2)\n"
        "v_mfma_f32_16x16x32_f16 v[92:95], v[160:163], v[236:239], v[92:95]\n"
        "ds_read_b128 v[232:235], %[brd] offset:43008\n"
        "v_mfma_f32_16x16x32_f16 v[124:127], v[164:167], v[236:239], v[124:127]\n"
        "s_waitcnt lgkmcnt(2)\n"
        "v_mfma_f32_16x16x32_f16 v[64:67], v[168:171], v[224:227], v[64:67]\n"
        "ds_read_b128 v[236:239], %[brd] offset:44032\n"
        "v_mfma_f32_16x16x32_f16 v[96:99], v[172:175], v[224:227], v[96:99]\n"
        "s_waitcnt lgkmcnt(2)\n"
        "v_mfma_f32_16x16x32_f16 v[68:71], v[168:171], v[228:231], v[68:71]\n"
        "ds_read_b128 v[224:227], %[brd] offset:45056\n"
        "v_mfma_f32_16x16x32_f16 v[100:103], v[172:175], v[228:231], v[100:103]\n"
        "s_setprio 2\n"
        "s_lshl_b32 s81, s77, 16\n"
        "s_add_u32 s81, s81, s62\n"
        "s_add_u32 s81, s81, 2048\n"
        "s_add_u32 s87, s81, 4096\n"
        "buffer_load_dwordx4 v[144:147], %[avoff], %[rsA], s81 offen offset:0\n"
        "buffer_load_dwordx4 v[148:151], %[avoff], %[rsA], s81 offen offset:1024\n"
        "buffer_load_dwordx4 v[152:155], %[avoff], %[rsA], s87 offen offset:0\n"
        "buffer_load_dwordx4 v[156:159], %[avoff], %[rsA], s87 offen offset:1024\n"
        "s_setprio 0\n"
        "s_mov_b32 s88, 2228224\n"
        "v_max_f32 v200, 0, v33\n"
        "v_max_f32 v201, 0, v37\n"
        "v_max_f32 v202, 0, v41\n"
        "v_max_f32 v203, 0, v45\n"
        "buffer_store_dwordx4 v[200:203], %[ooff], %[rsO], s88 offen offset:0 nt\n"
        "s_waitcnt lgkmcnt(2)\n"
        "v_mfma_f32_16x16x32_f16 v[72:75], v[168:171], v[232:235], v[72:75]\n"
        "ds_read_b128 v[228:231], %[brd] offset:46080\n"
        "v_mfma_f32_16x16x32_f16 v[104:107], v[172:175], v[232:235], v[104:107]\n"
        "s_waitcnt lgkmcnt(2)\n"
        "v_mfma_f32_16x16x32_f16 v[76:79], v[168:171], v[236:239], v[76:79]\n"
        "ds_read_b128 v[232:235], %[brd] offset:47104\n"
        "v_mfma_f32_16x16x32_f16 v[108:111], v[172:175], v[236:239], v[108:111]\n"
        "s_waitcnt lgkmcnt(2)\n"
        "v_mfma_f32_16x16x32_f16 v[80:83], v[168:171], v[224:227], v[80:83]\n"
        "ds_read_b128 v[236:239], %[brd] offset:48128\n"
        "v_mfma_f32_16x16x32_f16 v[112:115], v[172:175], v[224:227], v[112:115]\n"
        "s_waitcnt lgkmcnt(2)\n"
        "v_mfma_f32_16x16x32_f16 v[84:87], v[168:171], v[228:231], v[84:87]\n"
        "ds_read_b128 v[224:227], %[brd] offset:49152\n"
        "v_mfma_f32_16x16x32_f16 v[116:119], v[172:175], v[228:231], v[116:119]\n"
        "s_mov_b32 s89, 2228224\n"
        "v_max_f32 v204, 0, v49\n"
        "v_max_f32 v205, 0, v53\n"
        "v_max_f32 v206, 0, v57\n"
        "v_max_f32 v207, 0, v61\n"
        "buffer_store_dwordx4 v[204:207], %[ooff], %[rsO], s89 offen offset:256 nt\n"
        "s_waitcnt lgkmcnt(2)\n"
        "v_mfma_f32_16x16x32_f16 v[88:91], v[168:171], v[232:235], v[88:91]\n"
        "ds_read_b128 v[228:231], %[brd] offset:50176\n"
        "v_mfma_f32_16x16x32_f16 v[120:123], v[172:175], v[232:235], v[120:123]\n"
        "s_waitcnt lgkmcnt(2)\n"
        "v_mfma_f32_16x16x32_f16 v[92:95], v[168:171], v[236:239], v[92:95]\n"
        "ds_read_b128 v[232:235], %[brd] offset:51200\n"
        "v_mfma_f32_16x16x32_f16 v[124:127], v[172:175], v[236:239], v[124:127]\n"
        "s_waitcnt vmcnt(15) lgkmcnt(2)\n"
        "v_mfma_f32_16x16x32_f16 v[64:67], v[176:179], v[224:227], v[64:67]\n"
        "ds_read_b128 v[236:239], %[brd] offset:52224\n"
        "v_mfma_f32_16x16x32_f16 v[96:99], v[180:183], v[224:227], v[96:99]\n"
        "s_waitcnt lgkmcnt(2)\n"
        "v_mfma_f32_16x16x32_f16 v[68:71], v[176:179], v[228:231], v[68:71]\n"
        "ds_read_b128 v[224:227], %[brd] offset:53248\n"
        "v_mfma_f32_16x16x32_f16 v[100:103], v[180:183], v[228:231], v[100:103]\n"
        "s_mov_b32 s88, 2359296\n"
        "v_max_f32 v208, 0, v34\n"
        "v_max_f32 v209, 0, v38\n"
        "v_max_f32 v210, 0, v42\n"
        "v_max_f32 v211, 0, v46\n"
        "buffer_store_dwordx4 v[208:211], %[ooff], %[rsO], s88 offen offset:0 nt\n"
        "s_waitcnt lgkmcnt(2)\n"
        "v_mfma_f32_16x16x32_f16 v[72:75], v[176:179], v[232:235], v[72:75]\n"
        "ds_read_b128 v[228:231], %[brd] offset:54272\n"
        "v_mfma_f32_16x16x32_f16 v[104:107], v[180:183], v[232:235], v[104:107]\n"
        "s_waitcnt lgkmcnt(2)\n"
        "v_mfma_f32_16x16x32_f16 v[76:79], v[176:179], v[236:239], v[76:79]\n"
        "ds_read_b128 v[232:235], %[brd] offset:55296\n"
        "v_mfma_f32_16x16x32_f16 v[108:111], v[180:183], v[236:239], v[108:111]\n"
        "s_waitcnt lgkmcnt(2)\n"
        "v_mfma_f32_16x16x32_f16 v[80:83], v[176:179], v[224:227], v[80:83]\n"
        "ds_read_b128 v[236:239], %[brd] offset:56320\n"
        "v_mfma_f32_16x16x32_f16 v[112:115], v[180:183], v[224:227], v[112:115]\n"
        "s_waitcnt lgkmcnt(2)\n"
        "v_mfma_f32_16x16x32_f16 v[84:87], v[176:179], v[228:231], v[84:87]\n"
        "ds_read_b128 v[224:227], %[brd] offset:57344\n"
        "v_mfma_f32_16x16x32_f16 v[116:119], v[180:183], v[228:231], v[116:119]\n"
        "s_mov_b32 s89, 2359296\n"
        "v_max_f32 v212, 0, v50\n"
        "v_max_f32 v213, 0, v54\n"
        "v_max_f32 v214, 0, v58\n"
        "v_max_f32 v215, 0, v62\n"
        "buffer_store_dwordx4 v[212:215], %[ooff], %[rsO], s89 offen offset:256 nt\n"
        "s_waitcnt lgkmcnt(2)\n"
        "v_mfma_f32_16x16x32_f16 v[88:91], v[176:179], v[232:235], v[88:91]\n"
        "ds_read_b128 v[228:231], %[brd] offset:58368\n"
        "v_mfma_f32_16x16x32_f16 v[120:123], v[180:183], v[232:235], v[120:123]\n"
        "s_waitcnt lgkmcnt(2)\n"
        "v_mfma_f32_16x16x32_f16 v[92:95], v[176:179], v[236:239], v[92:95]\n"
        "ds_read_b128 v[232:235], %[brd] offset:59392\n"
        "v_mfma_f32_16x16x32_f16 v[124:127], v[180:183], v[236:239], v[124:127]\n"
        "s_waitcnt lgkmcnt(2)\n"
        "v_mfma_f32_16x16x32_f16 v[64:67], v[184:187], v[224:227], v[64:67]\n"
        "ds_read_b128 v[236:239], %[brd] offset:60416\n"
        "v_mfma_f32_16x16x32_f16 v[96:99], v[188:191], v[224:227], v[96:99]\n"
        "s_waitcnt lgkmcnt(2)\n"
        "v_mfma_f32_16x16x32_f16 v[68:71], v[184:187], v[228:231], v[68:71]\n"
        "ds_read_b128 v[224:227], %[brd] offset:61440\n"
        "v_mfma_f32_16x16x32_f16 v[100:103], v[188:191], v[228:231], v[100:103]\n"
        "s_mov_b32 s88, 2490368\n"
        "v_max_f32 v216, 0, v35\n"
        "v_max_f32 v217, 0, v39\n"
        "v_max_f32 v218, 0, v43\n"
        "v_max_f32 v219, 0, v47\n"
        "buffer_store_dwordx4 v[216:219], %[ooff], %[rsO], s88 offen offset:0 nt\n"
        "s_waitcnt lgkmcnt(2)\n"
        "v_mfma_f32_16x16x32_f16 v[72:75], v[184:187], v[232:235], v[72:75]\n"
        "ds_read_b128 v[228:231], %[brd] offset:62464\n"
        "v_mfma_f32_16x16x32_f16 v[104:107], v[188:191], v[232:235], v[104:107]\n"
        "s_waitcnt lgkmcnt(2)\n"
        "v_mfma_f32_16x16x32_f16 v[76:79], v[184:187], v[236:239], v[76:79]\n"
        "ds_read_b128 v[232:235], %[brd] offset:63488\n"
        "v_mfma_f32_16x16x32_f16 v[108:111], v[188:191], v[236:239], v[108:111]\n"
        "s_waitcnt lgkmcnt(2)\n"
        "v_mfma_f32_16x16x32_f16 v[80:83], v[184:187], v[224:227], v[80:83]\n"
        "ds_read_b128 v[236:239], %[brd] offset:64512\n"
        "v_mfma_f32_16x16x32_f16 v[112:115], v[188:191], v[224:227], v[112:115]\n"
        "s_waitcnt lgkmcnt(2)\n"
        "v_mfma_f32_16x16x32_f16 v[84:87], v[184:187], v[228:231], v[84:87]\n"
        "ds_read_b128 v[224:227], v244 offset:0\n"
        "v_mfma_f32_16x16x32_f16 v[116:119], v[188:191], v[228:231], v[116:119]\n"
        "s_mov_b32 s89, 2490368\n"
        "v_max_f32 v220, 0, v51\n"
        "v_max_f32 v221, 0, v55\n"
        "v_max_f32 v222, 0, v59\n"
        "v_max_f32 v223, 0, v63\n"
        "buffer_store_dwordx4 v[220:223], %[ooff], %[rsO], s89 offen offset:256 nt\n"
        "s_waitcnt lgkmcnt(2)\n"
        "v_mfma_f32_16x16x32_f16 v[88:91], v[184:187], v[232:235], v[88:91]\n"
        "ds_read_b128 v[228:231], v244 offset:1024\n"
        "v_mfma_f32_16x16x32_f16 v[120:123], v[188:191], v[232:235], v[120:123]\n"
        "s_waitcnt lgkmcnt(2)\n"
        "v_mfma_f32_16x16x32_f16 v[92:95], v[184:187], v[236:239], v[92:95]\n"
        "ds_read_b128 v[232:235], v244 offset:2048\n"
        "v_mfma_f32_16x16x32_f16 v[124:127], v[188:191], v[236:239], v[124:127]\n"
        "s_setprio 2\n"
        "s_lshl_b32 s80, s78, 16\n"
        "s_add_u32 s80, s80, s62\n"
        "s_add_u32 s80, s80, 2048\n"
        "s_add_u32 s86, s80, 4096\n"
        "buffer_load_dwordx4 v[160:163], %[avoff], %[rsA], s80 offen offset:0\n"
        "buffer_load_dwordx4 v[164:167], %[avoff], %[rsA], s80 offen offset:1024\n"
        "buffer_load_dwordx4 v[168:171], %[avoff], %[rsA], s86 offen offset:0\n"
        "buffer_load_dwordx4 v[172:175], %[avoff], %[rsA], s86 offen offset:1024\n"
        "s_lshl_b32 s81, s79, 16\n"
        "s_add_u32 s81, s81, s62\n"
        "s_add_u32 s81, s81, 2048\n"
        "s_add_u32 s87, s81, 4096\n"
        "buffer_load_dwordx4 v[176:179], %[avoff], %[rsA], s81 offen offset:0\n"
        "buffer_load_dwordx4 v[180:183], %[avoff], %[rsA], s81 offen offset:1024\n"
        "s_setprio 0\n"
        "s_waitcnt vmcnt(21) lgkmcnt(2)\n"
        "v_mfma_f32_16x16x32_f16 v[64:67], v[128:131], v[224:227], v[64:67]\n"
        "ds_read_b128 v[236:239], v244 offset:3072\n"
        "v_mfma_f32_16x16x32_f16 v[96:99], v[132:135], v[224:227], v[96:99]\n"
        "s_waitcnt lgkmcnt(2)\n"
        "v_mfma_f32_16x16x32_f16 v[68:71], v[128:131], v[228:231], v[68:71]\n"
        "ds_read_b128 v[224:227], v244 offset:4096\n"
        "v_mfma_f32_16x16x32_f16 v[100:103], v[132:135], v[228:231], v[100:103]\n"
        "s_waitcnt lgkmcnt(2)\n"
        "v_mfma_f32_16x16x32_f16 v[72:75], v[128:131], v[232:235], v[72:75]\n"
        "ds_read_b128 v[228:231], v244 offset:5120\n"
        "v_mfma_f32_16x16x32_f16 v[104:107], v[132:135], v[232:235], v[104:107]\n"
        "s_waitcnt lgkmcnt(2)\n"
        "v_mfma_f32_16x16x32_f16 v[76:79], v[128:131], v[236:239], v[76:79]\n"
        "ds_read_b128 v[232:235], v244 offset:6144\n"
        "v_mfma_f32_16x16x32_f16 v[108:111], v[132:135], v[236:239], v[108:111]\n"
        "s_waitcnt lgkmcnt(2)\n"
        "v_mfma_f32_16x16x32_f16 v[80:83], v[128:131], v[224:227], v[80:83]\n"
        "ds_read_b128 v[236:239], v244 offset:7168\n"
        "v_mfma_f32_16x16x32_f16 v[112:115], v[132:135], v[224:227], v[112:115]\n"
        "s_waitcnt lgkmcnt(2)\n"
        "v_mfma_f32_16x16x32_f16 v[84:87], v[128:131], v[228:231], v[84:87]\n"
        "ds_read_b128 v[224:227], v244 offset:8192\n"
        "v_mfma_f32_16x16x32_f16 v[116:119], v[132:135], v[228:231], v[116:119]\n"
        "s_waitcnt lgkmcnt(2)\n"
        "v_mfma_f32_16x16x32_f16 v[88:91], v[128:131], v[232:235], v[88:91]\n"
        "ds_read_b128 v[228:231], v244 offset:9216\n"
        "v_mfma_f32_16x16x32_f16 v[120:123], v[132:135], v[232:235], v[120:123]\n"
        "s_waitcnt lgkmcnt(2)\n"
        "v_mfma_f32_16x16x32_f16 v[92:95], v[128:131], v[236:239], v[92:95]\n"
        "ds_read_b128 v[232:235], v244 offset:10240\n"
        "v_mfma_f32_16x16x32_f16 v[124:127], v[132:135], v[236:239], v[124:127]\n"
        "s_waitcnt lgkmcnt(2)\n"
        "v_mfma_f32_16x16x32_f16 v[64:67], v[136:139], v[224:227], v[64:67]\n"
        "ds_read_b128 v[236:239], v244 offset:11264\n"
        "v_mfma_f32_16x16x32_f16 v[96:99], v[140:143], v[224:227], v[96:99]\n"
        "s_waitcnt lgkmcnt(2)\n"
        "v_mfma_f32_16x16x32_f16 v[68:71], v[136:139], v[228:231], v[68:71]\n"
        "ds_read_b128 v[224:227], v244 offset:12288\n"
        "v_mfma_f32_16x16x32_f16 v[100:103], v[140:143], v[228:231], v[100:103]\n"
        "s_waitcnt lgkmcnt(2)\n"
        "v_mfma_f32_16x16x32_f16 v[72:75], v[136:139], v[232:235], v[72:75]\n"
        "ds_read_b128 v[228:231], v244 offset:13312\n"
        "v_mfma_f32_16x16x32_f16 v[104:107], v[140:143], v[232:235], v[104:107]\n"
        "s_waitcnt lgkmcnt(2)\n"
        "v_mfma_f32_16x16x32_f16 v[76:79], v[136:139], v[236:239], v[76:79]\n"
        "ds_read_b128 v[232:235], v244 offset:14336\n"
        "v_mfma_f32_16x16x32_f16 v[108:111], v[140:143], v[236:239], v[108:111]\n"
        "s_waitcnt lgkmcnt(2)\n"
        "v_mfma_f32_16x16x32_f16 v[80:83], v[136:139], v[224:227], v[80:83]\n"
        "ds_read_b128 v[236:239], v244 offset:15360\n"
        "v_mfma_f32_16x16x32_f16 v[112:115], v[140:143], v[224:227], v[112:115]\n"
        "s_waitcnt lgkmcnt(2)\n"
        "v_mfma_f32_16x16x32_f16 v[84:87], v[136:139], v[228:231], v[84:87]\n"
        "ds_read_b128 v[224:227], v244 offset:16384\n"
        "v_mfma_f32_16x16x32_f16 v[116:119], v[140:143], v[228:231], v[116:119]\n"
        "s_waitcnt lgkmcnt(2)\n"
        "v_mfma_f32_16x16x32_f16 v[88:91], v[136:139], v[232:235], v[88:91]\n"
        "ds_read_b128 v[228:231], v244 offset:17408\n"
        "v_mfma_f32_16x16x32_f16 v[120:123], v[140:143], v[232:235], v[120:123]\n"
        "s_waitcnt lgkmcnt(2)\n"
        "v_mfma_f32_16x16x32_f16 v[92:95], v[136:139], v[236:239], v[92:95]\n"
        "ds_read_b128 v[232:235], v244 offset:18432\n"
        "v_mfma_f32_16x16x32_f16 v[124:127], v[140:143], v[236:239], v[124:127]\n"
        "s_waitcnt vmcnt(12) lgkmcnt(2)\n"
        "v_mfma_f32_16x16x32_f16 v[64:67], v[144:147], v[224:227], v[64:67]\n"
        "ds_read_b128 v[236:239], v244 offset:19456\n"
        "v_mfma_f32_16x16x32_f16 v[96:99], v[148:151], v[224:227], v[96:99]\n"
        "s_waitcnt lgkmcnt(2)\n"
        "v_mfma_f32_16x16x32_f16 v[68:71], v[144:147], v[228:231], v[68:71]\n"
        "ds_read_b128 v[224:227], v244 offset:20480\n"
        "v_mfma_f32_16x16x32_f16 v[100:103], v[148:151], v[228:231], v[100:103]\n"
        "s_waitcnt lgkmcnt(2)\n"
        "v_mfma_f32_16x16x32_f16 v[72:75], v[144:147], v[232:235], v[72:75]\n"
        "ds_read_b128 v[228:231], v244 offset:21504\n"
        "v_mfma_f32_16x16x32_f16 v[104:107], v[148:151], v[232:235], v[104:107]\n"
        "s_waitcnt lgkmcnt(2)\n"
        "v_mfma_f32_16x16x32_f16 v[76:79], v[144:147], v[236:239], v[76:79]\n"
        "ds_read_b128 v[232:235], v244 offset:22528\n"
        "v_mfma_f32_16x16x32_f16 v[108:111], v[148:151], v[236:239], v[108:111]\n"
        "s_waitcnt lgkmcnt(2)\n"
        "v_mfma_f32_16x16x32_f16 v[80:83], v[144:147], v[224:227], v[80:83]\n"
        "ds_read_b128 v[236:239], v244 offset:23552\n"
        "v_mfma_f32_16x16x32_f16 v[112:115], v[148:151], v[224:227], v[112:115]\n"
        "s_waitcnt lgkmcnt(2)\n"
        "v_mfma_f32_16x16x32_f16 v[84:87], v[144:147], v[228:231], v[84:87]\n"
        "ds_read_b128 v[224:227], v244 offset:24576\n"
        "v_mfma_f32_16x16x32_f16 v[116:119], v[148:151], v[228:231], v[116:119]\n"
        "s_setprio 2\n"
        "buffer_load_dwordx4 v[184:187], %[avoff], %[rsA], s87 offen offset:0\n"
        "buffer_load_dwordx4 v[188:191], %[avoff], %[rsA], s87 offen offset:1024\n"
        "s_setprio 0\n"
        "s_waitcnt lgkmcnt(2)\n"
        "v_mfma_f32_16x16x32_f16 v[88:91], v[144:147], v[232:235], v[88:91]\n"
        "ds_read_b128 v[228:231], v244 offset:25600\n"
        "v_mfma_f32_16x16x32_f16 v[120:123], v[148:151], v[232:235], v[120:123]\n"
        "s_waitcnt lgkmcnt(2)\n"
        "v_mfma_f32_16x16x32_f16 v[92:95], v[144:147], v[236:239], v[92:95]\n"
        "ds_read_b128 v[232:235], v244 offset:26624\n"
        "v_mfma_f32_16x16x32_f16 v[124:127], v[148:151], v[236:239], v[124:127]\n"
        "s_waitcnt lgkmcnt(2)\n"
        "v_mfma_f32_16x16x32_f16 v[64:67], v[152:155], v[224:227], v[64:67]\n"
        "ds_read_b128 v[236:239], v244 offset:27648\n"
        "v_mfma_f32_16x16x32_f16 v[96:99], v[156:159], v[224:227], v[96:99]\n"
        "s_waitcnt lgkmcnt(2)\n"
        "v_mfma_f32_16x16x32_f16 v[68:71], v[152:155], v[228:231], v[68:71]\n"
        "ds_read_b128 v[224:227], v244 offset:28672\n"
        "v_mfma_f32_16x16x32_f16 v[100:103], v[156:159], v[228:231], v[100:103]\n"
        "s_waitcnt lgkmcnt(2)\n"
        "v_mfma_f32_16x16x32_f16 v[72:75], v[152:155], v[232:235], v[72:75]\n"
        "ds_read_b128 v[228:231], v244 offset:29696\n"
        "v_mfma_f32_16x16x32_f16 v[104:107], v[156:159], v[232:235], v[104:107]\n"
        "s_waitcnt lgkmcnt(2)\n"
        "v_mfma_f32_16x16x32_f16 v[76:79], v[152:155], v[236:239], v[76:79]\n"
        "ds_read_b128 v[232:235], v244 offset:30720\n"
        "v_mfma_f32_16x16x32_f16 v[108:111], v[156:159], v[236:239], v[108:111]\n"
        "s_waitcnt lgkmcnt(2)\n"
        "v_mfma_f32_16x16x32_f16 v[80:83], v[152:155], v[224:227], v[80:83]\n"
        "ds_read_b128 v[236:239], v244 offset:31744\n"
        "v_mfma_f32_16x16x32_f16 v[112:115], v[156:159], v[224:227], v[112:115]\n"
        "s_waitcnt lgkmcnt(2)\n"
        "v_mfma_f32_16x16x32_f16 v[84:87], v[152:155], v[228:231], v[84:87]\n"
        "ds_read_b128 v[224:227], v244 offset:32768\n"
        "v_mfma_f32_16x16x32_f16 v[116:119], v[156:159], v[228:231], v[116:119]\n"
        "s_waitcnt lgkmcnt(2)\n"
        "v_mfma_f32_16x16x32_f16 v[88:91], v[152:155], v[232:235], v[88:91]\n"
        "ds_read_b128 v[228:231], v244 offset:33792\n"
        "v_mfma_f32_16x16x32_f16 v[120:123], v[156:159], v[232:235], v[120:123]\n"
        "s_waitcnt lgkmcnt(2)\n"
        "v_mfma_f32_16x16x32_f16 v[92:95], v[152:155], v[236:239], v[92:95]\n"
        "ds_read_b128 v[232:235], v244 offset:34816\n"
        "v_mfma_f32_16x16x32_f16 v[124:127], v[156:159], v[236:239], v[124:127]\n"
        "s_waitcnt vmcnt(4) lgkmcnt(2)\n"
        "v_mfma_f32_16x16x32_f16 v[64:67], v[160:163], v[224:227], v[64:67]\n"
        "ds_read_b128 v[236:239], v244 offset:35840\n"
        "v_mfma_f32_16x16x32_f16 v[96:99], v[164:167], v[224:227], v[96:99]\n"
        "s_waitcnt lgkmcnt(2)\n"
        "v_mfma_f32_16x16x32_f16 v[68:71], v[160:163], v[228:231], v[68:71]\n"
        "ds_read_b128 v[224:227], v244 offset:36864\n"
        "v_mfma_f32_16x16x32_f16 v[100:103], v[164:167], v[228:231], v[100:103]\n"
        "s_waitcnt lgkmcnt(2)\n"
        "v_mfma_f32_16x16x32_f16 v[72:75], v[160:163], v[232:235], v[72:75]\n"
        "ds_read_b128 v[228:231], v244 offset:37888\n"
        "v_mfma_f32_16x16x32_f16 v[104:107], v[164:167], v[232:235], v[104:107]\n"
        "s_waitcnt lgkmcnt(2)\n"
        "v_mfma_f32_16x16x32_f16 v[76:79], v[160:163], v[236:239], v[76:79]\n"
        "ds_read_b128 v[232:235], v244 offset:38912\n"
        "v_mfma_f32_16x16x32_f16 v[108:111], v[164:167], v[236:239], v[108:111]\n"
        "s_waitcnt lgkmcnt(2)\n"
        "v_mfma_f32_16x16x32_f16 v[80:83], v[160:163], v[224:227], v[80:83]\n"
        "ds_read_b128 v[236:239], v244 offset:39936\n"
        "v_mfma_f32_16x16x32_f16 v[112:115], v[164:167], v[224:227], v[112:115]\n"
        "s_waitcnt lgkmcnt(2)\n"
        "v_mfma_f32_16x16x32_f16 v[84:87], v[160:163], v[228:231], v[84:87]\n"
        "ds_read_b128 v[224:227], v244 offset:40960\n"
        "v_mfma_f32_16x16x32_f16 v[116:119], v[164:167], v[228:231], v[116:119]\n"
        "s_waitcnt lgkmcnt(2)\n"
        "v_mfma_f32_16x16x32_f16 v[88:91], v[160:163], v[232:235], v[88:91]\n"
        "ds_read_b128 v[228:231], v244 offset:41984\n"
        "v_mfma_f32_16x16x32_f16 v[120:123], v[164:167], v[232:235], v[120:123]\n"
        "s_waitcnt lgkmcnt(2)\n"
        "v_mfma_f32_16x16x32_f16 v[92:95], v[160:163], v[236:239], v[92:95]\n"
        "ds_read_b128 v[232:235], v244 offset:43008\n"
        "v_mfma_f32_16x16x32_f16 v[124:127], v[164:167], v[236:239], v[124:127]\n"
        "s_waitcnt lgkmcnt(2)\n"
        "v_mfma_f32_16x16x32_f16 v[64:67], v[168:171], v[224:227], v[64:67]\n"
        "ds_read_b128 v[236:239], v244 offset:44032\n"
        "v_mfma_f32_16x16x32_f16 v[96:99], v[172:175], v[224:227], v[96:99]\n"
        "s_waitcnt lgkmcnt(2)\n"
        "v_mfma_f32_16x16x32_f16 v[68:71], v[168:171], v[228:231], v[68:71]\n"
        "ds_read_b128 v[224:227], v244 offset:45056\n"
        "v_mfma_f32_16x16x32_f16 v[100:103], v[172:175], v[228:231], v[100:103]\n"
        "s_setprio 2\n"
        "s_setprio 0\n"
        "s_waitcnt lgkmcnt(2)\n"
        "v_mfma_f32_16x16x32_f16 v[72:75], v[168:171], v[232:235], v[72:75]\n"
        "ds_read_b128 v[228:231], v244 offset:46080\n"
        "v_mfma_f32_16x16x32_f16 v[104:107], v[172:175], v[232:235], v[104:107]\n"
        "s_waitcnt lgkmcnt(2)\n"
        "v_mfma_f32_16x16x32_f16 v[76:79], v[168:171], v[236:239], v[76:79]\n"
        "ds_read_b128 v[232:235], v244 offset:47104\n"
        "v_mfma_f32_16x16x32_f16 v[108:111], v[172:175], v[236:239], v[108:111]\n"
        "s_waitcnt lgkmcnt(2)\n"
        "v_mfma_f32_16x16x32_f16 v[80:83], v[168:171], v[224:227], v[80:83]\n"
        "ds_read_b128 v[236:239], v244 offset:48128\n"
        "v_mfma_f32_16x16x32_f16 v[112:115], v[172:175], v[224:227], v[112:115]\n"
        "s_waitcnt lgkmcnt(2)\n"
        "v_mfma_f32_16x16x32_f16 v[84:87], v[168:171], v[228:231], v[84:87]\n"
        "ds_read_b128 v[224:227], v244 offset:49152\n"
        "v_mfma_f32_16x16x32_f16 v[116:119], v[172:175], v[228:231], v[116:119]\n"
        "s_waitcnt lgkmcnt(2)\n"
        "v_mfma_f32_16x16x32_f16 v[88:91], v[168:171], v[232:235], v[88:91]\n"
        "ds_read_b128 v[228:231], v244 offset:50176\n"
        "v_mfma_f32_16x16x32_f16 v[120:123], v[172:175], v[232:235], v[120:123]\n"
        "s_waitcnt lgkmcnt(2)\n"
        "v_mfma_f32_16x16x32_f16 v[92:95], v[168:171], v[236:239], v[92:95]\n"
        "ds_read_b128 v[232:235], v244 offset:51200\n"
        "v_mfma_f32_16x16x32_f16 v[124:127], v[172:175], v[236:239], v[124:127]\n"
        "s_waitcnt vmcnt(0) lgkmcnt(2)\n"
        "v_mfma_f32_16x16x32_f16 v[64:67], v[176:179], v[224:227], v[64:67]\n"
        "ds_read_b128 v[236:239], v244 offset:52224\n"
        "v_mfma_f32_16x16x32_f16 v[96:99], v[180:183], v[224:227], v[96:99]\n"
        "s_waitcnt lgkmcnt(2)\n"
        "v_mfma_f32_16x16x32_f16 v[68:71], v[176:179], v[228:231], v[68:71]\n"
        "ds_read_b128 v[224:227], v244 offset:53248\n"
        "v_mfma_f32_16x16x32_f16 v[100:103], v[180:183], v[228:231], v[100:103]\n"
        "s_waitcnt lgkmcnt(2)\n"
        "v_mfma_f32_16x16x32_f16 v[72:75], v[176:179], v[232:235], v[72:75]\n"
        "ds_read_b128 v[228:231], v244 offset:54272\n"
        "v_mfma_f32_16x16x32_f16 v[104:107], v[180:183], v[232:235], v[104:107]\n"
        "s_waitcnt lgkmcnt(2)\n"
        "v_mfma_f32_16x16x32_f16 v[76:79], v[176:179], v[236:239], v[76:79]\n"
        "ds_read_b128 v[232:235], v244 offset:55296\n"
        "v_mfma_f32_16x16x32_f16 v[108:111], v[180:183], v[236:239], v[108:111]\n"
        "s_waitcnt lgkmcnt(2)\n"
        "v_mfma_f32_16x16x32_f16 v[80:83], v[176:179], v[224:227], v[80:83]\n"
        "ds_read_b128 v[236:239], v244 offset:56320\n"
        "v_mfma_f32_16x16x32_f16 v[112:115], v[180:183], v[224:227], v[112:115]\n"
        "s_waitcnt lgkmcnt(2)\n"
        "v_mfma_f32_16x16x32_f16 v[84:87], v[176:179], v[228:231], v[84:87]\n"
        "ds_read_b128 v[224:227], v244 offset:57344\n"
        "v_mfma_f32_16x16x32_f16 v[116:119], v[180:183], v[228:231], v[116:119]\n"
        "s_waitcnt lgkmcnt(2)\n"
        "v_mfma_f32_16x16x32_f16 v[88:91], v[176:179], v[232:235], v[88:91]\n"
        "ds_read_b128 v[228:231], v244 offset:58368\n"
        "v_mfma_f32_16x16x32_f16 v[120:123], v[180:183], v[232:235], v[120:123]\n"
        "s_waitcnt lgkmcnt(2)\n"
        "v_mfma_f32_16x16x32_f16 v[92:95], v[176:179], v[236:239], v[92:95]\n"
        "ds_read_b128 v[232:235], v244 offset:59392\n"
        "v_mfma_f32_16x16x32_f16 v[124:127], v[180:183], v[236:239], v[124:127]\n"
        "s_waitcnt lgkmcnt(2)\n"
        "v_mfma_f32_16x16x32_f16 v[64:67], v[184:187], v[224:227], v[64:67]\n"
        "ds_read_b128 v[236:239], v244 offset:60416\n"
        "v_mfma_f32_16x16x32_f16 v[96:99], v[188:191], v[224:227], v[96:99]\n"
        "s_waitcnt lgkmcnt(2)\n"
        "v_mfma_f32_16x16x32_f16 v[68:71], v[184:187], v[228:231], v[68:71]\n"
        "ds_read_b128 v[224:227], v244 offset:61440\n"
        "v_mfma_f32_16x16x32_f16 v[100:103], v[188:191], v[228:231], v[100:103]\n"
        "s_waitcnt lgkmcnt(2)\n"
        "v_mfma_f32_16x16x32_f16 v[72:75], v[184:187], v[232:235], v[72:75]\n"
        "ds_read_b128 v[228:231], v244 offset:62464\n"
        "v_mfma_f32_16x16x32_f16 v[104:107], v[188:191], v[232:235], v[104:107]\n"
        "s_waitcnt lgkmcnt(2)\n"
        "v_mfma_f32_16x16x32_f16 v[76:79], v[184:187], v[236:239], v[76:79]\n"
        "ds_read_b128 v[232:235], v244 offset:63488\n"
        "v_mfma_f32_16x16x32_f16 v[108:111], v[188:191], v[236:239], v[108:111]\n"
        "s_waitcnt lgkmcnt(2)\n"
        "v_mfma_f32_16x16x32_f16 v[80:83], v[184:187], v[224:227], v[80:83]\n"
        "ds_read_b128 v[236:239], v244 offset:64512\n"
        "v_mfma_f32_16x16x32_f16 v[112:115], v[188:191], v[224:227], v[112:115]\n"
        "s_waitcnt lgkmcnt(2)\n"
        "v_mfma_f32_16x16x32_f16 v[84:87], v[184:187], v[228:231], v[84:87]\n"
        "v_mfma_f32_16x16x32_f16 v[116:119], v[188:191], v[228:231], v[116:119]\n"
        "s_waitcnt lgkmcnt(1)\n"
        "v_mfma_f32_16x16x32_f16 v[88:91], v[184:187], v[232:235], v[88:91]\n"
        "v_mfma_f32_16x16x32_f16 v[120:123], v[188:191], v[232:235], v[120:123]\n"
        "s_waitcnt lgkmcnt(0)\n"
        "v_mfma_f32_16x16x32_f16 v[92:95], v[184:187], v[236:239], v[92:95]\n"
        "v_mfma_f32_16x16x32_f16 v[124:127], v[188:191], v[236:239], v[124:127]\n"
        "s_branch L_epi%=\n"
        "L_ypre%=:\n"
        "s_lshl_b32 s82, s75, 23\n"
        "buffer_load_dwordx4 v[192:195], %[wvoff], %[rsW], s82 offen nt\n"
        "s_lshl_b32 s82, s75, 23\n"
        "s_add_u32 s82, s82, 131072\n"
        "buffer_load_dwordx4 v[196:199], %[wvoff], %[rsW], s82 offen nt\n"
        "s_lshl_b32 s80, s72, 16\n"
        "s_add_u32 s80, s80, s62\n"
        "s_add_u32 s86, s80, 2048\n"
        "buffer_load_dwordx4 v[128:131], %[avoff], %[rsA], s80 offen offset:0\n"
        "buffer_load_dwordx4 v[132:135], %[avoff], %[rsA], s80 offen offset:1024\n"
        "buffer_load_dwordx4 v[136:139], %[avoff], %[rsA], s86 offen offset:0\n"
        "buffer_load_dwordx4 v[140:143], %[avoff], %[rsA], s86 offen offset:1024\n"
        "s_lshl_b32 s82, s75, 23\n"
        "s_add_u32 s82, s82, 262144\n"
        "buffer_load_dwordx4 v[200:203], %[wvoff], %[rsW], s82 offen nt\n"
        "s_lshl_b32 s82, s75, 23\n"
        "s_add_u32 s82, s82, 393216\n"
        "buffer_load_dwordx4 v[204:207], %[wvoff], %[rsW], s82 offen nt\n"
        "s_lshl_b32 s82, s76, 23\n"
        "buffer_load_dwordx4 v[208:211], %[wvoff], %[rsW], s82 offen nt\n"
        "s_lshl_b32 s81, s72, 16\n"
        "s_add_u32 s81, s81, s62\n"
        "s_add_u32 s81, s81, 4096\n"
        "s_add_u32 s87, s81, 2048\n"
        "buffer_load_dwordx4 v[144:147], %[avoff], %[rsA], s81 offen offset:0\n"
        "buffer_load_dwordx4 v[148:151], %[avoff], %[rsA], s81 offen offset:1024\n"
        "buffer_load_dwordx4 v[152:155], %[avoff], %[rsA], s87 offen offset:0\n"
        "buffer_load_dwordx4 v[156:159], %[avoff], %[rsA], s87 offen offset:1024\n"
        "s_lshl_b32 s82, s76, 23\n"
        "s_add_u32 s82, s82, 131072\n"
        "buffer_load_dwordx4 v[212:215], %[wvoff], %[rsW], s82 offen nt\n"
        "s_lshl_b32 s82, s76, 23\n"
        "s_add_u32 s82, s82, 262144\n"
        "buffer_load_dwordx4 v[216:219], %[wvoff], %[rsW], s82 offen nt\n"
        "s_lshl_b32 s82, s76, 23\n"
        "s_add_u32 s82, s82, 393216\n"
        "buffer_load_dwordx4 v[220:223], %[wvoff], %[rsW], s82 offen nt\n"
        "v_mov_b32 v0, 0\n"
        "v_mov_b32 v1, 0\n"
        "v_mov_b32 v2, 0\n"
        "v_mov_b32 v3, 0\n"
        "v_mov_b32 v4, 0\n"
        "v_mov_b32 v5, 0\n"
        "v_mov_b32 v6, 0\n"
        "v_mov_b32 v7, 0\n"
        "v_mov_b32 v8, 0\n"
        "v_mov_b32 v9, 0\n"
        "v_mov_b32 v10, 0\n"
        "v_mov_b32 v11, 0\n"
        "v_mov_b32 v12, 0\n"
        "v_mov_b32 v13, 0\n"
        "v_mov_b32 v14, 0\n"
        "v_mov_b32 v15, 0\n"
        "v_mov_b32 v16, 0\n"
        "v_mov_b32 v17, 0\n"
        "v_mov_b32 v18, 0\n"
        "v_mov_b32 v19, 0\n"
        "v_mov_b32 v20, 0\n"
        "v_mov_b32 v21, 0\n"
        "v_mov_b32 v22, 0\n"
        "v_mov_b32 v23, 0\n"
        "v_mov_b32 v24, 0\n"
        "v_mov_b32 v25, 0\n"
        "v_mov_b32 v26, 0\n"
        "v_mov_b32 v27, 0\n"
        "v_mov_b32 v28, 0\n"
        "v_mov_b32 v29, 0\n"
        "v_mov_b32 v30, 0\n"
        "v_mov_b32 v31, 0\n"
        "v_mov_b32 v32, 0\n"
        "v_mov_b32 v33, 0\n"
        "v_mov_b32 v34, 0\n"
        "v_mov_b32 v35, 0\n"
        "v_mov_b32 v36, 0\n"
        "v_mov_b32 v37, 0\n"
        "v_mov_b32 v38, 0\n"
        "v_mov_b32 v39, 0\n"
        "v_mov_b32 v40, 0\n"
        "v_mov_b32 v41, 0\n"
        "v_mov_b32 v42, 0\n"
        "v_mov_b32 v43, 0\n"
        "v_mov_b32 v44, 0\n"
        "v_mov_b32 v45, 0\n"
        "v_mov_b32 v46, 0\n"
        "v_mov_b32 v47, 0\n"
        "v_mov_b32 v48, 0\n"
        "v_mov_b32 v49, 0\n"
        "v_mov_b32 v50, 0\n"
        "v_mov_b32 v51, 0\n"
        "v_mov_b32 v52, 0\n"
        "v_mov_b32 v53, 0\n"
        "v_mov_b32 v54, 0\n"
        "v_mov_b32 v55, 0\n"
        "v_mov_b32 v56, 0\n"
        "v_mov_b32 v57, 0\n"
        "v_mov_b32 v58, 0\n"
        "v_mov_b32 v59, 0\n"
        "v_mov_b32 v60, 0\n"
        "v_mov_b32 v61, 0\n"
        "v_mov_b32 v62, 0\n"
        "v_mov_b32 v63, 0\n"
        "v_mov_b32 v64, 0\n"
        "v_mov_b32 v65, 0\n"
        "v_mov_b32 v66, 0\n"
        "v_mov_b32 v67, 0\n"
        "v_mov_b32 v68, 0\n"
        "v_mov_b32 v69, 0\n"
        "v_mov_b32 v70, 0\n"
        "v_mov_b32 v71, 0\n"
        "v_mov_b32 v72, 0\n"
        "v_mov_b32 v73, 0\n"
        "v_mov_b32 v74, 0\n"
        "v_mov_b32 v75, 0\n"
        "v_mov_b32 v76, 0\n"
        "v_mov_b32 v77, 0\n"
        "v_mov_b32 v78, 0\n"
        "v_mov_b32 v79, 0\n"
        "v_mov_b32 v80, 0\n"
        "v_mov_b32 v81, 0\n"
        "v_mov_b32 v82, 0\n"
        "v_mov_b32 v83, 0\n"
        "v_mov_b32 v84, 0\n"
        "v_mov_b32 v85, 0\n"
        "v_mov_b32 v86, 0\n"
        "v_mov_b32 v87, 0\n"
        "v_mov_b32 v88, 0\n"
        "v_mov_b32 v89, 0\n"
        "v_mov_b32 v90, 0\n"
        "v_mov_b32 v91, 0\n"
        "v_mov_b32 v92, 0\n"
        "v_mov_b32 v93, 0\n"
        "v_mov_b32 v94, 0\n"
        "v_mov_b32 v95, 0\n"
        "v_mov_b32 v96, 0\n"
        "v_mov_b32 v97, 0\n"
        "v_mov_b32 v98, 0\n"
        "v_mov_b32 v99, 0\n"
        "v_mov_b32 v100, 0\n"
        "v_mov_b32 v101, 0\n"
        "v_mov_b32 v102, 0\n"
        "v_mov_b32 v103, 0\n"
        "v_mov_b32 v104, 0\n"
        "v_mov_b32 v105, 0\n"
        "v_mov_b32 v106, 0\n"
        "v_mov_b32 v107, 0\n"
        "v_mov_b32 v108, 0\n"
        "v_mov_b32 v109, 0\n"
        "v_mov_b32 v110, 0\n"
        "v_mov_b32 v111, 0\n"
        "v_mov_b32 v112, 0\n"
        "v_mov_b32 v113, 0\n"
        "v_mov_b32 v114, 0\n"
        "v_mov_b32 v115, 0\n"
        "v_mov_b32 v116, 0\n"
        "v_mov_b32 v117, 0\n"
        "v_mov_b32 v118, 0\n"
        "v_mov_b32 v119, 0\n"
        "v_mov_b32 v120, 0\n"
        "v_mov_b32 v121, 0\n"
        "v_mov_b32 v122, 0\n"
        "v_mov_b32 v123, 0\n"
        "v_mov_b32 v124, 0\n"
        "v_mov_b32 v125, 0\n"
        "v_mov_b32 v126, 0\n"
        "v_mov_b32 v127, 0\n"
        "ds_read_b128 v[224:227], %[brd] offset:0\n"
        "ds_read_b128 v[228:231], %[brd] offset:1024\n"
        "ds_read_b128 v[232:235], %[brd] offset:2048\n"
        "s_setprio 1\n"
        "L_loopY%=:\n"
        "s_waitcnt vmcnt(8)\n"
        "v_cvt_pk_f16_f32 v240, v192, v196\n"
        "v_cvt_pk_f16_f32 v241, v200, v204\n"
        "ds_write_b64 %[bwr], v[240:241] offset:49152\n"
        "v_cvt_pk_f16_f32 v242, v193, v197\n"
        "v_cvt_pk_f16_f32 v243, v201, v205\n"
        "ds_write_b64 %[bwr], v[242:243] offset:50176\n"
        "v_cvt_pk_f16_f32 v240, v194, v198\n"
        "v_cvt_pk_f16_f32 v241, v202, v206\n"
        "ds_write_b64 %[bwr], v[240:241] offset:51200\n"
        "v_cvt_pk_f16_f32 v242, v195, v199\n"
        "v_cvt_pk_f16_f32 v243, v203, v207\n"
        "ds_write_b64 %[bwr], v[242:243] offset:52224\n"
        "s_lshl_b32 s80, s73, 16\n"
        "s_add_u32 s80, s80, s62\n"
        "s_add_u32 s86, s80, 2048\n"
        "buffer_load_dwordx4 v[160:163], %[avoff], %[rsA], s80 offen offset:0\n"
        "buffer_load_dwordx4 v[164:167], %[avoff], %[rsA], s80 offen offset:1024\n"
        "buffer_load_dwordx4 v[168:171], %[avoff], %[rsA], s86 offen offset:0\n"
        "buffer_load_dwordx4 v[172:175], %[avoff], %[rsA], s86 offen offset:1024\n"
        "s_lshl_b32 s81, s73, 16\n"
        "s_add_u32 s81, s81, s62\n"
        "s_add_u32 s81, s81, 4096\n"
        "s_add_u32 s87, s81, 2048\n"
        "buffer_load_dwordx4 v[176:179], %[avoff], %[rsA], s81 offen offset:0\n"
        "buffer_load_dwordx4 v[180:183], %[avoff], %[rsA], s81 offen offset:1024\n"
        "s_cmp_eq_u32 %[wave], 4\n"
        "s_cbranch_scc1 L_chkk2%=\n"
        "L_chkk2r%=:\n"
        "s_cmp_lg_u32 %[wave], 5\n"
        "s_cbranch_scc1 L_pc0%=\n"
        "s_sub_i32 s89, s58, 16\n"
        "s_max_i32 s89, s89, 0\n"
        "s_mul_i32 s89, s89, 4\n"
        "s_sub_i32 s89, 152, s89\n"
        "s_max_i32 s89, s89, s50\n"
        "s_cmp_lt_u32 s58, 4\n"
        "s_cselect_b32 s89, 100, s89\n"
        "s_add_u32 s58, s58, 1\n"
        "L_pace0%=:\n"
        "s_memrealtime s[92:93]\n"
        "s_waitcnt lgkmcnt(0)\n"
        "s_sub_u32 s93, s92, s59\n"
        "s_cmp_lt_u32 s93, s89\n"
        "s_cbranch_scc1 L_pace0%=\n"
        "s_mov_b32 s59, s92\n"
        "L_pc0%=:\n"
        "s_waitcnt lgkmcnt(0)\n"
        "s_barrier\n"
        "v_mfma_f32_16x16x32_f16 v[0:3], v[128:131], v[224:227], v[0:3]\n"
        "ds_read_b128 v[236:239], %[brd] offset:3072\n"
        "v_mfma_f32_16x16x32_f16 v[32:35], v[132:135], v[224:227], v[32:35]\n"
        "v_mfma_f32_16x16x32_f16 v[64:67], v[136:139], v[224:227], v[64:67]\n"
        "v_mfma_f32_16x16x32_f16 v[96:99], v[140:143], v[224:227], v[96:99]\n"
        "v_mfma_f32_16x16x32_f16 v[4:7], v[128:131], v[228:231], v[4:7]\n"
        "ds_read_b128 v[224:227], %[brd] offset:4096\n"
        "v_mfma_f32_16x16x32_f16 v[36:39], v[132:135], v[228:231], v[36:39]\n"
        "v_mfma_f32_16x16x32_f16 v[68:71], v[136:139], v[228:231], v[68:71]\n"
        "v_mfma_f32_16x16x32_f16 v[100:103], v[140:143], v[228:231], v[100:103]\n"
        "v_mfma_f32_16x16x32_f16 v[8:11], v[128:131], v[232:235], v[8:11]\n"
        "ds_read_b128 v[228:231], %[brd] offset:5120\n"
        "v_mfma_f32_16x16x32_f16 v[40:43], v[132:135], v[232:235], v[40:43]\n"
        "v_mfma_f32_16x16x32_f16 v[72:75], v[136:139], v[232:235], v[72:75]\n"
        "v_mfma_f32_16x16x32_f16 v[104:107], v[140:143], v[232:235], v[104:107]\n"
        "s_waitcnt lgkmcnt(2)\n"
        "v_mfma_f32_16x16x32_f16 v[12:15], v[128:131], v[236:239], v[12:15]\n"
        "ds_read_b128 v[232:235], %[brd] offset:6144\n"
        "v_mfma_f32_16x16x32_f16 v[44:47], v[132:135], v[236:239], v[44:47]\n"
        "v_mfma_f32_16x16x32_f16 v[76:79], v[136:139], v[236:239], v[76:79]\n"
        "v_mfma_f32_16x16x32_f16 v[108:111], v[140:143], v[236:239], v[108:111]\n"
        "s_waitcnt lgkmcnt(2)\n"
        "v_mfma_f32_16x16x32_f16 v[16:19], v[128:131], v[224:227], v[16:19]\n"
        "ds_read_b128 v[236:239], %[brd] offset:7168\n"
        "v_mfma_f32_16x16x32_f16 v[48:51], v[132:135], v[224:227], v[48:51]\n"
        "v_mfma_f32_16x16x32_f16 v[80:83], v[136:139], v[224:227], v[80:83]\n"
        "v_mfma_f32_16x16x32_f16 v[112:115], v[140:143], v[224:227], v[112:115]\n"
        "s_waitcnt lgkmcnt(2)\n"
        "v_mfma_f32_16x16x32_f16 v[20:23], v[128:131], v[228:231], v[20:23]\n"
        "ds_read_b128 v[224:227], %[brd] offset:8192\n"
        "v_mfma_f32_16x16x32_f16 v[52:55], v[132:135], v[228:231], v[52:55]\n"
        "v_mfma_f32_16x16x32_f16 v[84:87], v[136:139], v[228:231], v[84:87]\n"
        "v_mfma_f32_16x16x32_f16 v[116:119], v[140:143], v[228:231], v[116:119]\n"
        "s_barrier\n"
        "s_cmp_eq_u32 s66, 0\n"
        "s_cbranch_scc1 L_flagcY0%=\n"
        "L_flagretcY0%=:\n"
        "s_cmp_eq_u32 s65, 0\n"
        "s_cbranch_scc1 L_pushcY0%=\n"
        "L_pushretcY0%=:\n"
        "s_cmp_eq_u32 %[wave], 4\n"
        "s_cbranch_scc1 L_ctlcY0%=\n"
        "L_ctlcY0r%=:\n"
        "s_lshl_b32 s82, s77, 23\n"
        "buffer_load_dwordx4 v[192:195], %[wvoff], %[rsW], s82 offen nt\n"
        "s_lshl_b32 s82, s77, 23\n"
        "s_add_u32 s82, s82, 131072\n"
        "buffer_load_dwordx4 v[196:199], %[wvoff], %[rsW], s82 offen nt\n"
        "s_waitcnt lgkmcnt(2)\n"
        "v_mfma_f32_16x16x32_f16 v[24:27], v[128:131], v[232:235], v[24:27]\n"
        "ds_read_b128 v[228:231], %[brd] offset:9216\n"
        "v_mfma_f32_16x16x32_f16 v[56:59], v[132:135], v[232:235], v[56:59]\n"
        "v_mfma_f32_16x16x32_f16 v[88:91], v[136:139], v[232:235], v[88:91]\n"
        "v_mfma_f32_16x16x32_f16 v[120:123], v[140:143], v[232:235], v[120:123]\n"
        "s_waitcnt lgkmcnt(2)\n"
        "v_mfma_f32_16x16x32_f16 v[28:31], v[128:131], v[236:239], v[28:31]\n"
        "ds_read_b128 v[232:235], %[brd] offset:10240\n"
        "v_mfma_f32_16x16x32_f16 v[60:63], v[132:135], v[236:239], v[60:63]\n"
        "v_mfma_f32_16x16x32_f16 v[92:95], v[136:139], v[236:239], v[92:95]\n"
        "v_mfma_f32_16x16x32_f16 v[124:127], v[140:143], v[236:239], v[124:127]\n"
        "s_waitcnt vmcnt(11) lgkmcnt(2)\n"
        "v_mfma_f32_16x16x32_f16 v[0:3], v[144:147], v[224:227], v[0:3]\n"
        "ds_read_b128 v[236:239], %[brd] offset:11264\n"
        "v_mfma_f32_16x16x32_f16 v[32:35], v[148:151], v[224:227], v[32:35]\n"
        "v_mfma_f32_16x16x32_f16 v[64:67], v[152:155], v[224:227], v[64:67]\n"
        "v_mfma_f32_16x16x32_f16 v[96:99], v[156:159], v[224:227], v[96:99]\n"
        "s_waitcnt lgkmcnt(2)\n"
        "v_mfma_f32_16x16x32_f16 v[4:7], v[144:147], v[228:231], v[4:7]\n"
        "ds_read_b128 v[224:227], %[brd] offset:12288\n"
        "v_mfma_f32_16x16x32_f16 v[36:39], v[148:151], v[228:231], v[36:39]\n"
        "v_mfma_f32_16x16x32_f16 v[68:71], v[152:155], v[228:231], v[68:71]\n"
        "v_mfma_f32_16x16x32_f16 v[100:103], v[156:159], v[228:231], v[100:103]\n"
        "s_waitcnt lgkmcnt(2)\n"
        "v_mfma_f32_16x16x32_f16 v[8:11], v[144:147], v[232:235], v[8:11]\n"
        "ds_read_b128 v[228:231], %[brd] offset:13312\n"
        "v_mfma_f32_16x16x32_f16 v[40:43], v[148:151], v[232:235], v[40:43]\n"
        "v_mfma_f32_16x16x32_f16 v[72:75], v[152:155], v[232:235], v[72:75]\n"
        "v_mfma_f32_16x16x32_f16 v[104:107], v[156:159], v[232:235], v[104:107]\n"
        "s_waitcnt vmcnt(8)\n"
        "v_cvt_pk_f16_f32 v240, v208, v212\n"
        "v_cvt_pk_f16_f32 v241, v216, v220\n"
        "ds_write_b64 v245, v[240:241] offset:0\n"
        "v_cvt_pk_f16_f32 v242, v209, v213\n"
        "v_cvt_pk_f16_f32 v243, v217, v221\n"
        "ds_write_b64 v245, v[242:243] offset:1024\n"
        "v_cvt_pk_f16_f32 v240, v210, v214\n"
        "v_cvt_pk_f16_f32 v241, v218, v222\n"
        "ds_write_b64 v245, v[240:241] offset:2048\n"
        "v_cvt_pk_f16_f32 v242, v211, v215\n"
        "v_cvt_pk_f16_f32 v243, v219, v223\n"
        "ds_write_b64 v245, v[242:243] offset:3072\n"
        "buffer_load_dwordx4 v[184:187], %[avoff], %[rsA], s87 offen offset:0\n"
        "buffer_load_dwordx4 v[188:191], %[avoff], %[rsA], s87 offen offset:1024\n"
        "s_lshl_b32 s80, s74, 16\n"
        "s_add_u32 s80, s80, s62\n"
        "s_add_u32 s86, s80, 2048\n"
        "buffer_load_dwordx4 v[128:131], %[avoff], %[rsA], s80 offen offset:0\n"
        "buffer_load_dwordx4 v[132:135], %[avoff], %[rsA], s80 offen offset:1024\n"
        "buffer_load_dwordx4 v[136:139], %[avoff], %[rsA], s86 offen offset:0\n"
        "buffer_load_dwordx4 v[140:143], %[avoff], %[rsA], s86 offen offset:1024\n"
        "s_cmp_lg_u32 %[wave], 5\n"
        "s_cbranch_scc1 L_pc1%=\n"
        "s_sub_i32 s89, s58, 16\n"
        "s_max_i32 s89, s89, 0\n"
        "s_mul_i32 s89, s89, 4\n"
        "s_sub_i32 s89, 152, s89\n"
        "s_max_i32 s89, s89, s50\n"
        "s_cmp_lt_u32 s58, 4\n"
        "s_cselect_b32 s89, 100, s89\n"
        "s_add_u32 s58, s58, 1\n"
        "L_pace1%=:\n"
        "s_memrealtime s[92:93]\n"
        "s_waitcnt lgkmcnt(0)\n"
        "s_sub_u32 s93, s92, s59\n"
        "s_cmp_lt_u32 s93, s89\n"
        "s_cbranch_scc1 L_pace1%=\n"
        "s_mov_b32 s59, s92\n"
        "L_pc1%=:\n"
        "s_waitcnt lgkmcnt(0)\n"
        "s_barrier\n"
        "v_mfma_f32_16x16x32_f16 v[12:15], v[144:147], v[236:239], v[12:15]\n"
        "ds_read_b128 v[232:235], %[brd] offset:14336\n"
        "v_mfma_f32_16x16x32_f16 v[44:47], v[148:151], v[236:239], v[44:47]\n"
        "v_mov_b32 v247, 147456\n"
        "ds_read_b32 v240, v247\n"
        "v_mfma_f32_16x16x32_f16 v[76:79], v[152:155], v[236:239], v[76:79]\n"
        "v_mfma_f32_16x16x32_f16 v[108:111], v[156:159], v[236:239], v[108:111]\n"
        "v_mfma_f32_16x16x32_f16 v[16:19], v[144:147], v[224:227], v[16:19]\n"
        "ds_read_b128 v[236:239], %[brd] offset:15360\n"
        "v_mfma_f32_16x16x32_f16 v[48:51], v[148:151], v[224:227], v[48:51]\n"
        "v_mfma_f32_16x16x32_f16 v[80:83], v[152:155], v[224:227], v[80:83]\n"
        "v_mfma_f32_16x16x32_f16 v[112:115], v[156:159], v[224:227], v[112:115]\n"
        "v_mfma_f32_16x16x32_f16 v[20:23], v[144:147], v[228:231], v[20:23]\n"
        "ds_read_b128 v[224:227], %[brd] offset:16384\n"
        "v_mfma_f32_16x16x32_f16 v[52:55], v[148:151], v[228:231], v[52:55]\n"
        "v_mfma_f32_16x16x32_f16 v[84:87], v[152:155], v[228:231], v[84:87]\n"
        "v_mfma_f32_16x16x32_f16 v[116:119], v[156:159], v[228:231], v[116:119]\n"
        "s_waitcnt lgkmcnt(3)\n"
        "v_mfma_f32_16x16x32_f16 v[24:27], v[144:147], v[232:235], v[24:27]\n"
        "ds_read_b128 v[228:231], %[brd] offset:17408\n"
        "v_mfma_f32_16x16x32_f16 v[56:59], v[148:151], v[232:235], v[56:59]\n"
        "s_waitcnt lgkmcnt(3)\n"
        "v_readfirstlane_b32 s72, v240\n"
        "v_mfma_f32_16x16x32_f16 v[88:91], v[152:155], v[232:235], v[88:91]\n"
        "v_mfma_f32_16x16x32_f16 v[120:123], v[156:159], v[232:235], v[120:123]\n"
        "s_waitcnt lgkmcnt(2)\n"
        "v_mfma_f32_16x16x32_f16 v[28:31], v[144:147], v[236:239], v[28:31]\n"
        "ds_read_b128 v[232:235], %[brd] offset:18432\n"
        "v_mfma_f32_16x16x32_f16 v[60:63], v[148:151], v[236:239], v[60:63]\n"
        "v_mfma_f32_16x16x32_f16 v[92:95], v[152:155], v[236:239], v[92:95]\n"
        "v_mfma_f32_16x16x32_f16 v[124:127], v[156:159], v[236:239], v[124:127]\n"
        "s_barrier\n"
        "s_lshl_b32 s82, s77, 23\n"
        "s_add_u32 s82, s82, 262144\n"
        "buffer_load_dwordx4 v[200:203], %[wvoff], %[rsW], s82 offen nt\n"
        "s_lshl_b32 s82, s77, 23\n"
        "s_add_u32 s82, s82, 393216\n"
        "buffer_load_dwordx4 v[204:207], %[wvoff], %[rsW], s82 offen nt\n"
        "s_lshl_b32 s82, s78, 23\n"
        "buffer_load_dwordx4 v[208:211], %[wvoff], %[rsW], s82 offen nt\n"
        "s_waitcnt vmcnt(13) lgkmcnt(2)\n"
        "v_mfma_f32_16x16x32_f16 v[0:3], v[160:163], v[224:227], v[0:3]\n"
        "ds_read_b128 v[236:239], %[brd] offset:19456\n"
        "v_mfma_f32_16x16x32_f16 v[32:35], v[164:167], v[224:227], v[32:35]\n"
        "v_mfma_f32_16x16x32_f16 v[64:67], v[168:171], v[224:227], v[64:67]\n"
        "v_mfma_f32_16x16x32_f16 v[96:99], v[172:175], v[224:227], v[96:99]\n"
        "s_waitcnt lgkmcnt(2)\n"
        "v_mfma_f32_16x16x32_f16 v[4:7], v[160:163], v[228:231], v[4:7]\n"
        "ds_read_b128 v[224:227], %[brd] offset:20480\n"
        "v_mfma_f32_16x16x32_f16 v[36:39], v[164:167], v[228:231], v[36:39]\n"
        "v_mfma_f32_16x16x32_f16 v[68:71], v[168:171], v[228:231], v[68:71]\n"
        "v_mfma_f32_16x16x32_f16 v[100:103], v[172:175], v[228:231], v[100:103]\n"
        "s_waitcnt lgkmcnt(2)\n"
        "v_mfma_f32_16x16x32_f16 v[8:11], v[160:163], v[232:235], v[8:11]\n"
        "ds_read_b128 v[228:231], %[brd] offset:21504\n"
        "v_mfma_f32_16x16x32_f16 v[40:43], v[164:167], v[232:235], v[40:43]\n"
        "v_mfma_f32_16x16x32_f16 v[72:75], v[168:171], v[232:235], v[72:75]\n"
        "v_mfma_f32_16x16x32_f16 v[104:107], v[172:175], v[232:235], v[104:107]\n"
        "s_waitcnt lgkmcnt(2)\n"
        "v_mfma_f32_16x16x32_f16 v[12:15], v[160:163], v[236:239], v[12:15]\n"
        "ds_read_b128 v[232:235], %[brd] offset:22528\n"
        "v_mfma_f32_16x16x32_f16 v[44:47], v[164:167], v[236:239], v[44:47]\n"
        "v_mfma_f32_16x16x32_f16 v[76:79], v[168:171], v[236:239], v[76:79]\n"
        "v_mfma_f32_16x16x32_f16 v[108:111], v[172:175], v[236:239], v[108:111]\n"
        "s_waitcnt lgkmcnt(2)\n"
        "v_mfma_f32_16x16x32_f16 v[16:19], v[160:163], v[224:227], v[16:19]\n"
        "ds_read_b128 v[236:239], %[brd] offset:23552\n"
        "v_mfma_f32_16x16x32_f16 v[48:51], v[164:167], v[224:227], v[48:51]\n"
        "v_mfma_f32_16x16x32_f16 v[80:83], v[168:171], v[224:227], v[80:83]\n"
        "v_mfma_f32_16x16x32_f16 v[112:115], v[172:175], v[224:227], v[112:115]\n"
        "s_lshl_b32 s81, s74, 16\n"
        "s_add_u32 s81, s81, s62\n"
        "s_add_u32 s81, s81, 4096\n"
        "s_add_u32 s87, s81, 2048\n"
        "buffer_load_dwordx4 v[144:147], %[avoff], %[rsA], s81 offen offset:0\n"
        "buffer_load_dwordx4 v[148:151], %[avoff], %[rsA], s81 offen offset:1024\n"
        "buffer_load_dwordx4 v[152:155], %[avoff], %[rsA], s87 offen offset:0\n"
        "buffer_load_dwordx4 v[156:159], %[avoff], %[rsA], s87 offen offset:1024\n"
        "s_cmp_eq_u32 %[wave], 4\n"
        "s_cbranch_scc1 L_chkk3%=\n"
        "L_chkk3r%=:\n"
        "s_cmp_lg_u32 %[wave], 5\n"
        "s_cbranch_scc1 L_pc2%=\n"
        "s_sub_i32 s89, s58, 16\n"
        "s_max_i32 s89, s89, 0\n"
        "s_mul_i32 s89, s89, 4\n"
        "s_sub_i32 s89, 152, s89\n"
        "s_max_i32 s89, s89, s50\n"
        "s_cmp_lt_u32 s58, 4\n"
        "s_cselect_b32 s89, 100, s89\n"
        "s_add_u32 s58, s58, 1\n"
        "L_pace2%=:\n"
        "s_memrealtime s[92:93]\n"
        "s_waitcnt lgkmcnt(0)\n"
        "s_sub_u32 s93, s92, s59\n"
        "s_cmp_lt_u32 s93, s89\n"
        "s_cbranch_scc1 L_pace2%=\n"
        "s_mov_b32 s59, s92\n"
        "L_pc2%=:\n"
        "s_barrier\n"
        "s_waitcnt lgkmcnt(2)\n"
        "v_mfma_f32_16x16x32_f16 v[20:23], v[160:163], v[228:231], v[20:23]\n"
        "ds_read_b128 v[224:227], %[brd] offset:24576\n"
        "v_mfma_f32_16x16x32_f16 v[52:55], v[164:167], v[228:231], v[52:55]\n"
        "v_mfma_f32_16x16x32_f16 v[84:87], v[168:171], v[228:231], v[84:87]\n"
        "v_mfma_f32_16x16x32_f16 v[116:119], v[172:175], v[228:231], v[116:119]\n"
        "s_waitcnt lgkmcnt(2)\n"
        "v_mfma_f32_16x16x32_f16 v[24:27], v[160:163], v[232:235], v[24:27]\n"
        "ds_read_b128 v[228:231], %[brd] offset:25600\n"
        "v_mfma_f32_16x16x32_f16 v[56:59], v[164:167], v[232:235], v[56:59]\n"
        "v_mfma_f32_16x16x32_f16 v[88:91], v[168:171], v[232:235], v[88:91]\n"
        "v_mfma_f32_16x16x32_f16 v[120:123], v[172:175], v[232:235], v[120:123]\n"
        "s_waitcnt lgkmcnt(2)\n"
        "v_mfma_f32_16x16x32_f16 v[28:31], v[160:163], v[236:239], v[28:31]\n"
        "ds_read_b128 v[232:235], %[brd] offset:26624\n"
        "v_mfma_f32_16x16x32_f16 v[60:63], v[164:167], v[236:239], v[60:63]\n"
        "v_mfma_f32_16x16x32_f16 v[92:95], v[168:171], v[236:239], v[92:95]\n"
        "v_mfma_f32_16x16x32_f16 v[124:127], v[172:175], v[236:239], v[124:127]\n"
        "s_waitcnt vmcnt(11) lgkmcnt(2)\n"
        "v_mfma_f32_16x16x32_f16 v[0:3], v[176:179], v[224:227], v[0:3]\n"
        "ds_read_b128 v[236:239], %[brd] offset:27648\n"
        "v_mfma_f32_16x16x32_f16 v[32:35], v[180:183], v[224:227], v[32:35]\n"
        "v_mfma_f32_16x16x32_f16 v[64:67], v[184:187], v[224:227], v[64:67]\n"
        "v_mfma_f32_16x16x32_f16 v[96:99], v[188:191], v[224:227], v[96:99]\n"
        "s_waitcnt lgkmcnt(2)\n"
        "v_mfma_f32_16x16x32_f16 v[4:7], v[176:179], v[228:231], v[4:7]\n"
        "ds_read_b128 v[224:227], %[brd] offset:28672\n"
        "v_mfma_f32_16x16x32_f16 v[36:39], v[180:183], v[228:231], v[36:39]\n"
        "v_mov_b32 v247, 147460\n"
        "ds_read_b32 v240, v247\n"
        "v_mfma_f32_16x16x32_f16 v[68:71], v[184:187], v[228:231], v[68:71]\n"
        "v_mfma_f32_16x16x32_f16 v[100:103], v[188:191], v[228:231], v[100:103]\n"
        "s_waitcnt lgkmcnt(3)\n"
        "v_mfma_f32_16x16x32_f16 v[8:11], v[176:179], v[232:235], v[8:11]\n"
        "ds_read_b128 v[228:231], %[brd] offset:29696\n"
        "v_mfma_f32_16x16x32_f16 v[40:43], v[180:183], v[232:235], v[40:43]\n"
        "v_mfma_f32_16x16x32_f16 v[72:75], v[184:187], v[232:235], v[72:75]\n"
        "v_mfma_f32_16x16x32_f16 v[104:107], v[188:191], v[232:235], v[104:107]\n"
        "s_barrier\n"
        "s_lshl_b32 s82, s78, 23\n"
        "s_add_u32 s82, s82, 131072\n"
        "buffer_load_dwordx4 v[212:215], %[wvoff], %[rsW], s82 offen nt\n"
        "s_lshl_b32 s82, s78, 23\n"
        "s_add_u32 s82, s82, 262144\n"
        "buffer_load_dwordx4 v[216:219], %[wvoff], %[rsW], s82 offen nt\n"
        "s_lshl_b32 s82, s78, 23\n"
        "s_add_u32 s82, s82, 393216\n"
        "buffer_load_dwordx4 v[220:223], %[wvoff], %[rsW], s82 offen nt\n"
        "s_waitcnt lgkmcnt(3)\n"
        "v_mfma_f32_16x16x32_f16 v[12:15], v[176:179], v[236:239], v[12:15]\n"
        "ds_read_b128 v[232:235], %[brd] offset:30720\n"
        "v_mfma_f32_16x16x32_f16 v[44:47], v[180:183], v[236:239], v[44:47]\n"
        "v_mfma_f32_16x16x32_f16 v[76:79], v[184:187], v[236:239], v[76:79]\n"
        "v_mfma_f32_16x16x32_f16 v[108:111], v[188:191], v[236:239], v[108:111]\n"
        "s_waitcnt lgkmcnt(3)\n"
        "v_mfma_f32_16x16x32_f16 v[16:19], v[176:179], v[224:227], v[16:19]\n"
        "ds_read_b128 v[236:239], %[brd] offset:31744\n"
        "v_mfma_f32_16x16x32_f16 v[48:51], v[180:183], v[224:227], v[48:51]\n"
        "s_waitcnt lgkmcnt(3)\n"
        "v_readfirstlane_b32 s73, v240\n"
        "v_mfma_f32_16x16x32_f16 v[80:83], v[184:187], v[224:227], v[80:83]\n"
        "v_mfma_f32_16x16x32_f16 v[112:115], v[188:191], v[224:227], v[112:115]\n"
        "s_waitcnt lgkmcnt(2)\n"
        "v_mfma_f32_16x16x32_f16 v[20:23], v[176:179], v[228:231], v[20:23]\n"
        "ds_read_b128 v[224:227], %[brd] offset:32768\n"
        "v_mfma_f32_16x16x32_f16 v[52:55], v[180:183], v[228:231], v[52:55]\n"
        "v_mfma_f32_16x16x32_f16 v[84:87], v[184:187], v[228:231], v[84:87]\n"
        "v_mfma_f32_16x16x32_f16 v[116:119], v[188:191], v[228:231], v[116:119]\n"
        "s_waitcnt lgkmcnt(2)\n"
        "v_mfma_f32_16x16x32_f16 v[24:27], v[176:179], v[232:235], v[24:27]\n"
        "ds_read_b128 v[228:231], %[brd] offset:33792\n"
        "v_mfma_f32_16x16x32_f16 v[56:59], v[180:183], v[232:235], v[56:59]\n"
        "v_mfma_f32_16x16x32_f16 v[88:91], v[184:187], v[232:235], v[88:91]\n"
        "v_mfma_f32_16x16x32_f16 v[120:123], v[188:191], v[232:235], v[120:123]\n"
        "s_waitcnt lgkmcnt(2)\n"
        "v_mfma_f32_16x16x32_f16 v[28:31], v[176:179], v[236:239], v[28:31]\n"
        "ds_read_b128 v[232:235], %[brd] offset:34816\n"
        "v_mfma_f32_16x16x32_f16 v[60:63], v[180:183], v[236:239], v[60:63]\n"
        "v_mfma_f32_16x16x32_f16 v[92:95], v[184:187], v[236:239], v[92:95]\n"
        "v_mfma_f32_16x16x32_f16 v[124:127], v[188:191], v[236:239], v[124:127]\n"
        "s_waitcnt vmcnt(8)\n"
        "v_cvt_pk_f16_f32 v240, v192, v196\n"
        "v_cvt_pk_f16_f32 v241, v200, v204\n"
        "ds_write_b64 v245, v[240:241] offset:16384\n"
        "v_cvt_pk_f16_f32 v242, v193, v197\n"
        "v_cvt_pk_f16_f32 v243, v201, v205\n"
        "ds_write_b64 v245, v[242:243] offset:17408\n"
        "v_cvt_pk_f16_f32 v240, v194, v198\n"
        "v_cvt_pk_f16_f32 v241, v202, v206\n"
        "ds_write_b64 v245, v[240:241] offset:18432\n"
        "v_cvt_pk_f16_f32 v242, v195, v199\n"
        "v_cvt_pk_f16_f32 v243, v203, v207\n"
        "ds_write_b64 v245, v[242:243] offset:19456\n"
        "s_lshl_b32 s80, s75, 16\n"
        "s_add_u32 s80, s80, s62\n"
        "s_add_u32 s86, s80, 2048\n"
        "buffer_load_dwordx4 v[160:163], %[avoff], %[rsA], s80 offen offset:0\n"
        "buffer_load_dwordx4 v[164:167], %[avoff], %[rsA], s80 offen offset:1024\n"
        "buffer_load_dwordx4 v[168:171], %[avoff], %[rsA], s86 offen offset:0\n"
        "buffer_load_dwordx4 v[172:175], %[avoff], %[rsA], s86 offen offset:1024\n"
        "s_lshl_b32 s81, s75, 16\n"
        "s_add_u32 s81, s81, s62\n"
        "s_add_u32 s81, s81, 4096\n"
        "s_add_u32 s87, s81, 2048\n"
        "buffer_load_dwordx4 v[176:179], %[avoff], %[rsA], s81 offen offset:0\n"
        "buffer_load_dwordx4 v[180:183], %[avoff], %[rsA], s81 offen offset:1024\n"
        "s_cmp_eq_u32 %[wave], 4\n"
        "s_cbranch_scc1 L_chkk4%=\n"
        "L_chkk4r%=:\n"
        "s_cmp_lg_u32 %[wave], 5\n"
        "s_cbranch_scc1 L_pc3%=\n"
        "s_sub_i32 s89, s58, 16\n"
        "s_max_i32 s89, s89, 0\n"
        "s_mul_i32 s89, s89, 4\n"
        "s_sub_i32 s89, 152, s89\n"
        "s_max_i32 s89, s89, s50\n"
        "s_cmp_lt_u32 s58, 4\n"
        "s_cselect_b32 s89, 100, s89\n"
        "s_add_u32 s58, s58, 1\n"
        "L_pace3%=:\n"
        "s_memrealtime s[92:93]\n"
        "s_waitcnt lgkmcnt(0)\n"
        "s_sub_u32 s93, s92, s59\n"
        "s_cmp_lt_u32 s93, s89\n"
        "s_cbranch_scc1 L_pace3%=\n"
        "s_mov_b32 s59, s92\n"
        "L_pc3%=:\n"
        "s_waitcnt lgkmcnt(0)\n"
        "s_barrier\n"
        "v_mfma_f32_16x16x32_f16 v[0:3], v[128:131], v[224:227], v[0:3]\n"
        "ds_read_b128 v[236:239], %[brd] offset:35840\n"
        "v_mfma_f32_16x16x32_f16 v[32:35], v[132:135], v[224:227], v[32:35]\n"
        "v_mfma_f32_16x16x32_f16 v[64:67], v[136:139], v[224:227], v[64:67]\n"
        "v_mfma_f32_16x16x32_f16 v[96:99], v[140:143], v[224:227], v[96:99]\n"
        "v_mfma_f32_16x16x32_f16 v[4:7], v[128:131], v[228:231], v[4:7]\n"
        "ds_read_b128 v[224:227], %[brd] offset:36864\n"
        "v_mfma_f32_16x16x32_f16 v[36:39], v[132:135], v[228:231], v[36:39]\n"
        "v_mfma_f32_16x16x32_f16 v[68:71], v[136:139], v[228:231], v[68:71]\n"
        "v_mfma_f32_16x16x32_f16 v[100:103], v[140:143], v[228:231], v[100:103]\n"
        "v_mfma_f32_16x16x32_f16 v[8:11], v[128:131], v[232:235], v[8:11]\n"
        "ds_read_b128 v[228:231], %[brd] offset:37888\n"
        "v_mfma_f32_16x16x32_f16 v[40:43], v[132:135], v[232:235], v[40:43]\n"
        "v_mfma_f32_16x16x32_f16 v[72:75], v[136:139], v[232:235], v[72:75]\n"
        "v_mfma_f32_16x16x32_f16 v[104:107], v[140:143], v[232:235], v[104:107]\n"
        "s_waitcnt lgkmcnt(2)\n"
        "v_mfma_f32_16x16x32_f16 v[12:15], v[128:131], v[236:239], v[12:15]\n"
        "ds_read_b128 v[232:235], %[brd] offset:38912\n"
        "v_mfma_f32_16x16x32_f16 v[44:47], v[132:135], v[236:239], v[44:47]\n"
        "v_mfma_f32_16x16x32_f16 v[76:79], v[136:139], v[236:239], v[76:79]\n"
        "v_mfma_f32_16x16x32_f16 v[108:111], v[140:143], v[236:239], v[108:111]\n"
        "s_waitcnt lgkmcnt(2)\n"
        "v_mfma_f32_16x16x32_f16 v[16:19], v[128:131], v[224:227], v[16:19]\n"
        "ds_read_b128 v[236:239], %[brd] offset:39936\n"
        "v_mfma_f32_16x16x32_f16 v[48:51], v[132:135], v[224:227], v[48:51]\n"
        "v_mfma_f32_16x16x32_f16 v[80:83], v[136:139], v[224:227], v[80:83]\n"
        "v_mfma_f32_16x16x32_f16 v[112:115], v[140:143], v[224:227], v[112:115]\n"
        "s_waitcnt lgkmcnt(2)\n"
        "v_mfma_f32_16x16x32_f16 v[20:23], v[128:131], v[228:231], v[20:23]\n"
        "ds_read_b128 v[224:227], %[brd] offset:40960\n"
        "v_mfma_f32_16x16x32_f16 v[52:55], v[132:135], v[228:231], v[52:55]\n"
        "v_mfma_f32_16x16x32_f16 v[84:87], v[136:139], v[228:231], v[84:87]\n"
        "v_mfma_f32_16x16x32_f16 v[116:119], v[140:143], v[228:231], v[116:119]\n"
        "s_barrier\n"
        "s_cmp_eq_u32 s66, 2\n"
        "s_cbranch_scc1 L_flagcY1%=\n"
        "L_flagretcY1%=:\n"
        "s_cmp_eq_u32 s65, 2\n"
        "s_cbranch_scc1 L_pushcY1%=\n"
        "L_pushretcY1%=:\n"
        "s_cmp_eq_u32 %[wave], 4\n"
        "s_cbranch_scc1 L_ctlcY1%=\n"
        "L_ctlcY1r%=:\n"
        "s_lshl_b32 s82, s79, 23\n"
        "buffer_load_dwordx4 v[192:195], %[wvoff], %[rsW], s82 offen nt\n"
        "s_lshl_b32 s82, s79, 23\n"
        "s_add_u32 s82, s82, 131072\n"
        "buffer_load_dwordx4 v[196:199], %[wvoff], %[rsW], s82 offen nt\n"
        "s_waitcnt lgkmcnt(2)\n"
        "v_mfma_f32_16x16x32_f16 v[24:27], v[128:131], v[232:235], v[24:27]\n"
        "ds_read_b128 v[228:231], %[brd] offset:41984\n"
        "v_mfma_f32_16x16x32_f16 v[56:59], v[132:135], v[232:235], v[56:59]\n"
        "v_mfma_f32_16x16x32_f16 v[88:91], v[136:139], v[232:235], v[88:91]\n"
        "v_mfma_f32_16x16x32_f16 v[120:123], v[140:143], v[232:235], v[120:123]\n"
        "s_waitcnt lgkmcnt(2)\n"
        "v_mfma_f32_16x16x32_f16 v[28:31], v[128:131], v[236:239], v[28:31]\n"
        "ds_read_b128 v[232:235], %[brd] offset:43008\n"
        "v_mfma_f32_16x16x32_f16 v[60:63], v[132:135], v[236:239], v[60:63]\n"
        "v_mfma_f32_16x16x32_f16 v[92:95], v[136:139], v[236:239], v[92:95]\n"
        "v_mfma_f32_16x16x32_f16 v[124:127], v[140:143], v[236:239], v[124:127]\n"
        "s_waitcnt vmcnt(11) lgkmcnt(2)\n"
        "v_mfma_f32_16x16x32_f16 v[0:3], v[144:147], v[224:227], v[0:3]\n"
        "ds_read_b128 v[236:239], %[brd] offset:44032\n"
        "v_mfma_f32_16x16x32_f16 v[32:35], v[148:151], v[224:227], v[32:35]\n"
        "v_mfma_f32_16x16x32_f16 v[64:67], v[152:155], v[224:227], v[64:67]\n"
        "v_mfma_f32_16x16x32_f16 v[96:99], v[156:159], v[224:227], v[96:99]\n"
        "s_waitcnt lgkmcnt(2)\n"
        "v_mfma_f32_16x16x32_f16 v[4:7], v[144:147], v[228:231], v[4:7]\n"
        "ds_read_b128 v[224:227], %[brd] offset:45056\n"
        "v_mfma_f32_16x16x32_f16 v[36:39], v[148:151], v[228:231], v[36:39]\n"
        "v_mfma_f32_16x16x32_f16 v[68:71], v[152:155], v[228:231], v[68:71]\n"
        "v_mfma_f32_16x16x32_f16 v[100:103], v[156:159], v[228:231], v[100:103]\n"
        "s_waitcnt lgkmcnt(2)\n"
        "v_mfma_f32_16x16x32_f16 v[8:11], v[144:147], v[232:235], v[8:11]\n"
        "ds_read_b128 v[228:231], %[brd] offset:46080\n"
        "v_mfma_f32_16x16x32_f16 v[40:43], v[148:151], v[232:235], v[40:43]\n"
        "v_mfma_f32_16x16x32_f16 v[72:75], v[152:155], v[232:235], v[72:75]\n"
        "v_mfma_f32_16x16x32_f16 v[104:107], v[156:159], v[232:235], v[104:107]\n"
        "s_waitcnt vmcnt(8)\n"
        "v_cvt_pk_f16_f32 v240, v208, v212\n"
        "v_cvt_pk_f16_f32 v241, v216, v220\n"
        "ds_write_b64 v245, v[240:241] offset:32768\n"
        "v_cvt_pk_f16_f32 v242, v209, v213\n"
        "v_cvt_pk_f16_f32 v243, v217, v221\n"
        "ds_write_b64 v245, v[242:243] offset:33792\n"
        "v_cvt_pk_f16_f32 v240, v210, v214\n"
        "v_cvt_pk_f16_f32 v241, v218, v222\n"
        "ds_write_b64 v245, v[240:241] offset:34816\n"
        "v_cvt_pk_f16_f32 v242, v211, v215\n"
        "v_cvt_pk_f16_f32 v243, v219, v223\n"
        "ds_write_b64 v245, v[242:243] offset:35840\n"
        "buffer_load_dwordx4 v[184:187], %[avoff], %[rsA], s87 offen offset:0\n"
        "buffer_load_dwordx4 v[188:191], %[avoff], %[rsA], s87 offen offset:1024\n"
        "s_lshl_b32 s80, s76, 16\n"
        "s_add_u32 s80, s80, s62\n"
        "s_add_u32 s86, s80, 2048\n"
        "buffer_load_dwordx4 v[128:131], %[avoff], %[rsA], s80 offen offset:0\n"
        "buffer_load_dwordx4 v[132:135], %[avoff], %[rsA], s80 offen offset:1024\n"
        "buffer_load_dwordx4 v[136:139], %[avoff], %[rsA], s86 offen offset:0\n"
        "buffer_load_dwordx4 v[140:143], %[avoff], %[rsA], s86 offen offset:1024\n"
        "s_cmp_lg_u32 %[wave], 5\n"
        "s_cbranch_scc1 L_pc4%=\n"
        "s_sub_i32 s89, s58, 16\n"
        "s_max_i32 s89, s89, 0\n"
        "s_mul_i32 s89, s89, 4\n"
        "s_sub_i32 s89, 152, s89\n"
        "s_max_i32 s89, s89, s50\n"
        "s_cmp_lt_u32 s58, 4\n"
        "s_cselect_b32 s89, 100, s89\n"
        "s_add_u32 s58, s58, 1\n"
        "L_pace4%=:\n"
        "s_memrealtime s[92:93]\n"
        "s_waitcnt lgkmcnt(0)\n"
        "s_sub_u32 s93, s92, s59\n"
        "s_cmp_lt_u32 s93, s89\n"
        "s_cbranch_scc1 L_pace4%=\n"
        "s_mov_b32 s59, s92\n"
        "L_pc4%=:\n"
        "s_waitcnt lgkmcnt(0)\n"
        "s_barrier\n"
        "v_mfma_f32_16x16x32_f16 v[12:15], v[144:147], v[236:239], v[12:15]\n"
        "ds_read_b128 v[232:235], %[brd] offset:47104\n"
        "v_mfma_f32_16x16x32_f16 v[44:47], v[148:151], v[236:239], v[44:47]\n"
        "v_mov_b32 v247, 147464\n"
        "ds_read_b32 v240, v247\n"
        "v_mfma_f32_16x16x32_f16 v[76:79], v[152:155], v[236:239], v[76:79]\n"
        "v_mfma_f32_16x16x32_f16 v[108:111], v[156:159], v[236:239], v[108:111]\n"
        "v_mfma_f32_16x16x32_f16 v[16:19], v[144:147], v[224:227], v[16:19]\n"
        "ds_read_b128 v[236:239], %[brd] offset:48128\n"
        "v_mfma_f32_16x16x32_f16 v[48:51], v[148:151], v[224:227], v[48:51]\n"
        "v_mfma_f32_16x16x32_f16 v[80:83], v[152:155], v[224:227], v[80:83]\n"
        "v_mfma_f32_16x16x32_f16 v[112:115], v[156:159], v[224:227], v[112:115]\n"
        "v_mfma_f32_16x16x32_f16 v[20:23], v[144:147], v[228:231], v[20:23]\n"
        "ds_read_b128 v[224:227], %[brd] offset:49152\n"
        "v_mfma_f32_16x16x32_f16 v[52:55], v[148:151], v[228:231], v[52:55]\n"
        "v_mfma_f32_16x16x32_f16 v[84:87], v[152:155], v[228:231], v[84:87]\n"
        "v_mfma_f32_16x16x32_f16 v[116:119], v[156:159], v[228:231], v[116:119]\n"
        "s_waitcnt lgkmcnt(3)\n"
        "v_mfma_f32_16x16x32_f16 v[24:27], v[144:147], v[232:235], v[24:27]\n"
        "ds_read_b128 v[228:231], %[brd] offset:50176\n"
        "v_mfma_f32_16x16x32_f16 v[56:59], v[148:151], v[232:235], v[56:59]\n"
        "s_waitcnt lgkmcnt(3)\n"
        "v_readfirstlane_b32 s74, v240\n"
        "v_mfma_f32_16x16x32_f16 v[88:91], v[152:155], v[232:235], v[88:91]\n"
        "v_mfma_f32_16x16x32_f16 v[120:123], v[156:159], v[232:235], v[120:123]\n"
        "s_waitcnt lgkmcnt(2)\n"
        "v_mfma_f32_16x16x32_f16 v[28:31], v[144:147], v[236:239], v[28:31]\n"
        "ds_read_b128 v[232:235], %[brd] offset:51200\n"
        "v_mfma_f32_16x16x32_f16 v[60:63], v[148:151], v[236:239], v[60:63]\n"
        "v_mfma_f32_16x16x32_f16 v[92:95], v[152:155], v[236:239], v[92:95]\n"
        "v_mfma_f32_16x16x32_f16 v[124:127], v[156:159], v[236:239], v[124:127]\n"
        "s_barrier\n"
        "s_lshl_b32 s82, s79, 23\n"
        "s_add_u32 s82, s82, 262144\n"
        "buffer_load_dwordx4 v[200:203], %[wvoff], %[rsW], s82 offen nt\n"
        "s_lshl_b32 s82, s79, 23\n"
        "s_add_u32 s82, s82, 393216\n"
        "buffer_load_dwordx4 v[204:207], %[wvoff], %[rsW], s82 offen nt\n"
        "s_lshl_b32 s82, s72, 23\n"
        "buffer_load_dwordx4 v[208:211], %[wvoff], %[rsW], s82 offen nt\n"
        "s_waitcnt vmcnt(13) lgkmcnt(2)\n"
        "v_mfma_f32_16x16x32_f16 v[0:3], v[160:163], v[224:227], v[0:3]\n"
        "ds_read_b128 v[236:239], %[brd] offset:52224\n"
        "v_mfma_f32_16x16x32_f16 v[32:35], v[164:167], v[224:227], v[32:35]\n"
        "v_mfma_f32_16x16x32_f16 v[64:67], v[168:171], v[224:227], v[64:67]\n"
        "v_mfma_f32_16x16x32_f16 v[96:99], v[172:175], v[224:227], v[96:99]\n"
        "s_waitcnt lgkmcnt(2)\n"
        "v_mfma_f32_16x16x32_f16 v[4:7], v[160:163], v[228:231], v[4:7]\n"
        "ds_read_b128 v[224:227], %[brd] offset:53248\n"
        "v_mfma_f32_16x16x32_f16 v[36:39], v[164:167], v[228:231], v[36:39]\n"
        "v_mfma_f32_16x16x32_f16 v[68:71], v[168:171], v[228:231], v[68:71]\n"
        "v_mfma_f32_16x16x32_f16 v[100:103], v[172:175], v[228:231], v[100:103]\n"
        "s_waitcnt lgkmcnt(2)\n"
        "v_mfma_f32_16x16x32_f16 v[8:11], v[160:163], v[232:235], v[8:11]\n"
        "ds_read_b128 v[228:231], %[brd] offset:54272\n"
        "v_mfma_f32_16x16x32_f16 v[40:43], v[164:167], v[232:235], v[40:43]\n"
        "v_mfma_f32_16x16x32_f16 v[72:75], v[168:171], v[232:235], v[72:75]\n"
        "v_mfma_f32_16x16x32_f16 v[104:107], v[172:175], v[232:235], v[104:107]\n"
        "s_waitcnt lgkmcnt(2)\n"
        "v_mfma_f32_16x16x32_f16 v[12:15], v[160:163], v[236:239], v[12:15]\n"
        "ds_read_b128 v[232:235], %[brd] offset:55296\n"
        "v_mfma_f32_16x16x32_f16 v[44:47], v[164:167], v[236:239], v[44:47]\n"
        "v_mfma_f32_16x16x32_f16 v[76:79], v[168:171], v[236:239], v[76:79]\n"
        "v_mfma_f32_16x16x32_f16 v[108:111], v[172:175], v[236:239], v[108:111]\n"
        "s_waitcnt lgkmcnt(2)\n"
        "v_mfma_f32_16x16x32_f16 v[16:19], v[160:163], v[224:227], v[16:19]\n"
        "ds_read_b128 v[236:239], %[brd] offset:56320\n"
        "v_mfma_f32_16x16x32_f16 v[48:51], v[164:167], v[224:227], v[48:51]\n"
        "v_mfma_f32_16x16x32_f16 v[80:83], v[168:171], v[224:227], v[80:83]\n"
        "v_mfma_f32_16x16x32_f16 v[112:115], v[172:175], v[224:227], v[112:115]\n"
        "s_lshl_b32 s81, s76, 16\n"
        "s_add_u32 s81, s81, s62\n"
        "s_add_u32 s81, s81, 4096\n"
        "s_add_u32 s87, s81, 2048\n"
        "buffer_load_dwordx4 v[144:147], %[avoff], %[rsA], s81 offen offset:0\n"
        "buffer_load_dwordx4 v[148:151], %[avoff], %[rsA], s81 offen offset:1024\n"
        "buffer_load_dwordx4 v[152:155], %[avoff], %[rsA], s87 offen offset:0\n"
        "buffer_load_dwordx4 v[156:159], %[avoff], %[rsA], s87 offen offset:1024\n"
        "s_cmp_eq_u32 %[wave], 4\n"
        "s_cbranch_scc1 L_chkk5%=\n"
        "L_chkk5r%=:\n"
        "s_cmp_lg_u32 %[wave], 5\n"
        "s_cbranch_scc1 L_pc5%=\n"
        "s_sub_i32 s89, s58, 16\n"
        "s_max_i32 s89, s89, 0\n"
        "s_mul_i32 s89, s89, 4\n"
        "s_sub_i32 s89, 152, s89\n"
        "s_max_i32 s89, s89, s50\n"
        "s_cmp_lt_u32 s58, 4\n"
        "s_cselect_b32 s89, 100, s89\n"
        "s_add_u32 s58, s58, 1\n"
        "L_pace5%=:\n"
        "s_memrealtime s[92:93]\n"
        "s_waitcnt lgkmcnt(0)\n"
        "s_sub_u32 s93, s92, s59\n"
        "s_cmp_lt_u32 s93, s89\n"
        "s_cbranch_scc1 L_pace5%=\n"
        "s_mov_b32 s59, s92\n"
        "L_pc5%=:\n"
        "s_barrier\n"
        "s_waitcnt lgkmcnt(2)\n"
        "v_mfma_f32_16x16x32_f16 v[20:23], v[160:163], v[228:231], v[20:23]\n"
        "ds_read_b128 v[224:227], %[brd] offset:57344\n"
        "v_mfma_f32_16x16x32_f16 v[52:55], v[164:167], v[228:231], v[52:55]\n"
        "v_mfma_f32_16x16x32_f16 v[84:87], v[168:171], v[228:231], v[84:87]\n"
        "v_mfma_f32_16x16x32_f16 v[116:119], v[172:175], v[228:231], v[116:119]\n"
        "s_waitcnt lgkmcnt(2)\n"
        "v_mfma_f32_16x16x32_f16 v[24:27], v[160:163], v[232:235], v[24:27]\n"
        "ds_read_b128 v[228:231], %[brd] offset:58368\n"
        "v_mfma_f32_16x16x32_f16 v[56:59], v[164:167], v[232:235], v[56:59]\n"
        "v_mfma_f32_16x16x32_f16 v[88:91], v[168:171], v[232:235], v[88:91]\n"
        "v_mfma_f32_16x16x32_f16 v[120:123], v[172:175], v[232:235], v[120:123]\n"
        "s_waitcnt lgkmcnt(2)\n"
        "v_mfma_f32_16x16x32_f16 v[28:31], v[160:163], v[236:239], v[28:31]\n"
        "ds_read_b128 v[232:235], %[brd] offset:59392\n"
        "v_mfma_f32_16x16x32_f16 v[60:63], v[164:167], v[236:239], v[60:63]\n"
        "v_mfma_f32_16x16x32_f16 v[92:95], v[168:171], v[236:239], v[92:95]\n"
        "v_mfma_f32_16x16x32_f16 v[124:127], v[172:175], v[236:239], v[124:127]\n"
        "s_waitcnt vmcnt(11) lgkmcnt(2)\n"
        "v_mfma_f32_16x16x32_f16 v[0:3], v[176:179], v[224:227], v[0:3]\n"
        "ds_read_b128 v[236:239], %[brd] offset:60416\n"
        "v_mfma_f32_16x16x32_f16 v[32:35], v[180:183], v[224:227], v[32:35]\n"
        "v_mfma_f32_16x16x32_f16 v[64:67], v[184:187], v[224:227], v[64:67]\n"
        "v_mfma_f32_16x16x32_f16 v[96:99], v[188:191], v[224:227], v[96:99]\n"
        "s_waitcnt lgkmcnt(2)\n"
        "v_mfma_f32_16x16x32_f16 v[4:7], v[176:179], v[228:231], v[4:7]\n"
        "ds_read_b128 v[224:227], %[brd] offset:61440\n"
        "v_mfma_f32_16x16x32_f16 v[36:39], v[180:183], v[228:231], v[36:39]\n"
        "v_mov_b32 v247, 147468\n"
        "ds_read_b32 v240, v247\n"
        "v_mfma_f32_16x16x32_f16 v[68:71], v[184:187], v[228:231], v[68:71]\n"
        "v_mfma_f32_16x16x32_f16 v[100:103], v[188:191], v[228:231], v[100:103]\n"
        "s_waitcnt lgkmcnt(3)\n"
        "v_mfma_f32_16x16x32_f16 v[8:11], v[176:179], v[232:235], v[8:11]\n"
        "ds_read_b128 v[228:231], %[brd] offset:62464\n"
        "v_mfma_f32_16x16x32_f16 v[40:43], v[180:183], v[232:235], v[40:43]\n"
        "v_mfma_f32_16x16x32_f16 v[72:75], v[184:187], v[232:235], v[72:75]\n"
        "v_mfma_f32_16x16x32_f16 v[104:107], v[188:191], v[232:235], v[104:107]\n"
        "s_barrier\n"
        "s_lshl_b32 s82, s72, 23\n"
        "s_add_u32 s82, s82, 131072\n"
        "buffer_load_dwordx4 v[212:215], %[wvoff], %[rsW], s82 offen nt\n"
        "s_lshl_b32 s82, s72, 23\n"
        "s_add_u32 s82, s82, 262144\n"
        "buffer_load_dwordx4 v[216:219], %[wvoff], %[rsW], s82 offen nt\n"
        "s_lshl_b32 s82, s72, 23\n"
        "s_add_u32 s82, s82, 393216\n"
        "buffer_load_dwordx4 v[220:223], %[wvoff], %[rsW], s82 offen nt\n"
        "s_waitcnt lgkmcnt(3)\n"
        "v_mfma_f32_16x16x32_f16 v[12:15], v[176:179], v[236:239], v[12:15]\n"
        "ds_read_b128 v[232:235], %[brd] offset:63488\n"
        "v_mfma_f32_16x16x32_f16 v[44:47], v[180:183], v[236:239], v[44:47]\n"
        "v_mfma_f32_16x16x32_f16 v[76:79], v[184:187], v[236:239], v[76:79]\n"
        "v_mfma_f32_16x16x32_f16 v[108:111], v[188:191], v[236:239], v[108:111]\n"
        "s_waitcnt lgkmcnt(3)\n"
        "v_mfma_f32_16x16x32_f16 v[16:19], v[176:179], v[224:227], v[16:19]\n"
        "ds_read_b128 v[236:239], %[brd] offset:64512\n"
        "v_mfma_f32_16x16x32_f16 v[48:51], v[180:183], v[224:227], v[48:51]\n"
        "s_waitcnt lgkmcnt(3)\n"
        "v_readfirstlane_b32 s75, v240\n"
        "v_mfma_f32_16x16x32_f16 v[80:83], v[184:187], v[224:227], v[80:83]\n"
        "v_mfma_f32_16x16x32_f16 v[112:115], v[188:191], v[224:227], v[112:115]\n"
        "s_waitcnt lgkmcnt(2)\n"
        "v_mfma_f32_16x16x32_f16 v[20:23], v[176:179], v[228:231], v[20:23]\n"
        "ds_read_b128 v[224:227], v244 offset:0\n"
        "v_mfma_f32_16x16x32_f16 v[52:55], v[180:183], v[228:231], v[52:55]\n"
        "v_mfma_f32_16x16x32_f16 v[84:87], v[184:187], v[228:231], v[84:87]\n"
        "v_mfma_f32_16x16x32_f16 v[116:119], v[188:191], v[228:231], v[116:119]\n"
        "s_waitcnt lgkmcnt(2)\n"
        "v_mfma_f32_16x16x32_f16 v[24:27], v[176:179], v[232:235], v[24:27]\n"
        "ds_read_b128 v[228:231], v244 offset:1024\n"
        "v_mfma_f32_16x16x32_f16 v[56:59], v[180:183], v[232:235], v[56:59]\n"
        "v_mfma_f32_16x16x32_f16 v[88:91], v[184:187], v[232:235], v[88:91]\n"
        "v_mfma_f32_16x16x32_f16 v[120:123], v[188:191], v[232:235], v[120:123]\n"
        "s_waitcnt lgkmcnt(2)\n"
        "v_mfma_f32_16x16x32_f16 v[28:31], v[176:179], v[236:239], v[28:31]\n"
        "ds_read_b128 v[232:235], v244 offset:2048\n"
        "v_mfma_f32_16x16x32_f16 v[60:63], v[180:183], v[236:239], v[60:63]\n"
        "v_mfma_f32_16x16x32_f16 v[92:95], v[184:187], v[236:239], v[92:95]\n"
        "v_mfma_f32_16x16x32_f16 v[124:127], v[188:191], v[236:239], v[124:127]\n"
        "s_waitcnt vmcnt(8)\n"
        "v_cvt_pk_f16_f32 v240, v192, v196\n"
        "v_cvt_pk_f16_f32 v241, v200, v204\n"
        "ds_write_b64 v245, v[240:241] offset:49152\n"
        "v_cvt_pk_f16_f32 v242, v193, v197\n"
        "v_cvt_pk_f16_f32 v243, v201, v205\n"
        "ds_write_b64 v245, v[242:243] offset:50176\n"
        "v_cvt_pk_f16_f32 v240, v194, v198\n"
        "v_cvt_pk_f16_f32 v241, v202, v206\n"
        "ds_write_b64 v245, v[240:241] offset:51200\n"
        "v_cvt_pk_f16_f32 v242, v195, v199\n"
        "v_cvt_pk_f16_f32 v243, v203, v207\n"
        "ds_write_b64 v245, v[242:243] offset:52224\n"
        "s_lshl_b32 s80, s77, 16\n"
        "s_add_u32 s80, s80, s62\n"
        "s_add_u32 s86, s80, 2048\n"
        "buffer_load_dwordx4 v[160:163], %[avoff], %[rsA], s80 offen offset:0\n"
        "buffer_load_dwordx4 v[164:167], %[avoff], %[rsA], s80 offen offset:1024\n"
        "buffer_load_dwordx4 v[168:171], %[avoff], %[rsA], s86 offen offset:0\n"
        "buffer_load_dwordx4 v[172:175], %[avoff], %[rsA], s86 offen offset:1024\n"
        "s_lshl_b32 s81, s77, 16\n"
        "s_add_u32 s81, s81, s62\n"
        "s_add_u32 s81, s81, 4096\n"
        "s_add_u32 s87, s81, 2048\n"
        "buffer_load_dwordx4 v[176:179], %[avoff], %[rsA], s81 offen offset:0\n"
        "buffer_load_dwordx4 v[180:183], %[avoff], %[rsA], s81 offen offset:1024\n"
        "s_cmp_eq_u32 %[wave], 4\n"
        "s_cbranch_scc1 L_chkk6%=\n"
        "L_chkk6r%=:\n"
        "s_cmp_lg_u32 %[wave], 5\n"
        "s_cbranch_scc1 L_pc6%=\n"
        "s_sub_i32 s89, s58, 16\n"
        "s_max_i32 s89, s89, 0\n"
        "s_mul_i32 s89, s89, 4\n"
        "s_sub_i32 s89, 152, s89\n"
        "s_max_i32 s89, s89, s50\n"
        "s_cmp_lt_u32 s58, 4\n"
        "s_cselect_b32 s89, 100, s89\n"
        "s_add_u32 s58, s58, 1\n"
        "L_pace6%=:\n"
        "s_memrealtime s[92:93]\n"
        "s_waitcnt lgkmcnt(0)\n"
        "s_sub_u32 s93, s92, s59\n"
        "s_cmp_lt_u32 s93, s89\n"
        "s_cbranch_scc1 L_pace6%=\n"
        "s_mov_b32 s59, s92\n"
        "L_pc6%=:\n"
        "s_waitcnt lgkmcnt(0)\n"
        "s_barrier\n"
        "v_mfma_f32_16x16x32_f16 v[0:3], v[128:131], v[224:227], v[0:3]\n"
        "ds_read_b128 v[236:239], v244 offset:3072\n"
        "v_mfma_f32_16x16x32_f16 v[32:35], v[132:135], v[224:227], v[32:35]\n"
        "v_mfma_f32_16x16x32_f16 v[64:67], v[136:139], v[224:227], v[64:67]\n"
        "v_mfma_f32_16x16x32_f16 v[96:99], v[140:143], v[224:227], v[96:99]\n"
        "v_mfma_f32_16x16x32_f16 v[4:7], v[128:131], v[228:231], v[4:7]\n"
        "ds_read_b128 v[224:227], v244 offset:4096\n"
        "v_mfma_f32_16x16x32_f16 v[36:39], v[132:135], v[228:231], v[36:39]\n"
        "v_mfma_f32_16x16x32_f16 v[68:71], v[136:139], v[228:231], v[68:71]\n"
        "v_mfma_f32_16x16x32_f16 v[100:103], v[140:143], v[228:231], v[100:103]\n"
        "v_mfma_f32_16x16x32_f16 v[8:11], v[128:131], v[232:235], v[8:11]\n"
        "ds_read_b128 v[228:231], v244 offset:5120\n"
        "v_mfma_f32_16x16x32_f16 v[40:43], v[132:135], v[232:235], v[40:43]\n"
        "v_mfma_f32_16x16x32_f16 v[72:75], v[136:139], v[232:235], v[72:75]\n"
        "v_mfma_f32_16x16x32_f16 v[104:107], v[140:143], v[232:235], v[104:107]\n"
        "s_waitcnt lgkmcnt(2)\n"
        "v_mfma_f32_16x16x32_f16 v[12:15], v[128:131], v[236:239], v[12:15]\n"
        "ds_read_b128 v[232:235], v244 offset:6144\n"
        "v_mfma_f32_16x16x32_f16 v[44:47], v[132:135], v[236:239], v[44:47]\n"
        "v_mfma_f32_16x16x32_f16 v[76:79], v[136:139], v[236:239], v[76:79]\n"
        "v_mfma_f32_16x16x32_f16 v[108:111], v[140:143], v[236:239], v[108:111]\n"
        "s_waitcnt lgkmcnt(2)\n"
        "v_mfma_f32_16x16x32_f16 v[16:19], v[128:131], v[224:227], v[16:19]\n"
        "ds_read_b128 v[236:239], v244 offset:7168\n"
        "v_mfma_f32_16x16x32_f16 v[48:51], v[132:135], v[224:227], v[48:51]\n"
        "v_mfma_f32_16x16x32_f16 v[80:83], v[136:139], v[224:227], v[80:83]\n"
        "v_mfma_f32_16x16x32_f16 v[112:115], v[140:143], v[224:227], v[112:115]\n"
        "s_waitcnt lgkmcnt(2)\n"
        "v_mfma_f32_16x16x32_f16 v[20:23], v[128:131], v[228:231], v[20:23]\n"
        "ds_read_b128 v[224:227], v244 offset:8192\n"
        "v_mfma_f32_16x16x32_f16 v[52:55], v[132:135], v[228:231], v[52:55]\n"
        "v_mfma_f32_16x16x32_f16 v[84:87], v[136:139], v[228:231], v[84:87]\n"
        "v_mfma_f32_16x16x32_f16 v[116:119], v[140:143], v[228:231], v[116:119]\n"
        "s_barrier\n"
        "s_cmp_eq_u32 s66, 4\n"
        "s_cbranch_scc1 L_flagcY2%=\n"
        "L_flagretcY2%=:\n"
        "s_cmp_eq_u32 s65, 4\n"
        "s_cbranch_scc1 L_pushcY2%=\n"
        "L_pushretcY2%=:\n"
        "s_cmp_eq_u32 %[wave], 4\n"
        "s_cbranch_scc1 L_ctlcY2%=\n"
        "L_ctlcY2r%=:\n"
        "s_lshl_b32 s82, s73, 23\n"
        "buffer_load_dwordx4 v[192:195], %[wvoff], %[rsW], s82 offen nt\n"
        "s_lshl_b32 s82, s73, 23\n"
        "s_add_u32 s82, s82, 131072\n"
        "buffer_load_dwordx4 v[196:199], %[wvoff], %[rsW], s82 offen nt\n"
        "s_waitcnt lgkmcnt(2)\n"
        "v_mfma_f32_16x16x32_f16 v[24:27], v[128:131], v[232:235], v[24:27]\n"
        "ds_read_b128 v[228:231], v244 offset:9216\n"
        "v_mfma_f32_16x16x32_f16 v[56:59], v[132:135], v[232:235], v[56:59]\n"
        "v_mfma_f32_16x16x32_f16 v[88:91], v[136:139], v[232:235], v[88:91]\n"
        "v_mfma_f32_16x16x32_f16 v[120:123], v[140:143], v[232:235], v[120:123]\n"
        "s_waitcnt lgkmcnt(2)\n"
        "v_mfma_f32_16x16x32_f16 v[28:31], v[128:131], v[236:239], v[28:31]\n"
        "ds_read_b128 v[232:235], v244 offset:10240\n"
        "v_mfma_f32_16x16x32_f16 v[60:63], v[132:135], v[236:239], v[60:63]\n"
        "v_mfma_f32_16x16x32_f16 v[92:95], v[136:139], v[236:239], v[92:95]\n"
        "v_mfma_f32_16x16x32_f16 v[124:127], v[140:143], v[236:239], v[124:127]\n"
        "s_waitcnt vmcnt(11) lgkmcnt(2)\n"
        "v_mfma_f32_16x16x32_f16 v[0:3], v[144:147], v[224:227], v[0:3]\n"
        "ds_read_b128 v[236:239], v244 offset:11264\n"
        "v_mfma_f32_16x16x32_f16 v[32:35], v[148:151], v[224:227], v[32:35]\n"
        "v_mfma_f32_16x16x32_f16 v[64:67], v[152:155], v[224:227], v[64:67]\n"
        "v_mfma_f32_16x16x32_f16 v[96:99], v[156:159], v[224:227], v[96:99]\n"
        "s_waitcnt lgkmcnt(2)\n"
        "v_mfma_f32_16x16x32_f16 v[4:7], v[144:147], v[228:231], v[4:7]\n"
        "ds_read_b128 v[224:227], v244 offset:12288\n"
        "v_mfma_f32_16x16x32_f16 v[36:39], v[148:151], v[228:231], v[36:39]\n"
        "v_mfma_f32_16x16x32_f16 v[68:71], v[152:155], v[228:231], v[68:71]\n"
        "v_mfma_f32_16x16x32_f16 v[100:103], v[156:159], v[228:231], v[100:103]\n"
        "s_waitcnt lgkmcnt(2)\n"
        "v_mfma_f32_16x16x32_f16 v[8:11], v[144:147], v[232:235], v[8:11]\n"
        "ds_read_b128 v[228:231], v244 offset:13312\n"
        "v_mfma_f32_16x16x32_f16 v[40:43], v[148:151], v[232:235], v[40:43]\n"
        "v_mfma_f32_16x16x32_f16 v[72:75], v[152:155], v[232:235], v[72:75]\n"
        "v_mfma_f32_16x16x32_f16 v[104:107], v[156:159], v[232:235], v[104:107]\n"
        "s_waitcnt vmcnt(8)\n"
        "v_cvt_pk_f16_f32 v240, v208, v212\n"
        "v_cvt_pk_f16_f32 v241, v216, v220\n"
        "ds_write_b64 %[bwr], v[240:241] offset:0\n"
        "v_cvt_pk_f16_f32 v242, v209, v213\n"
        "v_cvt_pk_f16_f32 v243, v217, v221\n"
        "ds_write_b64 %[bwr], v[242:243] offset:1024\n"
        "v_cvt_pk_f16_f32 v240, v210, v214\n"
        "v_cvt_pk_f16_f32 v241, v218, v222\n"
        "ds_write_b64 %[bwr], v[240:241] offset:2048\n"
        "v_cvt_pk_f16_f32 v242, v211, v215\n"
        "v_cvt_pk_f16_f32 v243, v219, v223\n"
        "ds_write_b64 %[bwr], v[242:243] offset:3072\n"
        "buffer_load_dwordx4 v[184:187], %[avoff], %[rsA], s87 offen offset:0\n"
        "buffer_load_dwordx4 v[188:191], %[avoff], %[rsA], s87 offen offset:1024\n"
        "s_lshl_b32 s80, s78, 16\n"
        "s_add_u32 s80, s80, s62\n"
        "s_add_u32 s86, s80, 2048\n"
        "buffer_load_dwordx4 v[128:131], %[avoff], %[rsA], s80 offen offset:0\n"
        "buffer_load_dwordx4 v[132:135], %[avoff], %[rsA], s80 offen offset:1024\n"
        "buffer_load_dwordx4 v[136:139], %[avoff], %[rsA], s86 offen offset:0\n"
        "buffer_load_dwordx4 v[140:143], %[avoff], %[rsA], s86 offen offset:1024\n"
        "s_cmp_lg_u32 %[wave], 5\n"
        "s_cbranch_scc1 L_pc7%=\n"
        "s_sub_i32 s89, s58, 16\n"
        "s_max_i32 s89, s89, 0\n"
        "s_mul_i32 s89, s89, 4\n"
        "s_sub_i32 s89, 152, s89\n"
        "s_max_i32 s89, s89, s50\n"
        "s_cmp_lt_u32 s58, 4\n"
        "s_cselect_b32 s89, 100, s89\n"
        "s_add_u32 s58, s58, 1\n"
        "L_pace7%=:\n"
        "s_memrealtime s[92:93]\n"
        "s_waitcnt lgkmcnt(0)\n"
        "s_sub_u32 s93, s92, s59\n"
        "s_cmp_lt_u32 s93, s89\n"
        "s_cbranch_scc1 L_pace7%=\n"
        "s_mov_b32 s59, s92\n"
        "L_pc7%=:\n"
        "s_waitcnt lgkmcnt(0)\n"
        "s_barrier\n"
        "v_mfma_f32_16x16x32_f16 v[12:15], v[144:147], v[236:239], v[12:15]\n"
        "ds_read_b128 v[232:235], v244 offset:14336\n"
        "v_mfma_f32_16x16x32_f16 v[44:47], v[148:151], v[236:239], v[44:47]\n"
        "v_mov_b32 v247, 147472\n"
        "ds_read_b32 v240, v247\n"
        "v_mfma_f32_16x16x32_f16 v[76:79], v[152:155], v[236:239], v[76:79]\n"
        "v_mfma_f32_16x16x32_f16 v[108:111], v[156:159], v[236:239], v[108:111]\n"
        "v_mfma_f32_16x16x32_f16 v[16:19], v[144:147], v[224:227], v[16:19]\n"
        "ds_read_b128 v[236:239], v244 offset:15360\n"
        "v_mfma_f32_16x16x32_f16 v[48:51], v[148:151], v[224:227], v[48:51]\n"
        "v_mfma_f32_16x16x32_f16 v[80:83], v[152:155], v[224:227], v[80:83]\n"
        "v_mfma_f32_16x16x32_f16 v[112:115], v[156:159], v[224:227], v[112:115]\n"
        "v_mfma_f32_16x16x32_f16 v[20:23], v[144:147], v[228:231], v[20:23]\n"
        "ds_read_b128 v[224:227], v244 offset:16384\n"
        "v_mfma_f32_16x16x32_f16 v[52:55], v[148:151], v[228:231], v[52:55]\n"
        "v_mfma_f32_16x16x32_f16 v[84:87], v[152:155], v[228:231], v[84:87]\n"
        "v_mfma_f32_16x16x32_f16 v[116:119], v[156:159], v[228:231], v[116:119]\n"
        "s_waitcnt lgkmcnt(3)\n"
        "v_mfma_f32_16x16x32_f16 v[24:27], v[144:147], v[232:235], v[24:27]\n"
        "ds_read_b128 v[228:231], v244 offset:17408\n"
        "v_mfma_f32_16x16x32_f16 v[56:59], v[148:151], v[232:235], v[56:59]\n"
        "s_waitcnt lgkmcnt(3)\n"
        "v_readfirstlane_b32 s76, v240\n"
        "v_mfma_f32_16x16x32_f16 v[88:91], v[152:155], v[232:235], v[88:91]\n"
        "v_mfma_f32_16x16x32_f16 v[120:123], v[156:159], v[232:235], v[120:123]\n"
        "s_waitcnt lgkmcnt(2)\n"
        "v_mfma_f32_16x16x32_f16 v[28:31], v[144:147], v[236:239], v[28:31]\n"
        "ds_read_b128 v[232:235], v244 offset:18432\n"
        "v_mfma_f32_16x16x32_f16 v[60:63], v[148:151], v[236:239], v[60:63]\n"
        "v_mfma_f32_16x16x32_f16 v[92:95], v[152:155], v[236:239], v[92:95]\n"
        "v_mfma_f32_16x16x32_f16 v[124:127], v[156:159], v[236:239], v[124:127]\n"
        "s_barrier\n"
        "s_lshl_b32 s82, s73, 23\n"
        "s_add_u32 s82, s82, 262144\n"
        "buffer_load_dwordx4 v[200:203], %[wvoff], %[rsW], s82 offen nt\n"
        "s_lshl_b32 s82, s73, 23\n"
        "s_add_u32 s82, s82, 393216\n"
        "buffer_load_dwordx4 v[204:207], %[wvoff], %[rsW], s82 offen nt\n"
        "s_lshl_b32 s82, s74, 23\n"
        "buffer_load_dwordx4 v[208:211], %[wvoff], %[rsW], s82 offen nt\n"
        "s_waitcnt vmcnt(13) lgkmcnt(2)\n"
        "v_mfma_f32_16x16x32_f16 v[0:3], v[160:163], v[224:227], v[0:3]\n"
        "ds_read_b128 v[236:239], v244 offset:19456\n"
        "v_mfma_f32_16x16x32_f16 v[32:35], v[164:167], v[224:227], v[32:35]\n"
        "v_mfma_f32_16x16x32_f16 v[64:67], v[168:171], v[224:227], v[64:67]\n"
        "v_mfma_f32_16x16x32_f16 v[96:99], v[172:175], v[224:227], v[96:99]\n"
        "s_waitcnt lgkmcnt(2)\n"
        "v_mfma_f32_16x16x32_f16 v[4:7], v[160:163], v[228:231], v[4:7]\n"
        "ds_read_b128 v[224:227], v244 offset:20480\n"
        "v_mfma_f32_16x16x32_f16 v[36:39], v[164:167], v[228:231], v[36:39]\n"
        "v_mfma_f32_16x16x32_f16 v[68:71], v[168:171], v[228:231], v[68:71]\n"
        "v_mfma_f32_16x16x32_f16 v[100:103], v[172:175], v[228:231], v[100:103]\n"
        "s_waitcnt lgkmcnt(2)\n"
        "v_mfma_f32_16x16x32_f16 v[8:11], v[160:163], v[232:235], v[8:11]\n"
        "ds_read_b128 v[228:231], v244 offset:21504\n"
        "v_mfma_f32_16x16x32_f16 v[40:43], v[164:167], v[232:235], v[40:43]\n"
        "v_mfma_f32_16x16x32_f16 v[72:75], v[168:171], v[232:235], v[72:75]\n"
        "v_mfma_f32_16x16x32_f16 v[104:107], v[172:175], v[232:235], v[104:107]\n"
        "s_waitcnt lgkmcnt(2)\n"
        "v_mfma_f32_16x16x32_f16 v[12:15], v[160:163], v[236:239], v[12:15]\n"
        "ds_read_b128 v[232:235], v244 offset:22528\n"
        "v_mfma_f32_16x16x32_f16 v[44:47], v[164:167], v[236:239], v[44:47]\n"
        "v_mfma_f32_16x16x32_f16 v[76:79], v[168:171], v[236:239], v[76:79]\n"
        "v_mfma_f32_16x16x32_f16 v[108:111], v[172:175], v[236:239], v[108:111]\n"
        "s_waitcnt lgkmcnt(2)\n"
        "v_mfma_f32_16x16x32_f16 v[16:19], v[160:163], v[224:227], v[16:19]\n"
        "ds_read_b128 v[236:239], v244 offset:23552\n"
        "v_mfma_f32_16x16x32_f16 v[48:51], v[164:167], v[224:227], v[48:51]\n"
        "v_mfma_f32_16x16x32_f16 v[80:83], v[168:171], v[224:227], v[80:83]\n"
        "v_mfma_f32_16x16x32_f16 v[112:115], v[172:175], v[224:227], v[112:115]\n"
        "s_lshl_b32 s81, s78, 16\n"
        "s_add_u32 s81, s81, s62\n"
        "s_add_u32 s81, s81, 4096\n"
        "s_add_u32 s87, s81, 2048\n"
        "buffer_load_dwordx4 v[144:147], %[avoff], %[rsA], s81 offen offset:0\n"
        "buffer_load_dwordx4 v[148:151], %[avoff], %[rsA], s81 offen offset:1024\n"
        "buffer_load_dwordx4 v[152:155], %[avoff], %[rsA], s87 offen offset:0\n"
        "buffer_load_dwordx4 v[156:159], %[avoff], %[rsA], s87 offen offset:1024\n"
        "s_cmp_eq_u32 %[wave], 4\n"
        "s_cbranch_scc1 L_chkk7%=\n"
        "L_chkk7r%=:\n"
        "s_cmp_lg_u32 %[wave], 5\n"
        "s_cbranch_scc1 L_pc8%=\n"
        "s_sub_i32 s89, s58, 16\n"
        "s_max_i32 s89, s89, 0\n"
        "s_mul_i32 s89, s89, 4\n"
        "s_sub_i32 s89, 152, s89\n"
        "s_max_i32 s89, s89, s50\n"
        "s_cmp_lt_u32 s58, 4\n"
        "s_cselect_b32 s89, 100, s89\n"
        "s_add_u32 s58, s58, 1\n"
        "L_pace8%=:\n"
        "s_memrealtime s[92:93]\n"
        "s_waitcnt lgkmcnt(0)\n"
        "s_sub_u32 s93, s92, s59\n"
        "s_cmp_lt_u32 s93, s89\n"
        "s_cbranch_scc1 L_pace8%=\n"
        "s_mov_b32 s59, s92\n"
        "L_pc8%=:\n"
        "s_barrier\n"
        "s_waitcnt lgkmcnt(2)\n"
        "v_mfma_f32_16x16x32_f16 v[20:23], v[160:163], v[228:231], v[20:23]\n"
        "ds_read_b128 v[224:227], v244 offset:24576\n"
        "v_mfma_f32_16x16x32_f16 v[52:55], v[164:167], v[228:231], v[52:55]\n"
        "v_mfma_f32_16x16x32_f16 v[84:87], v[168:171], v[228:231], v[84:87]\n"
        "v_mfma_f32_16x16x32_f16 v[116:119], v[172:175], v[228:231], v[116:119]\n"
        "s_waitcnt lgkmcnt(2)\n"
        "v_mfma_f32_16x16x32_f16 v[24:27], v[160:163], v[232:235], v[24:27]\n"
        "ds_read_b128 v[228:231], v244 offset:25600\n"
        "v_mfma_f32_16x16x32_f16 v[56:59], v[164:167], v[232:235], v[56:59]\n"
        "v_mfma_f32_16x16x32_f16 v[88:91], v[168:171], v[232:235], v[88:91]\n"
        "v_mfma_f32_16x16x32_f16 v[120:123], v[172:175], v[232:235], v[120:123]\n"
        "s_waitcnt lgkmcnt(2)\n"
        "v_mfma_f32_16x16x32_f16 v[28:31], v[160:163], v[236:239], v[28:31]\n"
        "ds_read_b128 v[232:235], v244 offset:26624\n"
        "v_mfma_f32_16x16x32_f16 v[60:63], v[164:167], v[236:239], v[60:63]\n"
        "v_mfma_f32_16x16x32_f16 v[92:95], v[168:171], v[236:239], v[92:95]\n"
        "v_mfma_f32_16x16x32_f16 v[124:127], v[172:175], v[236:239], v[124:127]\n"
        "s_waitcnt vmcnt(11) lgkmcnt(2)\n"
        "v_mfma_f32_16x16x32_f16 v[0:3], v[176:179], v[224:227], v[0:3]\n"
        "ds_read_b128 v[236:239], v244 offset:27648\n"
        "v_mfma_f32_16x16x32_f16 v[32:35], v[180:183], v[224:227], v[32:35]\n"
        "v_mfma_f32_16x16x32_f16 v[64:67], v[184:187], v[224:227], v[64:67]\n"
        "v_mfma_f32_16x16x32_f16 v[96:99], v[188:191], v[224:227], v[96:99]\n"
        "s_waitcnt lgkmcnt(2)\n"
        "v_mfma_f32_16x16x32_f16 v[4:7], v[176:179], v[228:231], v[4:7]\n"
        "ds_read_b128 v[224:227], v244 offset:28672\n"
        "v_mfma_f32_16x16x32_f16 v[36:39], v[180:183], v[228:231], v[36:39]\n"
        "v_mov_b32 v247, 147476\n"
        "ds_read_b32 v240, v247\n"
        "v_mfma_f32_16x16x32_f16 v[68:71], v[184:187], v[228:231], v[68:71]\n"
        "v_mfma_f32_16x16x32_f16 v[100:103], v[188:191], v[228:231], v[100:103]\n"
        "s_waitcnt lgkmcnt(3)\n"
        "v_mfma_f32_16x16x32_f16 v[8:11], v[176:179], v[232:235], v[8:11]\n"
        "ds_read_b128 v[228:231], v244 offset:29696\n"
        "v_mfma_f32_16x16x32_f16 v[40:43], v[180:183], v[232:235], v[40:43]\n"
        "v_mfma_f32_16x16x32_f16 v[72:75], v[184:187], v[232:235], v[72:75]\n"
        "v_mfma_f32_16x16x32_f16 v[104:107], v[188:191], v[232:235], v[104:107]\n"
        "s_barrier\n"
        "s_lshl_b32 s82, s74, 23\n"
        "s_add_u32 s82, s82, 131072\n"
        "buffer_load_dwordx4 v[212:215], %[wvoff], %[rsW], s82 offen nt\n"
        "s_lshl_b32 s82, s74, 23\n"
        "s_add_u32 s82, s82, 262144\n"
        "buffer_load_dwordx4 v[216:219], %[wvoff], %[rsW], s82 offen nt\n"
        "s_lshl_b32 s82, s74, 23\n"
        "s_add_u32 s82, s82, 393216\n"
        "buffer_load_dwordx4 v[220:223], %[wvoff], %[rsW], s82 offen nt\n"
        "s_waitcnt lgkmcnt(3)\n"
        "v_mfma_f32_16x16x32_f16 v[12:15], v[176:179], v[236:239], v[12:15]\n"
        "ds_read_b128 v[232:235], v244 offset:30720\n"
        "v_mfma_f32_16x16x32_f16 v[44:47], v[180:183], v[236:239], v[44:47]\n"
        "v_mfma_f32_16x16x32_f16 v[76:79], v[184:187], v[236:239], v[76:79]\n"
        "v_mfma_f32_16x16x32_f16 v[108:111], v[188:191], v[236:239], v[108:111]\n"
        "s_waitcnt lgkmcnt(3)\n"
        "v_mfma_f32_16x16x32_f16 v[16:19], v[176:179], v[224:227], v[16:19]\n"
        "ds_read_b128 v[236:239], v244 offset:31744\n"
        "v_mfma_f32_16x16x32_f16 v[48:51], v[180:183], v[224:227], v[48:51]\n"
        "s_waitcnt lgkmcnt(3)\n"
        "v_readfirstlane_b32 s77, v240\n"
        "v_mfma_f32_16x16x32_f16 v[80:83], v[184:187], v[224:227], v[80:83]\n"
        "v_mfma_f32_16x16x32_f16 v[112:115], v[188:191], v[224:227], v[112:115]\n"
        "s_waitcnt lgkmcnt(2)\n"
        "v_mfma_f32_16x16x32_f16 v[20:23], v[176:179], v[228:231], v[20:23]\n"
        "ds_read_b128 v[224:227], v244 offset:32768\n"
        "v_mfma_f32_16x16x32_f16 v[52:55], v[180:183], v[228:231], v[52:55]\n"
        "v_mfma_f32_16x16x32_f16 v[84:87], v[184:187], v[228:231], v[84:87]\n"
        "v_mfma_f32_16x16x32_f16 v[116:119], v[188:191], v[228:231], v[116:119]\n"
        "s_waitcnt lgkmcnt(2)\n"
        "v_mfma_f32_16x16x32_f16 v[24:27], v[176:179], v[232:235], v[24:27]\n"
        "ds_read_b128 v[228:231], v244 offset:33792\n"
        "v_mfma_f32_16x16x32_f16 v[56:59], v[180:183], v[232:235], v[56:59]\n"
        "v_mfma_f32_16x16x32_f16 v[88:91], v[184:187], v[232:235], v[88:91]\n"
        "v_mfma_f32_16x16x32_f16 v[120:123], v[188:191], v[232:235], v[120:123]\n"
        "s_waitcnt lgkmcnt(2)\n"
        "v_mfma_f32_16x16x32_f16 v[28:31], v[176:179], v[236:239], v[28:31]\n"
        "ds_read_b128 v[232:235], v244 offset:34816\n"
        "v_mfma_f32_16x16x32_f16 v[60:63], v[180:183], v[236:239], v[60:63]\n"
        "v_mfma_f32_16x16x32_f16 v[92:95], v[184:187], v[236:239], v[92:95]\n"
        "v_mfma_f32_16x16x32_f16 v[124:127], v[188:191], v[236:239], v[124:127]\n"
        "s_waitcnt vmcnt(8)\n"
        "v_cvt_pk_f16_f32 v240, v192, v196\n"
        "v_cvt_pk_f16_f32 v241, v200, v204\n"
        "ds_write_b64 %[bwr], v[240:241] offset:16384\n"
        "v_cvt_pk_f16_f32 v242, v193, v197\n"
        "v_cvt_pk_f16_f32 v243, v201, v205\n"
        "ds_write_b64 %[bwr], v[242:243] offset:17408\n"
        "v_cvt_pk_f16_f32 v240, v194, v198\n"
        "v_cvt_pk_f16_f32 v241, v202, v206\n"
        "ds_write_b64 %[bwr], v[240:241] offset:18432\n"
        "v_cvt_pk_f16_f32 v242, v195, v199\n"
        "v_cvt_pk_f16_f32 v243, v203, v207\n"
        "ds_write_b64 %[bwr], v[242:243] offset:19456\n"
        "s_lshl_b32 s80, s79, 16\n"
        "s_add_u32 s80, s80, s62\n"
        "s_add_u32 s86, s80, 2048\n"
        "buffer_load_dwordx4 v[160:163], %[avoff], %[rsA], s80 offen offset:0\n"
        "buffer_load_dwordx4 v[164:167], %[avoff], %[rsA], s80 offen offset:1024\n"
        "buffer_load_dwordx4 v[168:171], %[avoff], %[rsA], s86 offen offset:0\n"
        "buffer_load_dwordx4 v[172:175], %[avoff], %[rsA], s86 offen offset:1024\n"
        "s_lshl_b32 s81, s79, 16\n"
        "s_add_u32 s81, s81, s62\n"
        "s_add_u32 s81, s81, 4096\n"
        "s_add_u32 s87, s81, 2048\n"
        "buffer_load_dwordx4 v[176:179], %[avoff], %[rsA], s81 offen offset:0\n"
        "buffer_load_dwordx4 v[180:183], %[avoff], %[rsA], s81 offen offset:1024\n"
        "s_cmp_eq_u32 %[wave], 4\n"
        "s_cbranch_scc1 L_chkk8%=\n"
        "L_chkk8r%=:\n"
        "s_cmp_lg_u32 %[wave], 5\n"
        "s_cbranch_scc1 L_pc9%=\n"
        "s_sub_i32 s89, s58, 16\n"
        "s_max_i32 s89, s89, 0\n"
        "s_mul_i32 s89, s89, 4\n"
        "s_sub_i32 s89, 152, s89\n"
        "s_max_i32 s89, s89, s50\n"
        "s_cmp_lt_u32 s58, 4\n"
        "s_cselect_b32 s89, 100, s89\n"
        "s_add_u32 s58, s58, 1\n"
        "L_pace9%=:\n"
        "s_memrealtime s[92:93]\n"
        "s_waitcnt lgkmcnt(0)\n"
        "s_sub_u32 s93, s92, s59\n"
        "s_cmp_lt_u32 s93, s89\n"
        "s_cbranch_scc1 L_pace9%=\n"
        "s_mov_b32 s59, s92\n"
        "L_pc9%=:\n"
        "s_waitcnt lgkmcnt(0)\n"
        "s_barrier\n"
        "v_mfma_f32_16x16x32_f16 v[0:3], v[128:131], v[224:227], v[0:3]\n"
        "ds_read_b128 v[236:239], v244 offset:35840\n"
        "v_mfma_f32_16x16x32_f16 v[32:35], v[132:135], v[224:227], v[32:35]\n"
        "v_mfma_f32_16x16x32_f16 v[64:67], v[136:139], v[224:227], v[64:67]\n"
        "v_mfma_f32_16x16x32_f16 v[96:99], v[140:143], v[224:227], v[96:99]\n"
        "v_mfma_f32_16x16x32_f16 v[4:7], v[128:131], v[228:231], v[4:7]\n"
        "ds_read_b128 v[224:227], v244 offset:36864\n"
        "v_mfma_f32_16x16x32_f16 v[36:39], v[132:135], v[228:231], v[36:39]\n"
        "v_mfma_f32_16x16x32_f16 v[68:71], v[136:139], v[228:231], v[68:71]\n"
        "v_mfma_f32_16x16x32_f16 v[100:103], v[140:143], v[228:231], v[100:103]\n"
        "v_mfma_f32_16x16x32_f16 v[8:11], v[128:131], v[232:235], v[8:11]\n"
        "ds_read_b128 v[228:231], v244 offset:37888\n"
        "v_mfma_f32_16x16x32_f16 v[40:43], v[132:135], v[232:235], v[40:43]\n"
        "v_mfma_f32_16x16x32_f16 v[72:75], v[136:139], v[232:235], v[72:75]\n"
        "v_mfma_f32_16x16x32_f16 v[104:107], v[140:143], v[232:235], v[104:107]\n"
        "s_waitcnt lgkmcnt(2)\n"
        "v_mfma_f32_16x16x32_f16 v[12:15], v[128:131], v[236:239], v[12:15]\n"
        "ds_read_b128 v[232:235], v244 offset:38912\n"
        "v_mfma_f32_16x16x32_f16 v[44:47], v[132:135], v[236:239], v[44:47]\n"
        "v_mfma_f32_16x16x32_f16 v[76:79], v[136:139], v[236:239], v[76:79]\n"
        "v_mfma_f32_16x16x32_f16 v[108:111], v[140:143], v[236:239], v[108:111]\n"
        "s_waitcnt lgkmcnt(2)\n"
        "v_mfma_f32_16x16x32_f16 v[16:19], v[128:131], v[224:227], v[16:19]\n"
        "ds_read_b128 v[236:239], v244 offset:39936\n"
        "v_mfma_f32_16x16x32_f16 v[48:51], v[132:135], v[224:227], v[48:51]\n"
        "v_mfma_f32_16x16x32_f16 v[80:83], v[136:139], v[224:227], v[80:83]\n"
        "v_mfma_f32_16x16x32_f16 v[112:115], v[140:143], v[224:227], v[112:115]\n"
        "s_waitcnt lgkmcnt(2)\n"
        "v_mfma_f32_16x16x32_f16 v[20:23], v[128:131], v[228:231], v[20:23]\n"
        "ds_read_b128 v[224:227], v244 offset:40960\n"
        "v_mfma_f32_16x16x32_f16 v[52:55], v[132:135], v[228:231], v[52:55]\n"
        "v_mfma_f32_16x16x32_f16 v[84:87], v[136:139], v[228:231], v[84:87]\n"
        "v_mfma_f32_16x16x32_f16 v[116:119], v[140:143], v[228:231], v[116:119]\n"
        "s_barrier\n"
        "s_cmp_eq_u32 s66, 6\n"
        "s_cbranch_scc1 L_flagcY3%=\n"
        "L_flagretcY3%=:\n"
        "s_cmp_eq_u32 s65, 6\n"
        "s_cbranch_scc1 L_pushcY3%=\n"
        "L_pushretcY3%=:\n"
        "s_cmp_eq_u32 %[wave], 4\n"
        "s_cbranch_scc1 L_ctlcY3%=\n"
        "L_ctlcY3r%=:\n"
        "s_lshl_b32 s82, s75, 23\n"
        "buffer_load_dwordx4 v[192:195], %[wvoff], %[rsW], s82 offen nt\n"
        "s_lshl_b32 s82, s75, 23\n"
        "s_add_u32 s82, s82, 131072\n"
        "buffer_load_dwordx4 v[196:199], %[wvoff], %[rsW], s82 offen nt\n"
        "s_waitcnt lgkmcnt(2)\n"
        "v_mfma_f32_16x16x32_f16 v[24:27], v[128:131], v[232:235], v[24:27]\n"
        "ds_read_b128 v[228:231], v244 offset:41984\n"
        "v_mfma_f32_16x16x32_f16 v[56:59], v[132:135], v[232:235], v[56:59]\n"
        "v_mfma_f32_16x16x32_f16 v[88:91], v[136:139], v[232:235], v[88:91]\n"
        "v_mfma_f32_16x16x32_f16 v[120:123], v[140:143], v[232:235], v[120:123]\n"
        "s_waitcnt lgkmcnt(2)\n"
        "v_mfma_f32_16x16x32_f16 v[28:31], v[128:131], v[236:239], v[28:31]\n"
        "ds_read_b128 v[232:235], v244 offset:43008\n"
        "v_mfma_f32_16x16x32_f16 v[60:63], v[132:135], v[236:239], v[60:63]\n"
        "v_mfma_f32_16x16x32_f16 v[92:95], v[136:139], v[236:239], v[92:95]\n"
        "v_mfma_f32_16x16x32_f16 v[124:127], v[140:143], v[236:239], v[124:127]\n"
        "s_waitcnt vmcnt(11) lgkmcnt(2)\n"
        "v_mfma_f32_16x16x32_f16 v[0:3], v[144:147], v[224:227], v[0:3]\n"
        "ds_read_b128 v[236:239], v244 offset:44032\n"
        "v_mfma_f32_16x16x32_f16 v[32:35], v[148:151], v[224:227], v[32:35]\n"
        "v_mfma_f32_16x16x32_f16 v[64:67], v[152:155], v[224:227], v[64:67]\n"
        "v_mfma_f32_16x16x32_f16 v[96:99], v[156:159], v[224:227], v[96:99]\n"
        "s_waitcnt lgkmcnt(2)\n"
        "v_mfma_f32_16x16x32_f16 v[4:7], v[144:147], v[228:231], v[4:7]\n"
        "ds_read_b128 v[224:227], v244 offset:45056\n"
        "v_mfma_f32_16x16x32_f16 v[36:39], v[148:151], v[228:231], v[36:39]\n"
        "v_mfma_f32_16x16x32_f16 v[68:71], v[152:155], v[228:231], v[68:71]\n"
        "v_mfma_f32_16x16x32_f16 v[100:103], v[156:159], v[228:231], v[100:103]\n"
        "s_waitcnt lgkmcnt(2)\n"
        "v_mfma_f32_16x16x32_f16 v[8:11], v[144:147], v[232:235], v[8:11]\n"
        "ds_read_b128 v[228:231], v244 offset:46080\n"
        "v_mfma_f32_16x16x32_f16 v[40:43], v[148:151], v[232:235], v[40:43]\n"
        "v_mfma_f32_16x16x32_f16 v[72:75], v[152:155], v[232:235], v[72:75]\n"
        "v_mfma_f32_16x16x32_f16 v[104:107], v[156:159], v[232:235], v[104:107]\n"
        "s_waitcnt vmcnt(8)\n"
        "v_cvt_pk_f16_f32 v240, v208, v212\n"
        "v_cvt_pk_f16_f32 v241, v216, v220\n"
        "ds_write_b64 %[bwr], v[240:241] offset:32768\n"
        "v_cvt_pk_f16_f32 v242, v209, v213\n"
        "v_cvt_pk_f16_f32 v243, v217, v221\n"
        "ds_write_b64 %[bwr], v[242:243] offset:33792\n"
        "v_cvt_pk_f16_f32 v240, v210, v214\n"
        "v_cvt_pk_f16_f32 v241, v218, v222\n"
        "ds_write_b64 %[bwr], v[240:241] offset:34816\n"
        "v_cvt_pk_f16_f32 v242, v211, v215\n"
        "v_cvt_pk_f16_f32 v243, v219, v223\n"
        "ds_write_b64 %[bwr], v[242:243] offset:35840\n"
        "buffer_load_dwordx4 v[184:187], %[avoff], %[rsA], s87 offen offset:0\n"
        "buffer_load_dwordx4 v[188:191], %[avoff], %[rsA], s87 offen offset:1024\n"
        "s_lshl_b32 s80, s72, 16\n"
        "s_lshl_b32 s86, s72, 16\n"
        "s_cmp_eq_u32 s64, 48\n"
        "s_cselect_b32 s80, s86, s80\n"
        "s_cselect_b32 s86, 2, 1\n"
        "s_lshl_b32 s86, s86, 11\n"
        "s_add_u32 s80, s80, s62\n"
        "s_add_u32 s86, s86, s80\n"
        "buffer_load_dwordx4 v[128:131], %[avoff], %[rsA], s80 offen offset:0\n"
        "buffer_load_dwordx4 v[132:135], %[avoff], %[rsA], s80 offen offset:1024\n"
        "buffer_load_dwordx4 v[136:139], %[avoff], %[rsA], s86 offen offset:0\n"
        "buffer_load_dwordx4 v[140:143], %[avoff], %[rsA], s86 offen offset:1024\n"
        "s_cmp_eq_u32 %[wave], 4\n"
        "s_cbranch_scc1 L_chkkNone%=\n"
        "L_chkkNoner%=:\n"
        "s_cmp_lg_u32 %[wave], 5\n"
        "s_cbranch_scc1 L_pc10%=\n"
        "s_sub_i32 s89, s58, 16\n"
        "s_max_i32 s89, s89, 0\n"
        "s_mul_i32 s89, s89, 4\n"
        "s_sub_i32 s89, 152, s89\n"
        "s_max_i32 s89, s89, s50\n"
        "s_cmp_lt_u32 s58, 4\n"
        "s_cselect_b32 s89, 100, s89\n"
        "s_add_u32 s58, s58, 1\n"
        "L_pace10%=:\n"
        "s_memrealtime s[92:93]\n"
        "s_waitcnt lgkmcnt(0)\n"
        "s_sub_u32 s93, s92, s59\n"
        "s_cmp_lt_u32 s93, s89\n"
        "s_cbranch_scc1 L_pace10%=\n"
        "s_mov_b32 s59, s92\n"
        "L_pc10%=:\n"
        "s_waitcnt lgkmcnt(0)\n"
        "s_barrier\n"
        "v_mfma_f32_16x16x32_f16 v[12:15], v[144:147], v[236:239], v[12:15]\n"
        "ds_read_b128 v[232:235], v244 offset:47104\n"
        "v_mfma_f32_16x16x32_f16 v[44:47], v[148:151], v[236:239], v[44:47]\n"
        "v_mov_b32 v247, 147480\n"
        "ds_read_b32 v240, v247\n"
        "v_mfma_f32_16x16x32_f16 v[76:79], v[152:155], v[236:239], v[76:79]\n"
        "v_mfma_f32_16x16x32_f16 v[108:111], v[156:159], v[236:239], v[108:111]\n"
        "v_mfma_f32_16x16x32_f16 v[16:19], v[144:147], v[224:227], v[16:19]\n"
        "ds_read_b128 v[236:239], v244 offset:48128\n"
        "v_mfma_f32_16x16x32_f16 v[48:51], v[148:151], v[224:227], v[48:51]\n"
        "v_mfma_f32_16x16x32_f16 v[80:83], v[152:155], v[224:227], v[80:83]\n"
        "v_mfma_f32_16x16x32_f16 v[112:115], v[156:159], v[224:227], v[112:115]\n"
        "v_mfma_f32_16x16x32_f16 v[20:23], v[144:147], v[228:231], v[20:23]\n"
        "ds_read_b128 v[224:227], v244 offset:49152\n"
        "v_mfma_f32_16x16x32_f16 v[52:55], v[148:151], v[228:231], v[52:55]\n"
        "v_mfma_f32_16x16x32_f16 v[84:87], v[152:155], v[228:231], v[84:87]\n"
        "v_mfma_f32_16x16x32_f16 v[116:119], v[156:159], v[228:231], v[116:119]\n"
        "s_waitcnt lgkmcnt(3)\n"
        "v_mfma_f32_16x16x32_f16 v[24:27], v[144:147], v[232:235], v[24:27]\n"
        "ds_read_b128 v[228:231], v244 offset:50176\n"
        "v_mfma_f32_16x16x32_f16 v[56:59], v[148:151], v[232:235], v[56:59]\n"
        "s_waitcnt lgkmcnt(3)\n"
        "v_readfirstlane_b32 s78, v240\n"
        "v_mfma_f32_16x16x32_f16 v[88:91], v[152:155], v[232:235], v[88:91]\n"
        "v_mfma_f32_16x16x32_f16 v[120:123], v[156:159], v[232:235], v[120:123]\n"
        "s_waitcnt lgkmcnt(2)\n"
        "v_mfma_f32_16x16x32_f16 v[28:31], v[144:147], v[236:239], v[28:31]\n"
        "ds_read_b128 v[232:235], v244 offset:51200\n"
        "v_mfma_f32_16x16x32_f16 v[60:63], v[148:151], v[236:239], v[60:63]\n"
        "v_mfma_f32_16x16x32_f16 v[92:95], v[152:155], v[236:239], v[92:95]\n"
        "v_mfma_f32_16x16x32_f16 v[124:127], v[156:159], v[236:239], v[124:127]\n"
        "s_barrier\n"
        "s_lshl_b32 s82, s75, 23\n"
        "s_add_u32 s82, s82, 262144\n"
        "buffer_load_dwordx4 v[200:203], %[wvoff], %[rsW], s82 offen nt\n"
        "s_lshl_b32 s82, s75, 23\n"
        "s_add_u32 s82, s82, 393216\n"
        "buffer_load_dwordx4 v[204:207], %[wvoff], %[rsW], s82 offen nt\n"
        "s_lshl_b32 s82, s76, 23\n"
        "buffer_load_dwordx4 v[208:211], %[wvoff], %[rsW], s82 offen nt\n"
        "s_waitcnt vmcnt(13) lgkmcnt(2)\n"
        "v_mfma_f32_16x16x32_f16 v[0:3], v[160:163], v[224:227], v[0:3]\n"
        "ds_read_b128 v[236:239], v244 offset:52224\n"
        "v_mfma_f32_16x16x32_f16 v[32:35], v[164:167], v[224:227], v[32:35]\n"
        "v_mfma_f32_16x16x32_f16 v[64:67], v[168:171], v[224:227], v[64:67]\n"
        "v_mfma_f32_16x16x32_f16 v[96:99], v[172:175], v[224:227], v[96:99]\n"
        "s_waitcnt lgkmcnt(2)\n"
        "v_mfma_f32_16x16x32_f16 v[4:7], v[160:163], v[228:231], v[4:7]\n"
        "ds_read_b128 v[224:227], v244 offset:53248\n"
        "v_mfma_f32_16x16x32_f16 v[36:39], v[164:167], v[228:231], v[36:39]\n"
        "v_mfma_f32_16x16x32_f16 v[68:71], v[168:171], v[228:231], v[68:71]\n"
        "v_mfma_f32_16x16x32_f16 v[100:103], v[172:175], v[228:231], v[100:103]\n"
        "s_waitcnt lgkmcnt(2)\n"
        "v_mfma_f32_16x16x32_f16 v[8:11], v[160:163], v[232:235], v[8:11]\n"
        "ds_read_b128 v[228:231], v244 offset:54272\n"
        "v_mfma_f32_16x16x32_f16 v[40:43], v[164:167], v[232:235], v[40:43]\n"
        "v_mfma_f32_16x16x32_f16 v[72:75], v[168:171], v[232:235], v[72:75]\n"
        "v_mfma_f32_16x16x32_f16 v[104:107], v[172:175], v[232:235], v[104:107]\n"
        "s_waitcnt lgkmcnt(2)\n"
        "v_mfma_f32_16x16x32_f16 v[12:15], v[160:163], v[236:239], v[12:15]\n"
        "ds_read_b128 v[232:235], v244 offset:55296\n"
        "v_mfma_f32_16x16x32_f16 v[44:47], v[164:167], v[236:239], v[44:47]\n"
        "v_mfma_f32_16x16x32_f16 v[76:79], v[168:171], v[236:239], v[76:79]\n"
        "v_mfma_f32_16x16x32_f16 v[108:111], v[172:175], v[236:239], v[108:111]\n"
        "s_waitcnt lgkmcnt(2)\n"
        "v_mfma_f32_16x16x32_f16 v[16:19], v[160:163], v[224:227], v[16:19]\n"
        "ds_read_b128 v[236:239], v244 offset:56320\n"
        "v_mfma_f32_16x16x32_f16 v[48:51], v[164:167], v[224:227], v[48:51]\n"
        "v_mfma_f32_16x16x32_f16 v[80:83], v[168:171], v[224:227], v[80:83]\n"
        "v_mfma_f32_16x16x32_f16 v[112:115], v[172:175], v[224:227], v[112:115]\n"
        "s_lshl_b32 s81, s72, 16\n"
        "s_add_u32 s81, s81, 4096\n"
        "s_lshl_b32 s87, s73, 16\n"
        "s_cmp_eq_u32 s64, 48\n"
        "s_cselect_b32 s81, s87, s81\n"
        "s_cselect_b32 s87, 2, 1\n"
        "s_lshl_b32 s87, s87, 11\n"
        "s_add_u32 s81, s81, s62\n"
        "s_add_u32 s87, s87, s81\n"
        "buffer_load_dwordx4 v[144:147], %[avoff], %[rsA], s81 offen offset:0\n"
        "buffer_load_dwordx4 v[148:151], %[avoff], %[rsA], s81 offen offset:1024\n"
        "buffer_load_dwordx4 v[152:155], %[avoff], %[rsA], s87 offen offset:0\n"
        "buffer_load_dwordx4 v[156:159], %[avoff], %[rsA], s87 offen offset:1024\n"
        "s_cmp_eq_u32 %[wave], 4\n"
        "s_cbranch_scc1 L_chkk9%=\n"
        "L_chkk9r%=:\n"
        "s_cmp_lg_u32 %[wave], 5\n"
        "s_cbranch_scc1 L_pc11%=\n"
        "s_sub_i32 s89, s58, 16\n"
        "s_max_i32 s89, s89, 0\n"
        "s_mul_i32 s89, s89, 4\n"
        "s_sub_i32 s89, 152, s89\n"
        "s_max_i32 s89, s89, s50\n"
        "s_cmp_lt_u32 s58, 4\n"
        "s_cselect_b32 s89, 100, s89\n"
        "s_add_u32 s58, s58, 1\n"
        "L_pace11%=:\n"
        "s_memrealtime s[92:93]\n"
        "s_waitcnt lgkmcnt(0)\n"
        "s_sub_u32 s93, s92, s59\n"
        "s_cmp_lt_u32 s93, s89\n"
        "s_cbranch_scc1 L_pace11%=\n"
        "s_mov_b32 s59, s92\n"
        "L_pc11%=:\n"
        "s_barrier\n"
        "s_waitcnt lgkmcnt(2)\n"
        "v_mfma_f32_16x16x32_f16 v[20:23], v[160:163], v[228:231], v[20:23]\n"
        "ds_read_b128 v[224:227], v244 offset:57344\n"
        "v_mfma_f32_16x16x32_f16 v[52:55], v[164:167], v[228:231], v[52:55]\n"
        "v_mfma_f32_16x16x32_f16 v[84:87], v[168:171], v[228:231], v[84:87]\n"
        "v_mfma_f32_16x16x32_f16 v[116:119], v[172:175], v[228:231], v[116:119]\n"
        "s_waitcnt lgkmcnt(2)\n"
        "v_mfma_f32_16x16x32_f16 v[24:27], v[160:163], v[232:235], v[24:27]\n"
        "ds_read_b128 v[228:231], v244 offset:58368\n"
        "v_mfma_f32_16x16x32_f16 v[56:59], v[164:167], v[232:235], v[56:59]\n"
        "v_mfma_f32_16x16x32_f16 v[88:91], v[168:171], v[232:235], v[88:91]\n"
        "v_mfma_f32_16x16x32_f16 v[120:123], v[172:175], v[232:235], v[120:123]\n"
        "s_waitcnt lgkmcnt(2)\n"
        "v_mfma_f32_16x16x32_f16 v[28:31], v[160:163], v[236:239], v[28:31]\n"
        "ds_read_b128 v[232:235], v244 offset:59392\n"
        "v_mfma_f32_16x16x32_f16 v[60:63], v[164:167], v[236:239], v[60:63]\n"
        "v_mfma_f32_16x16x32_f16 v[92:95], v[168:171], v[236:239], v[92:95]\n"
        "v_mfma_f32_16x16x32_f16 v[124:127], v[172:175], v[236:239], v[124:127]\n"
        "s_waitcnt vmcnt(11) lgkmcnt(2)\n"
        "v_mfma_f32_16x16x32_f16 v[0:3], v[176:179], v[224:227], v[0:3]\n"
        "ds_read_b128 v[236:239], v244 offset:60416\n"
        "v_mfma_f32_16x16x32_f16 v[32:35], v[180:183], v[224:227], v[32:35]\n"
        "v_mfma_f32_16x16x32_f16 v[64:67], v[184:187], v[224:227], v[64:67]\n"
        "v_mfma_f32_16x16x32_f16 v[96:99], v[188:191], v[224:227], v[96:99]\n"
        "s_waitcnt lgkmcnt(2)\n"
        "v_mfma_f32_16x16x32_f16 v[4:7], v[176:179], v[228:231], v[4:7]\n"
        "ds_read_b128 v[224:227], v244 offset:61440\n"
        "v_mfma_f32_16x16x32_f16 v[36:39], v[180:183], v[228:231], v[36:39]\n"
        "v_mov_b32 v247, 147484\n"
        "ds_read_b32 v240, v247\n"
        "v_mfma_f32_16x16x32_f16 v[68:71], v[184:187], v[228:231], v[68:71]\n"
        "v_mfma_f32_16x16x32_f16 v[100:103], v[188:191], v[228:231], v[100:103]\n"
        "s_waitcnt lgkmcnt(3)\n"
        "v_mfma_f32_16x16x32_f16 v[8:11], v[176:179], v[232:235], v[8:11]\n"
        "ds_read_b128 v[228:231], v244 offset:62464\n"
        "v_mfma_f32_16x16x32_f16 v[40:43], v[180:183], v[232:235], v[40:43]\n"
        "v_mfma_f32_16x16x32_f16 v[72:75], v[184:187], v[232:235], v[72:75]\n"
        "v_mfma_f32_16x16x32_f16 v[104:107], v[188:191], v[232:235], v[104:107]\n"
        "s_barrier\n"
        "s_lshl_b32 s82, s76, 23\n"
        "s_add_u32 s82, s82, 131072\n"
        "buffer_load_dwordx4 v[212:215], %[wvoff], %[rsW], s82 offen nt\n"
        "s_lshl_b32 s82, s76, 23\n"
        "s_add_u32 s82, s82, 262144\n"
        "buffer_load_dwordx4 v[216:219], %[wvoff], %[rsW], s82 offen nt\n"
        "s_lshl_b32 s82, s76, 23\n"
        "s_add_u32 s82, s82, 393216\n"
        "buffer_load_dwordx4 v[220:223], %[wvoff], %[rsW], s82 offen nt\n"
        "s_waitcnt lgkmcnt(3)\n"
        "v_mfma_f32_16x16x32_f16 v[12:15], v[176:179], v[236:239], v[12:15]\n"
        "ds_read_b128 v[232:235], v244 offset:63488\n"
        "v_mfma_f32_16x16x32_f16 v[44:47], v[180:183], v[236:239], v[44:47]\n"
        "v_mfma_f32_16x16x32_f16 v[76:79], v[184:187], v[236:239], v[76:79]\n"
        "v_mfma_f32_16x16x32_f16 v[108:111], v[188:191], v[236:239], v[108:111]\n"
        "s_waitcnt lgkmcnt(3)\n"
        "v_mfma_f32_16x16x32_f16 v[16:19], v[176:179], v[224:227], v[16:19]\n"
        "ds_read_b128 v[236:239], v244 offset:64512\n"
        "v_mfma_f32_16x16x32_f16 v[48:51], v[180:183], v[224:227], v[48:51]\n"
        "s_waitcnt lgkmcnt(3)\n"
        "v_readfirstlane_b32 s79, v240\n"
        "v_mfma_f32_16x16x32_f16 v[80:83], v[184:187], v[224:227], v[80:83]\n"
        "v_mfma_f32_16x16x32_f16 v[112:115], v[188:191], v[224:227], v[112:115]\n"
        "s_waitcnt lgkmcnt(2)\n"
        "v_mfma_f32_16x16x32_f16 v[20:23], v[176:179], v[228:231], v[20:23]\n"
        "ds_read_b128 v[224:227], %[brd] offset:0\n"
        "v_mfma_f32_16x16x32_f16 v[52:55], v[180:183], v[228:231], v[52:55]\n"
        "v_mfma_f32_16x16x32_f16 v[84:87], v[184:187], v[228:231], v[84:87]\n"
        "v_mfma_f32_16x16x32_f16 v[116:119], v[188:191], v[228:231], v[116:119]\n"
        "s_waitcnt lgkmcnt(2)\n"
        "v_mfma_f32_16x16x32_f16 v[24:27], v[176:179], v[232:235], v[24:27]\n"
        "ds_read_b128 v[228:231], %[brd] offset:1024\n"
        "v_mfma_f32_16x16x32_f16 v[56:59], v[180:183], v[232:235], v[56:59]\n"
        "v_mfma_f32_16x16x32_f16 v[88:91], v[184:187], v[232:235], v[88:91]\n"
        "v_mfma_f32_16x16x32_f16 v[120:123], v[188:191], v[232:235], v[120:123]\n"
        "s_waitcnt lgkmcnt(2)\n"
        "v_mfma_f32_16x16x32_f16 v[28:31], v[176:179], v[236:239], v[28:31]\n"
        "ds_read_b128 v[232:235], %[brd] offset:2048\n"
        "v_mfma_f32_16x16x32_f16 v[60:63], v[180:183], v[236:239], v[60:63]\n"
        "v_mfma_f32_16x16x32_f16 v[92:95], v[184:187], v[236:239], v[92:95]\n"
        "v_mfma_f32_16x16x32_f16 v[124:127], v[188:191], v[236:239], v[124:127]\n"
        "s_add_u32 s64, s64, 8\n"
        "s_sub_u32 s65, s65, 8\n"
        "s_sub_u32 s66, s66, 8\n"
        "s_cmp_lt_u32 s64, 56\n"
        "s_cbranch_scc1 L_loopY%=\n"
        "s_waitcnt vmcnt(8)\n"
        "v_cvt_pk_f16_f32 v240, v192, v196\n"
        "v_cvt_pk_f16_f32 v241, v200, v204\n"
        "ds_write_b64 %[bwr], v[240:241] offset:49152\n"
        "v_cvt_pk_f16_f32 v242, v193, v197\n"
        "v_cvt_pk_f16_f32 v243, v201, v205\n"
        "ds_write_b64 %[bwr], v[242:243] offset:50176\n"
        "v_cvt_pk_f16_f32 v240, v194, v198\n"
        "v_cvt_pk_f16_f32 v241, v202, v206\n"
        "ds_write_b64 %[bwr], v[240:241] offset:51200\n"
        "v_cvt_pk_f16_f32 v242, v195, v199\n"
        "v_cvt_pk_f16_f32 v243, v203, v207\n"
        "ds_write_b64 %[bwr], v[242:243] offset:52224\n"
        "s_lshl_b32 s80, s74, 16\n"
        "s_add_u32 s80, s80, s62\n"
        "s_add_u32 s86, s80, 4096\n"
        "buffer_load_dwordx4 v[160:163], %[avoff], %[rsA], s80 offen offset:0\n"
        "buffer_load_dwordx4 v[164:167], %[avoff], %[rsA], s80 offen offset:1024\n"
        "buffer_load_dwordx4 v[168:171], %[avoff], %[rsA], s86 offen offset:0\n"
        "buffer_load_dwordx4 v[172:175], %[avoff], %[rsA], s86 offen offset:1024\n"
        "s_lshl_b32 s81, s75, 16\n"
        "s_add_u32 s81, s81, s62\n"
        "s_add_u32 s87, s81, 4096\n"
        "buffer_load_dwordx4 v[176:179], %[avoff], %[rsA], s81 offen offset:0\n"
        "buffer_load_dwordx4 v[180:183], %[avoff], %[rsA], s81 offen offset:1024\n"
        "s_cmp_lg_u32 %[wave], 5\n"
        "s_cbranch_scc1 L_pc0f%=\n"
        "s_sub_i32 s89, s58, 16\n"
        "s_max_i32 s89, s89, 0\n"
        "s_mul_i32 s89, s89, 4\n"
        "s_sub_i32 s89, 152, s89\n"
        "s_max_i32 s89, s89, s50\n"
        "s_cmp_lt_u32 s58, 4\n"
        "s_cselect_b32 s89, 100, s89\n"
        "s_add_u32 s58, s58, 1\n"
        "L_pace0f%=:\n"
        "s_memrealtime s[92:93]\n"
        "s_waitcnt lgkmcnt(0)\n"
        "s_sub_u32 s93, s92, s59\n"
        "s_cmp_lt_u32 s93, s89\n"
        "s_cbranch_scc1 L_pace0f%=\n"
        "s_mov_b32 s59, s92\n"
        "L_pc0f%=:\n"
        "s_waitcnt lgkmcnt(0)\n"
        "s_barrier\n"
        "v_mfma_f32_16x16x32_f16 v[0:3], v[128:131], v[224:227], v[0:3]\n"
        "ds_read_b128 v[236:239], %[brd] offset:3072\n"
        "v_mfma_f32_16x16x32_f16 v[32:35], v[132:135], v[224:227], v[32:35]\n"
        "v_mfma_f32_16x16x32_f16 v[4:7], v[128:131], v[228:231], v[4:7]\n"
        "ds_read_b128 v[224:227], %[brd] offset:4096\n"
        "v_mfma_f32_16x16x32_f16 v[36:39], v[132:135], v[228:231], v[36:39]\n"
        "v_mfma_f32_16x16x32_f16 v[8:11], v[128:131], v[232:235], v[8:11]\n"
        "ds_read_b128 v[228:231], %[brd] offset:5120\n"
        "v_mfma_f32_16x16x32_f16 v[40:43], v[132:135], v[232:235], v[40:43]\n"
        "s_waitcnt lgkmcnt(2)\n"
        "v_mfma_f32_16x16x32_f16 v[12:15], v[128:131], v[236:239], v[12:15]\n"
        "ds_read_b128 v[232:235], %[brd] offset:6144\n"
        "v_mfma_f32_16x16x32_f16 v[44:47], v[132:135], v[236:239], v[44:47]\n"
        "s_waitcnt lgkmcnt(2)\n"
        "v_mfma_f32_16x16x32_f16 v[16:19], v[128:131], v[224:227], v[16:19]\n"
        "ds_read_b128 v[236:239], %[brd] offset:7168\n"
        "v_mfma_f32_16x16x32_f16 v[48:51], v[132:135], v[224:227], v[48:51]\n"
        "s_waitcnt lgkmcnt(2)\n"
        "v_mfma_f32_16x16x32_f16 v[20:23], v[128:131], v[228:231], v[20:23]\n"
        "ds_read_b128 v[224:227], %[brd] offset:8192\n"
        "v_mfma_f32_16x16x32_f16 v[52:55], v[132:135], v[228:231], v[52:55]\n"
        "s_waitcnt lgkmcnt(2)\n"
        "v_mfma_f32_16x16x32_f16 v[24:27], v[128:131], v[232:235], v[24:27]\n"
        "ds_read_b128 v[228:231], %[brd] offset:9216\n"
        "v_mfma_f32_16x16x32_f16 v[56:59], v[132:135], v[232:235], v[56:59]\n"
        "s_waitcnt lgkmcnt(2)\n"
        "v_mfma_f32_16x16x32_f16 v[28:31], v[128:131], v[236:239], v[28:31]\n"
        "ds_read_b128 v[232:235], %[brd] offset:10240\n"
        "v_mfma_f32_16x16x32_f16 v[60:63], v[132:135], v[236:239], v[60:63]\n"
        "s_waitcnt lgkmcnt(2)\n"
        "v_mfma_f32_16x16x32_f16 v[0:3], v[136:139], v[224:227], v[0:3]\n"
        "ds_read_b128 v[236:239], %[brd] offset:11264\n"
        "v_mfma_f32_16x16x32_f16 v[32:35], v[140:143], v[224:227], v[32:35]\n"
        "s_waitcnt lgkmcnt(2)\n"
        "v_mfma_f32_16x16x32_f16 v[4:7], v[136:139], v[228:231], v[4:7]\n"
        "ds_read_b128 v[224:227], %[brd] offset:12288\n"
        "v_mfma_f32_16x16x32_f16 v[36:39], v[140:143], v[228:231], v[36:39]\n"
        "s_waitcnt lgkmcnt(2)\n"
        "v_mfma_f32_16x16x32_f16 v[8:11], v[136:139], v[232:235], v[8:11]\n"
        "ds_read_b128 v[228:231], %[brd] offset:13312\n"
        "v_mfma_f32_16x16x32_f16 v[40:43], v[140:143], v[232:235], v[40:43]\n"
        "s_waitcnt lgkmcnt(2)\n"
        "v_mfma_f32_16x16x32_f16 v[12:15], v[136:139], v[236:239], v[12:15]\n"
        "ds_read_b128 v[232:235], %[brd] offset:14336\n"
        "v_mfma_f32_16x16x32_f16 v[44:47], v[140:143], v[236:239], v[44:47]\n"
        "s_barrier\n"
        "s_lshl_b32 s82, s77, 23\n"
        "buffer_load_dwordx4 v[192:195], %[wvoff], %[rsW], s82 offen nt\n"
        "s_lshl_b32 s82, s77, 23\n"
        "s_add_u32 s82, s82, 131072\n"
        "buffer_load_dwordx4 v[196:199], %[wvoff], %[rsW], s82 offen nt\n"
        "s_lshl_b32 s82, s77, 23\n"
        "s_add_u32 s82, s82, 262144\n"
        "buffer_load_dwordx4 v[200:203], %[wvoff], %[rsW], s82 offen nt\n"
        "s_waitcnt lgkmcnt(2)\n"
        "v_mfma_f32_16x16x32_f16 v[16:19], v[136:139], v[224:227], v[16:19]\n"
        "ds_read_b128 v[236:239], %[brd] offset:15360\n"
        "v_mfma_f32_16x16x32_f16 v[48:51], v[140:143], v[224:227], v[48:51]\n"
        "s_waitcnt lgkmcnt(2)\n"
        "v_mfma_f32_16x16x32_f16 v[20:23], v[136:139], v[228:231], v[20:23]\n"
        "ds_read_b128 v[224:227], %[brd] offset:16384\n"
        "v_mfma_f32_16x16x32_f16 v[52:55], v[140:143], v[228:231], v[52:55]\n"
        "s_waitcnt lgkmcnt(2)\n"
        "v_mfma_f32_16x16x32_f16 v[24:27], v[136:139], v[232:235], v[24:27]\n"
        "ds_read_b128 v[228:231], %[brd] offset:17408\n"
        "v_mfma_f32_16x16x32_f16 v[56:59], v[140:143], v[232:235], v[56:59]\n"
        "s_waitcnt lgkmcnt(2)\n"
        "v_mfma_f32_16x16x32_f16 v[28:31], v[136:139], v[236:239], v[28:31]\n"
        "ds_read_b128 v[232:235], %[brd] offset:18432\n"
        "v_mfma_f32_16x16x32_f16 v[60:63], v[140:143], v[236:239], v[60:63]\n"
        "s_waitcnt vmcnt(12) lgkmcnt(2)\n"
        "v_mfma_f32_16x16x32_f16 v[0:3], v[144:147], v[224:227], v[0:3]\n"
        "ds_read_b128 v[236:239], %[brd] offset:19456\n"
        "v_mfma_f32_16x16x32_f16 v[32:35], v[148:151], v[224:227], v[32:35]\n"
        "s_waitcnt lgkmcnt(2)\n"
        "v_mfma_f32_16x16x32_f16 v[4:7], v[144:147], v[228:231], v[4:7]\n"
        "ds_read_b128 v[224:227], %[brd] offset:20480\n"
        "v_mfma_f32_16x16x32_f16 v[36:39], v[148:151], v[228:231], v[36:39]\n"
        "s_waitcnt lgkmcnt(2)\n"
        "v_mfma_f32_16x16x32_f16 v[8:11], v[144:147], v[232:235], v[8:11]\n"
        "ds_read_b128 v[228:231], %[brd] offset:21504\n"
        "v_mfma_f32_16x16x32_f16 v[40:43], v[148:151], v[232:235], v[40:43]\n"
        "s_waitcnt lgkmcnt(2)\n"
        "v_mfma_f32_16x16x32_f16 v[12:15], v[144:147], v[236:239], v[12:15]\n"
        "ds_read_b128 v[232:235], %[brd] offset:22528\n"
        "v_mfma_f32_16x16x32_f16 v[44:47], v[148:151], v[236:239], v[44:47]\n"
        "s_waitcnt lgkmcnt(2)\n"
        "v_mfma_f32_16x16x32_f16 v[16:19], v[144:147], v[224:227], v[16:19]\n"
        "ds_read_b128 v[236:239], %[brd] offset:23552\n"
        "v_mfma_f32_16x16x32_f16 v[48:51], v[148:151], v[224:227], v[48:51]\n"
        "s_waitcnt lgkmcnt(2)\n"
        "v_mfma_f32_16x16x32_f16 v[20:23], v[144:147], v[228:231], v[20:23]\n"
        "ds_read_b128 v[224:227], %[brd] offset:24576\n"
        "v_mfma_f32_16x16x32_f16 v[52:55], v[148:151], v[228:231], v[52:55]\n"
        "s_waitcnt vmcnt(9)\n"
        "v_cvt_pk_f16_f32 v240, v208, v212\n"
        "v_cvt_pk_f16_f32 v241, v216, v220\n"
        "ds_write_b64 v245, v[240:241] offset:0\n"
        "v_cvt_pk_f16_f32 v242, v209, v213\n"
        "v_cvt_pk_f16_f32 v243, v217, v221\n"
        "ds_write_b64 v245, v[242:243] offset:1024\n"
        "v_cvt_pk_f16_f32 v240, v210, v214\n"
        "v_cvt_pk_f16_f32 v241, v218, v222\n"
        "ds_write_b64 v245, v[240:241] offset:2048\n"
        "v_cvt_pk_f16_f32 v242, v211, v215\n"
        "v_cvt_pk_f16_f32 v243, v219, v223\n"
        "ds_write_b64 v245, v[242:243] offset:3072\n"
        "buffer_load_dwordx4 v[184:187], %[avoff], %[rsA], s87 offen offset:0\n"
        "buffer_load_dwordx4 v[188:191], %[avoff], %[rsA], s87 offen offset:1024\n"
        "s_lshl_b32 s80, s76, 16\n"
        "s_add_u32 s80, s80, s62\n"
        "s_add_u32 s86, s80, 4096\n"
        "buffer_load_dwordx4 v[128:131], %[avoff], %[rsA], s80 offen offset:0\n"
        "buffer_load_dwordx4 v[132:135], %[avoff], %[rsA], s80 offen offset:1024\n"
        "buffer_load_dwordx4 v[136:139], %[avoff], %[rsA], s86 offen offset:0\n"
        "buffer_load_dwordx4 v[140:143], %[avoff], %[rsA], s86 offen offset:1024\n"
        "s_cmp_lg_u32 %[wave], 5\n"
        "s_cbranch_scc1 L_pc1f%=\n"
        "s_sub_i32 s89, s58, 16\n"
        "s_max_i32 s89, s89, 0\n"
        "s_mul_i32 s89, s89, 4\n"
        "s_sub_i32 s89, 152, s89\n"
        "s_max_i32 s89, s89, s50\n"
        "s_cmp_lt_u32 s58, 4\n"
        "s_cselect_b32 s89, 100, s89\n"
        "s_add_u32 s58, s58, 1\n"
        "L_pace1f%=:\n"
        "s_memrealtime s[92:93]\n"
        "s_waitcnt lgkmcnt(0)\n"
        "s_sub_u32 s93, s92, s59\n"
        "s_cmp_lt_u32 s93, s89\n"
        "s_cbranch_scc1 L_pace1f%=\n"
        "s_mov_b32 s59, s92\n"
        "L_pc1f%=:\n"
        "s_waitcnt lgkmcnt(0)\n"
        "s_barrier\n"
        "v_mfma_f32_16x16x32_f16 v[24:27], v[144:147], v[232:235], v[24:27]\n"
        "ds_read_b128 v[228:231], %[brd] offset:25600\n"
        "v_mfma_f32_16x16x32_f16 v[56:59], v[148:151], v[232:235], v[56:59]\n"
        "v_mfma_f32_16x16x32_f16 v[28:31], v[144:147], v[236:239], v[28:31]\n"
        "ds_read_b128 v[232:235], %[brd] offset:26624\n"
        "v_mfma_f32_16x16x32_f16 v[60:63], v[148:151], v[236:239], v[60:63]\n"
        "v_mfma_f32_16x16x32_f16 v[0:3], v[152:155], v[224:227], v[0:3]\n"
        "ds_read_b128 v[236:239], %[brd] offset:27648\n"
        "v_mfma_f32_16x16x32_f16 v[32:35], v[156:159], v[224:227], v[32:35]\n"
        "s_waitcnt lgkmcnt(2)\n"
        "v_mfma_f32_16x16x32_f16 v[4:7], v[152:155], v[228:231], v[4:7]\n"
        "ds_read_b128 v[224:227], %[brd] offset:28672\n"
        "v_mfma_f32_16x16x32_f16 v[36:39], v[156:159], v[228:231], v[36:39]\n"
        "s_waitcnt lgkmcnt(2)\n"
        "v_mfma_f32_16x16x32_f16 v[8:11], v[152:155], v[232:235], v[8:11]\n"
        "ds_read_b128 v[228:231], %[brd] offset:29696\n"
        "v_mfma_f32_16x16x32_f16 v[40:43], v[156:159], v[232:235], v[40:43]\n"
        "s_waitcnt lgkmcnt(2)\n"
        "v_mfma_f32_16x16x32_f16 v[12:15], v[152:155], v[236:239], v[12:15]\n"
        "ds_read_b128 v[232:235], %[brd] offset:30720\n"
        "v_mfma_f32_16x16x32_f16 v[44:47], v[156:159], v[236:239], v[44:47]\n"
        "s_waitcnt lgkmcnt(2)\n"
        "v_mfma_f32_16x16x32_f16 v[16:19], v[152:155], v[224:227], v[16:19]\n"
        "ds_read_b128 v[236:239], %[brd] offset:31744\n"
        "v_mfma_f32_16x16x32_f16 v[48:51], v[156:159], v[224:227], v[48:51]\n"
        "s_waitcnt lgkmcnt(2)\n"
        "v_mfma_f32_16x16x32_f16 v[20:23], v[152:155], v[228:231], v[20:23]\n"
        "ds_read_b128 v[224:227], %[brd] offset:32768\n"
        "v_mfma_f32_16x16x32_f16 v[52:55], v[156:159], v[228:231], v[52:55]\n"
        "s_waitcnt lgkmcnt(2)\n"
        "v_mfma_f32_16x16x32_f16 v[24:27], v[152:155], v[232:235], v[24:27]\n"
        "ds_read_b128 v[228:231], %[brd] offset:33792\n"
        "v_mfma_f32_16x16x32_f16 v[56:59], v[156:159], v[232:235], v[56:59]\n"
        "s_waitcnt lgkmcnt(2)\n"
        "v_mfma_f32_16x16x32_f16 v[28:31], v[152:155], v[236:239], v[28:31]\n"
        "ds_read_b128 v[232:235], %[brd] offset:34816\n"
        "v_mfma_f32_16x16x32_f16 v[60:63], v[156:159], v[236:239], v[60:63]\n"
        "s_barrier\n"
        "s_lshl_b32 s82, s77, 23\n"
        "s_add_u32 s82, s82, 393216\n"
        "buffer_load_dwordx4 v[204:207], %[wvoff], %[rsW], s82 offen nt\n"
        "s_lshl_b32 s82, s78, 23\n"
        "buffer_load_dwordx4 v[208:211], %[wvoff], %[rsW], s82 offen nt\n"
        "s_lshl_b32 s82, s78, 23\n"
        "s_add_u32 s82, s82, 131072\n"
        "buffer_load_dwordx4 v[212:215], %[wvoff], %[rsW], s82 offen nt\n"
        "s_waitcnt vmcnt(14) lgkmcnt(2)\n"
        "v_mfma_f32_16x16x32_f16 v[0:3], v[160:163], v[224:227], v[0:3]\n"
        "ds_read_b128 v[236:239], %[brd] offset:35840\n"
        "v_mfma_f32_16x16x32_f16 v[32:35], v[164:167], v[224:227], v[32:35]\n"
        "s_waitcnt lgkmcnt(2)\n"
        "v_mfma_f32_16x16x32_f16 v[4:7], v[160:163], v[228:231], v[4:7]\n"
        "ds_read_b128 v[224:227], %[brd] offset:36864\n"
        "v_mfma_f32_16x16x32_f16 v[36:39], v[164:167], v[228:231], v[36:39]\n"
        "s_waitcnt lgkmcnt(2)\n"
        "v_mfma_f32_16x16x32_f16 v[8:11], v[160:163], v[232:235], v[8:11]\n"
        "ds_read_b128 v[228:231], %[brd] offset:37888\n"
        "v_mfma_f32_16x16x32_f16 v[40:43], v[164:167], v[232:235], v[40:43]\n"
        "s_waitcnt lgkmcnt(2)\n"
        "v_mfma_f32_16x16x32_f16 v[12:15], v[160:163], v[236:239], v[12:15]\n"
        "ds_read_b128 v[232:235], %[brd] offset:38912\n"
        "v_mfma_f32_16x16x32_f16 v[44:47], v[164:167], v[236:239], v[44:47]\n"
        "s_waitcnt lgkmcnt(2)\n"
        "v_mfma_f32_16x16x32_f16 v[16:19], v[160:163], v[224:227], v[16:19]\n"
        "ds_read_b128 v[236:239], %[brd] offset:39936\n"
        "v_mfma_f32_16x16x32_f16 v[48:51], v[164:167], v[224:227], v[48:51]\n"
        "s_waitcnt lgkmcnt(2)\n"
        "v_mfma_f32_16x16x32_f16 v[20:23], v[160:163], v[228:231], v[20:23]\n"
        "ds_read_b128 v[224:227], %[brd] offset:40960\n"
        "v_mfma_f32_16x16x32_f16 v[52:55], v[164:167], v[228:231], v[52:55]\n"
        "s_waitcnt lgkmcnt(2)\n"
        "v_mfma_f32_16x16x32_f16 v[24:27], v[160:163], v[232:235], v[24:27]\n"
        "ds_read_b128 v[228:231], %[brd] offset:41984\n"
        "v_mfma_f32_16x16x32_f16 v[56:59], v[164:167], v[232:235], v[56:59]\n"
        "s_waitcnt lgkmcnt(2)\n"
        "v_mfma_f32_16x16x32_f16 v[28:31], v[160:163], v[236:239], v[28:31]\n"
        "ds_read_b128 v[232:235], %[brd] offset:43008\n"
        "v_mfma_f32_16x16x32_f16 v[60:63], v[164:167], v[236:239], v[60:63]\n"
        "s_waitcnt lgkmcnt(2)\n"
        "v_mfma_f32_16x16x32_f16 v[0:3], v[168:171], v[224:227], v[0:3]\n"
        "ds_read_b128 v[236:239], %[brd] offset:44032\n"
        "v_mfma_f32_16x16x32_f16 v[32:35], v[172:175], v[224:227], v[32:35]\n"
        "s_waitcnt lgkmcnt(2)\n"
        "v_mfma_f32_16x16x32_f16 v[4:7], v[168:171], v[228:231], v[4:7]\n"
        "ds_read_b128 v[224:227], %[brd] offset:45056\n"
        "v_mfma_f32_16x16x32_f16 v[36:39], v[172:175], v[228:231], v[36:39]\n"
        "s_lshl_b32 s81, s77, 16\n"
        "s_add_u32 s81, s81, s62\n"
        "s_add_u32 s87, s81, 4096\n"
        "buffer_load_dwordx4 v[144:147], %[avoff], %[rsA], s81 offen offset:0\n"
        "buffer_load_dwordx4 v[148:151], %[avoff], %[rsA], s81 offen offset:1024\n"
        "buffer_load_dwordx4 v[152:155], %[avoff], %[rsA], s87 offen offset:0\n"
        "buffer_load_dwordx4 v[156:159], %[avoff], %[rsA], s87 offen offset:1024\n"
        "s_cmp_lg_u32 %[wave], 5\n"
        "s_cbranch_scc1 L_pc2f%=\n"
        "s_sub_i32 s89, s58, 16\n"
        "s_max_i32 s89, s89, 0\n"
        "s_mul_i32 s89, s89, 4\n"
        "s_sub_i32 s89, 152, s89\n"
        "s_max_i32 s89, s89, s50\n"
        "s_cmp_lt_u32 s58, 4\n"
        "s_cselect_b32 s89, 100, s89\n"
        "s_add_u32 s58, s58, 1\n"
        "L_pace2f%=:\n"
        "s_memrealtime s[92:93]\n"
        "s_waitcnt lgkmcnt(0)\n"
        "s_sub_u32 s93, s92, s59\n"
        "s_cmp_lt_u32 s93, s89\n"
        "s_cbranch_scc1 L_pace2f%=\n"
        "s_mov_b32 s59, s92\n"
        "L_pc2f%=:\n"
        "s_barrier\n"
        "s_waitcnt lgkmcnt(2)\n"
        "v_mfma_f32_16x16x32_f16 v[8:11], v[168:171], v[232:235], v[8:11]\n"
        "ds_read_b128 v[228:231], %[brd] offset:46080\n"
        "v_mfma_f32_16x16x32_f16 v[40:43], v[172:175], v[232:235], v[40:43]\n"
        "s_waitcnt lgkmcnt(2)\n"
        "v_mfma_f32_16x16x32_f16 v[12:15], v[168:171], v[236:239], v[12:15]\n"
        "ds_read_b128 v[232:235], %[brd] offset:47104\n"
        "v_mfma_f32_16x16x32_f16 v[44:47], v[172:175], v[236:239], v[44:47]\n"
        "s_waitcnt lgkmcnt(2)\n"
        "v_mfma_f32_16x16x32_f16 v[16:19], v[168:171], v[224:227], v[16:19]\n"
        "ds_read_b128 v[236:239], %[brd] offset:48128\n"
        "v_mfma_f32_16x16x32_f16 v[48:51], v[172:175], v[224:227], v[48:51]\n"
        "s_waitcnt lgkmcnt(2)\n"
        "v_mfma_f32_16x16x32_f16 v[20:23], v[168:171], v[228:231], v[20:23]\n"
        "ds_read_b128 v[224:227], %[brd] offset:49152\n"
        "v_mfma_f32_16x16x32_f16 v[52:55], v[172:175], v[228:231], v[52:55]\n"
        "s_waitcnt lgkmcnt(2)\n"
        "v_mfma_f32_16x16x32_f16 v[24:27], v[168:171], v[232:235], v[24:27]\n"
        "ds_read_b128 v[228:231], %[brd] offset:50176\n"
        "v_mfma_f32_16x16x32_f16 v[56:59], v[172:175], v[232:235], v[56:59]\n"
        "s_waitcnt lgkmcnt(2)\n"
        "v_mfma_f32_16x16x32_f16 v[28:31], v[168:171], v[236:239], v[28:31]\n"
        "ds_read_b128 v[232:235], %[brd] offset:51200\n"
        "v_mfma_f32_16x16x32_f16 v[60:63], v[172:175], v[236:239], v[60:63]\n"
        "s_waitcnt vmcnt(11) lgkmcnt(2)\n"
        "v_mfma_f32_16x16x32_f16 v[0:3], v[176:179], v[224:227], v[0:3]\n"
        "ds_read_b128 v[236:239], %[brd] offset:52224\n"
        "v_mfma_f32_16x16x32_f16 v[32:35], v[180:183], v[224:227], v[32:35]\n"
        "s_waitcnt lgkmcnt(2)\n"
        "v_mfma_f32_16x16x32_f16 v[4:7], v[176:179], v[228:231], v[4:7]\n"
        "ds_read_b128 v[224:227], %[brd] offset:53248\n"
        "v_mfma_f32_16x16x32_f16 v[36:39], v[180:183], v[228:231], v[36:39]\n"
        "s_waitcnt lgkmcnt(2)\n"
        "v_mfma_f32_16x16x32_f16 v[8:11], v[176:179], v[232:235], v[8:11]\n"
        "ds_read_b128 v[228:231], %[brd] offset:54272\n"
        "v_mfma_f32_16x16x32_f16 v[40:43], v[180:183], v[232:235], v[40:43]\n"
        "s_waitcnt lgkmcnt(2)\n"
        "v_mfma_f32_16x16x32_f16 v[12:15], v[176:179], v[236:239], v[12:15]\n"
        "ds_read_b128 v[232:235], %[brd] offset:55296\n"
        "v_mfma_f32_16x16x32_f16 v[44:47], v[180:183], v[236:239], v[44:47]\n"
        "s_waitcnt lgkmcnt(2)\n"
        "v_mfma_f32_16x16x32_f16 v[16:19], v[176:179], v[224:227], v[16:19]\n"
        "ds_read_b128 v[236:239], %[brd] offset:56320\n"
        "v_mfma_f32_16x16x32_f16 v[48:51], v[180:183], v[224:227], v[48:51]\n"
        "s_waitcnt lgkmcnt(2)\n"
        "v_mfma_f32_16x16x32_f16 v[20:23], v[176:179], v[228:231], v[20:23]\n"
        "ds_read_b128 v[224:227], %[brd] offset:57344\n"
        "v_mfma_f32_16x16x32_f16 v[52:55], v[180:183], v[228:231], v[52:55]\n"
        "s_waitcnt vmcnt(6)\n"
        "v_cvt_pk_f16_f32 v240, v192, v196\n"
        "v_cvt_pk_f16_f32 v241, v200, v204\n"
        "ds_write_b64 v245, v[240:241] offset:16384\n"
        "v_cvt_pk_f16_f32 v242, v193, v197\n"
        "v_cvt_pk_f16_f32 v243, v201, v205\n"
        "ds_write_b64 v245, v[242:243] offset:17408\n"
        "v_cvt_pk_f16_f32 v240, v194, v198\n"
        "v_cvt_pk_f16_f32 v241, v202, v206\n"
        "ds_write_b64 v245, v[240:241] offset:18432\n"
        "v_cvt_pk_f16_f32 v242, v195, v199\n"
        "v_cvt_pk_f16_f32 v243, v203, v207\n"
        "ds_write_b64 v245, v[242:243] offset:19456\n"
        "s_barrier\n"
        "s_lshl_b32 s82, s78, 23\n"
        "s_add_u32 s82, s82, 262144\n"
        "buffer_load_dwordx4 v[216:219], %[wvoff], %[rsW], s82 offen nt\n"
        "s_lshl_b32 s82, s78, 23\n"
        "s_add_u32 s82, s82, 393216\n"
        "buffer_load_dwordx4 v[220:223], %[wvoff], %[rsW], s82 offen nt\n"
        "s_lshl_b32 s82, s79, 23\n"
        "buffer_load_dwordx4 v[192:195], %[wvoff], %[rsW], s82 offen nt\n"
        "s_waitcnt lgkmcnt(6)\n"
        "v_mfma_f32_16x16x32_f16 v[24:27], v[176:179], v[232:235], v[24:27]\n"
        "ds_read_b128 v[228:231], %[brd] offset:58368\n"
        "v_mfma_f32_16x16x32_f16 v[56:59], v[180:183], v[232:235], v[56:59]\n"
        "s_waitcnt lgkmcnt(6)\n"
        "v_mfma_f32_16x16x32_f16 v[28:31], v[176:179], v[236:239], v[28:31]\n"
        "ds_read_b128 v[232:235], %[brd] offset:59392\n"
        "v_mfma_f32_16x16x32_f16 v[60:63], v[180:183], v[236:239], v[60:63]\n"
        "s_waitcnt lgkmcnt(6)\n"
        "v_mfma_f32_16x16x32_f16 v[0:3], v[184:187], v[224:227], v[0:3]\n"
        "ds_read_b128 v[236:239], %[brd] offset:60416\n"
        "v_mfma_f32_16x16x32_f16 v[32:35], v[188:191], v[224:227], v[32:35]\n"
        "s_waitcnt lgkmcnt(2)\n"
        "v_mfma_f32_16x16x32_f16 v[4:7], v[184:187], v[228:231], v[4:7]\n"
        "ds_read_b128 v[224:227], %[brd] offset:61440\n"
        "v_mfma_f32_16x16x32_f16 v[36:39], v[188:191], v[228:231], v[36:39]\n"
        "s_waitcnt lgkmcnt(2)\n"
        "v_mfma_f32_16x16x32_f16 v[8:11], v[184:187], v[232:235], v[8:11]\n"
        "ds_read_b128 v[228:231], %[brd] offset:62464\n"
        "v_mfma_f32_16x16x32_f16 v[40:43], v[188:191], v[232:235], v[40:43]\n"
        "s_waitcnt lgkmcnt(2)\n"
        "v_mfma_f32_16x16x32_f16 v[12:15], v[184:187], v[236:239], v[12:15]\n"
        "ds_read_b128 v[232:235], %[brd] offset:63488\n"
        "v_mfma_f32_16x16x32_f16 v[44:47], v[188:191], v[236:239], v[44:47]\n"
        "s_waitcnt lgkmcnt(2)\n"
        "v_mfma_f32_16x16x32_f16 v[16:19], v[184:187], v[224:227], v[16:19]\n"
        "ds_read_b128 v[236:239], %[brd] offset:64512\n"
        "v_mfma_f32_16x16x32_f16 v[48:51], v[188:191], v[224:227], v[48:51]\n"
        "s_waitcnt lgkmcnt(2)\n"
        "v_mfma_f32_16x16x32_f16 v[20:23], v[184:187], v[228:231], v[20:23]\n"
        "ds_read_b128 v[224:227], v244 offset:0\n"
        "v_mfma_f32_16x16x32_f16 v[52:55], v[188:191], v[228:231], v[52:55]\n"
        "s_waitcnt lgkmcnt(2)\n"
        "v_mfma_f32_16x16x32_f16 v[24:27], v[184:187], v[232:235], v[24:27]\n"
        "ds_read_b128 v[228:231], v244 offset:1024\n"
        "v_mfma_f32_16x16x32_f16 v[56:59], v[188:191], v[232:235], v[56:59]\n"
        "s_waitcnt lgkmcnt(2)\n"
        "v_mfma_f32_16x16x32_f16 v[28:31], v[184:187], v[236:239], v[28:31]\n"
        "ds_read_b128 v[232:235], v244 offset:2048\n"
        "v_mfma_f32_16x16x32_f16 v[60:63], v[188:191], v[236:239], v[60:63]\n"
        "s_lshl_b32 s80, s78, 16\n"
        "s_add_u32 s80, s80, s62\n"
        "s_add_u32 s86, s80, 4096\n"
        "buffer_load_dwordx4 v[160:163], %[avoff], %[rsA], s80 offen offset:0\n"
        "buffer_load_dwordx4 v[164:167], %[avoff], %[rsA], s80 offen offset:1024\n"
        "buffer_load_dwordx4 v[168:171], %[avoff], %[rsA], s86 offen offset:0\n"
        "buffer_load_dwordx4 v[172:175], %[avoff], %[rsA], s86 offen offset:1024\n"
        "s_lshl_b32 s81, s79, 16\n"
        "s_add_u32 s81, s81, s62\n"
        "s_add_u32 s87, s81, 4096\n"
        "buffer_load_dwordx4 v[176:179], %[avoff], %[rsA], s81 offen offset:0\n"
        "buffer_load_dwordx4 v[180:183], %[avoff], %[rsA], s81 offen offset:1024\n"
        "s_cmp_lg_u32 %[wave], 5\n"
        "s_cbranch_scc1 L_pc3f%=\n"
        "s_sub_i32 s89, s58, 16\n"
        "s_max_i32 s89, s89, 0\n"
        "s_mul_i32 s89, s89, 4\n"
        "s_sub_i32 s89, 152, s89\n"
        "s_max_i32 s89, s89, s50\n"
        "s_cmp_lt_u32 s58, 4\n"
        "s_cselect_b32 s89, 100, s89\n"
        "s_add_u32 s58, s58, 1\n"
        "L_pace3f%=:\n"
        "s_memrealtime s[92:93]\n"
        "s_waitcnt lgkmcnt(0)\n"
        "s_sub_u32 s93, s92, s59\n"
        "s_cmp_lt_u32 s93, s89\n"
        "s_cbranch_scc1 L_pace3f%=\n"
        "s_mov_b32 s59, s92\n"
        "L_pc3f%=:\n"
        "s_barrier\n"
        "s_waitcnt lgkmcnt(2)\n"
        "v_mfma_f32_16x16x32_f16 v[0:3], v[128:131], v[224:227], v[0:3]\n"
        "ds_read_b128 v[236:239], v244 offset:3072\n"
        "v_mfma_f32_16x16x32_f16 v[32:35], v[132:135], v[224:227], v[32:35]\n"
        "s_waitcnt lgkmcnt(2)\n"
        "v_mfma_f32_16x16x32_f16 v[4:7], v[128:131], v[228:231], v[4:7]\n"
        "ds_read_b128 v[224:227], v244 offset:4096\n"
        "v_mfma_f32_16x16x32_f16 v[36:39], v[132:135], v[228:231], v[36:39]\n"
        "s_waitcnt lgkmcnt(2)\n"
        "v_mfma_f32_16x16x32_f16 v[8:11], v[128:131], v[232:235], v[8:11]\n"
        "ds_read_b128 v[228:231], v244 offset:5120\n"
        "v_mfma_f32_16x16x32_f16 v[40:43], v[132:135], v[232:235], v[40:43]\n"
        "s_waitcnt lgkmcnt(2)\n"
        "v_mfma_f32_16x16x32_f16 v[12:15], v[128:131], v[236:239], v[12:15]\n"
        "ds_read_b128 v[232:235], v244 offset:6144\n"
        "v_mfma_f32_16x16x32_f16 v[44:47], v[132:135], v[236:239], v[44:47]\n"
        "s_waitcnt lgkmcnt(2)\n"
        "v_mfma_f32_16x16x32_f16 v[16:19], v[128:131], v[224:227], v[16:19]\n"
        "ds_read_b128 v[236:239], v244 offset:7168\n"
        "v_mfma_f32_16x16x32_f16 v[48:51], v[132:135], v[224:227], v[48:51]\n"
        "s_waitcnt lgkmcnt(2)\n"
        "v_mfma_f32_16x16x32_f16 v[20:23], v[128:131], v[228:231], v[20:23]\n"
        "ds_read_b128 v[224:227], v244 offset:8192\n"
        "v_mfma_f32_16x16x32_f16 v[52:55], v[132:135], v[228:231], v[52:55]\n"
        "s_waitcnt lgkmcnt(2)\n"
        "v_mfma_f32_16x16x32_f16 v[24:27], v[128:131], v[232:235], v[24:27]\n"
        "ds_read_b128 v[228:231], v244 offset:9216\n"
        "v_mfma_f32_16x16x32_f16 v[56:59], v[132:135], v[232:235], v[56:59]\n"
        "s_waitcnt lgkmcnt(2)\n"
        "v_mfma_f32_16x16x32_f16 v[28:31], v[128:131], v[236:239], v[28:31]\n"
        "ds_read_b128 v[232:235], v244 offset:10240\n"
        "v_mfma_f32_16x16x32_f16 v[60:63], v[132:135], v[236:239], v[60:63]\n"
        "s_waitcnt lgkmcnt(2)\n"
        "v_mfma_f32_16x16x32_f16 v[0:3], v[136:139], v[224:227], v[0:3]\n"
        "ds_read_b128 v[236:239], v244 offset:11264\n"
        "v_mfma_f32_16x16x32_f16 v[32:35], v[140:143], v[224:227], v[32:35]\n"
        "s_waitcnt lgkmcnt(2)\n"
        "v_mfma_f32_16x16x32_f16 v[4:7], v[136:139], v[228:231], v[4:7]\n"
        "ds_read_b128 v[224:227], v244 offset:12288\n"
        "v_mfma_f32_16x16x32_f16 v[36:39], v[140:143], v[228:231], v[36:39]\n"
        "s_waitcnt lgkmcnt(2)\n"
        "v_mfma_f32_16x16x32_f16 v[8:11], v[136:139], v[232:235], v[8:11]\n"
        "ds_read_b128 v[228:231], v244 offset:13312\n"
        "v_mfma_f32_16x16x32_f16 v[40:43], v[140:143], v[232:235], v[40:43]\n"
        "s_waitcnt lgkmcnt(2)\n"
        "v_mfma_f32_16x16x32_f16 v[12:15], v[136:139], v[236:239], v[12:15]\n"
        "ds_read_b128 v[232:235], v244 offset:14336\n"
        "v_mfma_f32_16x16x32_f16 v[44:47], v[140:143], v[236:239], v[44:47]\n"
        "s_waitcnt vmcnt(7)\n"
        "v_cvt_pk_f16_f32 v240, v208, v212\n"
        "v_cvt_pk_f16_f32 v241, v216, v220\n"
        "ds_write_b64 v245, v[240:241] offset:32768\n"
        "v_cvt_pk_f16_f32 v242, v209, v213\n"
        "v_cvt_pk_f16_f32 v243, v217, v221\n"
        "ds_write_b64 v245, v[242:243] offset:33792\n"
        "v_cvt_pk_f16_f32 v240, v210, v214\n"
        "v_cvt_pk_f16_f32 v241, v218, v222\n"
        "ds_write_b64 v245, v[240:241] offset:34816\n"
        "v_cvt_pk_f16_f32 v242, v211, v215\n"
        "v_cvt_pk_f16_f32 v243, v219, v223\n"
        "ds_write_b64 v245, v[242:243] offset:35840\n"
        "s_barrier\n"
        "s_lshl_b32 s82, s79, 23\n"
        "s_add_u32 s82, s82, 131072\n"
        "buffer_load_dwordx4 v[196:199], %[wvoff], %[rsW], s82 offen nt\n"
        "s_lshl_b32 s82, s79, 23\n"
        "s_add_u32 s82, s82, 262144\n"
        "buffer_load_dwordx4 v[200:203], %[wvoff], %[rsW], s82 offen nt\n"
        "s_lshl_b32 s82, s79, 23\n"
        "s_add_u32 s82, s82, 393216\n"
        "buffer_load_dwordx4 v[204:207], %[wvoff], %[rsW], s82 offen nt\n"
        "s_waitcnt lgkmcnt(6)\n"
        "v_mfma_f32_16x16x32_f16 v[16:19], v[136:139], v[224:227], v[16:19]\n"
        "ds_read_b128 v[236:239], v244 offset:15360\n"
        "v_mfma_f32_16x16x32_f16 v[48:51], v[140:143], v[224:227], v[48:51]\n"
        "s_waitcnt lgkmcnt(6)\n"
        "v_mfma_f32_16x16x32_f16 v[20:23], v[136:139], v[228:231], v[20:23]\n"
        "ds_read_b128 v[224:227], v244 offset:16384\n"
        "v_mfma_f32_16x16x32_f16 v[52:55], v[140:143], v[228:231], v[52:55]\n"
        "s_waitcnt lgkmcnt(6)\n"
        "v_mfma_f32_16x16x32_f16 v[24:27], v[136:139], v[232:235], v[24:27]\n"
        "ds_read_b128 v[228:231], v244 offset:17408\n"
        "v_mfma_f32_16x16x32_f16 v[56:59], v[140:143], v[232:235], v[56:59]\n"
        "s_waitcnt lgkmcnt(2)\n"
        "v_mfma_f32_16x16x32_f16 v[28:31], v[136:139], v[236:239], v[28:31]\n"
        "ds_read_b128 v[232:235], v244 offset:18432\n"
        "v_mfma_f32_16x16x32_f16 v[60:63], v[140:143], v[236:239], v[60:63]\n"
        "s_waitcnt lgkmcnt(2)\n"
        "v_mfma_f32_16x16x32_f16 v[0:3], v[144:147], v[224:227], v[0:3]\n"
        "ds_read_b128 v[236:239], v244 offset:19456\n"
        "v_mfma_f32_16x16x32_f16 v[32:35], v[148:151], v[224:227], v[32:35]\n"
        "s_waitcnt lgkmcnt(2)\n"
        "v_mfma_f32_16x16x32_f16 v[4:7], v[144:147], v[228:231], v[4:7]\n"
        "ds_read_b128 v[224:227], v244 offset:20480\n"
        "v_mfma_f32_16x16x32_f16 v[36:39], v[148:151], v[228:231], v[36:39]\n"
        "s_waitcnt lgkmcnt(2)\n"
        "v_mfma_f32_16x16x32_f16 v[8:11], v[144:147], v[232:235], v[8:11]\n"
        "ds_read_b128 v[228:231], v244 offset:21504\n"
        "v_mfma_f32_16x16x32_f16 v[40:43], v[148:151], v[232:235], v[40:43]\n"
        "s_waitcnt lgkmcnt(2)\n"
        "v_mfma_f32_16x16x32_f16 v[12:15], v[144:147], v[236:239], v[12:15]\n"
        "ds_read_b128 v[232:235], v244 offset:22528\n"
        "v_mfma_f32_16x16x32_f16 v[44:47], v[148:151], v[236:239], v[44:47]\n"
        "s_waitcnt lgkmcnt(2)\n"
        "v_mfma_f32_16x16x32_f16 v[16:19], v[144:147], v[224:227], v[16:19]\n"
        "ds_read_b128 v[236:239], v244 offset:23552\n"
        "v_mfma_f32_16x16x32_f16 v[48:51], v[148:151], v[224:227], v[48:51]\n"
        "s_waitcnt lgkmcnt(2)\n"
        "v_mfma_f32_16x16x32_f16 v[20:23], v[144:147], v[228:231], v[20:23]\n"
        "ds_read_b128 v[224:227], v244 offset:24576\n"
        "v_mfma_f32_16x16x32_f16 v[52:55], v[148:151], v[228:231], v[52:55]\n"
        "buffer_load_dwordx4 v[184:187], %[avoff], %[rsA], s87 offen offset:0\n"
        "buffer_load_dwordx4 v[188:191], %[avoff], %[rsA], s87 offen offset:1024\n"
        "s_lshl_b32 s80, s72, 16\n"
        "s_add_u32 s80, s80, s62\n"
        "s_add_u32 s80, s80, 2048\n"
        "s_add_u32 s86, s80, 4096\n"
        "buffer_load_dwordx4 v[128:131], %[avoff], %[rsA], s80 offen offset:0\n"
        "buffer_load_dwordx4 v[132:135], %[avoff], %[rsA], s80 offen offset:1024\n"
        "buffer_load_dwordx4 v[136:139], %[avoff], %[rsA], s86 offen offset:0\n"
        "buffer_load_dwordx4 v[140:143], %[avoff], %[rsA], s86 offen offset:1024\n"
        "s_cmp_lg_u32 %[wave], 5\n"
        "s_cbranch_scc1 L_pc4f%=\n"
        "s_sub_i32 s89, s58, 16\n"
        "s_max_i32 s89, s89, 0\n"
        "s_mul_i32 s89, s89, 4\n"
        "s_sub_i32 s89, 152, s89\n"
        "s_max_i32 s89, s89, s50\n"
        "s_cmp_lt_u32 s58, 4\n"
        "s_cselect_b32 s89, 100, s89\n"
        "s_add_u32 s58, s58, 1\n"
        "L_pace4f%=:\n"
        "s_memrealtime s[92:93]\n"
        "s_waitcnt lgkmcnt(0)\n"
        "s_sub_u32 s93, s92, s59\n"
        "s_cmp_lt_u32 s93, s89\n"
        "s_cbranch_scc1 L_pace4f%=\n"
        "s_mov_b32 s59, s92\n"
        "L_pc4f%=:\n"
        "s_barrier\n"
        "s_waitcnt lgkmcnt(2)\n"
        "v_mfma_f32_16x16x32_f16 v[24:27], v[144:147], v[232:235], v[24:27]\n"
        "ds_read_b128 v[228:231], v244 offset:25600\n"
        "v_mfma_f32_16x16x32_f16 v[56:59], v[148:151], v[232:235], v[56:59]\n"
        "s_waitcnt lgkmcnt(2)\n"
        "v_mfma_f32_16x16x32_f16 v[28:31], v[144:147], v[236:239], v[28:31]\n"
        "ds_read_b128 v[232:235], v244 offset:26624\n"
        "v_mfma_f32_16x16x32_f16 v[60:63], v[148:151], v[236:239], v[60:63]\n"
        "s_waitcnt lgkmcnt(2)\n"
        "v_mfma_f32_16x16x32_f16 v[0:3], v[152:155], v[224:227], v[0:3]\n"
        "ds_read_b128 v[236:239], v244 offset:27648\n"
        "v_mfma_f32_16x16x32_f16 v[32:35], v[156:159], v[224:227], v[32:35]\n"
        "s_waitcnt lgkmcnt(2)\n"
        "v_mfma_f32_16x16x32_f16 v[4:7], v[152:155], v[228:231], v[4:7]\n"
        "ds_read_b128 v[224:227], v244 offset:28672\n"
        "v_mfma_f32_16x16x32_f16 v[36:39], v[156:159], v[228:231], v[36:39]\n"
        "s_waitcnt lgkmcnt(2)\n"
        "v_mfma_f32_16x16x32_f16 v[8:11], v[152:155], v[232:235], v[8:11]\n"
        "ds_read_b128 v[228:231], v244 offset:29696\n"
        "v_mfma_f32_16x16x32_f16 v[40:43], v[156:159], v[232:235], v[40:43]\n"
        "s_waitcnt lgkmcnt(2)\n"
        "v_mfma_f32_16x16x32_f16 v[12:15], v[152:155], v[236:239], v[12:15]\n"
        "ds_read_b128 v[232:235], v244 offset:30720\n"
        "v_mfma_f32_16x16x32_f16 v[44:47], v[156:159], v[236:239], v[44:47]\n"
        "s_waitcnt lgkmcnt(2)\n"
        "v_mfma_f32_16x16x32_f16 v[16:19], v[152:155], v[224:227], v[16:19]\n"
        "ds_read_b128 v[236:239], v244 offset:31744\n"
        "v_mfma_f32_16x16x32_f16 v[48:51], v[156:159], v[224:227], v[48:51]\n"
        "s_waitcnt lgkmcnt(2)\n"
        "v_mfma_f32_16x16x32_f16 v[20:23], v[152:155], v[228:231], v[20:23]\n"
        "ds_read_b128 v[224:227], v244 offset:32768\n"
        "v_mfma_f32_16x16x32_f16 v[52:55], v[156:159], v[228:231], v[52:55]\n"
        "s_waitcnt lgkmcnt(2)\n"
        "v_mfma_f32_16x16x32_f16 v[24:27], v[152:155], v[232:235], v[24:27]\n"
        "ds_read_b128 v[228:231], v244 offset:33792\n"
        "v_mfma_f32_16x16x32_f16 v[56:59], v[156:159], v[232:235], v[56:59]\n"
        "s_waitcnt lgkmcnt(2)\n"
        "v_mfma_f32_16x16x32_f16 v[28:31], v[152:155], v[236:239], v[28:31]\n"
        "ds_read_b128 v[232:235], v244 offset:34816\n"
        "v_mfma_f32_16x16x32_f16 v[60:63], v[156:159], v[236:239], v[60:63]\n"
        "s_waitcnt vmcnt(6)\n"
        "v_cvt_pk_f16_f32 v240, v192, v196\n"
        "v_cvt_pk_f16_f32 v241, v200, v204\n"
        "ds_write_b64 v245, v[240:241] offset:49152\n"
        "v_cvt_pk_f16_f32 v242, v193, v197\n"
        "v_cvt_pk_f16_f32 v243, v201, v205\n"
        "ds_write_b64 v245, v[242:243] offset:50176\n"
        "v_cvt_pk_f16_f32 v240, v194, v198\n"
        "v_cvt_pk_f16_f32 v241, v202, v206\n"
        "ds_write_b64 v245, v[240:241] offset:51200\n"
        "v_cvt_pk_f16_f32 v242, v195, v199\n"
        "v_cvt_pk_f16_f32 v243, v203, v207\n"
        "ds_write_b64 v245, v[242:243] offset:52224\n"
        "s_waitcnt lgkmcnt(6)\n"
        "v_mfma_f32_16x16x32_f16 v[0:3], v[160:163], v[224:227], v[0:3]\n"
        "ds_read_b128 v[236:239], v244 offset:35840\n"
        "v_mfma_f32_16x16x32_f16 v[32:35], v[164:167], v[224:227], v[32:35]\n"
        "s_waitcnt lgkmcnt(6)\n"
        "v_mfma_f32_16x16x32_f16 v[4:7], v[160:163], v[228:231], v[4:7]\n"
        "ds_read_b128 v[224:227], v244 offset:36864\n"
        "v_mfma_f32_16x16x32_f16 v[36:39], v[164:167], v[228:231], v[36:39]\n"
        "s_waitcnt lgkmcnt(6)\n"
        "v_mfma_f32_16x16x32_f16 v[8:11], v[160:163], v[232:235], v[8:11]\n"
        "ds_read_b128 v[228:231], v244 offset:37888\n"
        "v_mfma_f32_16x16x32_f16 v[40:43], v[164:167], v[232:235], v[40:43]\n"
        "s_waitcnt lgkmcnt(2)\n"
        "v_mfma_f32_16x16x32_f16 v[12:15], v[160:163], v[236:239], v[12:15]\n"
        "ds_read_b128 v[232:235], v244 offset:38912\n"
        "v_mfma_f32_16x16x32_f16 v[44:47], v[164:167], v[236:239], v[44:47]\n"
        "s_waitcnt lgkmcnt(2)\n"
        "v_mfma_f32_16x16x32_f16 v[16:19], v[160:163], v[224:227], v[16:19]\n"
        "ds_read_b128 v[236:239], v244 offset:39936\n"
        "v_mfma_f32_16x16x32_f16 v[48:51], v[164:167], v[224:227], v[48:51]\n"
        "s_waitcnt lgkmcnt(2)\n"
        "v_mfma_f32_16x16x32_f16 v[20:23], v[160:163], v[228:231], v[20:23]\n"
        "ds_read_b128 v[224:227], v244 offset:40960\n"
        "v_mfma_f32_16x16x32_f16 v[52:55], v[164:167], v[228:231], v[52:55]\n"
        "s_waitcnt lgkmcnt(2)\n"
        "v_mfma_f32_16x16x32_f16 v[24:27], v[160:163], v[232:235], v[24:27]\n"
        "ds_read_b128 v[228:231], v244 offset:41984\n"
        "v_mfma_f32_16x16x32_f16 v[56:59], v[164:167], v[232:235], v[56:59]\n"
        "s_waitcnt lgkmcnt(2)\n"
        "v_mfma_f32_16x16x32_f16 v[28:31], v[160:163], v[236:239], v[28:31]\n"
        "ds_read_b128 v[232:235], v244 offset:43008\n"
        "v_mfma_f32_16x16x32_f16 v[60:63], v[164:167], v[236:239], v[60:63]\n"
        "s_waitcnt lgkmcnt(2)\n"
        "v_mfma_f32_16x16x32_f16 v[0:3], v[168:171], v[224:227], v[0:3]\n"
        "ds_read_b128 v[236:239], v244 offset:44032\n"
        "v_mfma_f32_16x16x32_f16 v[32:35], v[172:175], v[224:227], v[32:35]\n"
        "s_waitcnt lgkmcnt(2)\n"
        "v_mfma_f32_16x16x32_f16 v[4:7], v[168:171], v[228:231], v[4:7]\n"
        "ds_read_b128 v[224:227], v244 offset:45056\n"
        "v_mfma_f32_16x16x32_f16 v[36:39], v[172:175], v[228:231], v[36:39]\n"
        "s_lshl_b32 s81, s73, 16\n"
        "s_add_u32 s81, s81, s62\n"
        "s_add_u32 s81, s81, 2048\n"
        "s_add_u32 s87, s81, 4096\n"
        "buffer_load_dwordx4 v[144:147], %[avoff], %[rsA], s81 offen offset:0\n"
        "buffer_load_dwordx4 v[148:151], %[avoff], %[rsA], s81 offen offset:1024\n"
        "buffer_load_dwordx4 v[152:155], %[avoff], %[rsA], s87 offen offset:0\n"
        "buffer_load_dwordx4 v[156:159], %[avoff], %[rsA], s87 offen offset:1024\n"
        "s_cmp_lg_u32 %[wave], 5\n"
        "s_cbranch_scc1 L_pc5f%=\n"
        "s_sub_i32 s89, s58, 16\n"
        "s_max_i32 s89, s89, 0\n"
        "s_mul_i32 s89, s89, 4\n"
        "s_sub_i32 s89, 152, s89\n"
        "s_max_i32 s89, s89, s50\n"
        "s_cmp_lt_u32 s58, 4\n"
        "s_cselect_b32 s89, 100, s89\n"
        "s_add_u32 s58, s58, 1\n"
        "L_pace5f%=:\n"
        "s_memrealtime s[92:93]\n"
        "s_waitcnt lgkmcnt(0)\n"
        "s_sub_u32 s93, s92, s59\n"
        "s_cmp_lt_u32 s93, s89\n"
        "s_cbranch_scc1 L_pace5f%=\n"
        "s_mov_b32 s59, s92\n"
        "L_pc5f%=:\n"
        "s_barrier\n"
        "s_waitcnt lgkmcnt(2)\n"
        "v_mfma_f32_16x16x32_f16 v[8:11], v[168:171], v[232:235], v[8:11]\n"
        "ds_read_b128 v[228:231], v244 offset:46080\n"
        "v_mfma_f32_16x16x32_f16 v[40:43], v[172:175], v[232:235], v[40:43]\n"
        "s_waitcnt lgkmcnt(2)\n"
        "v_mfma_f32_16x16x32_f16 v[12:15], v[168:171], v[236:239], v[12:15]\n"
        "ds_read_b128 v[232:235], v244 offset:47104\n"
        "v_mfma_f32_16x16x32_f16 v[44:47], v[172:175], v[236:239], v[44:47]\n"
        "s_waitcnt lgkmcnt(2)\n"
        "v_mfma_f32_16x16x32_f16 v[16:19], v[168:171], v[224:227], v[16:19]\n"
        "ds_read_b128 v[236:239], v244 offset:48128\n"
        "v_mfma_f32_16x16x32_f16 v[48:51], v[172:175], v[224:227], v[48:51]\n"
        "s_waitcnt lgkmcnt(2)\n"
        "v_mfma_f32_16x16x32_f16 v[20:23], v[168:171], v[228:231], v[20:23]\n"
        "ds_read_b128 v[224:227], v244 offset:49152\n"
        "v_mfma_f32_16x16x32_f16 v[52:55], v[172:175], v[228:231], v[52:55]\n"
        "s_waitcnt lgkmcnt(2)\n"
        "v_mfma_f32_16x16x32_f16 v[24:27], v[168:171], v[232:235], v[24:27]\n"
        "ds_read_b128 v[228:231], v244 offset:50176\n"
        "v_mfma_f32_16x16x32_f16 v[56:59], v[172:175], v[232:235], v[56:59]\n"
        "s_waitcnt lgkmcnt(2)\n"
        "v_mfma_f32_16x16x32_f16 v[28:31], v[168:171], v[236:239], v[28:31]\n"
        "ds_read_b128 v[232:235], v244 offset:51200\n"
        "v_mfma_f32_16x16x32_f16 v[60:63], v[172:175], v[236:239], v[60:63]\n"
        "s_waitcnt vmcnt(8) lgkmcnt(2)\n"
        "v_mfma_f32_16x16x32_f16 v[0:3], v[176:179], v[224:227], v[0:3]\n"
        "ds_read_b128 v[236:239], v244 offset:52224\n"
        "v_mfma_f32_16x16x32_f16 v[32:35], v[180:183], v[224:227], v[32:35]\n"
        "s_waitcnt lgkmcnt(2)\n"
        "v_mfma_f32_16x16x32_f16 v[4:7], v[176:179], v[228:231], v[4:7]\n"
        "ds_read_b128 v[224:227], v244 offset:53248\n"
        "v_mfma_f32_16x16x32_f16 v[36:39], v[180:183], v[228:231], v[36:39]\n"
        "s_waitcnt lgkmcnt(2)\n"
        "v_mfma_f32_16x16x32_f16 v[8:11], v[176:179], v[232:235], v[8:11]\n"
        "ds_read_b128 v[228:231], v244 offset:54272\n"
        "v_mfma_f32_16x16x32_f16 v[40:43], v[180:183], v[232:235], v[40:43]\n"
        "s_waitcnt lgkmcnt(2)\n"
        "v_mfma_f32_16x16x32_f16 v[12:15], v[176:179], v[236:239], v[12:15]\n"
        "ds_read_b128 v[232:235], v244 offset:55296\n"
        "v_mfma_f32_16x16x32_f16 v[44:47], v[180:183], v[236:239], v[44:47]\n"
        "s_waitcnt lgkmcnt(2)\n"
        "v_mfma_f32_16x16x32_f16 v[16:19], v[176:179], v[224:227], v[16:19]\n"
        "ds_read_b128 v[236:239], v244 offset:56320\n"
        "v_mfma_f32_16x16x32_f16 v[48:51], v[180:183], v[224:227], v[48:51]\n"
        "s_waitcnt lgkmcnt(2)\n"
        "v_mfma_f32_16x16x32_f16 v[20:23], v[176:179], v[228:231], v[20:23]\n"
        "ds_read_b128 v[224:227], v244 offset:57344\n"
        "v_mfma_f32_16x16x32_f16 v[52:55], v[180:183], v[228:231], v[52:55]\n"
        "s_waitcnt lgkmcnt(2)\n"
        "v_mfma_f32_16x16x32_f16 v[24:27], v[176:179], v[232:235], v[24:27]\n"
        "ds_read_b128 v[228:231], v244 offset:58368\n"
        "v_mfma_f32_16x16x32_f16 v[56:59], v[180:183], v[232:235], v[56:59]\n"
        "s_waitcnt lgkmcnt(2)\n"
        "v_mfma_f32_16x16x32_f16 v[28:31], v[176:179], v[236:239], v[28:31]\n"
        "ds_read_b128 v[232:235], v244 offset:59392\n"
        "v_mfma_f32_16x16x32_f16 v[60:63], v[180:183], v[236:239], v[60:63]\n"
        "s_waitcnt lgkmcnt(2)\n"
        "v_mfma_f32_16x16x32_f16 v[0:3], v[184:187], v[224:227], v[0:3]\n"
        "ds_read_b128 v[236:239], v244 offset:60416\n"
        "v_mfma_f32_16x16x32_f16 v[32:35], v[188:191], v[224:227], v[32:35]\n"
        "s_waitcnt lgkmcnt(2)\n"
        "v_mfma_f32_16x16x32_f16 v[4:7], v[184:187], v[228:231], v[4:7]\n"
        "ds_read_b128 v[224:227], v244 offset:61440\n"
        "v_mfma_f32_16x16x32_f16 v[36:39], v[188:191], v[228:231], v[36:39]\n"
        "s_waitcnt lgkmcnt(2)\n"
        "v_mfma_f32_16x16x32_f16 v[8:11], v[184:187], v[232:235], v[8:11]\n"
        "ds_read_b128 v[228:231], v244 offset:62464\n"
        "v_mfma_f32_16x16x32_f16 v[40:43], v[188:191], v[232:235], v[40:43]\n"
        "s_waitcnt lgkmcnt(2)\n"
        "v_mfma_f32_16x16x32_f16 v[12:15], v[184:187], v[236:239], v[12:15]\n"
        "ds_read_b128 v[232:235], v244 offset:63488\n"
        "v_mfma_f32_16x16x32_f16 v[44:47], v[188:191], v[236:239], v[44:47]\n"
        "s_waitcnt lgkmcnt(2)\n"
        "v_mfma_f32_16x16x32_f16 v[16:19], v[184:187], v[224:227], v[16:19]\n"
        "ds_read_b128 v[236:239], v244 offset:64512\n"
        "v_mfma_f32_16x16x32_f16 v[48:51], v[188:191], v[224:227], v[48:51]\n"
        "s_waitcnt lgkmcnt(2)\n"
        "v_mfma_f32_16x16x32_f16 v[20:23], v[184:187], v[228:231], v[20:23]\n"
        "ds_read_b128 v[224:227], %[brd] offset:0\n"
        "v_mfma_f32_16x16x32_f16 v[52:55], v[188:191], v[228:231], v[52:55]\n"
        "s_waitcnt lgkmcnt(2)\n"
        "v_mfma_f32_16x16x32_f16 v[24:27], v[184:187], v[232:235], v[24:27]\n"
        "ds_read_b128 v[228:231], %[brd] offset:1024\n"
        "v_mfma_f32_16x16x32_f16 v[56:59], v[188:191], v[232:235], v[56:59]\n"
        "s_waitcnt lgkmcnt(2)\n"
        "v_mfma_f32_16x16x32_f16 v[28:31], v[184:187], v[236:239], v[28:31]\n"
        "ds_read_b128 v[232:235], %[brd] offset:2048\n"
        "v_mfma_f32_16x16x32_f16 v[60:63], v[188:191], v[236:239], v[60:63]\n"
        "s_lshl_b32 s80, s74, 16\n"
        "s_add_u32 s80, s80, s62\n"
        "s_add_u32 s80, s80, 2048\n"
        "s_add_u32 s86, s80, 4096\n"
        "buffer_load_dwordx4 v[160:163], %[avoff], %[rsA], s80 offen offset:0\n"
        "buffer_load_dwordx4 v[164:167], %[avoff], %[rsA], s80 offen offset:1024\n"
        "buffer_load_dwordx4 v[168:171], %[avoff], %[rsA], s86 offen offset:0\n"
        "buffer_load_dwordx4 v[172:175], %[avoff], %[rsA], s86 offen offset:1024\n"
        "s_lshl_b32 s81, s75, 16\n"
        "s_add_u32 s81, s81, s62\n"
        "s_add_u32 s81, s81, 2048\n"
        "s_add_u32 s87, s81, 4096\n"
        "buffer_load_dwordx4 v[176:179], %[avoff], %[rsA], s81 offen offset:0\n"
        "buffer_load_dwordx4 v[180:183], %[avoff], %[rsA], s81 offen offset:1024\n"
        "s_waitcnt vmcnt(10) lgkmcnt(2)\n"
        "v_mfma_f32_16x16x32_f16 v[64:67], v[128:131], v[224:227], v[64:67]\n"
        "ds_read_b128 v[236:239], %[brd] offset:3072\n"
        "v_mfma_f32_16x16x32_f16 v[96:99], v[132:135], v[224:227], v[96:99]\n"
        "s_waitcnt lgkmcnt(2)\n"
        "v_mfma_f32_16x16x32_f16 v[68:71], v[128:131], v[228:231], v[68:71]\n"
        "ds_read_b128 v[224:227], %[brd] offset:4096\n"
        "v_mfma_f32_16x16x32_f16 v[100:103], v[132:135], v[228:231], v[100:103]\n"
        "s_waitcnt lgkmcnt(2)\n"
        "v_mfma_f32_16x16x32_f16 v[72:75], v[128:131], v[232:235], v[72:75]\n"
        "ds_read_b128 v[228:231], %[brd] offset:5120\n"
        "v_mfma_f32_16x16x32_f16 v[104:107], v[132:135], v[232:235], v[104:107]\n"
        "s_waitcnt lgkmcnt(2)\n"
        "v_mfma_f32_16x16x32_f16 v[76:79], v[128:131], v[236:239], v[76:79]\n"
        "ds_read_b128 v[232:235], %[brd] offset:6144\n"
        "v_mfma_f32_16x16x32_f16 v[108:111], v[132:135], v[236:239], v[108:111]\n"
        "s_mov_b32 s88, 0\n"
        "v_max_f32 v192, 0, v0\n"
        "v_max_f32 v193, 0, v4\n"
        "v_max_f32 v194, 0, v8\n"
        "v_max_f32 v195, 0, v12\n"
        "buffer_store_dwordx4 v[192:195], %[ooff], %[rsO], s88 offen offset:0 nt\n"
        "s_waitcnt lgkmcnt(2)\n"
        "v_mfma_f32_16x16x32_f16 v[80:83], v[128:131], v[224:227], v[80:83]\n"
        "ds_read_b128 v[236:239], %[brd] offset:7168\n"
        "v_mfma_f32_16x16x32_f16 v[112:115], v[132:135], v[224:227], v[112:115]\n"
        "s_waitcnt lgkmcnt(2)\n"
        "v_mfma_f32_16x16x32_f16 v[84:87], v[128:131], v[228:231], v[84:87]\n"
        "ds_read_b128 v[224:227], %[brd] offset:8192\n"
        "v_mfma_f32_16x16x32_f16 v[116:119], v[132:135], v[228:231], v[116:119]\n"
        "s_waitcnt lgkmcnt(2)\n"
        "v_mfma_f32_16x16x32_f16 v[88:91], v[128:131], v[232:235], v[88:91]\n"
        "ds_read_b128 v[228:231], %[brd] offset:9216\n"
        "v_mfma_f32_16x16x32_f16 v[120:123], v[132:135], v[232:235], v[120:123]\n"
        "s_waitcnt lgkmcnt(2)\n"
        "v_mfma_f32_16x16x32_f16 v[92:95], v[128:131], v[236:239], v[92:95]\n"
        "ds_read_b128 v[232:235], %[brd] offset:10240\n"
        "v_mfma_f32_16x16x32_f16 v[124:127], v[132:135], v[236:239], v[124:127]\n"
        "s_mov_b32 s89, 0\n"
        "v_max_f32 v196, 0, v16\n"
        "v_max_f32 v197, 0, v20\n"
        "v_max_f32 v198, 0, v24\n"
        "v_max_f32 v199, 0, v28\n"
        "buffer_store_dwordx4 v[196:199], %[ooff], %[rsO], s89 offen offset:256 nt\n"
        "s_waitcnt lgkmcnt(2)\n"
        "v_mfma_f32_16x16x32_f16 v[64:67], v[136:139], v[224:227], v[64:67]\n"
        "ds_read_b128 v[236:239], %[brd] offset:11264\n"
        "v_mfma_f32_16x16x32_f16 v[96:99], v[140:143], v[224:227], v[96:99]\n"
        "s_waitcnt lgkmcnt(2)\n"
        "v_mfma_f32_16x16x32_f16 v[68:71], v[136:139], v[228:231], v[68:71]\n"
        "ds_read_b128 v[224:227], %[brd] offset:12288\n"
        "v_mfma_f32_16x16x32_f16 v[100:103], v[140:143], v[228:231], v[100:103]\n"
        "s_waitcnt lgkmcnt(2)\n"
        "v_mfma_f32_16x16x32_f16 v[72:75], v[136:139], v[232:235], v[72:75]\n"
        "ds_read_b128 v[228:231], %[brd] offset:13312\n"
        "v_mfma_f32_16x16x32_f16 v[104:107], v[140:143], v[232:235], v[104:107]\n"
        "s_waitcnt lgkmcnt(2)\n"
        "v_mfma_f32_16x16x32_f16 v[76:79], v[136:139], v[236:239], v[76:79]\n"
        "ds_read_b128 v[232:235], %[brd] offset:14336\n"
        "v_mfma_f32_16x16x32_f16 v[108:111], v[140:143], v[236:239], v[108:111]\n"
        "s_mov_b32 s88, 131072\n"
        "v_max_f32 v200, 0, v1\n"
        "v_max_f32 v201, 0, v5\n"
        "v_max_f32 v202, 0, v9\n"
        "v_max_f32 v203, 0, v13\n"
        "buffer_store_dwordx4 v[200:203], %[ooff], %[rsO], s88 offen offset:0 nt\n"
        "s_waitcnt lgkmcnt(2)\n"
        "v_mfma_f32_16x16x32_f16 v[80:83], v[136:139], v[224:227], v[80:83]\n"
        "ds_read_b128 v[236:239], %[brd] offset:15360\n"
        "v_mfma_f32_16x16x32_f16 v[112:115], v[140:143], v[224:227], v[112:115]\n"
        "s_waitcnt lgkmcnt(2)\n"
        "v_mfma_f32_16x16x32_f16 v[84:87], v[136:139], v[228:231], v[84:87]\n"
        "ds_read_b128 v[224:227], %[brd] offset:16384\n"
        "v_mfma_f32_16x16x32_f16 v[116:119], v[140:143], v[228:231], v[116:119]\n"
        "s_waitcnt lgkmcnt(2)\n"
        "v_mfma_f32_16x16x32_f16 v[88:91], v[136:139], v[232:235], v[88:91]\n"
        "ds_read_b128 v[228:231], %[brd] offset:17408\n"
        "v_mfma_f32_16x16x32_f16 v[120:123], v[140:143], v[232:235], v[120:123]\n"
        "s_waitcnt lgkmcnt(2)\n"
        "v_mfma_f32_16x16x32_f16 v[92:95], v[136:139], v[236:239], v[92:95]\n"
        "ds_read_b128 v[232:235], %[brd] offset:18432\n"
        "v_mfma_f32_16x16x32_f16 v[124:127], v[140:143], v[236:239], v[124:127]\n"
        "s_mov_b32 s89, 131072\n"
        "v_max_f32 v204, 0, v17\n"
        "v_max_f32 v205, 0, v21\n"
        "v_max_f32 v206, 0, v25\n"
        "v_max_f32 v207, 0, v29\n"
        "buffer_store_dwordx4 v[204:207], %[ooff], %[rsO], s89 offen offset:256 nt\n"
        "s_waitcnt vmcnt(10) lgkmcnt(2)\n"
        "v_mfma_f32_16x16x32_f16 v[64:67], v[144:147], v[224:227], v[64:67]\n"
        "ds_read_b128 v[236:239], %[brd] offset:19456\n"
        "v_mfma_f32_16x16x32_f16 v[96:99], v[148:151], v[224:227], v[96:99]\n"
        "s_waitcnt lgkmcnt(2)\n"
        "v_mfma_f32_16x16x32_f16 v[68:71], v[144:147], v[228:231], v[68:71]\n"
        "ds_read_b128 v[224:227], %[brd] offset:20480\n"
        "v_mfma_f32_16x16x32_f16 v[100:103], v[148:151], v[228:231], v[100:103]\n"
        "s_waitcnt lgkmcnt(2)\n"
        "v_mfma_f32_16x16x32_f16 v[72:75], v[144:147], v[232:235], v[72:75]\n"
        "ds_read_b128 v[228:231], %[brd] offset:21504\n"
        "v_mfma_f32_16x16x32_f16 v[104:107], v[148:151], v[232:235], v[104:107]\n"
        "s_waitcnt lgkmcnt(2)\n"
        "v_mfma_f32_16x16x32_f16 v[76:79], v[144:147], v[236:239], v[76:79]\n"
        "ds_read_b128 v[232:235], %[brd] offset:22528\n"
        "v_mfma_f32_16x16x32_f16 v[108:111], v[148:151], v[236:239], v[108:111]\n"
        "s_mov_b32 s88, 262144\n"
        "v_max_f32 v208, 0, v2\n"
        "v_max_f32 v209, 0, v6\n"
        "v_max_f32 v210, 0, v10\n"
        "v_max_f32 v211, 0, v14\n"
        "buffer_store_dwordx4 v[208:211], %[ooff], %[rsO], s88 offen offset:0 nt\n"
        "s_waitcnt lgkmcnt(2)\n"
        "v_mfma_f32_16x16x32_f16 v[80:83], v[144:147], v[224:227], v[80:83]\n"
        "ds_read_b128 v[236:239], %[brd] offset:23552\n"
        "v_mfma_f32_16x16x32_f16 v[112:115], v[148:151], v[224:227], v[112:115]\n"
        "s_waitcnt lgkmcnt(2)\n"
        "v_mfma_f32_16x16x32_f16 v[84:87], v[144:147], v[228:231], v[84:87]\n"
        "ds_read_b128 v[224:227], %[brd] offset:24576\n"
        "v_mfma_f32_16x16x32_f16 v[116:119], v[148:151], v[228:231], v[116:119]\n"
        "buffer_load_dwordx4 v[184:187], %[avoff], %[rsA], s87 offen offset:0\n"
        "buffer_load_dwordx4 v[188:191], %[avoff], %[rsA], s87 offen offset:1024\n"
        "s_lshl_b32 s80, s76, 16\n"
        "s_add_u32 s80, s80, s62\n"
        "s_add_u32 s80, s80, 2048\n"
        "s_add_u32 s86, s80, 4096\n"
        "buffer_load_dwordx4 v[128:131], %[avoff], %[rsA], s80 offen offset:0\n"
        "buffer_load_dwordx4 v[132:135], %[avoff], %[rsA], s80 offen offset:1024\n"
        "buffer_load_dwordx4 v[136:139], %[avoff], %[rsA], s86 offen offset:0\n"
        "buffer_load_dwordx4 v[140:143], %[avoff], %[rsA], s86 offen offset:1024\n"
        "s_waitcnt lgkmcnt(2)\n"
        "v_mfma_f32_16x16x32_f16 v[88:91], v[144:147], v[232:235], v[88:91]\n"
        "ds_read_b128 v[228:231], %[brd] offset:25600\n"
        "v_mfma_f32_16x16x32_f16 v[120:123], v[148:151], v[232:235], v[120:123]\n"
        "s_waitcnt lgkmcnt(2)\n"
        "v_mfma_f32_16x16x32_f16 v[92:95], v[144:147], v[236:239], v[92:95]\n"
        "ds_read_b128 v[232:235], %[brd] offset:26624\n"
        "v_mfma_f32_16x16x32_f16 v[124:127], v[148:151], v[236:239], v[124:127]\n"
        "s_mov_b32 s89, 262144\n"
        "v_max_f32 v212, 0, v18\n"
        "v_max_f32 v213, 0, v22\n"
        "v_max_f32 v214, 0, v26\n"
        "v_max_f32 v215, 0, v30\n"
        "buffer_store_dwordx4 v[212:215], %[ooff], %[rsO], s89 offen offset:256 nt\n"
        "s_waitcnt lgkmcnt(2)\n"
        "v_mfma_f32_16x16x32_f16 v[64:67], v[152:155], v[224:227], v[64:67]\n"
        "ds_read_b128 v[236:239], %[brd] offset:27648\n"
        "v_mfma_f32_16x16x32_f16 v[96:99], v[156:159], v[224:227], v[96:99]\n"
        "s_waitcnt lgkmcnt(2)\n"
        "v_mfma_f32_16x16x32_f16 v[68:71], v[152:155], v[228:231], v[68:71]\n"
        "ds_read_b128 v[224:227], %[brd] offset:28672\n"
        "v_mfma_f32_16x16x32_f16 v[100:103], v[156:159], v[228:231], v[100:103]\n"
        "s_waitcnt lgkmcnt(2)\n"
        "v_mfma_f32_16x16x32_f16 v[72:75], v[152:155], v[232:235], v[72:75]\n"
        "ds_read_b128 v[228:231], %[brd] offset:29696\n"
        "v_mfma_f32_16x16x32_f16 v[104:107], v[156:159], v[232:235], v[104:107]\n"
        "s_waitcnt lgkmcnt(2)\n"
        "v_mfma_f32_16x16x32_f16 v[76:79], v[152:155], v[236:239], v[76:79]\n"
        "ds_read_b128 v[232:235], %[brd] offset:30720\n"
        "v_mfma_f32_16x16x32_f16 v[108:111], v[156:159], v[236:239], v[108:111]\n"
        "s_mov_b32 s88, 393216\n"
        "v_max_f32 v216, 0, v3\n"
        "v_max_f32 v217, 0, v7\n"
        "v_max_f32 v218, 0, v11\n"
        "v_max_f32 v219, 0, v15\n"
        "buffer_store_dwordx4 v[216:219], %[ooff], %[rsO], s88 offen offset:0 nt\n"
        "s_waitcnt lgkmcnt(2)\n"
        "v_mfma_f32_16x16x32_f16 v[80:83], v[152:155], v[224:227], v[80:83]\n"
        "ds_read_b128 v[236:239], %[brd] offset:31744\n"
        "v_mfma_f32_16x16x32_f16 v[112:115], v[156:159], v[224:227], v[112:115]\n"
        "s_waitcnt lgkmcnt(2)\n"
        "v_mfma_f32_16x16x32_f16 v[84:87], v[152:155], v[228:231], v[84:87]\n"
        "ds_read_b128 v[224:227], %[brd] offset:32768\n"
        "v_mfma_f32_16x16x32_f16 v[116:119], v[156:159], v[228:231], v[116:119]\n"
        "s_waitcnt lgkmcnt(2)\n"
        "v_mfma_f32_16x16x32_f16 v[88:91], v[152:155], v[232:235], v[88:91]\n"
        "ds_read_b128 v[228:231], %[brd] offset:33792\n"
        "v_mfma_f32_16x16x32_f16 v[120:123], v[156:159], v[232:235], v[120:123]\n"
        "s_waitcnt lgkmcnt(2)\n"
        "v_mfma_f32_16x16x32_f16 v[92:95], v[152:155], v[236:239], v[92:95]\n"
        "ds_read_b128 v[232:235], %[brd] offset:34816\n"
        "v_mfma_f32_16x16x32_f16 v[124:127], v[156:159], v[236:239], v[124:127]\n"
        "s_mov_b32 s89, 393216\n"
        "v_max_f32 v220, 0, v19\n"
        "v_max_f32 v221, 0, v23\n"
        "v_max_f32 v222, 0, v27\n"
        "v_max_f32 v223, 0, v31\n"
        "buffer_store_dwordx4 v[220:223], %[ooff], %[rsO], s89 offen offset:256 nt\n"
        "s_waitcnt vmcnt(16) lgkmcnt(2)\n"
        "v_mfma_f32_16x16x32_f16 v[64:67], v[160:163], v[224:227], v[64:67]\n"
        "ds_read_b128 v[236:239], %[brd] offset:35840\n"
        "v_mfma_f32_16x16x32_f16 v[96:99], v[164:167], v[224:227], v[96:99]\n"
        "s_waitcnt lgkmcnt(2)\n"
        "v_mfma_f32_16x16x32_f16 v[68:71], v[160:163], v[228:231], v[68:71]\n"
        "ds_read_b128 v[224:227], %[brd] offset:36864\n"
        "v_mfma_f32_16x16x32_f16 v[100:103], v[164:167], v[228:231], v[100:103]\n"
        "s_waitcnt lgkmcnt(2)\n"
        "v_mfma_f32_16x16x32_f16 v[72:75], v[160:163], v[232:235], v[72:75]\n"
        "ds_read_b128 v[228:231], %[brd] offset:37888\n"
        "v_mfma_f32_16x16x32_f16 v[104:107], v[164:167], v[232:235], v[104:107]\n"
        "s_waitcnt lgkmcnt(2)\n"
        "v_mfma_f32_16x16x32_f16 v[76:79], v[160:163], v[236:239], v[76:79]\n"
        "ds_read_b128 v[232:235], %[brd] offset:38912\n"
        "v_mfma_f32_16x16x32_f16 v[108:111], v[164:167], v[236:239], v[108:111]\n"
        "s_mov_b32 s88, 2097152\n"
        "v_max_f32 v192, 0, v32\n"
        "v_max_f32 v193, 0, v36\n"
        "v_max_f32 v194, 0, v40\n"
        "v_max_f32 v195, 0, v44\n"
        "buffer_store_dwordx4 v[192:195], %[ooff], %[rsO], s88 offen offset:0 nt\n"
        "s_waitcnt lgkmcnt(2)\n"
        "v_mfma_f32_16x16x32_f16 v[80:83], v[160:163], v[224:227], v[80:83]\n"
        "ds_read_b128 v[236:239], %[brd] offset:39936\n"
        "v_mfma_f32_16x16x32_f16 v[112:115], v[164:167], v[224:227], v[112:115]\n"
        "s_waitcnt lgkmcnt(2)\n"
        "v_mfma_f32_16x16x32_f16 v[84:87], v[160:163], v[228:231], v[84:87]\n"
        "ds_read_b128 v[224:227], %[brd] offset:40960\n"
        "v_mfma_f32_16x16x32_f16 v[116:119], v[164:167], v[228:231], v[116:119]\n"
        "s_waitcnt lgkmcnt(2)\n"
        "v_mfma_f32_16x16x32_f16 v[88:91], v[160:163], v[232:235], v[88:91]\n"
        "ds_read_b128 v[228:231], %[brd] offset:41984\n"
        "v_mfma_f32_16x16x32_f16 v[120:123], v[164:167], v[232:235], v[120:123]\n"
        "s_waitcnt lgkmcnt(2)\n"
        "v_mfma_f32_16x16x32_f16 v[92:95], v[160:163], v[236:239], v[92:95]\n"
        "ds_read_b128 v[232:235], %[brd] offset:43008\n"
        "v_mfma_f32_16x16x32_f16 v[124:127], v[164:167], v[236:239], v[124:127]\n"
        "s_mov_b32 s89, 2097152\n"
        "v_max_f32 v196, 0, v48\n"
        "v_max_f32 v197, 0, v52\n"
        "v_max_f32 v198, 0, v56\n"
        "v_max_f32 v199, 0, v60\n"
        "buffer_store_dwordx4 v[196:199], %[ooff], %[rsO], s89 offen offset:256 nt\n"
        "s_waitcnt lgkmcnt(2)\n"
        "v_mfma_f32_16x16x32_f16 v[64:67], v[168:171], v[224:227], v[64:67]\n"
        "ds_read_b128 v[236:239], %[brd] offset:44032\n"
        "v_mfma_f32_16x16x32_f16 v[96:99], v[172:175], v[224:227], v[96:99]\n"
        "s_waitcnt lgkmcnt(2)\n"
        "v_mfma_f32_16x16x32_f16 v[68:71], v[168:171], v[228:231], v[68:71]\n"
        "ds_read_b128 v[224:227], %[brd] offset:45056\n"
        "v_mfma_f32_16x16x32_f16 v[100:103], v[172:175], v[228:231], v[100:103]\n"
        "s_lshl_b32 s81, s77, 16\n"
        "s_add_u32 s81, s81, s62\n"
        "s_add_u32 s81, s81, 2048\n"
        "s_add_u32 s87, s81, 4096\n"
        "buffer_load_dwordx4 v[144:147], %[avoff], %[rsA], s81 offen offset:0\n"
        "buffer_load_dwordx4 v[148:151], %[avoff], %[rsA], s81 offen offset:1024\n"
        "buffer_load_dwordx4 v[152:155], %[avoff], %[rsA], s87 offen offset:0\n"
        "buffer_load_dwordx4 v[156:159], %[avoff], %[rsA], s87 offen offset:1024\n"
        "s_waitcnt lgkmcnt(2)\n"
        "v_mfma_f32_16x16x32_f16 v[72:75], v[168:171], v[232:235], v[72:75]\n"
        "ds_read_b128 v[228:231], %[brd] offset:46080\n"
        "v_mfma_f32_16x16x32_f16 v[104:107], v[172:175], v[232:235], v[104:107]\n"
        "s_waitcnt lgkmcnt(2)\n"
        "v_mfma_f32_16x16x32_f16 v[76:79], v[168:171], v[236:239], v[76:79]\n"
        "ds_read_b128 v[232:235], %[brd] offset:47104\n"
        "v_mfma_f32_16x16x32_f16 v[108:111], v[172:175], v[236:239], v[108:111]\n"
        "s_mov_b32 s88, 2228224\n"
        "v_max_f32 v200, 0, v33\n"
        "v_max_f32 v201, 0, v37\n"
        "v_max_f32 v202, 0, v41\n"
        "v_max_f32 v203, 0, v45\n"
        "buffer_store_dwordx4 v[200:203], %[ooff], %[rsO], s88 offen offset:0 nt\n"
        "s_waitcnt lgkmcnt(2)\n"
        "v_mfma_f32_16x16x32_f16 v[80:83], v[168:171], v[224:227], v[80:83]\n"
        "ds_read_b128 v[236:239], %[brd] offset:48128\n"
        "v_mfma_f32_16x16x32_f16 v[112:115], v[172:175], v[224:227], v[112:115]\n"
        "s_waitcnt lgkmcnt(2)\n"
        "v_mfma_f32_16x16x32_f16 v[84:87], v[168:171], v[228:231], v[84:87]\n"
        "ds_read_b128 v[224:227], %[brd] offset:49152\n"
        "v_mfma_f32_16x16x32_f16 v[116:119], v[172:175], v[228:231], v[116:119]\n"
        "s_waitcnt lgkmcnt(2)\n"
        "v_mfma_f32_16x16x32_f16 v[88:91], v[168:171], v[232:235], v[88:91]\n"
        "ds_read_b128 v[228:231], %[brd] offset:50176\n"
        "v_mfma_f32_16x16x32_f16 v[120:123], v[172:175], v[232:235], v[120:123]\n"
        "s_waitcnt lgkmcnt(2)\n"
        "v_mfma_f32_16x16x32_f16 v[92:95], v[168:171], v[236:239], v[92:95]\n"
        "ds_read_b128 v[232:235], %[brd] offset:51200\n"
        "v_mfma_f32_16x16x32_f16 v[124:127], v[172:175], v[236:239], v[124:127]\n"
        "s_mov_b32 s89, 2228224\n"
        "v_max_f32 v204, 0, v49\n"
        "v_max_f32 v205, 0, v53\n"
        "v_max_f32 v206, 0, v57\n"
        "v_max_f32 v207, 0, v61\n"
        "buffer_store_dwordx4 v[204:207], %[ooff], %[rsO], s89 offen offset:256 nt\n"
        "s_waitcnt vmcnt(15) lgkmcnt(2)\n"
        "v_mfma_f32_16x16x32_f16 v[64:67], v[176:179], v[224:227], v[64:67]\n"
        "ds_read_b128 v[236:239], %[brd] offset:52224\n"
        "v_mfma_f32_16x16x32_f16 v[96:99], v[180:183], v[224:227], v[96:99]\n"
        "s_waitcnt lgkmcnt(2)\n"
        "v_mfma_f32_16x16x32_f16 v[68:71], v[176:179], v[228:231], v[68:71]\n"
        "ds_read_b128 v[224:227], %[brd] offset:53248\n"
        "v_mfma_f32_16x16x32_f16 v[100:103], v[180:183], v[228:231], v[100:103]\n"
        "s_waitcnt lgkmcnt(2)\n"
        "v_mfma_f32_16x16x32_f16 v[72:75], v[176:179], v[232:235], v[72:75]\n"
        "ds_read_b128 v[228:231], %[brd] offset:54272\n"
        "v_mfma_f32_16x16x32_f16 v[104:107], v[180:183], v[232:235], v[104:107]\n"
        "s_waitcnt lgkmcnt(2)\n"
        "v_mfma_f32_16x16x32_f16 v[76:79], v[176:179], v[236:239], v[76:79]\n"
        "ds_read_b128 v[232:235], %[brd] offset:55296\n"
        "v_mfma_f32_16x16x32_f16 v[108:111], v[180:183], v[236:239], v[108:111]\n"
        "s_mov_b32 s88, 2359296\n"
        "v_max_f32 v208, 0, v34\n"
        "v_max_f32 v209, 0, v38\n"
        "v_max_f32 v210, 0, v42\n"
        "v_max_f32 v211, 0, v46\n"
        "buffer_store_dwordx4 v[208:211], %[ooff], %[rsO], s88 offen offset:0 nt\n"
        "s_waitcnt lgkmcnt(2)\n"
        "v_mfma_f32_16x16x32_f16 v[80:83], v[176:179], v[224:227], v[80:83]\n"
        "ds_read_b128 v[236:239], %[brd] offset:56320\n"
        "v_mfma_f32_16x16x32_f16 v[112:115], v[180:183], v[224:227], v[112:115]\n"
        "s_waitcnt lgkmcnt(2)\n"
        "v_mfma_f32_16x16x32_f16 v[84:87], v[176:179], v[228:231], v[84:87]\n"
        "ds_read_b128 v[224:227], %[brd] offset:57344\n"
        "v_mfma_f32_16x16x32_f16 v[116:119], v[180:183], v[228:231], v[116:119]\n"
        "s_waitcnt lgkmcnt(2)\n"
        "v_mfma_f32_16x16x32_f16 v[88:91], v[176:179], v[232:235], v[88:91]\n"
        "ds_read_b128 v[228:231], %[brd] offset:58368\n"
        "v_mfma_f32_16x16x32_f16 v[120:123], v[180:183], v[232:235], v[120:123]\n"
        "s_waitcnt lgkmcnt(2)\n"
        "v_mfma_f32_16x16x32_f16 v[92:95], v[176:179], v[236:239], v[92:95]\n"
        "ds_read_b128 v[232:235], %[brd] offset:59392\n"
        "v_mfma_f32_16x16x32_f16 v[124:127], v[180:183], v[236:239], v[124:127]\n"
        "s_mov_b32 s89, 2359296\n"
        "v_max_f32 v212, 0, v50\n"
        "v_max_f32 v213, 0, v54\n"
        "v_max_f32 v214, 0, v58\n"
        "v_max_f32 v215, 0, v62\n"
        "buffer_store_dwordx4 v[212:215], %[ooff], %[rsO], s89 offen offset:256 nt\n"
        "s_waitcnt lgkmcnt(2)\n"
        "v_mfma_f32_16x16x32_f16 v[64:67], v[184:187], v[224:227], v[64:67]\n"
        "ds_read_b128 v[236:239], %[brd] offset:60416\n"
        "v_mfma_f32_16x16x32_f16 v[96:99], v[188:191], v[224:227], v[96:99]\n"
        "s_waitcnt lgkmcnt(2)\n"
        "v_mfma_f32_16x16x32_f16 v[68:71], v[184:187], v[228:231], v[68:71]\n"
        "ds_read_b128 v[224:227], %[brd] offset:61440\n"
        "v_mfma_f32_16x16x32_f16 v[100:103], v[188:191], v[228:231], v[100:103]\n"
        "s_waitcnt lgkmcnt(2)\n"
        "v_mfma_f32_16x16x32_f16 v[72:75], v[184:187], v[232:235], v[72:75]\n"
        "ds_read_b128 v[228:231], %[brd] offset:62464\n"
        "v_mfma_f32_16x16x32_f16 v[104:107], v[188:191], v[232:235], v[104:107]\n"
        "s_waitcnt lgkmcnt(2)\n"
        "v_mfma_f32_16x16x32_f16 v[76:79], v[184:187], v[236:239], v[76:79]\n"
        "ds_read_b128 v[232:235], %[brd] offset:63488\n"
        "v_mfma_f32_16x16x32_f16 v[108:111], v[188:191], v[236:239], v[108:111]\n"
        "s_mov_b32 s88, 2490368\n"
        "v_max_f32 v216, 0, v35\n"
        "v_max_f32 v217, 0, v39\n"
        "v_max_f32 v218, 0, v43\n"
        "v_max_f32 v219, 0, v47\n"
        "buffer_store_dwordx4 v[216:219], %[ooff], %[rsO], s88 offen offset:0 nt\n"
        "s_waitcnt lgkmcnt(2)\n"
        "v_mfma_f32_16x16x32_f16 v[80:83], v[184:187], v[224:227], v[80:83]\n"
        "ds_read_b128 v[236:239], %[brd] offset:64512\n"
        "v_mfma_f32_16x16x32_f16 v[112:115], v[188:191], v[224:227], v[112:115]\n"
        "s_waitcnt lgkmcnt(2)\n"
        "v_mfma_f32_16x16x32_f16 v[84:87], v[184:187], v[228:231], v[84:87]\n"
        "ds_read_b128 v[224:227], v244 offset:0\n"
        "v_mfma_f32_16x16x32_f16 v[116:119], v[188:191], v[228:231], v[116:119]\n"
        "s_waitcnt lgkmcnt(2)\n"
        "v_mfma_f32_16x16x32_f16 v[88:91], v[184:187], v[232:235], v[88:91]\n"
        "ds_read_b128 v[228:231], v244 offset:1024\n"
        "v_mfma_f32_16x16x32_f16 v[120:123], v[188:191], v[232:235], v[120:123]\n"
        "s_waitcnt lgkmcnt(2)\n"
        "v_mfma_f32_16x16x32_f16 v[92:95], v[184:187], v[236:239], v[92:95]\n"
        "ds_read_b128 v[232:235], v244 offset:2048\n"
        "v_mfma_f32_16x16x32_f16 v[124:127], v[188:191], v[236:239], v[124:127]\n"
        "s_lshl_b32 s80, s78, 16\n"
        "s_add_u32 s80, s80, s62\n"
        "s_add_u32 s80, s80, 2048\n"
        "s_add_u32 s86, s80, 4096\n"
        "buffer_load_dwordx4 v[160:163], %[avoff], %[rsA], s80 offen offset:0\n"
        "buffer_load_dwordx4 v[164:167], %[avoff], %[rsA], s80 offen offset:1024\n"
        "buffer_load_dwordx4 v[168:171], %[avoff], %[rsA], s86 offen offset:0\n"
        "buffer_load_dwordx4 v[172:175], %[avoff], %[rsA], s86 offen offset:1024\n"
        "s_lshl_b32 s81, s79, 16\n"
        "s_add_u32 s81, s81, s62\n"
        "s_add_u32 s81, s81, 2048\n"
        "s_add_u32 s87, s81, 4096\n"
        "buffer_load_dwordx4 v[176:179], %[avoff], %[rsA], s81 offen offset:0\n"
        "buffer_load_dwordx4 v[180:183], %[avoff], %[rsA], s81 offen offset:1024\n"
        "s_mov_b32 s89, 2490368\n"
        "v_max_f32 v220, 0, v51\n"
        "v_max_f32 v221, 0, v55\n"
        "v_max_f32 v222, 0, v59\n"
        "v_max_f32 v223, 0, v63\n"
        "buffer_store_dwordx4 v[220:223], %[ooff], %[rsO], s89 offen offset:256 nt\n"
        "s_waitcnt vmcnt(21) lgkmcnt(2)\n"
        "v_mfma_f32_16x16x32_f16 v[64:67], v[128:131], v[224:227], v[64:67]\n"
        "ds_read_b128 v[236:239], v244 offset:3072\n"
        "v_mfma_f32_16x16x32_f16 v[96:99], v[132:135], v[224:227], v[96:99]\n"
        "s_waitcnt lgkmcnt(2)\n"
        "v_mfma_f32_16x16x32_f16 v[68:71], v[128:131], v[228:231], v[68:71]\n"
        "ds_read_b128 v[224:227], v244 offset:4096\n"
        "v_mfma_f32_16x16x32_f16 v[100:103], v[132:135], v[228:231], v[100:103]\n"
        "s_waitcnt lgkmcnt(2)\n"
        "v_mfma_f32_16x16x32_f16 v[72:75], v[128:131], v[232:235], v[72:75]\n"
        "ds_read_b128 v[228:231], v244 offset:5120\n"
        "v_mfma_f32_16x16x32_f16 v[104:107], v[132:135], v[232:235], v[104:107]\n"
        "s_waitcnt lgkmcnt(2)\n"
        "v_mfma_f32_16x16x32_f16 v[76:79], v[128:131], v[236:239], v[76:79]\n"
        "ds_read_b128 v[232:235], v244 offset:6144\n"
        "v_mfma_f32_16x16x32_f16 v[108:111], v[132:135], v[236:239], v[108:111]\n"
        "s_waitcnt lgkmcnt(2)\n"
        "v_mfma_f32_16x16x32_f16 v[80:83], v[128:131], v[224:227], v[80:83]\n"
        "ds_read_b128 v[236:239], v244 offset:7168\n"
        "v_mfma_f32_16x16x32_f16 v[112:115], v[132:135], v[224:227], v[112:115]\n"
        "s_waitcnt lgkmcnt(2)\n"
        "v_mfma_f32_16x16x32_f16 v[84:87], v[128:131], v[228:231], v[84:87]\n"
        "ds_read_b128 v[224:227], v244 offset:8192\n"
        "v_mfma_f32_16x16x32_f16 v[116:119], v[132:135], v[228:231], v[116:119]\n"
        "s_waitcnt lgkmcnt(2)\n"
        "v_mfma_f32_16x16x32_f16 v[88:91], v[128:131], v[232:235], v[88:91]\n"
        "ds_read_b128 v[228:231], v244 offset:9216\n"
        "v_mfma_f32_16x16x32_f16 v[120:123], v[132:135], v[232:235], v[120:123]\n"
        "s_waitcnt lgkmcnt(2)\n"
        "v_mfma_f32_16x16x32_f16 v[92:95], v[128:131], v[236:239], v[92:95]\n"
        "ds_read_b128 v[232:235], v244 offset:10240\n"
        "v_mfma_f32_16x16x32_f16 v[124:127], v[132:135], v[236:239], v[124:127]\n"
        "s_waitcnt lgkmcnt(2)\n"
        "v_mfma_f32_16x16x32_f16 v[64:67], v[136:139], v[224:227], v[64:67]\n"
        "ds_read_b128 v[236:239], v244 offset:11264\n"
        "v_mfma_f32_16x16x32_f16 v[96:99], v[140:143], v[224:227], v[96:99]\n"
        "s_waitcnt lgkmcnt(2)\n"
        "v_mfma_f32_16x16x32_f16 v[68:71], v[136:139], v[228:231], v[68:71]\n"
        "ds_read_b128 v[224:227], v244 offset:12288\n"
        "v_mfma_f32_16x16x32_f16 v[100:103], v[140:143], v[228:231], v[100:103]\n"
        "s_waitcnt lgkmcnt(2)\n"
        "v_mfma_f32_16x16x32_f16 v[72:75], v[136:139], v[232:235], v[72:75]\n"
        "ds_read_b128 v[228:231], v244 offset:13312\n"
        "v_mfma_f32_16x16x32_f16 v[104:107], v[140:143], v[232:235], v[104:107]\n"
        "s_waitcnt lgkmcnt(2)\n"
        "v_mfma_f32_16x16x32_f16 v[76:79], v[136:139], v[236:239], v[76:79]\n"
        "ds_read_b128 v[232:235], v244 offset:14336\n"
        "v_mfma_f32_16x16x32_f16 v[108:111], v[140:143], v[236:239], v[108:111]\n"
        "s_waitcnt lgkmcnt(2)\n"
        "v_mfma_f32_16x16x32_f16 v[80:83], v[136:139], v[224:227], v[80:83]\n"
        "ds_read_b128 v[236:239], v244 offset:15360\n"
        "v_mfma_f32_16x16x32_f16 v[112:115], v[140:143], v[224:227], v[112:115]\n"
        "s_waitcnt lgkmcnt(2)\n"
        "v_mfma_f32_16x16x32_f16 v[84:87], v[136:139], v[228:231], v[84:87]\n"
        "ds_read_b128 v[224:227], v244 offset:16384\n"
        "v_mfma_f32_16x16x32_f16 v[116:119], v[140:143], v[228:231], v[116:119]\n"
        "s_waitcnt lgkmcnt(2)\n"
        "v_mfma_f32_16x16x32_f16 v[88:91], v[136:139], v[232:235], v[88:91]\n"
        "ds_read_b128 v[228:231], v244 offset:17408\n"
        "v_mfma_f32_16x16x32_f16 v[120:123], v[140:143], v[232:235], v[120:123]\n"
        "s_waitcnt lgkmcnt(2)\n"
        "v_mfma_f32_16x16x32_f16 v[92:95], v[136:139], v[236:239], v[92:95]\n"
        "ds_read_b128 v[232:235], v244 offset:18432\n"
        "v_mfma_f32_16x16x32_f16 v[124:127], v[140:143], v[236:239], v[124:127]\n"
        "s_waitcnt vmcnt(12) lgkmcnt(2)\n"
        "v_mfma_f32_16x16x32_f16 v[64:67], v[144:147], v[224:227], v[64:67]\n"
        "ds_read_b128 v[236:239], v244 offset:19456\n"
        "v_mfma_f32_16x16x32_f16 v[96:99], v[148:151], v[224:227], v[96:99]\n"
        "s_waitcnt lgkmcnt(2)\n"
        "v_mfma_f32_16x16x32_f16 v[68:71], v[144:147], v[228:231], v[68:71]\n"
        "ds_read_b128 v[224:227], v244 offset:20480\n"
        "v_mfma_f32_16x16x32_f16 v[100:103], v[148:151], v[228:231], v[100:103]\n"
        "s_waitcnt lgkmcnt(2)\n"
        "v_mfma_f32_16x16x32_f16 v[72:75], v[144:147], v[232:235], v[72:75]\n"
        "ds_read_b128 v[228:231], v244 offset:21504\n"
        "v_mfma_f32_16x16x32_f16 v[104:107], v[148:151], v[232:235], v[104:107]\n"
        "s_waitcnt lgkmcnt(2)\n"
        "v_mfma_f32_16x16x32_f16 v[76:79], v[144:147], v[236:239], v[76:79]\n"
        "ds_read_b128 v[232:235], v244 offset:22528\n"
        "v_mfma_f32_16x16x32_f16 v[108:111], v[148:151], v[236:239], v[108:111]\n"
        "s_waitcnt lgkmcnt(2)\n"
        "v_mfma_f32_16x16x32_f16 v[80:83], v[144:147], v[224:227], v[80:83]\n"
        "ds_read_b128 v[236:239], v244 offset:23552\n"
        "v_mfma_f32_16x16x32_f16 v[112:115], v[148:151], v[224:227], v[112:115]\n"
        "s_waitcnt lgkmcnt(2)\n"
        "v_mfma_f32_16x16x32_f16 v[84:87], v[144:147], v[228:231], v[84:87]\n"
        "ds_read_b128 v[224:227], v244 offset:24576\n"
        "v_mfma_f32_16x16x32_f16 v[116:119], v[148:151], v[228:231], v[116:119]\n"
        "buffer_load_dwordx4 v[184:187], %[avoff], %[rsA], s87 offen offset:0\n"
        "buffer_load_dwordx4 v[188:191], %[avoff], %[rsA], s87 offen offset:1024\n"
        "s_waitcnt lgkmcnt(2)\n"
        "v_mfma_f32_16x16x32_f16 v[88:91], v[144:147], v[232:235], v[88:91]\n"
        "ds_read_b128 v[228:231], v244 offset:25600\n"
        "v_mfma_f32_16x16x32_f16 v[120:123], v[148:151], v[232:235], v[120:123]\n"
        "s_waitcnt lgkmcnt(2)\n"
        "v_mfma_f32_16x16x32_f16 v[92:95], v[144:147], v[236:239], v[92:95]\n"
        "ds_read_b128 v[232:235], v244 offset:26624\n"
        "v_mfma_f32_16x16x32_f16 v[124:127], v[148:151], v[236:239], v[124:127]\n"
        "s_waitcnt lgkmcnt(2)\n"
        "v_mfma_f32_16x16x32_f16 v[64:67], v[152:155], v[224:227], v[64:67]\n"
        "ds_read_b128 v[236:239], v244 offset:27648\n"
        "v_mfma_f32_16x16x32_f16 v[96:99], v[156:159], v[224:227], v[96:99]\n"
        "s_waitcnt lgkmcnt(2)\n"
        "v_mfma_f32_16x16x32_f16 v[68:71], v[152:155], v[228:231], v[68:71]\n"
        "ds_read_b128 v[224:227], v244 offset:28672\n"
        "v_mfma_f32_16x16x32_f16 v[100:103], v[156:159], v[228:231], v[100:103]\n"
        "s_waitcnt lgkmcnt(2)\n"
        "v_mfma_f32_16x16x32_f16 v[72:75], v[152:155], v[232:235], v[72:75]\n"
        "ds_read_b128 v[228:231], v244 offset:29696\n"
        "v_mfma_f32_16x16x32_f16 v[104:107], v[156:159], v[232:235], v[104:107]\n"
        "s_waitcnt lgkmcnt(2)\n"
        "v_mfma_f32_16x16x32_f16 v[76:79], v[152:155], v[236:239], v[76:79]\n"
        "ds_read_b128 v[232:235], v244 offset:30720\n"
        "v_mfma_f32_16x16x32_f16 v[108:111], v[156:159], v[236:239], v[108:111]\n"
        "s_waitcnt lgkmcnt(2)\n"
        "v_mfma_f32_16x16x32_f16 v[80:83], v[152:155], v[224:227], v[80:83]\n"
        "ds_read_b128 v[236:239], v244 offset:31744\n"
        "v_mfma_f32_16x16x32_f16 v[112:115], v[156:159], v[224:227], v[112:115]\n"
        "s_waitcnt lgkmcnt(2)\n"
        "v_mfma_f32_16x16x32_f16 v[84:87], v[152:155], v[228:231], v[84:87]\n"
        "ds_read_b128 v[224:227], v244 offset:32768\n"
        "v_mfma_f32_16x16x32_f16 v[116:119], v[156:159], v[228:231], v[116:119]\n"
        "s_waitcnt lgkmcnt(2)\n"
        "v_mfma_f32_16x16x32_f16 v[88:91], v[152:155], v[232:235], v[88:91]\n"
        "ds_read_b128 v[228:231], v244 offset:33792\n"
        "v_mfma_f32_16x16x32_f16 v[120:123], v[156:159], v[232:235], v[120:123]\n"
        "s_waitcnt lgkmcnt(2)\n"
        "v_mfma_f32_16x16x32_f16 v[92:95], v[152:155], v[236:239], v[92:95]\n"
        "ds_read_b128 v[232:235], v244 offset:34816\n"
        "v_mfma_f32_16x16x32_f16 v[124:127], v[156:159], v[236:239], v[124:127]\n"
        "s_waitcnt vmcnt(5) lgkmcnt(2)\n"
        "v_mfma_f32_16x16x32_f16 v[64:67], v[160:163], v[224:227], v[64:67]\n"
        "ds_read_b128 v[236:239], v244 offset:35840\n"
        "v_mfma_f32_16x16x32_f16 v[96:99], v[164:167], v[224:227], v[96:99]\n"
        "s_waitcnt lgkmcnt(2)\n"
        "v_mfma_f32_16x16x32_f16 v[68:71], v[160:163], v[228:231], v[68:71]\n"
        "ds_read_b128 v[224:227], v244 offset:36864\n"
        "v_mfma_f32_16x16x32_f16 v[100:103], v[164:167], v[228:231], v[100:103]\n"
        "s_waitcnt lgkmcnt(2)\n"
        "v_mfma_f32_16x16x32_f16 v[72:75], v[160:163], v[232:235], v[72:75]\n"
        "ds_read_b128 v[228:231], v244 offset:37888\n"
        "v_mfma_f32_16x16x32_f16 v[104:107], v[164:167], v[232:235], v[104:107]\n"
        "s_waitcnt lgkmcnt(2)\n"
        "v_mfma_f32_16x16x32_f16 v[76:79], v[160:163], v[236:239], v[76:79]\n"
        "ds_read_b128 v[232:235], v244 offset:38912\n"
        "v_mfma_f32_16x16x32_f16 v[108:111], v[164:167], v[236:239], v[108:111]\n"
        "s_waitcnt lgkmcnt(2)\n"
        "v_mfma_f32_16x16x32_f16 v[80:83], v[160:163], v[224:227], v[80:83]\n"
        "ds_read_b128 v[236:239], v244 offset:39936\n"
        "v_mfma_f32_16x16x32_f16 v[112:115], v[164:167], v[224:227], v[112:115]\n"
        "s_waitcnt lgkmcnt(2)\n"
        "v_mfma_f32_16x16x32_f16 v[84:87], v[160:163], v[228:231], v[84:87]\n"
        "ds_read_b128 v[224:227], v244 offset:40960\n"
        "v_mfma_f32_16x16x32_f16 v[116:119], v[164:167], v[228:231], v[116:119]\n"
        "s_waitcnt lgkmcnt(2)\n"
        "v_mfma_f32_16x16x32_f16 v[88:91], v[160:163], v[232:235], v[88:91]\n"
        "ds_read_b128 v[228:231], v244 offset:41984\n"
        "v_mfma_f32_16x16x32_f16 v[120:123], v[164:167], v[232:235], v[120:123]\n"
        "s_waitcnt lgkmcnt(2)\n"
        "v_mfma_f32_16x16x32_f16 v[92:95], v[160:163], v[236:239], v[92:95]\n"
        "ds_read_b128 v[232:235], v244 offset:43008\n"
        "v_mfma_f32_16x16x32_f16 v[124:127], v[164:167], v[236:239], v[124:127]\n"
        "s_waitcnt lgkmcnt(2)\n"
        "v_mfma_f32_16x16x32_f16 v[64:67], v[168:171], v[224:227], v[64:67]\n"
        "ds_read_b128 v[236:239], v244 offset:44032\n"
        "v_mfma_f32_16x16x32_f16 v[96:99], v[172:175], v[224:227], v[96:99]\n"
        "s_waitcnt lgkmcnt(2)\n"
        "v_mfma_f32_16x16x32_f16 v[68:71], v[168:171], v[228:231], v[68:71]\n"
        "ds_read_b128 v[224:227], v244 offset:45056\n"
        "v_mfma_f32_16x16x32_f16 v[100:103], v[172:175], v[228:231], v[100:103]\n"
        "s_waitcnt lgkmcnt(2)\n"
        "v_mfma_f32_16x16x32_f16 v[72:75], v[168:171], v[232:235], v[72:75]\n"
        "ds_read_b128 v[228:231], v244 offset:46080\n"
        "v_mfma_f32_16x16x32_f16 v[104:107], v[172:175], v[232:235], v[104:107]\n"
        "s_waitcnt lgkmcnt(2)\n"
        "v_mfma_f32_16x16x32_f16 v[76:79], v[168:171], v[236:239], v[76:79]\n"
        "ds_read_b128 v[232:235], v244 offset:47104\n"
        "v_mfma_f32_16x16x32_f16 v[108:111], v[172:175], v[236:239], v[108:111]\n"
        "s_waitcnt lgkmcnt(2)\n"
        "v_mfma_f32_16x16x32_f16 v[80:83], v[168:171], v[224:227], v[80:83]\n"
        "ds_read_b128 v[236:239], v244 offset:48128\n"
        "v_mfma_f32_16x16x32_f16 v[112:115], v[172:175], v[224:227], v[112:115]\n"
        "s_waitcnt lgkmcnt(2)\n"
        "v_mfma_f32_16x16x32_f16 v[84:87], v[168:171], v[228:231], v[84:87]\n"
        "ds_read_b128 v[224:227], v244 offset:49152\n"
        "v_mfma_f32_16x16x32_f16 v[116:119], v[172:175], v[228:231], v[116:119]\n"
        "s_waitcnt lgkmcnt(2)\n"
        "v_mfma_f32_16x16x32_f16 v[88:91], v[168:171], v[232:235], v[88:91]\n"
        "ds_read_b128 v[228:231], v244 offset:50176\n"
        "v_mfma_f32_16x16x32_f16 v[120:123], v[172:175], v[232:235], v[120:123]\n"
        "s_waitcnt lgkmcnt(2)\n"
        "v_mfma_f32_16x16x32_f16 v[92:95], v[168:171], v[236:239], v[92:95]\n"
        "ds_read_b128 v[232:235], v244 offset:51200\n"
        "v_mfma_f32_16x16x32_f16 v[124:127], v[172:175], v[236:239], v[124:127]\n"
        "s_waitcnt vmcnt(0) lgkmcnt(2)\n"
        "v_mfma_f32_16x16x32_f16 v[64:67], v[176:179], v[224:227], v[64:67]\n"
        "ds_read_b128 v[236:239], v244 offset:52224\n"
        "v_mfma_f32_16x16x32_f16 v[96:99], v[180:183], v[224:227], v[96:99]\n"
        "s_waitcnt lgkmcnt(2)\n"
        "v_mfma_f32_16x16x32_f16 v[68:71], v[176:179], v[228:231], v[68:71]\n"
        "ds_read_b128 v[224:227], v244 offset:53248\n"
        "v_mfma_f32_16x16x32_f16 v[100:103], v[180:183], v[228:231], v[100:103]\n"
        "s_waitcnt lgkmcnt(2)\n"
        "v_mfma_f32_16x16x32_f16 v[72:75], v[176:179], v[232:235], v[72:75]\n"
        "ds_read_b128 v[228:231], v244 offset:54272\n"
        "v_mfma_f32_16x16x32_f16 v[104:107], v[180:183], v[232:235], v[104:107]\n"
        "s_waitcnt lgkmcnt(2)\n"
        "v_mfma_f32_16x16x32_f16 v[76:79], v[176:179], v[236:239], v[76:79]\n"
        "ds_read_b128 v[232:235], v244 offset:55296\n"
        "v_mfma_f32_16x16x32_f16 v[108:111], v[180:183], v[236:239], v[108:111]\n"
        "s_waitcnt lgkmcnt(2)\n"
        "v_mfma_f32_16x16x32_f16 v[80:83], v[176:179], v[224:227], v[80:83]\n"
        "ds_read_b128 v[236:239], v244 offset:56320\n"
        "v_mfma_f32_16x16x32_f16 v[112:115], v[180:183], v[224:227], v[112:115]\n"
        "s_waitcnt lgkmcnt(2)\n"
        "v_mfma_f32_16x16x32_f16 v[84:87], v[176:179], v[228:231], v[84:87]\n"
        "ds_read_b128 v[224:227], v244 offset:57344\n"
        "v_mfma_f32_16x16x32_f16 v[116:119], v[180:183], v[228:231], v[116:119]\n"
        "s_waitcnt lgkmcnt(2)\n"
        "v_mfma_f32_16x16x32_f16 v[88:91], v[176:179], v[232:235], v[88:91]\n"
        "ds_read_b128 v[228:231], v244 offset:58368\n"
        "v_mfma_f32_16x16x32_f16 v[120:123], v[180:183], v[232:235], v[120:123]\n"
        "s_waitcnt lgkmcnt(2)\n"
        "v_mfma_f32_16x16x32_f16 v[92:95], v[176:179], v[236:239], v[92:95]\n"
        "ds_read_b128 v[232:235], v244 offset:59392\n"
        "v_mfma_f32_16x16x32_f16 v[124:127], v[180:183], v[236:239], v[124:127]\n"
        "s_waitcnt lgkmcnt(2)\n"
        "v_mfma_f32_16x16x32_f16 v[64:67], v[184:187], v[224:227], v[64:67]\n"
        "ds_read_b128 v[236:239], v244 offset:60416\n"
        "v_mfma_f32_16x16x32_f16 v[96:99], v[188:191], v[224:227], v[96:99]\n"
        "s_waitcnt lgkmcnt(2)\n"
        "v_mfma_f32_16x16x32_f16 v[68:71], v[184:187], v[228:231], v[68:71]\n"
        "ds_read_b128 v[224:227], v244 offset:61440\n"
        "v_mfma_f32_16x16x32_f16 v[100:103], v[188:191], v[228:231], v[100:103]\n"
        "s_waitcnt lgkmcnt(2)\n"
        "v_mfma_f32_16x16x32_f16 v[72:75], v[184:187], v[232:235], v[72:75]\n"
        "ds_read_b128 v[228:231], v244 offset:62464\n"
        "v_mfma_f32_16x16x32_f16 v[104:107], v[188:191], v[232:235], v[104:107]\n"
        "s_waitcnt lgkmcnt(2)\n"
        "v_mfma_f32_16x16x32_f16 v[76:79], v[184:187], v[236:239], v[76:79]\n"
        "ds_read_b128 v[232:235], v244 offset:63488\n"
        "v_mfma_f32_16x16x32_f16 v[108:111], v[188:191], v[236:239], v[108:111]\n"
        "s_waitcnt lgkmcnt(2)\n"
        "v_mfma_f32_16x16x32_f16 v[80:83], v[184:187], v[224:227], v[80:83]\n"
        "ds_read_b128 v[236:239], v244 offset:64512\n"
        "v_mfma_f32_16x16x32_f16 v[112:115], v[188:191], v[224:227], v[112:115]\n"
        "s_waitcnt lgkmcnt(2)\n"
        "v_mfma_f32_16x16x32_f16 v[84:87], v[184:187], v[228:231], v[84:87]\n"
        "v_mfma_f32_16x16x32_f16 v[116:119], v[188:191], v[228:231], v[116:119]\n"
        "s_waitcnt lgkmcnt(1)\n"
        "v_mfma_f32_16x16x32_f16 v[88:91], v[184:187], v[232:235], v[88:91]\n"
        "v_mfma_f32_16x16x32_f16 v[120:123], v[188:191], v[232:235], v[120:123]\n"
        "s_waitcnt lgkmcnt(0)\n"
        "v_mfma_f32_16x16x32_f16 v[92:95], v[184:187], v[236:239], v[92:95]\n"
        "v_mfma_f32_16x16x32_f16 v[124:127], v[188:191], v[236:239], v[124:127]\n"
        "L_epi%=:\n"
        "s_nop 7\n"
        "s_nop 7\n"
        "s_mov_b32 s88, 4194304\n"
        "v_max_f32 v128, 0, v64\n"
        "v_max_f32 v129, 0, v68\n"
        "v_max_f32 v130, 0, v72\n"
        "v_max_f32 v131, 0, v76\n"
        "buffer_store_dwordx4 v[128:131], %[ooff], %[rsO], s88 offen offset:0 nt\n"
        "s_mov_b32 s89, 4194304\n"
        "v_max_f32 v132, 0, v80\n"
        "v_max_f32 v133, 0, v84\n"
        "v_max_f32 v134, 0, v88\n"
        "v_max_f32 v135, 0, v92\n"
        "buffer_store_dwordx4 v[132:135], %[ooff], %[rsO], s89 offen offset:256 nt\n"
        "s_mov_b32 s88, 4325376\n"
        "v_max_f32 v136, 0, v65\n"
        "v_max_f32 v137, 0, v69\n"
        "v_max_f32 v138, 0, v73\n"
        "v_max_f32 v139, 0, v77\n"
        "buffer_store_dwordx4 v[136:139], %[ooff], %[rsO], s88 offen offset:0 nt\n"
        "s_mov_b32 s89, 4325376\n"
        "v_max_f32 v140, 0, v81\n"
        "v_max_f32 v141, 0, v85\n"
        "v_max_f32 v142, 0, v89\n"
        "v_max_f32 v143, 0, v93\n"
        "buffer_store_dwordx4 v[140:143], %[ooff], %[rsO], s89 offen offset:256 nt\n"
        "s_mov_b32 s88, 4456448\n"
        "v_max_f32 v144, 0, v66\n"
        "v_max_f32 v145, 0, v70\n"
        "v_max_f32 v146, 0, v74\n"
        "v_max_f32 v147, 0, v78\n"
        "buffer_store_dwordx4 v[144:147], %[ooff], %[rsO], s88 offen offset:0 nt\n"
        "s_mov_b32 s89, 4456448\n"
        "v_max_f32 v148, 0, v82\n"
        "v_max_f32 v149, 0, v86\n"
        "v_max_f32 v150, 0, v90\n"
        "v_max_f32 v151, 0, v94\n"
        "buffer_store_dwordx4 v[148:151], %[ooff], %[rsO], s89 offen offset:256 nt\n"
        "s_mov_b32 s88, 4587520\n"
        "v_max_f32 v152, 0, v67\n"
        "v_max_f32 v153, 0, v71\n"
        "v_max_f32 v154, 0, v75\n"
        "v_max_f32 v155, 0, v79\n"
        "buffer_store_dwordx4 v[152:155], %[ooff], %[rsO], s88 offen offset:0 nt\n"
        "s_mov_b32 s89, 4587520\n"
        "v_max_f32 v156, 0, v83\n"
        "v_max_f32 v157, 0, v87\n"
        "v_max_f32 v158, 0, v91\n"
        "v_max_f32 v159, 0, v95\n"
        "buffer_store_dwordx4 v[156:159], %[ooff], %[rsO], s89 offen offset:256 nt\n"
        "s_mov_b32 s88, 6291456\n"
        "v_max_f32 v128, 0, v96\n"
        "v_max_f32 v129, 0, v100\n"
        "v_max_f32 v130, 0, v104\n"
        "v_max_f32 v131, 0, v108\n"
        "buffer_store_dwordx4 v[128:131], %[ooff], %[rsO], s88 offen offset:0 nt\n"
        "s_mov_b32 s89, 6291456\n"
        "v_max_f32 v132, 0, v112\n"
        "v_max_f32 v133, 0, v116\n"
        "v_max_f32 v134, 0, v120\n"
        "v_max_f32 v135, 0, v124\n"
        "buffer_store_dwordx4 v[132:135], %[ooff], %[rsO], s89 offen offset:256 nt\n"
        "s_mov_b32 s88, 6422528\n"
        "v_max_f32 v136, 0, v97\n"
        "v_max_f32 v137, 0, v101\n"
        "v_max_f32 v138, 0, v105\n"
        "v_max_f32 v139, 0, v109\n"
        "buffer_store_dwordx4 v[136:139], %[ooff], %[rsO], s88 offen offset:0 nt\n"
        "s_mov_b32 s89, 6422528\n"
        "v_max_f32 v140, 0, v113\n"
        "v_max_f32 v141, 0, v117\n"
        "v_max_f32 v142, 0, v121\n"
        "v_max_f32 v143, 0, v125\n"
        "buffer_store_dwordx4 v[140:143], %[ooff], %[rsO], s89 offen offset:256 nt\n"
        "s_mov_b32 s88, 6553600\n"
        "v_max_f32 v144, 0, v98\n"
        "v_max_f32 v145, 0, v102\n"
        "v_max_f32 v146, 0, v106\n"
        "v_max_f32 v147, 0, v110\n"
        "buffer_store_dwordx4 v[144:147], %[ooff], %[rsO], s88 offen offset:0 nt\n"
        "s_mov_b32 s89, 6553600\n"
        "v_max_f32 v148, 0, v114\n"
        "v_max_f32 v149, 0, v118\n"
        "v_max_f32 v150, 0, v122\n"
        "v_max_f32 v151, 0, v126\n"
        "buffer_store_dwordx4 v[148:151], %[ooff], %[rsO], s89 offen offset:256 nt\n"
        "s_mov_b32 s88, 6684672\n"
        "v_max_f32 v152, 0, v99\n"
        "v_max_f32 v153, 0, v103\n"
        "v_max_f32 v154, 0, v107\n"
        "v_max_f32 v155, 0, v111\n"
        "buffer_store_dwordx4 v[152:155], %[ooff], %[rsO], s88 offen offset:0 nt\n"
        "s_mov_b32 s89, 6684672\n"
        "v_max_f32 v156, 0, v115\n"
        "v_max_f32 v157, 0, v119\n"
        "v_max_f32 v158, 0, v123\n"
        "v_max_f32 v159, 0, v127\n"
        "buffer_store_dwordx4 v[156:159], %[ooff], %[rsO], s89 offen offset:256 nt\n"
        "s_movk_i32 s91, 4096\n"
        "buffer_load_dword v246, off, %[rsC], s91 sc1\n"
        "s_waitcnt vmcnt(0)\n"
        "s_cmp_lg_u32 %[wave], 5\n"
        "s_cbranch_scc1 L_noend%=\n"
        "s_memrealtime s[92:93]\n"
        "s_waitcnt lgkmcnt(0)\n"
        "v_mov_b32 v240, s92\n"
        "s_mov_b64 exec, 1\n"
        "buffer_store_dword v240, off, %[rsC], s63 offset:4 sc1\n"
        "s_mov_b64 exec, -1\n"
        "L_noend%=:\n"
        "v_cmp_eq_u32 vcc, s60, v246\n"
        "s_cbranch_vccnz L_nan%=\n"
        "s_cmp_eq_u32 s61, 0\n"
        "s_cbranch_scc1 L_end%=\n"
        "L_nan%=:\n"
        "v_mov_b32 v0, 0x7fc00000\n"
        "v_mov_b32 v1, 0x7fc00000\n"
        "v_mov_b32 v2, 0x7fc00000\n"
        "v_mov_b32 v3, 0x7fc00000\n"
        "s_mov_b32 s80, 0\n"
        "L_nanloop%=:\n"
        "s_lshr_b32 s81, s80, 2\n"
        "s_lshl_b32 s81, s81, 4\n"
        "s_and_b32 s82, s80, 3\n"
        "s_add_u32 s81, s81, s82\n"
        "s_mul_i32 s81, s81, 131072\n"
        "buffer_store_dwordx4 v[0:3], %[ooff], %[rsO], s81 offen\n"
        "buffer_store_dwordx4 v[0:3], %[ooff], %[rsO], s81 offen offset:256\n"
        "s_add_u32 s80, s80, 1\n"
        "s_cmp_lt_u32 s80, 16\n"
        "s_cbranch_scc1 L_nanloop%=\n"
        "s_branch L_end%=\n"
        "L_poll0%=:\n"
        "s_memrealtime s[92:93]\n"
        "s_waitcnt vmcnt(0) lgkmcnt(0)\n"
        "L_poll0%=_loop:\n"
        "v_cmp_le_u32 vcc, s71, v246\n"
        "s_or_b64 s[44:45], s[44:45], vcc\n"
        "s_andn2_b64 s[98:99], s[48:49], s[44:45]\n"
        "s_cbranch_scc0 L_poll1%=\n"
        "s_andn2_b64 s[98:99], s[48:49], s[44:45]\n"
        "s_mov_b64 exec, s[98:99]\n"
        "s_movk_i32 s90, 8192\n"
        "buffer_load_dword v246, %[cvoff], %[rsC], s90 offen sc0 sc1\n"
        "s_mov_b64 exec, -1\n"
        "s_movk_i32 s91, 4096\n"
        "buffer_load_dword v247, off, %[rsC], s91 sc1\n"
        "s_memrealtime s[94:95]\n"
        "s_waitcnt vmcnt(0) lgkmcnt(0)\n"
        "v_cmp_eq_u32 vcc, s60, v247\n"
        "s_cbranch_vccnz L_poll0%=_fail\n"
        "s_sub_u32 s94, s94, s92\n"
        "s_subb_u32 s95, s95, s93\n"
        "s_cmp_lg_u32 s95, 0\n"
        "s_cbranch_scc1 L_poll0%=_fail\n"
        "s_cmp_gt_u32 s94, 40000000\n"
        "s_cbranch_scc1 L_poll0%=_fail\n"
        "s_sleep 1\n"
        "s_branch L_poll0%=_loop\n"
        "L_poll0%=_fail:\n"
        "v_mov_b32 v247, s60\n"
        "s_nop 1\n"
        "buffer_store_dword v247, off, %[rsC], s91 sc1\n"
        "s_mov_b32 s61, 1\n"
        "s_mov_b64 s[44:45], -1\n"
        "s_waitcnt vmcnt(0)\n"
        "s_branch L_poll1%=\n"
        "L_flagcX0%=:\n"
        "v_mov_b32 v247, 1\n"
        "s_mov_b64 exec, 1\n"
        "buffer_atomic_add v247, off, %[rsDC], s68 sc1\n"
        "s_mov_b64 exec, -1\n"
        "s_branch L_flagretcX0%=\n"
        "L_pushcX0%=:\n"
        "v_add_u32 v247, s69, %[brd]\n"
        "s_add_u32 s84, s67, 4096\n"
        "s_waitcnt lgkmcnt(0)\n"
        "ds_read_b128 v[240:243], v247\n"
        "s_waitcnt lgkmcnt(0)\n"
        "buffer_store_dwordx4 v[240:243], %[brd], %[rsD], s67 offen sc0 sc1\n"
        "s_nop 4\n"
        "ds_read_b128 v[240:243], v247 offset:1024\n"
        "s_waitcnt lgkmcnt(0)\n"
        "buffer_store_dwordx4 v[240:243], %[brd], %[rsD], s84 offen sc0 sc1\n"
        "s_nop 4\n"
        "s_branch L_pushretcX0%=\n"
        "L_flagcX1%=:\n"
        "v_mov_b32 v247, 1\n"
        "s_mov_b64 exec, 1\n"
        "buffer_atomic_add v247, off, %[rsDC], s68 sc1\n"
        "s_mov_b64 exec, -1\n"
        "s_branch L_flagretcX1%=\n"
        "L_pushcX1%=:\n"
        "v_add_u32 v247, s69, %[brd]\n"
        "s_add_u32 s84, s67, 4096\n"
        "s_waitcnt lgkmcnt(0)\n"
        "ds_read_b128 v[240:243], v247\n"
        "s_waitcnt lgkmcnt(0)\n"
        "buffer_store_dwordx4 v[240:243], %[brd], %[rsD], s67 offen sc0 sc1\n"
        "s_nop 4\n"
        "ds_read_b128 v[240:243], v247 offset:1024\n"
        "s_waitcnt lgkmcnt(0)\n"
        "buffer_store_dwordx4 v[240:243], %[brd], %[rsD], s84 offen sc0 sc1\n"
        "s_nop 4\n"
        "s_branch L_pushretcX1%=\n"
        "L_flagcX2%=:\n"
        "v_mov_b32 v247, 1\n"
        "s_mov_b64 exec, 1\n"
        "buffer_atomic_add v247, off, %[rsDC], s68 sc1\n"
        "s_mov_b64 exec, -1\n"
        "s_branch L_flagretcX2%=\n"
        "L_pushcX2%=:\n"
        "v_add_u32 v247, s69, %[brd]\n"
        "s_add_u32 s84, s67, 4096\n"
        "s_waitcnt lgkmcnt(0)\n"
        "ds_read_b128 v[240:243], v247\n"
        "s_waitcnt lgkmcnt(0)\n"
        "buffer_store_dwordx4 v[240:243], %[brd], %[rsD], s67 offen sc0 sc1\n"
        "s_nop 4\n"
        "ds_read_b128 v[240:243], v247 offset:1024\n"
        "s_waitcnt lgkmcnt(0)\n"
        "buffer_store_dwordx4 v[240:243], %[brd], %[rsD], s84 offen sc0 sc1\n"
        "s_nop 4\n"
        "s_branch L_pushretcX2%=\n"
        "L_flagcX3%=:\n"
        "v_mov_b32 v247, 1\n"
        "s_mov_b64 exec, 1\n"
        "buffer_atomic_add v247, off, %[rsDC], s68 sc1\n"
        "s_mov_b64 exec, -1\n"
        "s_branch L_flagretcX3%=\n"
        "L_pushcX3%=:\n"
        "v_add_u32 v247, s69, %[brd]\n"
        "s_add_u32 s84, s67, 4096\n"
        "s_waitcnt lgkmcnt(0)\n"
        "ds_read_b128 v[240:243], v247\n"
        "s_waitcnt lgkmcnt(0)\n"
        "buffer_store_dwordx4 v[240:243], %[brd], %[rsD], s67 offen sc0 sc1\n"
        "s_nop 4\n"
        "ds_read_b128 v[240:243], v247 offset:1024\n"
        "s_waitcnt lgkmcnt(0)\n"
        "buffer_store_dwordx4 v[240:243], %[brd], %[rsD], s84 offen sc0 sc1\n"
        "s_nop 4\n"
        "s_branch L_pushretcX3%=\n"
        "L_spink2%=:\n"
        "s_memrealtime s[92:93]\n"
        "s_waitcnt vmcnt(0) lgkmcnt(0)\n"
        "L_spink2%=_loop:\n"
        "v_cmp_le_u32 vcc, s71, v246\n"
        "s_or_b64 s[44:45], s[44:45], vcc\n"
        "s_andn2_b64 s[98:99], s[48:49], s[44:45]\n"
        "s_cbranch_scc0 L_chkk2r%=\n"
        "s_andn2_b64 s[98:99], s[48:49], s[44:45]\n"
        "s_mov_b64 exec, s[98:99]\n"
        "s_movk_i32 s90, 8192\n"
        "buffer_load_dword v246, %[cvoff], %[rsC], s90 offen sc0 sc1\n"
        "s_mov_b64 exec, -1\n"
        "s_movk_i32 s91, 4096\n"
        "buffer_load_dword v247, off, %[rsC], s91 sc1\n"
        "s_memrealtime s[94:95]\n"
        "s_waitcnt vmcnt(0) lgkmcnt(0)\n"
        "v_cmp_eq_u32 vcc, s60, v247\n"
        "s_cbranch_vccnz L_spink2%=_fail\n"
        "s_sub_u32 s94, s94, s92\n"
        "s_subb_u32 s95, s95, s93\n"
        "s_cmp_lg_u32 s95, 0\n"
        "s_cbranch_scc1 L_spink2%=_fail\n"
        "s_cmp_gt_u32 s94, 40000000\n"
        "s_cbranch_scc1 L_spink2%=_fail\n"
        "s_sleep 1\n"
        "s_branch L_spink2%=_loop\n"
        "L_spink2%=_fail:\n"
        "v_mov_b32 v247, s60\n"
        "s_nop 1\n"
        "buffer_store_dword v247, off, %[rsC], s91 sc1\n"
        "s_mov_b32 s61, 1\n"
        "s_mov_b64 s[44:45], -1\n"
        "s_waitcnt vmcnt(0)\n"
        "s_branch L_chkk2r%=\n"
        "L_chkk2%=:\n"
        "s_bitset1_b64 s[48:49], s52\n"
        "s_andn2_b64 s[98:99], s[48:49], s[44:45]\n"
        "s_cbranch_scc1 L_spink2%=\n"
        "s_branch L_chkk2r%=\n"
        "L_flagcY0%=:\n"
        "v_mov_b32 v247, 1\n"
        "s_mov_b64 exec, 1\n"
        "buffer_atomic_add v247, off, %[rsDC], s68 sc1\n"
        "s_mov_b64 exec, -1\n"
        "s_branch L_flagretcY0%=\n"
        "L_pushcY0%=:\n"
        "v_add_u32 v247, s69, %[brd]\n"
        "s_add_u32 s84, s67, 4096\n"
        "s_waitcnt lgkmcnt(0)\n"
        "ds_read_b128 v[240:243], v247\n"
        "s_waitcnt lgkmcnt(0)\n"
        "buffer_store_dwordx4 v[240:243], %[brd], %[rsD], s67 offen sc0 sc1\n"
        "s_nop 4\n"
        "ds_read_b128 v[240:243], v247 offset:1024\n"
        "s_waitcnt lgkmcnt(0)\n"
        "buffer_store_dwordx4 v[240:243], %[brd], %[rsD], s84 offen sc0 sc1\n"
        "s_nop 4\n"
        "s_branch L_pushretcY0%=\n"
        "L_ctlcY0%=:\n"
        "v_cmp_le_u32 vcc, s71, v246\n"
        "s_or_b64 s[44:45], s[44:45], vcc\n"
        "s_waitcnt lgkmcnt(0)\n"
        "s_andn2_b64 s[98:99], s[44:45], s[46:47]\n"
        "s_cbranch_scc1 L_pickcY00%=\n"
        "s_not_b64 s[98:99], s[46:47]\n"
        "L_pickcY00%=:\n"
        "s_ff1_i32_b64 s50, s[98:99]\n"
        "s_bitset1_b64 s[46:47], s50\n"
        "v_readlane_b32 s89, %[cvoff], s50\n"
        "s_nop 3\n"
        "s_lshr_b32 s89, s89, 7\n"
        "s_cmp_lt_i32 s50, 0\n"
        "s_cselect_b32 s89, 128, s89\n"
        "s_cselect_b32 s50, 0, s50\n"
        "v_mov_b32 v240, s89\n"
        "s_andn2_b64 s[98:99], s[44:45], s[46:47]\n"
        "s_cbranch_scc1 L_pickcY01%=\n"
        "s_not_b64 s[98:99], s[46:47]\n"
        "L_pickcY01%=:\n"
        "s_ff1_i32_b64 s51, s[98:99]\n"
        "s_bitset1_b64 s[46:47], s51\n"
        "v_readlane_b32 s89, %[cvoff], s51\n"
        "s_nop 3\n"
        "s_lshr_b32 s89, s89, 7\n"
        "s_cmp_lt_i32 s51, 0\n"
        "s_cselect_b32 s89, 128, s89\n"
        "s_cselect_b32 s51, 0, s51\n"
        "v_mov_b32 v241, s89\n"
        "v_mov_b32 v247, 147456\n"
        "ds_write_b64 v247, v[240:241]\n"
        "s_bitcmp1_b64 s[44:45], s54\n"
        "s_cbranch_scc0 L_nopfcY04%=\n"
        "s_lshl_b32 s85, s76, 16\n"
        "s_add_u32 s85, s85, s70\n"
        "buffer_load_dword %[xvoff], %[avoff], %[rsA], s85 offen\n"
        "buffer_load_dword %[xvoff], %[avoff], %[rsA], s85 offen offset:1024\n"
        "L_nopfcY04%=:\n"
        "s_bitcmp1_b64 s[44:45], s55\n"
        "s_cbranch_scc0 L_nopfcY05%=\n"
        "s_lshl_b32 s85, s77, 16\n"
        "s_add_u32 s85, s85, s70\n"
        "buffer_load_dword %[xvoff], %[avoff], %[rsA], s85 offen\n"
        "buffer_load_dword %[xvoff], %[avoff], %[rsA], s85 offen offset:1024\n"
        "L_nopfcY05%=:\n"
        "s_max_u32 s90, s58, 15\n"
        "s_bfm_b64 s[98:99], s90, 0\n"
        "s_cmp_ge_u32 s58, 64\n"
        "s_cselect_b64 s[98:99], -1, s[98:99]\n"
        "s_add_u32 s58, s58, 7\n"
        "s_andn2_b64 s[98:99], s[98:99], s[44:45]\n"
        "s_cbranch_scc0 L_nosmpcY0%=\n"
        "s_mov_b64 exec, s[98:99]\n"
        "s_movk_i32 s90, 8192\n"
        "buffer_load_dword v246, %[cvoff], %[rsC], s90 offen sc0 sc1\n"
        "s_mov_b64 exec, -1\n"
        "L_nosmpcY0%=:\n"
        "s_waitcnt lgkmcnt(0)\n"
        "s_branch L_ctlcY0r%=\n"
        "L_spink3%=:\n"
        "s_memrealtime s[92:93]\n"
        "s_waitcnt vmcnt(0) lgkmcnt(0)\n"
        "L_spink3%=_loop:\n"
        "v_cmp_le_u32 vcc, s71, v246\n"
        "s_or_b64 s[44:45], s[44:45], vcc\n"
        "s_andn2_b64 s[98:99], s[48:49], s[44:45]\n"
        "s_cbranch_scc0 L_chkk3r%=\n"
        "s_andn2_b64 s[98:99], s[48:49], s[44:45]\n"
        "s_mov_b64 exec, s[98:99]\n"
        "s_movk_i32 s90, 8192\n"
        "buffer_load_dword v246, %[cvoff], %[rsC], s90 offen sc0 sc1\n"
        "s_mov_b64 exec, -1\n"
        "s_movk_i32 s91, 4096\n"
        "buffer_load_dword v247, off, %[rsC], s91 sc1\n"
        "s_memrealtime s[94:95]\n"
        "s_waitcnt vmcnt(0) lgkmcnt(0)\n"
        "v_cmp_eq_u32 vcc, s60, v247\n"
        "s_cbranch_vccnz L_spink3%=_fail\n"
        "s_sub_u32 s94, s94, s92\n"
        "s_subb_u32 s95, s95, s93\n"
        "s_cmp_lg_u32 s95, 0\n"
        "s_cbranch_scc1 L_spink3%=_fail\n"
        "s_cmp_gt_u32 s94, 40000000\n"
        "s_cbranch_scc1 L_spink3%=_fail\n"
        "s_sleep 1\n"
        "s_branch L_spink3%=_loop\n"
        "L_spink3%=_fail:\n"
        "v_mov_b32 v247, s60\n"
        "s_nop 1\n"
        "buffer_store_dword v247, off, %[rsC], s91 sc1\n"
        "s_mov_b32 s61, 1\n"
        "s_mov_b64 s[44:45], -1\n"
        "s_waitcnt vmcnt(0)\n"
        "s_branch L_chkk3r%=\n"
        "L_chkk3%=:\n"
        "s_bitset1_b64 s[48:49], s53\n"
        "s_andn2_b64 s[98:99], s[48:49], s[44:45]\n"
        "s_cbranch_scc1 L_spink3%=\n"
        "s_branch L_chkk3r%=\n"
        "L_spink4%=:\n"
        "s_memrealtime s[92:93]\n"
        "s_waitcnt vmcnt(0) lgkmcnt(0)\n"
        "L_spink4%=_loop:\n"
        "v_cmp_le_u32 vcc, s71, v246\n"
        "s_or_b64 s[44:45], s[44:45], vcc\n"
        "s_andn2_b64 s[98:99], s[48:49], s[44:45]\n"
        "s_cbranch_scc0 L_chkk4r%=\n"
        "s_andn2_b64 s[98:99], s[48:49], s[44:45]\n"
        "s_mov_b64 exec, s[98:99]\n"
        "s_movk_i32 s90, 8192\n"
        "buffer_load_dword v246, %[cvoff], %[rsC], s90 offen sc0 sc1\n"
        "s_mov_b64 exec, -1\n"
        "s_movk_i32 s91, 4096\n"
        "buffer_load_dword v247, off, %[rsC], s91 sc1\n"
        "s_memrealtime s[94:95]\n"
        "s_waitcnt vmcnt(0) lgkmcnt(0)\n"
        "v_cmp_eq_u32 vcc, s60, v247\n"
        "s_cbranch_vccnz L_spink4%=_fail\n"
        "s_sub_u32 s94, s94, s92\n"
        "s_subb_u32 s95, s95, s93\n"
        "s_cmp_lg_u32 s95, 0\n"
        "s_cbranch_scc1 L_spink4%=_fail\n"
        "s_cmp_gt_u32 s94, 40000000\n"
        "s_cbranch_scc1 L_spink4%=_fail\n"
        "s_sleep 1\n"
        "s_branch L_spink4%=_loop\n"
        "L_spink4%=_fail:\n"
        "v_mov_b32 v247, s60\n"
        "s_nop 1\n"
        "buffer_store_dword v247, off, %[rsC], s91 sc1\n"
        "s_mov_b32 s61, 1\n"
        "s_mov_b64 s[44:45], -1\n"
        "s_waitcnt vmcnt(0)\n"
        "s_branch L_chkk4r%=\n"
        "L_chkk4%=:\n"
        "s_bitset1_b64 s[48:49], s54\n"
        "s_andn2_b64 s[98:99], s[48:49], s[44:45]\n"
        "s_cbranch_scc1 L_spink4%=\n"
        "s_branch L_chkk4r%=\n"
        "L_flagcY1%=:\n"
        "v_mov_b32 v247, 1\n"
        "s_mov_b64 exec, 1\n"
        "buffer_atomic_add v247, off, %[rsDC], s68 sc1\n"
        "s_mov_b64 exec, -1\n"
        "s_branch L_flagretcY1%=\n"
        "L_pushcY1%=:\n"
        "v_add_u32 v247, s69, %[brd]\n"
        "s_add_u32 s84, s67, 4096\n"
        "s_waitcnt lgkmcnt(0)\n"
        "ds_read_b128 v[240:243], v247\n"
        "s_waitcnt lgkmcnt(0)\n"
        "buffer_store_dwordx4 v[240:243], %[brd], %[rsD], s67 offen sc0 sc1\n"
        "s_nop 4\n"
        "ds_read_b128 v[240:243], v247 offset:1024\n"
        "s_waitcnt lgkmcnt(0)\n"
        "buffer_store_dwordx4 v[240:243], %[brd], %[rsD], s84 offen sc0 sc1\n"
        "s_nop 4\n"
        "s_branch L_pushretcY1%=\n"
        "L_ctlcY1%=:\n"
        "v_cmp_le_u32 vcc, s71, v246\n"
        "s_or_b64 s[44:45], s[44:45], vcc\n"
        "s_waitcnt lgkmcnt(0)\n"
        "s_andn2_b64 s[98:99], s[44:45], s[46:47]\n"
        "s_cbranch_scc1 L_pickcY10%=\n"
        "s_not_b64 s[98:99], s[46:47]\n"
        "L_pickcY10%=:\n"
        "s_ff1_i32_b64 s52, s[98:99]\n"
        "s_bitset1_b64 s[46:47], s52\n"
        "v_readlane_b32 s89, %[cvoff], s52\n"
        "s_nop 3\n"
        "s_lshr_b32 s89, s89, 7\n"
        "s_cmp_lt_i32 s52, 0\n"
        "s_cselect_b32 s89, 128, s89\n"
        "s_cselect_b32 s52, 0, s52\n"
        "v_mov_b32 v240, s89\n"
        "s_andn2_b64 s[98:99], s[44:45], s[46:47]\n"
        "s_cbranch_scc1 L_pickcY11%=\n"
        "s_not_b64 s[98:99], s[46:47]\n"
        "L_pickcY11%=:\n"
        "s_ff1_i32_b64 s53, s[98:99]\n"
        "s_bitset1_b64 s[46:47], s53\n"
        "v_readlane_b32 s89, %[cvoff], s53\n"
        "s_nop 3\n"
        "s_lshr_b32 s89, s89, 7\n"
        "s_cmp_lt_i32 s53, 0\n"
        "s_cselect_b32 s89, 128, s89\n"
        "s_cselect_b32 s53, 0, s53\n"
        "v_mov_b32 v241, s89\n"
        "v_mov_b32 v247, 147464\n"
        "ds_write_b64 v247, v[240:241]\n"
        "s_bitcmp1_b64 s[44:45], s56\n"
        "s_cbranch_scc0 L_nopfcY14%=\n"
        "s_lshl_b32 s85, s78, 16\n"
        "s_add_u32 s85, s85, s70\n"
        "buffer_load_dword %[xvoff], %[avoff], %[rsA], s85 offen\n"
        "buffer_load_dword %[xvoff], %[avoff], %[rsA], s85 offen offset:1024\n"
        "L_nopfcY14%=:\n"
        "s_bitcmp1_b64 s[44:45], s57\n"
        "s_cbranch_scc0 L_nopfcY15%=\n"
        "s_lshl_b32 s85, s79, 16\n"
        "s_add_u32 s85, s85, s70\n"
        "buffer_load_dword %[xvoff], %[avoff], %[rsA], s85 offen\n"
        "buffer_load_dword %[xvoff], %[avoff], %[rsA], s85 offen offset:1024\n"
        "L_nopfcY15%=:\n"
        "s_max_u32 s90, s58, 15\n"
        "s_bfm_b64 s[98:99], s90, 0\n"
        "s_cmp_ge_u32 s58, 64\n"
        "s_cselect_b64 s[98:99], -1, s[98:99]\n"
        "s_add_u32 s58, s58, 7\n"
        "s_andn2_b64 s[98:99], s[98:99], s[44:45]\n"
        "s_cbranch_scc0 L_nosmpcY1%=\n"
        "s_mov_b64 exec, s[98:99]\n"
        "s_movk_i32 s90, 8192\n"
        "buffer_load_dword v246, %[cvoff], %[rsC], s90 offen sc0 sc1\n"
        "s_mov_b64 exec, -1\n"
        "L_nosmpcY1%=:\n"
        "s_waitcnt lgkmcnt(0)\n"
        "s_branch L_ctlcY1r%=\n"
        "L_spink5%=:\n"
        "s_memrealtime s[92:93]\n"
        "s_waitcnt vmcnt(0) lgkmcnt(0)\n"
        "L_spink5%=_loop:\n"
        "v_cmp_le_u32 vcc, s71, v246\n"
        "s_or_b64 s[44:45], s[44:45], vcc\n"
        "s_andn2_b64 s[98:99], s[48:49], s[44:45]\n"
        "s_cbranch_scc0 L_chkk5r%=\n"
        "s_andn2_b64 s[98:99], s[48:49], s[44:45]\n"
        "s_mov_b64 exec, s[98:99]\n"
        "s_movk_i32 s90, 8192\n"
        "buffer_load_dword v246, %[cvoff], %[rsC], s90 offen sc0 sc1\n"
        "s_mov_b64 exec, -1\n"
        "s_movk_i32 s91, 4096\n"
        "buffer_load_dword v247, off, %[rsC], s91 sc1\n"
        "s_memrealtime s[94:95]\n"
        "s_waitcnt vmcnt(0) lgkmcnt(0)\n"
        "v_cmp_eq_u32 vcc, s60, v247\n"
        "s_cbranch_vccnz L_spink5%=_fail\n"
        "s_sub_u32 s94, s94, s92\n"
        "s_subb_u32 s95, s95, s93\n"
        "s_cmp_lg_u32 s95, 0\n"
        "s_cbranch_scc1 L_spink5%=_fail\n"
        "s_cmp_gt_u32 s94, 40000000\n"
        "s_cbranch_scc1 L_spink5%=_fail\n"
        "s_sleep 1\n"
        "s_branch L_spink5%=_loop\n"
        "L_spink5%=_fail:\n"
        "v_mov_b32 v247, s60\n"
        "s_nop 1\n"
        "buffer_store_dword v247, off, %[rsC], s91 sc1\n"
        "s_mov_b32 s61, 1\n"
        "s_mov_b64 s[44:45], -1\n"
        "s_waitcnt vmcnt(0)\n"
        "s_branch L_chkk5r%=\n"
        "L_chkk5%=:\n"
        "s_bitset1_b64 s[48:49], s55\n"
        "s_andn2_b64 s[98:99], s[48:49], s[44:45]\n"
        "s_cbranch_scc1 L_spink5%=\n"
        "s_branch L_chkk5r%=\n"
        "L_spink6%=:\n"
        "s_memrealtime s[92:93]\n"
        "s_waitcnt vmcnt(0) lgkmcnt(0)\n"
        "L_spink6%=_loop:\n"
        "v_cmp_le_u32 vcc, s71, v246\n"
        "s_or_b64 s[44:45], s[44:45], vcc\n"
        "s_andn2_b64 s[98:99], s[48:49], s[44:45]\n"
        "s_cbranch_scc0 L_chkk6r%=\n"
        "s_andn2_b64 s[98:99], s[48:49], s[44:45]\n"
        "s_mov_b64 exec, s[98:99]\n"
        "s_movk_i32 s90, 8192\n"
        "buffer_load_dword v246, %[cvoff], %[rsC], s90 offen sc0 sc1\n"
        "s_mov_b64 exec, -1\n"
        "s_movk_i32 s91, 4096\n"
        "buffer_load_dword v247, off, %[rsC], s91 sc1\n"
        "s_memrealtime s[94:95]\n"
        "s_waitcnt vmcnt(0) lgkmcnt(0)\n"
        "v_cmp_eq_u32 vcc, s60, v247\n"
        "s_cbranch_vccnz L_spink6%=_fail\n"
        "s_sub_u32 s94, s94, s92\n"
        "s_subb_u32 s95, s95, s93\n"
        "s_cmp_lg_u32 s95, 0\n"
        "s_cbranch_scc1 L_spink6%=_fail\n"
        "s_cmp_gt_u32 s94, 40000000\n"
        "s_cbranch_scc1 L_spink6%=_fail\n"
        "s_sleep 1\n"
        "s_branch L_spink6%=_loop\n"
        "L_spink6%=_fail:\n"
        "v_mov_b32 v247, s60\n"
        "s_nop 1\n"
        "buffer_store_dword v247, off, %[rsC], s91 sc1\n"
        "s_mov_b32 s61, 1\n"
        "s_mov_b64 s[44:45], -1\n"
        "s_waitcnt vmcnt(0)\n"
        "s_branch L_chkk6r%=\n"
        "L_chkk6%=:\n"
        "s_bitset1_b64 s[48:49], s56\n"
        "s_andn2_b64 s[98:99], s[48:49], s[44:45]\n"
        "s_cbranch_scc1 L_spink6%=\n"
        "s_branch L_chkk6r%=\n"
        "L_flagcY2%=:\n"
        "v_mov_b32 v247, 1\n"
        "s_mov_b64 exec, 1\n"
        "buffer_atomic_add v247, off, %[rsDC], s68 sc1\n"
        "s_mov_b64 exec, -1\n"
        "s_branch L_flagretcY2%=\n"
        "L_pushcY2%=:\n"
        "v_add_u32 v247, s69, %[brd]\n"
        "s_add_u32 s84, s67, 4096\n"
        "s_waitcnt lgkmcnt(0)\n"
        "ds_read_b128 v[240:243], v247\n"
        "s_waitcnt lgkmcnt(0)\n"
        "buffer_store_dwordx4 v[240:243], %[brd], %[rsD], s67 offen sc0 sc1\n"
        "s_nop 4\n"
        "ds_read_b128 v[240:243], v247 offset:1024\n"
        "s_waitcnt lgkmcnt(0)\n"
        "buffer_store_dwordx4 v[240:243], %[brd], %[rsD], s84 offen sc0 sc1\n"
        "s_nop 4\n"
        "s_branch L_pushretcY2%=\n"
        "L_ctlcY2%=:\n"
        "v_cmp_le_u32 vcc, s71, v246\n"
        "s_or_b64 s[44:45], s[44:45], vcc\n"
        "s_waitcnt lgkmcnt(0)\n"
        "s_andn2_b64 s[98:99], s[44:45], s[46:47]\n"
        "s_cbranch_scc1 L_pickcY20%=\n"
        "s_not_b64 s[98:99], s[46:47]\n"
        "L_pickcY20%=:\n"
        "s_ff1_i32_b64 s54, s[98:99]\n"
        "s_bitset1_b64 s[46:47], s54\n"
        "v_readlane_b32 s89, %[cvoff], s54\n"
        "s_nop 3\n"
        "s_lshr_b32 s89, s89, 7\n"
        "s_cmp_lt_i32 s54, 0\n"
        "s_cselect_b32 s89, 128, s89\n"
        "s_cselect_b32 s54, 0, s54\n"
        "v_mov_b32 v240, s89\n"
        "s_andn2_b64 s[98:99], s[44:45], s[46:47]\n"
        "s_cbranch_scc1 L_pickcY21%=\n"
        "s_not_b64 s[98:99], s[46:47]\n"
        "L_pickcY21%=:\n"
        "s_ff1_i32_b64 s55, s[98:99]\n"
        "s_bitset1_b64 s[46:47], s55\n"
        "v_readlane_b32 s89, %[cvoff], s55\n"
        "s_nop 3\n"
        "s_lshr_b32 s89, s89, 7\n"
        "s_cmp_lt_i32 s55, 0\n"
        "s_cselect_b32 s89, 128, s89\n"
        "s_cselect_b32 s55, 0, s55\n"
        "v_mov_b32 v241, s89\n"
        "v_mov_b32 v247, 147472\n"
        "ds_write_b64 v247, v[240:241]\n"
        "s_bitcmp1_b64 s[44:45], s50\n"
        "s_cbranch_scc0 L_nopfcY24%=\n"
        "s_lshl_b32 s85, s72, 16\n"
        "s_add_u32 s85, s85, s70\n"
        "buffer_load_dword %[xvoff], %[avoff], %[rsA], s85 offen\n"
        "buffer_load_dword %[xvoff], %[avoff], %[rsA], s85 offen offset:1024\n"
        "L_nopfcY24%=:\n"
        "s_bitcmp1_b64 s[44:45], s51\n"
        "s_cbranch_scc0 L_nopfcY25%=\n"
        "s_lshl_b32 s85, s73, 16\n"
        "s_add_u32 s85, s85, s70\n"
        "buffer_load_dword %[xvoff], %[avoff], %[rsA], s85 offen\n"
        "buffer_load_dword %[xvoff], %[avoff], %[rsA], s85 offen offset:1024\n"
        "L_nopfcY25%=:\n"
        "s_max_u32 s90, s58, 15\n"
        "s_bfm_b64 s[98:99], s90, 0\n"
        "s_cmp_ge_u32 s58, 64\n"
        "s_cselect_b64 s[98:99], -1, s[98:99]\n"
        "s_add_u32 s58, s58, 7\n"
        "s_andn2_b64 s[98:99], s[98:99], s[44:45]\n"
        "s_cbranch_scc0 L_nosmpcY2%=\n"
        "s_mov_b64 exec, s[98:99]\n"
        "s_movk_i32 s90, 8192\n"
        "buffer_load_dword v246, %[cvoff], %[rsC], s90 offen sc0 sc1\n"
        "s_mov_b64 exec, -1\n"
        "L_nosmpcY2%=:\n"
        "s_waitcnt lgkmcnt(0)\n"
        "s_branch L_ctlcY2r%=\n"
        "L_spink7%=:\n"
        "s_memrealtime s[92:93]\n"
        "s_waitcnt vmcnt(0) lgkmcnt(0)\n"
        "L_spink7%=_loop:\n"
        "v_cmp_le_u32 vcc, s71, v246\n"
        "s_or_b64 s[44:45], s[44:45], vcc\n"
        "s_andn2_b64 s[98:99], s[48:49], s[44:45]\n"
        "s_cbranch_scc0 L_chkk7r%=\n"
        "s_andn2_b64 s[98:99], s[48:49], s[44:45]\n"
        "s_mov_b64 exec, s[98:99]\n"
        "s_movk_i32 s90, 8192\n"
        "buffer_load_dword v246, %[cvoff], %[rsC], s90 offen sc0 sc1\n"
        "s_mov_b64 exec, -1\n"
        "s_movk_i32 s91, 4096\n"
        "buffer_load_dword v247, off, %[rsC], s91 sc1\n"
        "s_memrealtime s[94:95]\n"
        "s_waitcnt vmcnt(0) lgkmcnt(0)\n"
        "v_cmp_eq_u32 vcc, s60, v247\n"
        "s_cbranch_vccnz L_spink7%=_fail\n"
        "s_sub_u32 s94, s94, s92\n"
        "s_subb_u32 s95, s95, s93\n"
        "s_cmp_lg_u32 s95, 0\n"
        "s_cbranch_scc1 L_spink7%=_fail\n"
        "s_cmp_gt_u32 s94, 40000000\n"
        "s_cbranch_scc1 L_spink7%=_fail\n"
        "s_sleep 1\n"
        "s_branch L_spink7%=_loop\n"
        "L_spink7%=_fail:\n"
        "v_mov_b32 v247, s60\n"
        "s_nop 1\n"
        "buffer_store_dword v247, off, %[rsC], s91 sc1\n"
        "s_mov_b32 s61, 1\n"
        "s_mov_b64 s[44:45], -1\n"
        "s_waitcnt vmcnt(0)\n"
        "s_branch L_chkk7r%=\n"
        "L_chkk7%=:\n"
        "s_bitset1_b64 s[48:49], s57\n"
        "s_andn2_b64 s[98:99], s[48:49], s[44:45]\n"
        "s_cbranch_scc1 L_spink7%=\n"
        "s_branch L_chkk7r%=\n"
        "L_spink8%=:\n"
        "s_memrealtime s[92:93]\n"
        "s_waitcnt vmcnt(0) lgkmcnt(0)\n"
        "L_spink8%=_loop:\n"
        "v_cmp_le_u32 vcc, s71, v246\n"
        "s_or_b64 s[44:45], s[44:45], vcc\n"
        "s_andn2_b64 s[98:99], s[48:49], s[44:45]\n"
        "s_cbranch_scc0 L_chkk8r%=\n"
        "s_andn2_b64 s[98:99], s[48:49], s[44:45]\n"
        "s_mov_b64 exec, s[98:99]\n"
        "s_movk_i32 s90, 8192\n"
        "buffer_load_dword v246, %[cvoff], %[rsC], s90 offen sc0 sc1\n"
        "s_mov_b64 exec, -1\n"
        "s_movk_i32 s91, 4096\n"
        "buffer_load_dword v247, off, %[rsC], s91 sc1\n"
        "s_memrealtime s[94:95]\n"
        "s_waitcnt vmcnt(0) lgkmcnt(0)\n"
        "v_cmp_eq_u32 vcc, s60, v247\n"
        "s_cbranch_vccnz L_spink8%=_fail\n"
        "s_sub_u32 s94, s94, s92\n"
        "s_subb_u32 s95, s95, s93\n"
        "s_cmp_lg_u32 s95, 0\n"
        "s_cbranch_scc1 L_spink8%=_fail\n"
        "s_cmp_gt_u32 s94, 40000000\n"
        "s_cbranch_scc1 L_spink8%=_fail\n"
        "s_sleep 1\n"
        "s_branch L_spink8%=_loop\n"
        "L_spink8%=_fail:\n"
        "v_mov_b32 v247, s60\n"
        "s_nop 1\n"
        "buffer_store_dword v247, off, %[rsC], s91 sc1\n"
        "s_mov_b32 s61, 1\n"
        "s_mov_b64 s[44:45], -1\n"
        "s_waitcnt vmcnt(0)\n"
        "s_branch L_chkk8r%=\n"
        "L_chkk8%=:\n"
        "s_bitset1_b64 s[48:49], s50\n"
        "s_andn2_b64 s[98:99], s[48:49], s[44:45]\n"
        "s_cbranch_scc1 L_spink8%=\n"
        "s_branch L_chkk8r%=\n"
        "L_flagcY3%=:\n"
        "v_mov_b32 v247, 1\n"
        "s_mov_b64 exec, 1\n"
        "buffer_atomic_add v247, off, %[rsDC], s68 sc1\n"
        "s_mov_b64 exec, -1\n"
        "s_branch L_flagretcY3%=\n"
        "L_pushcY3%=:\n"
        "v_add_u32 v247, s69, %[brd]\n"
        "s_add_u32 s84, s67, 4096\n"
        "s_waitcnt lgkmcnt(0)\n"
        "ds_read_b128 v[240:243], v247\n"
        "s_waitcnt lgkmcnt(0)\n"
        "buffer_store_dwordx4 v[240:243], %[brd], %[rsD], s67 offen sc0 sc1\n"
        "s_nop 4\n"
        "ds_read_b128 v[240:243], v247 offset:1024\n"
        "s_waitcnt lgkmcnt(0)\n"
        "buffer_store_dwordx4 v[240:243], %[brd], %[rsD], s84 offen sc0 sc1\n"
        "s_nop 4\n"
        "s_branch L_pushretcY3%=\n"
        "L_ctlcY3%=:\n"
        "v_cmp_le_u32 vcc, s71, v246\n"
        "s_or_b64 s[44:45], s[44:45], vcc\n"
        "s_waitcnt lgkmcnt(0)\n"
        "s_andn2_b64 s[98:99], s[44:45], s[46:47]\n"
        "s_cbranch_scc1 L_pickcY30%=\n"
        "s_not_b64 s[98:99], s[46:47]\n"
        "L_pickcY30%=:\n"
        "s_ff1_i32_b64 s56, s[98:99]\n"
        "s_bitset1_b64 s[46:47], s56\n"
        "v_readlane_b32 s89, %[cvoff], s56\n"
        "s_nop 3\n"
        "s_lshr_b32 s89, s89, 7\n"
        "s_cmp_lt_i32 s56, 0\n"
        "s_cselect_b32 s89, 128, s89\n"
        "s_cselect_b32 s56, 0, s56\n"
        "v_mov_b32 v240, s89\n"
        "s_andn2_b64 s[98:99], s[44:45], s[46:47]\n"
        "s_cbranch_scc1 L_pickcY31%=\n"
        "s_not_b64 s[98:99], s[46:47]\n"
        "L_pickcY31%=:\n"
        "s_ff1_i32_b64 s57, s[98:99]\n"
        "s_bitset1_b64 s[46:47], s57\n"
        "v_readlane_b32 s89, %[cvoff], s57\n"
        "s_nop 3\n"
        "s_lshr_b32 s89, s89, 7\n"
        "s_cmp_lt_i32 s57, 0\n"
        "s_cselect_b32 s89, 128, s89\n"
        "s_cselect_b32 s57, 0, s57\n"
        "v_mov_b32 v241, s89\n"
        "v_mov_b32 v247, 147480\n"
        "ds_write_b64 v247, v[240:241]\n"
        "s_bitcmp1_b64 s[44:45], s52\n"
        "s_cbranch_scc0 L_nopfcY34%=\n"
        "s_lshl_b32 s85, s74, 16\n"
        "s_add_u32 s85, s85, s70\n"
        "buffer_load_dword %[xvoff], %[avoff], %[rsA], s85 offen\n"
        "buffer_load_dword %[xvoff], %[avoff], %[rsA], s85 offen offset:1024\n"
        "L_nopfcY34%=:\n"
        "s_bitcmp1_b64 s[44:45], s53\n"
        "s_cbranch_scc0 L_nopfcY35%=\n"
        "s_lshl_b32 s85, s75, 16\n"
        "s_add_u32 s85, s85, s70\n"
        "buffer_load_dword %[xvoff], %[avoff], %[rsA], s85 offen\n"
        "buffer_load_dword %[xvoff], %[avoff], %[rsA], s85 offen offset:1024\n"
        "L_nopfcY35%=:\n"
        "s_max_u32 s90, s58, 15\n"
        "s_bfm_b64 s[98:99], s90, 0\n"
        "s_cmp_ge_u32 s58, 64\n"
        "s_cselect_b64 s[98:99], -1, s[98:99]\n"
        "s_add_u32 s58, s58, 7\n"
        "s_andn2_b64 s[98:99], s[98:99], s[44:45]\n"
        "s_cbranch_scc0 L_nosmpcY3%=\n"
        "s_mov_b64 exec, s[98:99]\n"
        "s_movk_i32 s90, 8192\n"
        "buffer_load_dword v246, %[cvoff], %[rsC], s90 offen sc0 sc1\n"
        "s_mov_b64 exec, -1\n"
        "L_nosmpcY3%=:\n"
        "s_waitcnt lgkmcnt(0)\n"
        "s_branch L_ctlcY3r%=\n"
        "L_spinkNone%=:\n"
        "s_memrealtime s[92:93]\n"
        "s_waitcnt vmcnt(0) lgkmcnt(0)\n"
        "L_spinkNone%=_loop:\n"
        "v_cmp_le_u32 vcc, s71, v246\n"
        "s_or_b64 s[44:45], s[44:45], vcc\n"
        "s_andn2_b64 s[98:99], s[48:49], s[44:45]\n"
        "s_cbranch_scc0 L_chkkNoner%=\n"
        "s_andn2_b64 s[98:99], s[48:49], s[44:45]\n"
        "s_mov_b64 exec, s[98:99]\n"
        "s_movk_i32 s90, 8192\n"
        "buffer_load_dword v246, %[cvoff], %[rsC], s90 offen sc0 sc1\n"
        "s_mov_b64 exec, -1\n"
        "s_movk_i32 s91, 4096\n"
        "buffer_load_dword v247, off, %[rsC], s91 sc1\n"
        "s_memrealtime s[94:95]\n"
        "s_waitcnt vmcnt(0) lgkmcnt(0)\n"
        "v_cmp_eq_u32 vcc, s60, v247\n"
        "s_cbranch_vccnz L_spinkNone%=_fail\n"
        "s_sub_u32 s94, s94, s92\n"
        "s_subb_u32 s95, s95, s93\n"
        "s_cmp_lg_u32 s95, 0\n"
        "s_cbranch_scc1 L_spinkNone%=_fail\n"
        "s_cmp_gt_u32 s94, 40000000\n"
        "s_cbranch_scc1 L_spinkNone%=_fail\n"
        "s_sleep 1\n"
        "s_branch L_spinkNone%=_loop\n"
        "L_spinkNone%=_fail:\n"
        "v_mov_b32 v247, s60\n"
        "s_nop 1\n"
        "buffer_store_dword v247, off, %[rsC], s91 sc1\n"
        "s_mov_b32 s61, 1\n"
        "s_mov_b64 s[44:45], -1\n"
        "s_waitcnt vmcnt(0)\n"
        "s_branch L_chkkNoner%=\n"
        "L_chkkNone%=:\n"
        "s_cmp_eq_u32 s64, 48\n"
        "s_cselect_b64 s[48:49], -1, s[48:49]\n"
        "s_andn2_b64 s[98:99], s[48:49], s[44:45]\n"
        "s_cbranch_scc1 L_spinkNone%=\n"
        "s_branch L_chkkNoner%=\n"
        "L_spink9%=:\n"
        "s_memrealtime s[92:93]\n"
        "s_waitcnt vmcnt(0) lgkmcnt(0)\n"
        "L_spink9%=_loop:\n"
        "v_cmp_le_u32 vcc, s71, v246\n"
        "s_or_b64 s[44:45], s[44:45], vcc\n"
        "s_andn2_b64 s[98:99], s[48:49], s[44:45]\n"
        "s_cbranch_scc0 L_chkk9r%=\n"
        "s_andn2_b64 s[98:99], s[48:49], s[44:45]\n"
        "s_mov_b64 exec, s[98:99]\n"
        "s_movk_i32 s90, 8192\n"
        "buffer_load_dword v246, %[cvoff], %[rsC], s90 offen sc0 sc1\n"
        "s_mov_b64 exec, -1\n"
        "s_movk_i32 s91, 4096\n"
        "buffer_load_dword v247, off, %[rsC], s91 sc1\n"
        "s_memrealtime s[94:95]\n"
        "s_waitcnt vmcnt(0) lgkmcnt(0)\n"
        "v_cmp_eq_u32 vcc, s60, v247\n"
        "s_cbranch_vccnz L_spink9%=_fail\n"
        "s_sub_u32 s94, s94, s92\n"
        "s_subb_u32 s95, s95, s93\n"
        "s_cmp_lg_u32 s95, 0\n"
        "s_cbranch_scc1 L_spink9%=_fail\n"
        "s_cmp_gt_u32 s94, 40000000\n"
        "s_cbranch_scc1 L_spink9%=_fail\n"
        "s_sleep 1\n"
        "s_branch L_spink9%=_loop\n"
        "L_spink9%=_fail:\n"
        "v_mov_b32 v247, s60\n"
        "s_nop 1\n"
        "buffer_store_dword v247, off, %[rsC], s91 sc1\n"
        "s_mov_b32 s61, 1\n"
        "s_mov_b64 s[44:45], -1\n"
        "s_waitcnt vmcnt(0)\n"
        "s_branch L_chkk9r%=\n"
        "L_chkk9%=:\n"
        "s_bitset1_b64 s[48:49], s51\n"
        "s_andn2_b64 s[98:99], s[48:49], s[44:45]\n"
        "s_cbranch_scc1 L_spink9%=\n"
        "s_branch L_chkk9r%=\n"
        "L_end%=:\n"
        : [xvoff] "+v"(xvoff)
        : [rsW] "s"(rsW), [rsA] "s"(rsA), [rsC] "s"(rsC), [rsD] "s"(rsD), [rsDC] "s"(rsDC),
          [rsX] "s"(rsX), [rsO] "s"(rsO), [rank] "s"(rank), [bid] "s"(bid), [wave] "s"(wave),
          [wvoff] "v"(wvoff), [bwr] "v"(bwr), [brd] "v"(brd), [avoff] "v"(avoff),
          [ooff] "v"(ooff), [cvoff] "v"(cvoff)
        : "v0", "v1", "v2", "v3", "v4", "v5", "v6", "v7", "v8", "v9", "v10", "v11", "v12", "v13", "v14", "v15", "v16", "v17", "v18", "v19", "v20", "v21", "v22", "v23", "v24", "v25", "v26", "v27", "v28", "v29", "v30", "v31", "v32", "v33", "v34", "v35", "v36", "v37", "v38", "v39", "v40", "v41", "v42", "v43", "v44", "v45", "v46", "v47", "v48", "v49", "v50", "v51", "v52", "v53", "v54", "v55", "v56", "v57", "v58", "v59", "v60", "v61", "v62", "v63", "v64", "v65", "v66", "v67", "v68", "v69", "v70", "v71", "v72", "v73", "v74", "v75", "v76", "v77", "v78", "v79", "v80", "v81", "v82", "v83", "v84", "v85", "v86", "v87", "v88", "v89", "v90", "v91", "v92", "v93", "v94", "v95", "v96", "v97", "v98", "v99", "v100", "v101", "v102", "v103", "v104", "v105", "v106", "v107", "v108", "v109", "v110", "v111", "v112", "v113", "v114", "v115", "v116", "v117", "v118", "v119", "v120", "v121", "v122", "v123", "v124", "v125", "v126", "v127", "v128", "v129", "v130", "v131", "v132", "v133", "v134", "v135", "v136", "v137", "v138", "v139", "v140", "v141", "v142", "v143", "v144", "v145", "v146", "v147", "v148", "v149", "v150", "v151", "v152", "v153", "v154", "v155", "v156", "v157", "v158", "v159", "v160", "v161", "v162", "v163", "v164", "v165", "v166", "v167", "v168", "v169", "v170", "v171", "v172", "v173", "v174", "v175", "v176", "v177", "v178", "v179", "v180", "v181", "v182", "v183", "v184", "v185", "v186", "v187", "v188", "v189", "v190", "v191", "v192", "v193", "v194", "v195", "v196", "v197", "v198", "v199", "v200", "v201", "v202", "v203", "v204", "v205", "v206", "v207", "v208", "v209", "v210", "v211", "v212", "v213", "v214", "v215", "v216", "v217", "v218", "v219", "v220", "v221", "v222", "v223", "v224", "v225", "v226", "v227", "v228", "v229", "v230", "v231", "v232", "v233", "v234", "v235", "v236", "v237", "v238", "v239", "v240", "v241", "v242", "v243", "v244", "v245", "v246", "v247", "s44", "s45", "s46", "s47", "s48", "s49", "s50", "s51", "s52", "s53", "s54", "s55", "s56", "s57", "s58", "s59", "s60", "s61", "s62", "s63", "s64", "s65", "s66", "s67", "s68", "s69", "s70", "s71", "s72", "s73", "s74", "s75", "s76", "s77", "s78", "s79", "s80", "s81", "s82", "s83", "s84", "s85", "s86", "s87", "s88", "s89", "s90", "s91", "s92", "s93", "s94", "s95", "s96", "s97", "s98", "s99", "vcc", "scc", "memory");
}

}

extern "C" void kernel_launch(int rank, int nranks, ncclComm_t comm,
                              hipStream_t stream,
                              void* const* d_in, const int* in_sizes, int n_in,
                              void* d_out, int out_size,
                              const int* in_dtypes, int out_dtype,
                              void* const* d_out_all,
                              void* const* d_ws_all, size_t ws_bytes) {
    (void)comm; (void)in_sizes; (void)n_in; (void)out_size; (void)in_dtypes; (void)out_dtype; (void)d_out_all;
    if (nranks != NR || (unsigned)rank >= (unsigned)NR || ws_bytes < WS_NEED) {
        fprintf(stderr, "unsupported configuration\n");
        return;
    }
    Params p;
    p.x = (const float*)d_in[0];
    p.w = (const float*)d_in[1];
    p.out = (float*)d_out;
    for (int i = 0; i < NR; ++i) p.ws[i] = (char*)d_ws_all[(rank + i) % NR];
    p.rank = rank;
    a2a_gemm_kernel<<<dim3(NWG), dim3(NTHR), 0, stream>>>(p);
    HIP_CHECK(hipGetLastError());
}
```
